```python
import numpy as np

kernel_name = "gpu_gemm_a2a_m4096_k4096_n4096_e4m3e4m3_dqrelu"


ABSMAX_RTOL = 0.03

COMPUTE_DTYPE = "float32"
EPILOGUE = "dequant_relu"
IN_QUANT = None
OUT_QUANT = None
IN_FP8 = "e4m3e4m3"
IN_DTYPES = {"x": "e4m3", "w_mat": "e4m3",
             "scale_x": "float32", "scale_w": "float32"}
WORLD_SIZE = 8

def _snap_fp8(x, mant_bits, max_abs, min_normal):
    a = np.abs(x.astype(np.float64))
    _m, _e = np.frexp(a)
    q = np.where(a >= min_normal, np.ldexp(1.0, _e - 1 - mant_bits),
                 min_normal * 2.0 ** -mant_bits)
    snapped = np.minimum(np.round(a / q) * q, max_abs)
    return (np.sign(x) * snapped).astype(np.float32)

def setup_inputs():

    x = _snap_fp8(np.random.randn(4096, 4096).astype(np.float32), 3, 448.0, 2.0 ** -6)
    w_mat = _snap_fp8(np.random.randn(4096, 4096).astype(np.float32), 3, 448.0, 2.0 ** -6)
    scale_x = (0.02 * (1.0 + np.random.rand(1))).astype(np.float32)
    scale_w = (0.02 * (1.0 + np.random.rand(1))).astype(np.float32)
    return {"x": x, "w_mat": w_mat, "scale_x": scale_x, "scale_w": scale_w}

def reference(x, w_mat, scale_x, scale_w):
    acc = (x @ w_mat).astype(np.float32)
    y = (acc * (scale_x * scale_w)).astype(np.float32)
    return np.maximum(y, 0.0).astype(np.float32)
```

```cpp
#include <hip/hip_runtime.h>
#include <rccl/rccl.h>
#include <cstdio>

#define HIP_CHECK(expr) do { hipError_t _e = (expr); if (_e != hipSuccess) { \
    fprintf(stderr, "HIP error %s at %s:%d\n", hipGetErrorString(_e), __FILE__, __LINE__); \
    return; } } while (0)

typedef int v8i __attribute__((ext_vector_type(8)));
typedef int v4i __attribute__((ext_vector_type(4)));
typedef int v2i __attribute__((ext_vector_type(2)));
typedef float v16f __attribute__((ext_vector_type(16)));
typedef float v4f __attribute__((ext_vector_type(4)));
typedef unsigned u32x4 __attribute__((ext_vector_type(4)));
#define LAS __attribute__((address_space(3)))

constexpr int NR = 8;
constexpr int MS = 512;
constexpr int KK = 4096;
constexpr int NN = 4096;
constexpr int NS = 512;
constexpr int BM = 64, BN = 128, BK = 128;
constexpr int NSTAGE = KK / BK;
constexpr int DEPTH = 6;
constexpr int A_BYTES = BM * BK;
constexpr int B_BYTES = BK * BN;
constexpr int SLOT = A_BYTES + B_BYTES;
constexpr int LDS_BYTES = DEPTH * SLOT;
constexpr int GRID = 256;
constexpr unsigned long long DEADLINE_TICKS = 50000000ull;

constexpr size_t WS_CTR = 0;
constexpr size_t WS_FLAG = 32768;
constexpr size_t WS_TMO = 65536;

struct Params {
    const unsigned char* x;
    const unsigned char* w;
    const float* sx;
    const float* sw;
    float* out[NR];
    unsigned char* ws[NR];
    int rank;
    int pad;
};

#define WAIT_VM(n) asm volatile("s_waitcnt vmcnt(%0)" :: "i"(n) : "memory")
#define WAIT_LGKM0() asm volatile("s_waitcnt lgkmcnt(0)" ::: "memory")
#define BAR() __builtin_amdgcn_s_barrier()
#define SCHED() __builtin_amdgcn_sched_barrier(0)

__device__ __forceinline__ void glds16(unsigned voff, const unsigned char* sbase, unsigned lds_dst) {
    asm volatile("s_mov_b32 m0, %2\n\ts_nop 0\n\tglobal_load_lds_dwordx4 %0, %1"
                 :: "v"(voff), "s"(sbase), "s"(lds_dst) : "memory");
}

template <int I>
struct LoaderLoop {
    static __device__ __forceinline__ void run(const unsigned (&va)[2], const unsigned (&vb)[4], const unsigned char* xa,
                                               const unsigned char* wb, unsigned ldsA, unsigned ldsB, int rot) {
        constexpr int AFTER = (NSTAGE - 1 - I) < (DEPTH - 2) ? (NSTAGE - 1 - I) : (DEPTH - 2);
        WAIT_VM(6 * AFTER);
        BAR();
        if constexpr (I + DEPTH - 1 < NSTAGE) {
            constexpr int S = I + DEPTH - 1;
            const int kb = (S + rot) & (NSTAGE - 1);
            const unsigned char* pa = xa + kb * BK;
            const unsigned char* pb = wb + (size_t)kb * BK * NN;
            constexpr unsigned so = (S % DEPTH) * SLOT;
            glds16(vb[0], pb, ldsB + so);
            glds16(vb[1], pb, ldsB + so + 1024);
            glds16(va[0], pa, ldsA + so);
            glds16(vb[2], pb, ldsB + so + 2048);
            glds16(vb[3], pb, ldsB + so + 3072);
            glds16(va[1], pa, ldsA + so + 1024);
        }
        if constexpr (I + 1 < NSTAGE) LoaderLoop<I + 1>::run(va, vb, xa, wb, ldsA, ldsB, rot);
    }
};

struct Frag { v4i a[2][2][2]; v2i b[2][4]; };

__device__ __forceinline__ v8i cat8(v4i lo, v4i hi) { return __builtin_shufflevector(lo, hi, 0, 1, 2, 3, 4, 5, 6, 7); }
__device__ __forceinline__ v8i cat8b(v2i a, v2i b, v2i c, v2i d) {
    v4i lo = __builtin_shufflevector(a, b, 0, 1, 2, 3), hi = __builtin_shufflevector(c, d, 0, 1, 2, 3);
    return cat8(lo, hi);
}

__global__ void __launch_bounds__(512, 2) gemm_a2a(Params p) {
    __shared__ __attribute__((aligned(1024))) unsigned char lds[LDS_BYTES];
    const int tid = threadIdx.x, lane = tid & 63;
    const int wid = __builtin_amdgcn_readfirstlane(tid >> 6);
    const int b = blockIdx.x;
    const int dst = b & 7, j = b >> 3, tm = j >> 2, tn = j & 3;
    const unsigned lds0 = (unsigned)(size_t)(LAS unsigned char*)lds;
    unsigned char* myws = p.ws[p.rank];
    unsigned call = 0;
    v16f acc[2] = {};

    if (wid >= 4) {
        const int lw = wid - 4;
        unsigned va[2], vb[4];
#pragma unroll
        for (int i = 0; i < 2; ++i) {
            const int row = (lw * 2 + i) * 8 + (lane >> 3);
            va[i] = row * KK + (((lane & 7) ^ ((row >> 1) & 7)) << 4);
        }
#pragma unroll
        for (int i = 0; i < 4; ++i) {
            const int row = (lw * 4 + i) * 8 + (lane >> 3);
            vb[i] = row * NN + (((lane & 7) ^ (row & 6)) << 4);
        }
        const unsigned char* xa = p.x + (size_t)tm * BM * KK;
        const unsigned char* wb = p.w + dst * NS + tn * BN;
        const unsigned ldsA = lds0 + lw * 2048, ldsB = lds0 + A_BYTES + lw * 4096;
#define KROT 0
        const int rot = dst * KROT;
#pragma unroll
        for (int s = 0; s < DEPTH - 1; ++s) {
            const int kb = (s + rot) & (NSTAGE - 1);
            const unsigned char* pa = xa + kb * BK;
            const unsigned char* pb = wb + (size_t)kb * BK * NN;
            const unsigned so = s * SLOT;
            glds16(vb[0], pb, ldsB + so);
            glds16(vb[1], pb, ldsB + so + 1024);
            glds16(va[0], pa, ldsA + so);
            glds16(vb[2], pb, ldsB + so + 2048);
            glds16(vb[3], pb, ldsB + so + 3072);
            glds16(va[1], pa, ldsA + so + 1024);
        }
        LoaderLoop<0>::run(va, vb, xa, wb, ldsA, ldsB, rot);
    } else {
        if (wid == 0 && lane == 0)
            call = __hip_atomic_fetch_add((unsigned*)(myws + WS_CTR + b * 128), 1u, __ATOMIC_RELAXED, __HIP_MEMORY_SCOPE_AGENT) + 1u;
        const int r = lane & 31, h = lane >> 5;
        unsigned ao[2][2];
#pragma unroll
        for (int ks = 0; ks < 2; ++ks)
#pragma unroll
            for (int e = 0; e < 2; ++e) ao[ks][e] = r * 128 + (((4 * ks + 2 * h + e) ^ ((r >> 1) & 7)) << 4);
        const int q = (lane & 15) >> 1, pp = lane & 1, g1 = (lane >> 4) & 1;
        const unsigned bo = A_BYTES + (32 * h + q) * 128 + (((2 * wid + g1) ^ (q & 6)) << 4) + pp * 8;

        auto rd = [&](Frag& f, int slot) {
            const unsigned char LAS* base = (const unsigned char LAS*)lds + slot * SLOT;
#pragma unroll
            for (int ks = 0; ks < 2; ++ks) {
#pragma unroll
                for (int t = 0; t < 4; ++t)
                    f.b[ks][t] = __builtin_amdgcn_ds_read_tr8_b64_v2i32((LAS v2i*)(base + bo + (64 * ks + 8 * t) * 128));
#pragma unroll
                for (int mt = 0; mt < 2; ++mt)
#pragma unroll
                    for (int e = 0; e < 2; ++e) f.a[ks][mt][e] = *(const LAS v4i*)(base + ao[ks][e] + mt * 4096);
            }
        };
        auto mm = [&](const Frag& f) {
#pragma unroll
            for (int ks = 0; ks < 2; ++ks) {
                const v8i wf = cat8b(f.b[ks][0], f.b[ks][1], f.b[ks][2], f.b[ks][3]);
#pragma unroll
                for (int mt = 0; mt < 2; ++mt)
                    acc[mt] = __builtin_amdgcn_mfma_scale_f32_32x32x64_f8f6f4(wf, cat8(f.a[ks][mt][0], f.a[ks][mt][1]), acc[mt], 0, 0, 0, 0, 0, 0);
            }
        };
        Frag f0, f1;
        BAR();
        rd(f0, 0);
        WAIT_LGKM0();
        SCHED();
        int slot = 1;
#pragma unroll 1
        for (int i = 1; i < NSTAGE - 1; i += 2) {
            BAR();
            rd(f1, slot);
            mm(f0);
#pragma unroll
            for (int u = 0; u < 4; ++u) {
                __builtin_amdgcn_sched_group_barrier(0x8, 1, 0);
                __builtin_amdgcn_sched_group_barrier(0x100, 4, 0);
            }
            WAIT_LGKM0();
            SCHED();
            slot = slot + 1 == DEPTH ? 0 : slot + 1;
            BAR();
            rd(f0, slot);
            mm(f1);
#pragma unroll
            for (int u = 0; u < 4; ++u) {
                __builtin_amdgcn_sched_group_barrier(0x8, 1, 0);
                __builtin_amdgcn_sched_group_barrier(0x100, 4, 0);
            }
            WAIT_LGKM0();
            SCHED();
            slot = slot + 1 == DEPTH ? 0 : slot + 1;
        }
        BAR();
        rd(f1, slot);
        mm(f0);
        WAIT_LGKM0();
        SCHED();
        mm(f1);
    }
    BAR();
    if (wid < 4) {
        const float s = p.sx[0] * p.sw[0];
        const int r = lane & 31, h = lane >> 5;
#pragma unroll
        for (int mt = 0; mt < 2; ++mt)
#pragma unroll
            for (int g = 0; g < 4; ++g) {
                v4f v;
#pragma unroll
                for (int e = 0; e < 4; ++e) v[e] = fmaxf(acc[mt][4 * g + e] * s, 0.f);
                const int row = mt * 32 + r, ch = wid * 8 + 2 * g + h;
                *(LAS v4f*)((LAS unsigned char*)lds + row * 512 + ((ch ^ (row & 7)) << 4)) = v;
            }
        if (wid == 0 && lane == 0) *(LAS unsigned*)((LAS unsigned char*)lds + 32768) = call;
    }
    __syncthreads();
    call = *(LAS unsigned*)((LAS unsigned char*)lds + 32768);
    {
        float* ob = p.out[dst] + (size_t)(p.rank * MS + tm * BM) * NS + tn * BN;
#pragma unroll
        for (int u = 0; u < 4; ++u) {
            const int row = (wid * 4 + u) * 2 + (lane >> 5), ch = lane & 31;
            const u32x4 v = *(LAS u32x4*)((LAS unsigned char*)lds + row * 512 + ((ch ^ (row & 7)) << 4));
            float* ptr = ob + row * NS + ch * 4;
            asm volatile("global_store_dwordx4 %0, %1, off sc0 sc1\n\ts_nop 1" :: "v"(ptr), "v"(v) : "memory");
        }
    }
    WAIT_VM(0);
    __syncthreads();
    if (tid == 0 && dst != p.rank) {
        __hip_atomic_store((unsigned*)(p.ws[dst] + WS_FLAG + (p.rank * 32 + j) * 128), call, __ATOMIC_RELAXED, __HIP_MEMORY_SCOPE_SYSTEM);
        unsigned* flag = (unsigned*)(myws + WS_FLAG + (dst * 32 + j) * 128);
        unsigned* tmo = (unsigned*)(myws + WS_TMO);
        const unsigned long long t0 = __builtin_amdgcn_s_memrealtime();
        while (__hip_atomic_load(flag, __ATOMIC_RELAXED, __HIP_MEMORY_SCOPE_SYSTEM) < call) {
            if (__hip_atomic_load(tmo, __ATOMIC_RELAXED, __HIP_MEMORY_SCOPE_AGENT) == call ||
                __builtin_amdgcn_s_memrealtime() - t0 > DEADLINE_TICKS) {
                __hip_atomic_store(tmo, call, __ATOMIC_RELAXED, __HIP_MEMORY_SCOPE_AGENT);
                break;
            }
            __builtin_amdgcn_s_sleep(2);
        }
        if (__hip_atomic_load(tmo, __ATOMIC_RELAXED, __HIP_MEMORY_SCOPE_AGENT) == call)
            p.out[p.rank][(size_t)p.rank * MS * NS] = __builtin_nanf("");
    }
}

extern "C" void kernel_launch(int rank, int nranks, ncclComm_t comm, hipStream_t stream,
                              void* const* d_in, const int* in_sizes, int n_in,
                              void* d_out, int out_size, const int* in_dtypes, int out_dtype,
                              void* const* d_out_all, void* const* d_ws_all, size_t ws_bytes) {
    if (nranks != NR || n_in != 4 || in_sizes[0] != MS * KK || in_sizes[1] != KK * NN || out_size != NR * MS * NS ||
        (unsigned)rank >= (unsigned)NR || ws_bytes < (1u << 20)) {
        fprintf(stderr, "unexpected problem shape\n");
        return;
    }
    Params p;
    p.x = (const unsigned char*)d_in[0];
    p.w = (const unsigned char*)d_in[1];
    p.sx = (const float*)d_in[2];
    p.sw = (const float*)d_in[3];
    for (int i = 0; i < NR; ++i) {
        p.out[i] = (float*)d_out_all[i];
        p.ws[i] = (unsigned char*)d_ws_all[i];
    }
    p.rank = rank;
    p.pad = 0;
    gemm_a2a<<<GRID, 512, 0, stream>>>(p);
    HIP_CHECK(hipGetLastError());
}
```

```cpp
#include <hip/hip_runtime.h>
#include <rccl/rccl.h>
#include <cstdio>

#define HIP_CHECK(expr) do { hipError_t _e = (expr); if (_e != hipSuccess) { \
    fprintf(stderr, "HIP error %s at %s:%d\n", hipGetErrorString(_e), __FILE__, __LINE__); \
    return; } } while (0)

typedef int v8i __attribute__((ext_vector_type(8)));
typedef int v4i __attribute__((ext_vector_type(4)));
typedef int v2i __attribute__((ext_vector_type(2)));
typedef float v16f __attribute__((ext_vector_type(16)));
typedef float v4f __attribute__((ext_vector_type(4)));
typedef unsigned u32x4 __attribute__((ext_vector_type(4)));
#define LAS __attribute__((address_space(3)))

constexpr int NR = 8;
constexpr int MS = 512;
constexpr int KK = 4096;
constexpr int NN = 4096;
constexpr int NS = 512;
constexpr int BM = 64, BN = 128, BK = 128;
constexpr int NSTAGE = KK / BK;
#define DEPTH_ 6
constexpr int DEPTH = DEPTH_;
constexpr int A_BYTES = BM * BK;
constexpr int B_BYTES = BK * BN;
constexpr int SLOT = A_BYTES + B_BYTES;
constexpr int RING_BYTES = DEPTH * SLOT;
constexpr int LDS_MISC = RING_BYTES;
constexpr int LDS_BYTES = RING_BYTES + 16;
constexpr int PK_SLOT = 9216;
constexpr int LDS_TAB = 4 * PK_SLOT;
constexpr int GRID = 256;
#define POLL_SLEEP 2
#define OUTMODE 0
constexpr unsigned long long DEADLINE_TICKS = 50000000ull;

constexpr size_t WS_CTR = 0;
constexpr size_t WS_TMO = 32768;
constexpr size_t WS_STAMP = 65536;
constexpr size_t WS_FLAG = 131072;
constexpr size_t WS_STAGE = WS_FLAG + 2 * 1024 * 128;
constexpr size_t WS_END = WS_STAGE + (size_t)2 * 1024 * PK_SLOT;

struct Params {
    const unsigned char* x;
    const unsigned char* w;
    const float* sx;
    const float* sw;
    float* out;
    unsigned char* myws;
    unsigned char* ws[NR];
    int rank;
    int pad;
};

#define WAIT_VM(n) asm volatile("s_waitcnt vmcnt(%0)" :: "i"(n) : "memory")
#define WAIT_LGKM0() asm volatile("s_waitcnt lgkmcnt(0)" ::: "memory")
#define BAR() __builtin_amdgcn_s_barrier()
#define SCHED() __builtin_amdgcn_sched_barrier(0)

__device__ __forceinline__ void glds16(unsigned voff, const unsigned char* sbase, unsigned lds_dst) {
    asm volatile("s_mov_b32 m0, %2\n\ts_nop 0\n\tglobal_load_lds_dwordx4 %0, %1"
                 :: "v"(voff), "s"(sbase), "s"(lds_dst) : "memory");
}

template <int I>
struct LoaderLoop {
    static __device__ __forceinline__ void run(const unsigned (&va)[2], const unsigned (&vb)[4], const unsigned char* xa,
                                               const unsigned char* wb, unsigned ldsA, unsigned ldsB, int rot) {
        constexpr int AFTER = (NSTAGE - 1 - I) < (DEPTH - 2) ? (NSTAGE - 1 - I) : (DEPTH - 2);
        WAIT_VM(6 * AFTER);
        BAR();
        if constexpr (I + DEPTH - 1 < NSTAGE) {
            constexpr int S = I + DEPTH - 1;
            const int kb = (S + rot) & (NSTAGE - 1);
            const unsigned char* pa = xa + kb * BK;
            const unsigned char* pb = wb + (size_t)kb * BK * NN;
            constexpr unsigned so = (S % DEPTH) * SLOT;
            glds16(vb[0], pb, ldsB + so);
            glds16(vb[1], pb, ldsB + so + 1024);
            glds16(va[0], pa, ldsA + so);
            glds16(vb[2], pb, ldsB + so + 2048);
            glds16(vb[3], pb, ldsB + so + 3072);
            glds16(va[1], pa, ldsA + so + 1024);
        }
        if constexpr (I + 1 < NSTAGE) LoaderLoop<I + 1>::run(va, vb, xa, wb, ldsA, ldsB, rot);
    }
};

__device__ __forceinline__ unsigned wave_scan(unsigned v) {
    v += __builtin_amdgcn_update_dpp(0u, v, 0x111, 0xf, 0xf, true);
    v += __builtin_amdgcn_update_dpp(0u, v, 0x112, 0xf, 0xf, true);
    v += __builtin_amdgcn_update_dpp(0u, v, 0x114, 0xf, 0xf, true);
    v += __builtin_amdgcn_update_dpp(0u, v, 0x118, 0xf, 0xf, true);
    v += __builtin_amdgcn_update_dpp(0u, v, 0x142, 0xa, 0xf, false);
    v += __builtin_amdgcn_update_dpp(0u, v, 0x143, 0xc, 0xf, false);
    return v;
}

struct Frag { v4i a[2][2][2]; v2i b[2][4]; };

__device__ __forceinline__ v8i cat8(v4i lo, v4i hi) { return __builtin_shufflevector(lo, hi, 0, 1, 2, 3, 4, 5, 6, 7); }
__device__ __forceinline__ v8i cat8b(v2i a, v2i b, v2i c, v2i d) {
    v4i lo = __builtin_shufflevector(a, b, 0, 1, 2, 3), hi = __builtin_shufflevector(c, d, 0, 1, 2, 3);
    return cat8(lo, hi);
}

__global__ void __launch_bounds__(512, 2) gemm_a2a(Params p) {
    __shared__ __attribute__((aligned(1024))) unsigned char lds[LDS_BYTES];
    const int tid = threadIdx.x, lane = tid & 63;
    const int wid = __builtin_amdgcn_readfirstlane(tid >> 6);
    const int b = blockIdx.x;
    const int dst = b & 7, j = b >> 3, tm = j >> 2, tn = j & 3;
    const unsigned lds0 = (unsigned)(size_t)(LAS unsigned char*)lds;
    unsigned char* myws = p.myws;
    v16f acc[2] = {};
    float scale = 0.f;
    unsigned char* dws = nullptr;
    float* myout = p.out;

    if (wid >= 4) {
        const int lw = wid - 4;
        unsigned ticket = 0;
        if (lw == 0) {
            unsigned char* ctr = myws + WS_CTR + b * 128;
            asm volatile("s_mov_b64 exec, 1\n\tglobal_atomic_add %0, %1, %2, %3 sc0\n\ts_mov_b64 exec, -1"
                         : "=&v"(ticket) : "v"(0u), "v"(1u), "s"(ctr) : "memory");
        }
        unsigned va[2], vb[4];
#pragma unroll
        for (int i = 0; i < 2; ++i) {
            const int row = (lw * 2 + i) * 8 + (lane >> 3);
            va[i] = row * KK + (((lane & 7) ^ ((row >> 1) & 7)) << 4);
        }
#pragma unroll
        for (int i = 0; i < 4; ++i) {
            const int row = (lw * 4 + i) * 8 + (lane >> 3);
            vb[i] = row * NN + (((lane & 7) ^ (row & 6)) << 4);
        }
        const unsigned char* xa = p.x + (size_t)tm * BM * KK;
        const unsigned char* wb = p.w + dst * NS + tn * BN;
        const unsigned ldsA = lds0 + lw * 2048, ldsB = lds0 + A_BYTES + lw * 4096;
#define KROT 0
        const int rot = dst * KROT;
#pragma unroll
        for (int s = 0; s < DEPTH - 1; ++s) {
            const int kb = (s + rot) & (NSTAGE - 1);
            const unsigned char* pa = xa + kb * BK;
            const unsigned char* pb = wb + (size_t)kb * BK * NN;
            const unsigned so = s * SLOT;
            glds16(vb[0], pb, ldsB + so);
            glds16(vb[1], pb, ldsB + so + 1024);
            glds16(va[0], pa, ldsA + so);
            glds16(vb[2], pb, ldsB + so + 2048);
            glds16(vb[3], pb, ldsB + so + 3072);
            glds16(va[1], pa, ldsA + so + 1024);
        }
        LoaderLoop<0>::run(va, vb, xa, wb, ldsA, ldsB, rot);
        if (lw == 0) {
            asm volatile("" : "+v"(ticket));
            if (lane == 0) {
                *(LAS unsigned*)((LAS unsigned char*)lds + LDS_MISC) = ticket + 1u;
                *(LAS unsigned*)((LAS unsigned char*)lds + LDS_MISC + 4) = 0u;
            }
            WAIT_LGKM0();
        }
    } else {
        scale = __builtin_bit_cast(float, __builtin_amdgcn_readfirstlane(__builtin_bit_cast(int, p.sx[0] * p.sw[0])));
        dws = p.ws[dst];
        asm volatile("" : "+s"(scale), "+s"(dws));
        const int r = lane & 31, h = lane >> 5;
        unsigned ao[2][2];
#pragma unroll
        for (int ks = 0; ks < 2; ++ks)
#pragma unroll
            for (int e = 0; e < 2; ++e) ao[ks][e] = r * 128 + (((4 * ks + 2 * h + e) ^ ((r >> 1) & 7)) << 4);
        const int q = (lane & 15) >> 1, pp = lane & 1, g1 = (lane >> 4) & 1;
        const unsigned bo = A_BYTES + (32 * h + q) * 128 + (((2 * wid + g1) ^ (q & 6)) << 4) + pp * 8;

        auto rd = [&](Frag& f, int slot) {
            const unsigned char LAS* base = (const unsigned char LAS*)lds + slot * SLOT;
#pragma unroll
            for (int ks = 0; ks < 2; ++ks) {
#pragma unroll
                for (int t = 0; t < 4; ++t)
                    f.b[ks][t] = __builtin_amdgcn_ds_read_tr8_b64_v2i32((LAS v2i*)(base + bo + (64 * ks + 8 * t) * 128));
#pragma unroll
                for (int mt = 0; mt < 2; ++mt)
#pragma unroll
                    for (int e = 0; e < 2; ++e) f.a[ks][mt][e] = *(const LAS v4i*)(base + ao[ks][e] + mt * 4096);
            }
        };
        auto mm = [&](const Frag& f) {
#pragma unroll
            for (int ks = 0; ks < 2; ++ks) {
                const v8i wf = cat8b(f.b[ks][0], f.b[ks][1], f.b[ks][2], f.b[ks][3]);
#pragma unroll
                for (int mt = 0; mt < 2; ++mt)
                    acc[mt] = __builtin_amdgcn_mfma_scale_f32_32x32x64_f8f6f4(wf, cat8(f.a[ks][mt][0], f.a[ks][mt][1]), acc[mt], 0, 0, 0, 0, 0, 0);
            }
        };
        Frag f0, f1;
        BAR();
        rd(f0, 0);
        WAIT_LGKM0();
        SCHED();
        int slot = 1;
#pragma unroll 1
        for (int i = 1; i < NSTAGE - 1; i += 2) {
            BAR();
            rd(f1, slot);
            mm(f0);
#pragma unroll
            for (int u = 0; u < 4; ++u) {
                __builtin_amdgcn_sched_group_barrier(0x8, 1, 0);
                __builtin_amdgcn_sched_group_barrier(0x100, 4, 0);
            }
            WAIT_LGKM0();
            SCHED();
            slot = slot + 1 == DEPTH ? 0 : slot + 1;
            BAR();
            rd(f0, slot);
            mm(f1);
#pragma unroll
            for (int u = 0; u < 4; ++u) {
                __builtin_amdgcn_sched_group_barrier(0x8, 1, 0);
                __builtin_amdgcn_sched_group_barrier(0x100, 4, 0);
            }
            WAIT_LGKM0();
            SCHED();
            slot = slot + 1 == DEPTH ? 0 : slot + 1;
        }
        BAR();
        rd(f1, slot);
        mm(f0);
        WAIT_LGKM0();
        SCHED();
        mm(f1);
    }
    BAR();
    LAS unsigned char* L = (LAS unsigned char*)lds;
    if (dst == p.rank) {
        if (wid < 4) {
            const float s = scale;
            const int r = lane & 31, h = lane >> 5;
#pragma unroll
            for (int mt = 0; mt < 2; ++mt)
#pragma unroll
                for (int g = 0; g < 4; ++g) {
                    v4f v;
#pragma unroll
                    for (int e = 0; e < 4; ++e) { const float y = acc[mt][4 * g + e] * s; v[e] = y <= 0.f ? 0.f : y; }
                    const int row = mt * 32 + r, ch = wid * 8 + 2 * g + h;
                    *(LAS v4f*)(L + row * 512 + ((ch ^ (row & 7)) << 4)) = v;
                }
        }
        __syncthreads();
        float* ob = myout + (size_t)(p.rank * MS + tm * BM) * NS + tn * BN;
#pragma unroll
        for (int u = 0; u < 4; ++u) {
            const int row = (wid * 4 + u) * 2 + (lane >> 5), ch = lane & 31;
            *(v4f*)(ob + row * NS + ch * 4) = *(LAS v4f*)(L + row * 512 + ((ch ^ (row & 7)) << 4));
        }
    } else {
        const unsigned call = __builtin_amdgcn_readfirstlane(*(LAS unsigned*)(L + LDS_MISC));
        const unsigned par = call & 1u;
        if (wid < 4) {
            const float s = scale;
            float y[32];
            unsigned long long nz[32];
#pragma unroll
            for (int i = 0; i < 32; ++i) {
                y[i] = acc[i >> 4][i & 15] * s;
                nz[i] = __builtin_amdgcn_fcmpf(y[i], 0.f, 10  );
            }
            unsigned mask = 0;
#pragma unroll
            for (int q = 3; q >= 0; --q)
                asm volatile("s_nop 1\n\t"
                             "v_addc_co_u32_e64 %0, vcc, %0, %0, %8\n\tv_addc_co_u32_e64 %0, vcc, %0, %0, %7\n\t"
                             "v_addc_co_u32_e64 %0, vcc, %0, %0, %6\n\tv_addc_co_u32_e64 %0, vcc, %0, %0, %5\n\t"
                             "v_addc_co_u32_e64 %0, vcc, %0, %0, %4\n\tv_addc_co_u32_e64 %0, vcc, %0, %0, %3\n\t"
                             "v_addc_co_u32_e64 %0, vcc, %0, %0, %2\n\tv_addc_co_u32_e64 %0, vcc, %0, %0, %1"
                             : "+v"(mask)
                             : "s"(nz[8 * q]), "s"(nz[8 * q + 1]), "s"(nz[8 * q + 2]), "s"(nz[8 * q + 3]),
                               "s"(nz[8 * q + 4]), "s"(nz[8 * q + 5]), "s"(nz[8 * q + 6]), "s"(nz[8 * q + 7])
                             : "vcc");
            const unsigned cnt = __builtin_popcount(mask);
            const unsigned incl = wave_scan(cnt);
            const unsigned total = __builtin_amdgcn_readlane(incl, 63);
            const unsigned reg = lds0 + wid * PK_SLOT;
            *(LAS unsigned*)(L + wid * PK_SLOT + lane * 4) = mask;
            unsigned ptr = reg + 256 + (incl - cnt) * 4;
#define PK1(m, v) "s_mov_b64 exec, %" #m "\n\tds_write_b32 %0, %" #v "\n\tv_add_u32 %0, 4, %0\n\t"
#pragma unroll
            for (int q = 0; q < 4; ++q)
                asm volatile(PK1(1, 9) PK1(2, 10) PK1(3, 11) PK1(4, 12) PK1(5, 13) PK1(6, 14) PK1(7, 15) PK1(8, 16)
                             "s_mov_b64 exec, -1"
                             : "+v"(ptr)
                             : "s"(nz[8 * q]), "s"(nz[8 * q + 1]), "s"(nz[8 * q + 2]), "s"(nz[8 * q + 3]),
                               "s"(nz[8 * q + 4]), "s"(nz[8 * q + 5]), "s"(nz[8 * q + 6]), "s"(nz[8 * q + 7]),
                               "v"(y[8 * q]), "v"(y[8 * q + 1]), "v"(y[8 * q + 2]), "v"(y[8 * q + 3]),
                               "v"(y[8 * q + 4]), "v"(y[8 * q + 5]), "v"(y[8 * q + 6]), "v"(y[8 * q + 7])
                             : "memory");
            WAIT_LGKM0();
            const unsigned bytes = 256 + 4 * total;
            const unsigned sidx = ((par * 8 + p.rank) * 32 + j) * 4 + wid;
            unsigned char* dslot = dws + WS_STAGE + (size_t)sidx * PK_SLOT;
            const unsigned lo = lane * 16;
            u32x4 c[5];
#pragma unroll
            for (int u = 0; u < 5; ++u) c[u] = *(LAS u32x4*)(L + wid * PK_SLOT + u * 1024 + lo);
#pragma unroll
            for (int u = 0; u < 5; ++u)
                if (u * 1024 + lo < bytes)
                    asm volatile("global_store_dwordx4 %0, %1, %2 sc0 sc1\n\ts_nop 1" :: "v"(lo + u * 1024), "v"(c[u]), "s"(dslot) : "memory");
#pragma unroll 1
            for (unsigned o = 5 * 1024; o < bytes; o += 1024) {
                const u32x4 v = *(LAS u32x4*)(L + wid * PK_SLOT + o + lo);
                if (o + lo < bytes)
                    asm volatile("global_store_dwordx4 %0, %1, %2 sc0 sc1\n\ts_nop 1" :: "v"(lo + o), "v"(v), "s"(dslot) : "memory");
            }
            WAIT_VM(0);
            if (lane == 0)
                __hip_atomic_store((unsigned long long*)(dws + WS_FLAG + (size_t)sidx * 128),
                                   ((unsigned long long)call << 32) | total, __ATOMIC_RELAXED, __HIP_MEMORY_SCOPE_SYSTEM);
            const unsigned ridx = ((par * 8 + dst) * 32 + j) * 4 + wid;
            unsigned rtotal;
            unsigned* tmo = (unsigned*)(myws + WS_TMO);
            {
                const unsigned char* flag = myws + WS_FLAG + (size_t)ridx * 128;
                const unsigned long long t0 = __builtin_amdgcn_s_memrealtime();
                for (;;) {
                    v2i f; unsigned t;
                    asm volatile("global_load_dwordx2 %0, %2, %3 sc0 sc1\n\tglobal_load_dword %1, %2, %4 sc1\n\ts_waitcnt vmcnt(0)"
                                 : "=&v"(f), "=&v"(t) : "v"(0u), "s"(flag), "s"(tmo) : "memory");
                    rtotal = __builtin_amdgcn_readfirstlane(f[0]);
                    if ((unsigned)__builtin_amdgcn_readfirstlane(f[1]) >= call) break;
                    if (__builtin_amdgcn_readfirstlane(t) == call || __builtin_amdgcn_s_memrealtime() - t0 > DEADLINE_TICKS) {
                        if (lane == 0) __hip_atomic_store(tmo, call, __ATOMIC_RELAXED, __HIP_MEMORY_SCOPE_AGENT);
                        rtotal = 0xffffffffu;
                        break;
                    }
                    __builtin_amdgcn_s_sleep(POLL_SLEEP);
                }
            }
            if (rtotal <= 2048u) {
                const unsigned rbytes = 256 + 4 * rtotal;
                const unsigned char* sslot = myws + WS_STAGE + (size_t)ridx * PK_SLOT;
#pragma unroll 1
                for (unsigned o = 0; o < rbytes; o += 1024)
                    asm volatile("s_mov_b32 m0, %2\n\ts_nop 0\n\tglobal_load_lds_dwordx4 %0, %1 sc0 sc1"
                                 :: "v"(lo + o), "s"(sslot), "s"(reg + o) : "memory");
                unsigned t;
                asm volatile("global_load_dword %0, %1, %2 sc1\n\ts_waitcnt vmcnt(0)" : "=&v"(t) : "v"(0u), "s"(tmo) : "memory");
                if (t == call) *(LAS unsigned*)(L + LDS_MISC + 4) = 1u;
                const unsigned rm = *(LAS unsigned*)(L + wid * PK_SLOT + lane * 4);
                const unsigned rc = __builtin_popcount(rm);
                const unsigned rb = wave_scan(rc) - rc;
                v2i tb; tb[0] = rm; tb[1] = wid * PK_SLOT + 256 + rb * 4;
                *(LAS v2i*)(L + LDS_TAB + wid * 512 + lane * 8) = tb;
            } else {
                *(LAS unsigned*)(L + LDS_MISC + 4) = 1u;
                *(LAS v2i*)(L + LDS_TAB + wid * 512 + lane * 8) = (v2i){0, 0};
            }
        }
        __syncthreads();
        const int w = wid & 3, mt = wid >> 2;
        float* ob = myout + (size_t)(dst * MS + tm * BM + mt * 32 + (lane >> 3)) * NS + tn * BN + w * 32 + (lane & 7) * 4;
        const bool bad = *(LAS unsigned*)(L + LDS_MISC + 4) != 0u;
        const unsigned bp = 16 * mt + 4 * ((lane & 7) >> 1);
        const unsigned low = (1u << bp) - 1u;
        u32x4 o[4];
#pragma unroll
        for (int u = 0; u < 4; ++u) {
            const v2i t = *(LAS v2i*)(L + LDS_TAB + w * 512 + (8 * u + (lane >> 3) + 32 * (lane & 1)) * 8);
            const unsigned m = (unsigned)t[0];
            int addr = t[1] + 4 * __builtin_popcount(m & low);
            const int ms = (int)(m >> bp);
#pragma unroll
            for (int e = 0; e < 4; ++e) {
                const int sel = __builtin_amdgcn_sbfe(ms, e, 1);
                o[u][e] = *(LAS unsigned*)(L + addr) & (unsigned)sel;
                addr -= 4 * sel;
            }
        }
        if (bad) {
#pragma unroll
            for (int u = 0; u < 4; ++u) o[u] = (u32x4){0x7fc00000u, 0x7fc00000u, 0x7fc00000u, 0x7fc00000u};
        }
#pragma unroll
        for (int u = 0; u < 4; ++u) {
            *(u32x4*)(ob + u * 8 * NS) = o[u];
        }
    }
}

extern "C" void kernel_launch(int rank, int nranks, ncclComm_t comm, hipStream_t stream,
                              void* const* d_in, const int* in_sizes, int n_in,
                              void* d_out, int out_size, const int* in_dtypes, int out_dtype,
                              void* const* d_out_all, void* const* d_ws_all, size_t ws_bytes) {
    if (nranks != NR || n_in != 4 || in_sizes[0] != MS * KK || in_sizes[1] != KK * NN || out_size != NR * MS * NS ||
        (unsigned)rank >= (unsigned)NR || ws_bytes < WS_END) {
        fprintf(stderr, "unexpected problem shape\n");
        return;
    }
    Params p;
    p.x = (const unsigned char*)d_in[0];
    p.w = (const unsigned char*)d_in[1];
    p.sx = (const float*)d_in[2];
    p.sw = (const float*)d_in[3];
    p.out = (float*)d_out;
    p.myws = (unsigned char*)d_ws_all[rank];
    for (int i = 0; i < NR; ++i) p.ws[i] = (unsigned char*)d_ws_all[i];
    p.rank = rank;
    p.pad = 0;
    gemm_a2a<<<GRID, 512, 0, stream>>>(p);
    HIP_CHECK(hipGetLastError());
}
```

```cpp
#include <hip/hip_runtime.h>
#include <rccl/rccl.h>
#include <cstdio>

#define HIP_CHECK(expr) do { hipError_t _e = (expr); if (_e != hipSuccess) { \
    fprintf(stderr, "HIP error %s at %s:%d\n", hipGetErrorString(_e), __FILE__, __LINE__); \
    return; } } while (0)

typedef int v8i __attribute__((ext_vector_type(8)));
typedef int v4i __attribute__((ext_vector_type(4)));
typedef int v2i __attribute__((ext_vector_type(2)));
typedef float v16f __attribute__((ext_vector_type(16)));
typedef float v4f __attribute__((ext_vector_type(4)));
typedef unsigned u32x4 __attribute__((ext_vector_type(4)));
#define LAS __attribute__((address_space(3)))

constexpr int NR = 8;
constexpr int MS = 512;
constexpr int KK = 4096;
constexpr int NN = 4096;
constexpr int NS = 512;
constexpr int BM = 64, BN = 128, BK = 128;
constexpr int NSTAGE = KK / BK;
#define DEPTH_ 6
constexpr int DEPTH = DEPTH_;
constexpr int A_BYTES = BM * BK;
constexpr int B_BYTES = BK * BN;
constexpr int SLOT = A_BYTES + B_BYTES;
constexpr int RING_BYTES = DEPTH * SLOT;
constexpr int LDS_MISC = RING_BYTES;
constexpr int LDS_BYTES = RING_BYTES + 16;
constexpr int PK_SLOT = 9216;
constexpr int LDS_TAB = 4 * PK_SLOT;
constexpr int GRID = 256;
#define POLL_SLEEP 2
#define OUTMODE 0
constexpr unsigned long long DEADLINE_TICKS = 50000000ull;

constexpr size_t WS_CTR = 0;
constexpr size_t WS_TMO = 32768;
constexpr size_t WS_STAMP = 65536;
constexpr size_t WS_FLAG = 131072;
constexpr size_t WS_STAGE = WS_FLAG + 2 * 1024 * 128;
constexpr size_t WS_END = WS_STAGE + (size_t)2 * 1024 * PK_SLOT;

struct Params {
    const unsigned char* x;
    const unsigned char* w;
    const float* sx;
    const float* sw;
    float* out;
    unsigned char* myws;
    unsigned char* ws[NR];
    int rank;
    int pad;
};

#define WAIT_VM(n) asm volatile("s_waitcnt vmcnt(%0)" :: "i"(n) : "memory")
#define WAIT_LGKM0() asm volatile("s_waitcnt lgkmcnt(0)" ::: "memory")
#define BAR() __builtin_amdgcn_s_barrier()
#define SCHED() __builtin_amdgcn_sched_barrier(0)

__device__ __forceinline__ void glds16(unsigned voff, const unsigned char* sbase, unsigned lds_dst) {
    asm volatile("s_mov_b32 m0, %2\n\ts_nop 0\n\tglobal_load_lds_dwordx4 %0, %1"
                 :: "v"(voff), "s"(sbase), "s"(lds_dst) : "memory");
}

template <int I>
struct LoaderLoop {
    static __device__ __forceinline__ void run(const unsigned (&va)[2], const unsigned (&vb)[4], const unsigned char* xa,
                                               const unsigned char* wb, unsigned ldsA, unsigned ldsB, int rot) {
        constexpr int AFTER = (NSTAGE - 1 - I) < (DEPTH - 2) ? (NSTAGE - 1 - I) : (DEPTH - 2);
        WAIT_VM(6 * AFTER);
        BAR();
        if constexpr (I + DEPTH - 1 < NSTAGE) {
            constexpr int S = I + DEPTH - 1;
            const int kb = (S + rot) & (NSTAGE - 1);
            const unsigned char* pa = xa + kb * BK;
            const unsigned char* pb = wb + (size_t)kb * BK * NN;
            constexpr unsigned so = (S % DEPTH) * SLOT;
            glds16(vb[0], pb, ldsB + so);
            glds16(vb[1], pb, ldsB + so + 1024);
            glds16(va[0], pa, ldsA + so);
            glds16(vb[2], pb, ldsB + so + 2048);
            glds16(vb[3], pb, ldsB + so + 3072);
            glds16(va[1], pa, ldsA + so + 1024);
        }
        if constexpr (I + 1 < NSTAGE) LoaderLoop<I + 1>::run(va, vb, xa, wb, ldsA, ldsB, rot);
    }
};

__device__ __forceinline__ unsigned wave_scan(unsigned v) {
    v += __builtin_amdgcn_update_dpp(0u, v, 0x111, 0xf, 0xf, true);
    v += __builtin_amdgcn_update_dpp(0u, v, 0x112, 0xf, 0xf, true);
    v += __builtin_amdgcn_update_dpp(0u, v, 0x114, 0xf, 0xf, true);
    v += __builtin_amdgcn_update_dpp(0u, v, 0x118, 0xf, 0xf, true);
    v += __builtin_amdgcn_update_dpp(0u, v, 0x142, 0xa, 0xf, false);
    v += __builtin_amdgcn_update_dpp(0u, v, 0x143, 0xc, 0xf, false);
    return v;
}

struct Frag { v4i a[2][2][2]; v2i b[2][4]; };

__device__ __forceinline__ v8i cat8(v4i lo, v4i hi) { return __builtin_shufflevector(lo, hi, 0, 1, 2, 3, 4, 5, 6, 7); }
__device__ __forceinline__ v8i cat8b(v2i a, v2i b, v2i c, v2i d) {
    v4i lo = __builtin_shufflevector(a, b, 0, 1, 2, 3), hi = __builtin_shufflevector(c, d, 0, 1, 2, 3);
    return cat8(lo, hi);
}

__global__ void __launch_bounds__(512, 2) gemm_a2a(Params p) {
    __shared__ __attribute__((aligned(1024))) unsigned char lds[LDS_BYTES];
    const int tid = threadIdx.x, lane = tid & 63;
    const int wid = __builtin_amdgcn_readfirstlane(tid >> 6);
    const int b = blockIdx.x;
    const int dst = b & 7, j = b >> 3, tm = j >> 2, tn = j & 3;
    const unsigned lds0 = (unsigned)(size_t)(LAS unsigned char*)lds;
    unsigned char* myws = p.myws;
    v16f acc[2] = {};
    const unsigned long long d_start = __builtin_amdgcn_s_memrealtime();
    float scale = 0.f;
    unsigned char* dws = nullptr;
    float* myout = p.out;

    if (wid >= 4) {
        const int lw = wid - 4;
        unsigned ticket = 0;
        if (lw == 0) {
            unsigned char* ctr = myws + WS_CTR + b * 128;
            asm volatile("s_mov_b64 exec, 1\n\tglobal_atomic_add %0, %1, %2, %3 sc0\n\ts_mov_b64 exec, -1"
                         : "=&v"(ticket) : "v"(0u), "v"(1u), "s"(ctr) : "memory");
        }
        unsigned va[2], vb[4];
#pragma unroll
        for (int i = 0; i < 2; ++i) {
            const int row = (lw * 2 + i) * 8 + (lane >> 3);
            va[i] = row * KK + (((lane & 7) ^ ((row >> 1) & 7)) << 4);
        }
#pragma unroll
        for (int i = 0; i < 4; ++i) {
            const int row = (lw * 4 + i) * 8 + (lane >> 3);
            vb[i] = row * NN + (((lane & 7) ^ (row & 6)) << 4);
        }
        const unsigned char* xa = p.x + (size_t)tm * BM * KK;
        const unsigned char* wb = p.w + dst * NS + tn * BN;
        const unsigned ldsA = lds0 + lw * 2048, ldsB = lds0 + A_BYTES + lw * 4096;
#define KROT 0
        const int rot = dst * KROT;
#pragma unroll
        for (int s = 0; s < DEPTH - 1; ++s) {
            const int kb = (s + rot) & (NSTAGE - 1);
            const unsigned char* pa = xa + kb * BK;
            const unsigned char* pb = wb + (size_t)kb * BK * NN;
            const unsigned so = s * SLOT;
            glds16(vb[0], pb, ldsB + so);
            glds16(vb[1], pb, ldsB + so + 1024);
            glds16(va[0], pa, ldsA + so);
            glds16(vb[2], pb, ldsB + so + 2048);
            glds16(vb[3], pb, ldsB + so + 3072);
            glds16(va[1], pa, ldsA + so + 1024);
        }
        LoaderLoop<0>::run(va, vb, xa, wb, ldsA, ldsB, rot);
        if (lw == 0) {
            asm volatile("" : "+v"(ticket));
            if (lane == 0) {
                *(LAS unsigned*)((LAS unsigned char*)lds + LDS_MISC) = ticket + 1u;
                *(LAS unsigned*)((LAS unsigned char*)lds + LDS_MISC + 4) = 0u;
            }
            WAIT_LGKM0();
        }
    } else {
        scale = __builtin_bit_cast(float, __builtin_amdgcn_readfirstlane(__builtin_bit_cast(int, p.sx[0] * p.sw[0])));
        dws = p.ws[dst];
        asm volatile("" : "+s"(scale), "+s"(dws));
        const int r = lane & 31, h = lane >> 5;
        unsigned ao[2][2];
#pragma unroll
        for (int ks = 0; ks < 2; ++ks)
#pragma unroll
            for (int e = 0; e < 2; ++e) ao[ks][e] = r * 128 + (((4 * ks + 2 * h + e) ^ ((r >> 1) & 7)) << 4);
        const int q = (lane & 15) >> 1, pp = lane & 1, g1 = (lane >> 4) & 1;
        const unsigned bo = A_BYTES + (32 * h + q) * 128 + (((2 * wid + g1) ^ (q & 6)) << 4) + pp * 8;

        auto rd = [&](Frag& f, int slot) {
            const unsigned char LAS* base = (const unsigned char LAS*)lds + slot * SLOT;
#pragma unroll
            for (int ks = 0; ks < 2; ++ks) {
#pragma unroll
                for (int t = 0; t < 4; ++t)
                    f.b[ks][t] = __builtin_amdgcn_ds_read_tr8_b64_v2i32((LAS v2i*)(base + bo + (64 * ks + 8 * t) * 128));
#pragma unroll
                for (int mt = 0; mt < 2; ++mt)
#pragma unroll
                    for (int e = 0; e < 2; ++e) f.a[ks][mt][e] = *(const LAS v4i*)(base + ao[ks][e] + mt * 4096);
            }
        };
        auto mm = [&](const Frag& f) {
#pragma unroll
            for (int ks = 0; ks < 2; ++ks) {
                const v8i wf = cat8b(f.b[ks][0], f.b[ks][1], f.b[ks][2], f.b[ks][3]);
#pragma unroll
                for (int mt = 0; mt < 2; ++mt)
                    acc[mt] = __builtin_amdgcn_mfma_scale_f32_32x32x64_f8f6f4(wf, cat8(f.a[ks][mt][0], f.a[ks][mt][1]), acc[mt], 0, 0, 0, 0, 0, 0);
            }
        };
        Frag f0, f1;
        BAR();
        rd(f0, 0);
        WAIT_LGKM0();
        SCHED();
        int slot = 1;
#pragma unroll 1
        for (int i = 1; i < NSTAGE - 1; i += 2) {
            BAR();
            rd(f1, slot);
            mm(f0);
#pragma unroll
            for (int u = 0; u < 4; ++u) {
                __builtin_amdgcn_sched_group_barrier(0x8, 1, 0);
                __builtin_amdgcn_sched_group_barrier(0x100, 4, 0);
            }
            WAIT_LGKM0();
            SCHED();
            slot = slot + 1 == DEPTH ? 0 : slot + 1;
            BAR();
            rd(f0, slot);
            mm(f1);
#pragma unroll
            for (int u = 0; u < 4; ++u) {
                __builtin_amdgcn_sched_group_barrier(0x8, 1, 0);
                __builtin_amdgcn_sched_group_barrier(0x100, 4, 0);
            }
            WAIT_LGKM0();
            SCHED();
            slot = slot + 1 == DEPTH ? 0 : slot + 1;
        }
        BAR();
        rd(f1, slot);
        mm(f0);
        WAIT_LGKM0();
        SCHED();
        mm(f1);
    }
    BAR();
    {
        const unsigned long long dt = __builtin_amdgcn_s_memrealtime() - d_start;
        if (true) while (__builtin_amdgcn_s_memrealtime() - d_start < 20 * dt) __builtin_amdgcn_s_sleep(4);
        else while (__builtin_amdgcn_s_memrealtime() - d_start < 5000) __builtin_amdgcn_s_sleep(4);
        BAR();
    }
    LAS unsigned char* L = (LAS unsigned char*)lds;
    if (dst == p.rank) {
        if (wid < 4) {
            const float s = scale;
            const int r = lane & 31, h = lane >> 5;
#pragma unroll
            for (int mt = 0; mt < 2; ++mt)
#pragma unroll
                for (int g = 0; g < 4; ++g) {
                    v4f v;
#pragma unroll
                    for (int e = 0; e < 4; ++e) { const float y = acc[mt][4 * g + e] * s; v[e] = y <= 0.f ? 0.f : y; }
                    const int row = mt * 32 + r, ch = wid * 8 + 2 * g + h;
                    *(LAS v4f*)(L + row * 512 + ((ch ^ (row & 7)) << 4)) = v;
                }
        }
        __syncthreads();
        float* ob = myout + (size_t)(p.rank * MS + tm * BM) * NS + tn * BN;
#pragma unroll
        for (int u = 0; u < 4; ++u) {
            const int row = (wid * 4 + u) * 2 + (lane >> 5), ch = lane & 31;
            *(v4f*)(ob + row * NS + ch * 4) = *(LAS v4f*)(L + row * 512 + ((ch ^ (row & 7)) << 4));
        }
    } else {
        const unsigned call = __builtin_amdgcn_readfirstlane(*(LAS unsigned*)(L + LDS_MISC));
        const unsigned par = call & 1u;
        if (wid < 4) {
            const float s = scale;
            float y[32];
            unsigned long long nz[32];
#pragma unroll
            for (int i = 0; i < 32; ++i) {
                y[i] = acc[i >> 4][i & 15] * s;
                nz[i] = __builtin_amdgcn_fcmpf(y[i], 0.f, 10  );
            }
            unsigned mask = 0;
#pragma unroll
            for (int q = 3; q >= 0; --q)
                asm volatile("s_nop 1\n\t"
                             "v_addc_co_u32_e64 %0, vcc, %0, %0, %8\n\tv_addc_co_u32_e64 %0, vcc, %0, %0, %7\n\t"
                             "v_addc_co_u32_e64 %0, vcc, %0, %0, %6\n\tv_addc_co_u32_e64 %0, vcc, %0, %0, %5\n\t"
                             "v_addc_co_u32_e64 %0, vcc, %0, %0, %4\n\tv_addc_co_u32_e64 %0, vcc, %0, %0, %3\n\t"
                             "v_addc_co_u32_e64 %0, vcc, %0, %0, %2\n\tv_addc_co_u32_e64 %0, vcc, %0, %0, %1"
                             : "+v"(mask)
                             : "s"(nz[8 * q]), "s"(nz[8 * q + 1]), "s"(nz[8 * q + 2]), "s"(nz[8 * q + 3]),
                               "s"(nz[8 * q + 4]), "s"(nz[8 * q + 5]), "s"(nz[8 * q + 6]), "s"(nz[8 * q + 7])
                             : "vcc");
            const unsigned cnt = __builtin_popcount(mask);
            const unsigned incl = wave_scan(cnt);
            const unsigned total = __builtin_amdgcn_readlane(incl, 63);
            const unsigned reg = lds0 + wid * PK_SLOT;
            *(LAS unsigned*)(L + wid * PK_SLOT + lane * 4) = mask;
            unsigned ptr = reg + 256 + (incl - cnt) * 4;
#define PK1(m, v) "s_mov_b64 exec, %" #m "\n\tds_write_b32 %0, %" #v "\n\tv_add_u32 %0, 4, %0\n\t"
#pragma unroll
            for (int q = 0; q < 4; ++q)
                asm volatile(PK1(1, 9) PK1(2, 10) PK1(3, 11) PK1(4, 12) PK1(5, 13) PK1(6, 14) PK1(7, 15) PK1(8, 16)
                             "s_mov_b64 exec, -1"
                             : "+v"(ptr)
                             : "s"(nz[8 * q]), "s"(nz[8 * q + 1]), "s"(nz[8 * q + 2]), "s"(nz[8 * q + 3]),
                               "s"(nz[8 * q + 4]), "s"(nz[8 * q + 5]), "s"(nz[8 * q + 6]), "s"(nz[8 * q + 7]),
                               "v"(y[8 * q]), "v"(y[8 * q + 1]), "v"(y[8 * q + 2]), "v"(y[8 * q + 3]),
                               "v"(y[8 * q + 4]), "v"(y[8 * q + 5]), "v"(y[8 * q + 6]), "v"(y[8 * q + 7])
                             : "memory");
            WAIT_LGKM0();
            const unsigned bytes = 256 + 4 * total;
            const unsigned sidx = ((par * 8 + p.rank) * 32 + j) * 4 + wid;
            unsigned char* dslot = dws + WS_STAGE + (size_t)sidx * PK_SLOT;
            const unsigned lo = lane * 16;
            u32x4 c[5];
#pragma unroll
            for (int u = 0; u < 5; ++u) c[u] = *(LAS u32x4*)(L + wid * PK_SLOT + u * 1024 + lo);
#pragma unroll
            for (int u = 0; u < 5; ++u)
                if (u * 1024 + lo < bytes)
                    asm volatile("global_store_dwordx4 %0, %1, %2 sc0 sc1\n\ts_nop 1" :: "v"(lo + u * 1024), "v"(c[u]), "s"(dslot) : "memory");
#pragma unroll 1
            for (unsigned o = 5 * 1024; o < bytes; o += 1024) {
                const u32x4 v = *(LAS u32x4*)(L + wid * PK_SLOT + o + lo);
                if (o + lo < bytes)
                    asm volatile("global_store_dwordx4 %0, %1, %2 sc0 sc1\n\ts_nop 1" :: "v"(lo + o), "v"(v), "s"(dslot) : "memory");
            }
            WAIT_VM(0);
            if (lane == 0)
                __hip_atomic_store((unsigned long long*)(dws + WS_FLAG + (size_t)sidx * 128),
                                   ((unsigned long long)call << 32) | total, __ATOMIC_RELAXED, __HIP_MEMORY_SCOPE_SYSTEM);
            const unsigned ridx = ((par * 8 + dst) * 32 + j) * 4 + wid;
            unsigned rtotal;
            unsigned* tmo = (unsigned*)(myws + WS_TMO);
            {
                const unsigned char* flag = myws + WS_FLAG + (size_t)ridx * 128;
                const unsigned long long t0 = __builtin_amdgcn_s_memrealtime();
                for (;;) {
                    v2i f; unsigned t;
                    asm volatile("global_load_dwordx2 %0, %2, %3 sc0 sc1\n\tglobal_load_dword %1, %2, %4 sc1\n\ts_waitcnt vmcnt(0)"
                                 : "=&v"(f), "=&v"(t) : "v"(0u), "s"(flag), "s"(tmo) : "memory");
                    rtotal = __builtin_amdgcn_readfirstlane(f[0]);
                    if ((unsigned)__builtin_amdgcn_readfirstlane(f[1]) >= call) break;
                    if (__builtin_amdgcn_readfirstlane(t) == call || __builtin_amdgcn_s_memrealtime() - t0 > DEADLINE_TICKS) {
                        if (lane == 0) __hip_atomic_store(tmo, call, __ATOMIC_RELAXED, __HIP_MEMORY_SCOPE_AGENT);
                        rtotal = 0xffffffffu;
                        break;
                    }
                    __builtin_amdgcn_s_sleep(POLL_SLEEP);
                }
            }
            if (rtotal <= 2048u) {
                const unsigned rbytes = 256 + 4 * rtotal;
                const unsigned char* sslot = myws + WS_STAGE + (size_t)ridx * PK_SLOT;
#pragma unroll 1
                for (unsigned o = 0; o < rbytes; o += 1024)
                    asm volatile("s_mov_b32 m0, %2\n\ts_nop 0\n\tglobal_load_lds_dwordx4 %0, %1 sc0 sc1"
                                 :: "v"(lo + o), "s"(sslot), "s"(reg + o) : "memory");
                unsigned t;
                asm volatile("global_load_dword %0, %1, %2 sc1\n\ts_waitcnt vmcnt(0)" : "=&v"(t) : "v"(0u), "s"(tmo) : "memory");
                if (t == call) *(LAS unsigned*)(L + LDS_MISC + 4) = 1u;
                const unsigned rm = *(LAS unsigned*)(L + wid * PK_SLOT + lane * 4);
                const unsigned rc = __builtin_popcount(rm);
                const unsigned rb = wave_scan(rc) - rc;
                v2i tb; tb[0] = rm; tb[1] = wid * PK_SLOT + 256 + rb * 4;
                *(LAS v2i*)(L + LDS_TAB + wid * 512 + lane * 8) = tb;
            } else {
                *(LAS unsigned*)(L + LDS_MISC + 4) = 1u;
                *(LAS v2i*)(L + LDS_TAB + wid * 512 + lane * 8) = (v2i){0, 0};
            }
        }
        __syncthreads();
        const int w = wid & 3, mt = wid >> 2;
        float* ob = myout + (size_t)(dst * MS + tm * BM + mt * 32 + (lane >> 3)) * NS + tn * BN + w * 32 + (lane & 7) * 4;
        const bool bad = *(LAS unsigned*)(L + LDS_MISC + 4) != 0u;
        const unsigned bp = 16 * mt + 4 * ((lane & 7) >> 1);
        const unsigned low = (1u << bp) - 1u;
        u32x4 o[4];
#pragma unroll
        for (int u = 0; u < 4; ++u) {
            const v2i t = *(LAS v2i*)(L + LDS_TAB + w * 512 + (8 * u + (lane >> 3) + 32 * (lane & 1)) * 8);
            const unsigned m = (unsigned)t[0];
            int addr = t[1] + 4 * __builtin_popcount(m & low);
            const int ms = (int)(m >> bp);
#pragma unroll
            for (int e = 0; e < 4; ++e) {
                const int sel = __builtin_amdgcn_sbfe(ms, e, 1);
                o[u][e] = *(LAS unsigned*)(L + addr) & (unsigned)sel;
                addr -= 4 * sel;
            }
        }
        if (bad) {
#pragma unroll
            for (int u = 0; u < 4; ++u) o[u] = (u32x4){0x7fc00000u, 0x7fc00000u, 0x7fc00000u, 0x7fc00000u};
        }
#pragma unroll
        for (int u = 0; u < 4; ++u) {
            *(u32x4*)(ob + u * 8 * NS) = o[u];
        }
    }
}

extern "C" void kernel_launch(int rank, int nranks, ncclComm_t comm, hipStream_t stream,
                              void* const* d_in, const int* in_sizes, int n_in,
                              void* d_out, int out_size, const int* in_dtypes, int out_dtype,
                              void* const* d_out_all, void* const* d_ws_all, size_t ws_bytes) {
    if (nranks != NR || n_in != 4 || in_sizes[0] != MS * KK || in_sizes[1] != KK * NN || out_size != NR * MS * NS ||
        (unsigned)rank >= (unsigned)NR || ws_bytes < WS_END) {
        fprintf(stderr, "unexpected problem shape\n");
        return;
    }
    Params p;
    p.x = (const unsigned char*)d_in[0];
    p.w = (const unsigned char*)d_in[1];
    p.sx = (const float*)d_in[2];
    p.sw = (const float*)d_in[3];
    p.out = (float*)d_out;
    p.myws = (unsigned char*)d_ws_all[rank];
    for (int i = 0; i < NR; ++i) p.ws[i] = (unsigned char*)d_ws_all[i];
    p.rank = rank;
    p.pad = 0;
    gemm_a2a<<<GRID, 512, 0, stream>>>(p);
    HIP_CHECK(hipGetLastError());
}
```

```cpp
#include <hip/hip_runtime.h>
#include <rccl/rccl.h>
#include <cstdio>

#define HIP_CHECK(expr) do { hipError_t _e = (expr); if (_e != hipSuccess) { \
    fprintf(stderr, "HIP error %s at %s:%d\n", hipGetErrorString(_e), __FILE__, __LINE__); \
    return; } } while (0)

typedef int v8i __attribute__((ext_vector_type(8)));
typedef int v4i __attribute__((ext_vector_type(4)));
typedef int v2i __attribute__((ext_vector_type(2)));
typedef float v16f __attribute__((ext_vector_type(16)));
typedef float v4f __attribute__((ext_vector_type(4)));
typedef unsigned u32x4 __attribute__((ext_vector_type(4)));
#define LAS __attribute__((address_space(3)))

constexpr int NR = 8;
constexpr int MS = 512;
constexpr int KK = 4096;
constexpr int NN = 4096;
constexpr int NS = 512;
constexpr int BM = 64, BN = 128, BK = 128;
constexpr int NSTAGE = KK / BK;
#define DEPTH_ 6
constexpr int DEPTH = DEPTH_;
constexpr int A_BYTES = BM * BK;
constexpr int B_BYTES = BK * BN;
constexpr int SLOT = A_BYTES + B_BYTES;
constexpr int RING_BYTES = DEPTH * SLOT;
constexpr int LDS_MISC = RING_BYTES;
constexpr int LDS_BYTES = RING_BYTES + 16;
constexpr int PK_SLOT = 9216;
constexpr int LDS_TAB = 4 * PK_SLOT;
constexpr int GRID = 256;
#define POLL_SLEEP 2
#define OUTMODE 0
constexpr unsigned long long DEADLINE_TICKS = 50000000ull;

constexpr size_t WS_CTR = 0;
constexpr size_t WS_TMO = 32768;
constexpr size_t WS_STAMP = 65536;
constexpr size_t WS_FLAG = 131072;
constexpr size_t WS_STAGE = WS_FLAG + 2 * 1024 * 128;
constexpr size_t WS_END = WS_STAGE + (size_t)2 * 1024 * PK_SLOT;

struct Params {
    const unsigned char* x;
    const unsigned char* w;
    const float* sx;
    const float* sw;
    float* out;
    unsigned char* myws;
    unsigned char* ws[NR];
    int rank;
    int pad;
};

#define WAIT_VM(n) asm volatile("s_waitcnt vmcnt(%0)" :: "i"(n) : "memory")
#define WAIT_LGKM0() asm volatile("s_waitcnt lgkmcnt(0)" ::: "memory")
#define BAR() __builtin_amdgcn_s_barrier()
#define SCHED() __builtin_amdgcn_sched_barrier(0)

__device__ __forceinline__ void glds16(unsigned voff, const unsigned char* sbase, unsigned lds_dst) {
    asm volatile("s_mov_b32 m0, %2\n\ts_nop 0\n\tglobal_load_lds_dwordx4 %0, %1"
                 :: "v"(voff), "s"(sbase), "s"(lds_dst) : "memory");
}

#define RA 1
#define RB 2
#define RC 0
#define SS 8
__device__ __forceinline__ int korder(int s, int rot) { return (s & ~(SS - 1)) | ((s + rot) & (SS - 1)); }

template <int I>
struct LoaderLoop {
    static __device__ __forceinline__ void run(const unsigned (&va)[2], const unsigned (&vb)[4], const unsigned char* xa,
                                               const unsigned char* wb, unsigned ldsA, unsigned ldsB, int rot) {
        constexpr int AFTER = (NSTAGE - 1 - I) < (DEPTH - 2) ? (NSTAGE - 1 - I) : (DEPTH - 2);
        WAIT_VM(6 * AFTER);
        BAR();
        if constexpr (I + DEPTH - 1 < NSTAGE) {
            constexpr int S = I + DEPTH - 1;
            const int kb = korder(S, rot);
            const unsigned char* pa = xa + kb * BK;
            const unsigned char* pb = wb + (size_t)kb * BK * NN;
            constexpr unsigned so = (S % DEPTH) * SLOT;
            glds16(vb[0], pb, ldsB + so);
            glds16(vb[1], pb, ldsB + so + 1024);
            glds16(va[0], pa, ldsA + so);
            glds16(vb[2], pb, ldsB + so + 2048);
            glds16(vb[3], pb, ldsB + so + 3072);
            glds16(va[1], pa, ldsA + so + 1024);
        }
        if constexpr (I + 1 < NSTAGE) LoaderLoop<I + 1>::run(va, vb, xa, wb, ldsA, ldsB, rot);
    }
};

__device__ __forceinline__ unsigned wave_scan(unsigned v) {
    v += __builtin_amdgcn_update_dpp(0u, v, 0x111, 0xf, 0xf, true);
    v += __builtin_amdgcn_update_dpp(0u, v, 0x112, 0xf, 0xf, true);
    v += __builtin_amdgcn_update_dpp(0u, v, 0x114, 0xf, 0xf, true);
    v += __builtin_amdgcn_update_dpp(0u, v, 0x118, 0xf, 0xf, true);
    v += __builtin_amdgcn_update_dpp(0u, v, 0x142, 0xa, 0xf, false);
    v += __builtin_amdgcn_update_dpp(0u, v, 0x143, 0xc, 0xf, false);
    return v;
}

struct Frag { v4i a[2][2][2]; v2i b[2][4]; };

__device__ __forceinline__ v8i cat8(v4i lo, v4i hi) { return __builtin_shufflevector(lo, hi, 0, 1, 2, 3, 4, 5, 6, 7); }
__device__ __forceinline__ v8i cat8b(v2i a, v2i b, v2i c, v2i d) {
    v4i lo = __builtin_shufflevector(a, b, 0, 1, 2, 3), hi = __builtin_shufflevector(c, d, 0, 1, 2, 3);
    return cat8(lo, hi);
}

__global__ void __launch_bounds__(512, 2) gemm_a2a(Params p) {
    __shared__ __attribute__((aligned(1024))) unsigned char lds[LDS_BYTES];
    const int tid = threadIdx.x, lane = tid & 63;
    const int wid = __builtin_amdgcn_readfirstlane(tid >> 6);
    const int b = blockIdx.x;
    const int dst = b & 7, j = b >> 3, tm = j >> 2, tn = j & 3;
    const unsigned lds0 = (unsigned)(size_t)(LAS unsigned char*)lds;
    unsigned char* myws = p.myws;
    v16f acc[2] = {};
    float scale = 0.f;
    unsigned char* dws = nullptr;
    float* myout = p.out;

    if (wid >= 4) {
        const int lw = wid - 4;
        unsigned ticket = 0;
        if (lw == 0) {
            unsigned char* ctr = myws + WS_CTR + b * 128;
            asm volatile("s_mov_b64 exec, 1\n\tglobal_atomic_add %0, %1, %2, %3 sc0\n\ts_mov_b64 exec, -1"
                         : "=&v"(ticket) : "v"(0u), "v"(1u), "s"(ctr) : "memory");
        }
        unsigned va[2], vb[4];
#pragma unroll
        for (int i = 0; i < 2; ++i) {
            const int row = (lw * 2 + i) * 8 + (lane >> 3);
            va[i] = row * KK + (((lane & 7) ^ ((row >> 1) & 7)) << 4);
        }
#pragma unroll
        for (int i = 0; i < 4; ++i) {
            const int row = (lw * 4 + i) * 8 + (lane >> 3);
            vb[i] = row * NN + (((lane & 7) ^ (row & 6)) << 4);
        }
        const unsigned char* xa = p.x + (size_t)tm * BM * KK;
        const unsigned char* wb = p.w + dst * NS + tn * BN;
        const unsigned ldsA = lds0 + lw * 2048, ldsB = lds0 + A_BYTES + lw * 4096;
        const int rot = RA * tm + RB * tn + RC * dst;
#pragma unroll
        for (int s = 0; s < DEPTH - 1; ++s) {
            const int kb = korder(s, rot);
            const unsigned char* pa = xa + kb * BK;
            const unsigned char* pb = wb + (size_t)kb * BK * NN;
            const unsigned so = s * SLOT;
            glds16(vb[0], pb, ldsB + so);
            glds16(vb[1], pb, ldsB + so + 1024);
            glds16(va[0], pa, ldsA + so);
            glds16(vb[2], pb, ldsB + so + 2048);
            glds16(vb[3], pb, ldsB + so + 3072);
            glds16(va[1], pa, ldsA + so + 1024);
        }
        LoaderLoop<0>::run(va, vb, xa, wb, ldsA, ldsB, rot);
        if (lw == 0) {
            asm volatile("" : "+v"(ticket));
            if (lane == 0) {
                *(LAS unsigned*)((LAS unsigned char*)lds + LDS_MISC) = ticket + 1u;
                *(LAS unsigned*)((LAS unsigned char*)lds + LDS_MISC + 4) = 0u;
            }
            WAIT_LGKM0();
        }
    } else {
        scale = __builtin_bit_cast(float, __builtin_amdgcn_readfirstlane(__builtin_bit_cast(int, p.sx[0] * p.sw[0])));
        dws = p.ws[dst];
        asm volatile("" : "+s"(scale), "+s"(dws));
        const int r = lane & 31, h = lane >> 5;
        unsigned ao[2][2];
#pragma unroll
        for (int ks = 0; ks < 2; ++ks)
#pragma unroll
            for (int e = 0; e < 2; ++e) ao[ks][e] = r * 128 + (((4 * ks + 2 * h + e) ^ ((r >> 1) & 7)) << 4);
        const int q = (lane & 15) >> 1, pp = lane & 1, g1 = (lane >> 4) & 1;
        const unsigned bo = A_BYTES + (32 * h + q) * 128 + (((2 * wid + g1) ^ (q & 6)) << 4) + pp * 8;

        auto rd = [&](Frag& f, int slot) {
            const unsigned char LAS* base = (const unsigned char LAS*)lds + slot * SLOT;
#pragma unroll
            for (int ks = 0; ks < 2; ++ks) {
#pragma unroll
                for (int t = 0; t < 4; ++t)
                    f.b[ks][t] = __builtin_amdgcn_ds_read_tr8_b64_v2i32((LAS v2i*)(base + bo + (64 * ks + 8 * t) * 128));
#pragma unroll
                for (int mt = 0; mt < 2; ++mt)
#pragma unroll
                    for (int e = 0; e < 2; ++e) f.a[ks][mt][e] = *(const LAS v4i*)(base + ao[ks][e] + mt * 4096);
            }
        };
        auto mm = [&](const Frag& f) {
#pragma unroll
            for (int ks = 0; ks < 2; ++ks) {
                const v8i wf = cat8b(f.b[ks][0], f.b[ks][1], f.b[ks][2], f.b[ks][3]);
#pragma unroll
                for (int mt = 0; mt < 2; ++mt)
                    acc[mt] = __builtin_amdgcn_mfma_scale_f32_32x32x64_f8f6f4(wf, cat8(f.a[ks][mt][0], f.a[ks][mt][1]), acc[mt], 0, 0, 0, 0, 0, 0);
            }
        };
        Frag f0, f1;
        BAR();
        rd(f0, 0);
        WAIT_LGKM0();
        SCHED();
        int slot = 1;
#pragma unroll 1
        for (int i = 1; i < NSTAGE - 1; i += 2) {
            BAR();
            rd(f1, slot);
            mm(f0);
#pragma unroll
            for (int u = 0; u < 4; ++u) {
                __builtin_amdgcn_sched_group_barrier(0x8, 1, 0);
                __builtin_amdgcn_sched_group_barrier(0x100, 4, 0);
            }
            WAIT_LGKM0();
            SCHED();
            slot = slot + 1 == DEPTH ? 0 : slot + 1;
            BAR();
            rd(f0, slot);
            mm(f1);
#pragma unroll
            for (int u = 0; u < 4; ++u) {
                __builtin_amdgcn_sched_group_barrier(0x8, 1, 0);
                __builtin_amdgcn_sched_group_barrier(0x100, 4, 0);
            }
            WAIT_LGKM0();
            SCHED();
            slot = slot + 1 == DEPTH ? 0 : slot + 1;
        }
        BAR();
        rd(f1, slot);
        mm(f0);
        WAIT_LGKM0();
        SCHED();
        mm(f1);
    }
    BAR();
    LAS unsigned char* L = (LAS unsigned char*)lds;
    if (dst == p.rank) {
        if (wid < 4) {
            const float s = scale;
            const int r = lane & 31, h = lane >> 5;
#pragma unroll
            for (int mt = 0; mt < 2; ++mt)
#pragma unroll
                for (int g = 0; g < 4; ++g) {
                    v4f v;
#pragma unroll
                    for (int e = 0; e < 4; ++e) { const float y = acc[mt][4 * g + e] * s; v[e] = y <= 0.f ? 0.f : y; }
                    const int row = mt * 32 + r, ch = wid * 8 + 2 * g + h;
                    *(LAS v4f*)(L + row * 512 + ((ch ^ (row & 7)) << 4)) = v;
                }
        }
        __syncthreads();
        float* ob = myout + (size_t)(p.rank * MS + tm * BM) * NS + tn * BN;
#pragma unroll
        for (int u = 0; u < 4; ++u) {
            const int row = (wid * 4 + u) * 2 + (lane >> 5), ch = lane & 31;
            *(v4f*)(ob + row * NS + ch * 4) = *(LAS v4f*)(L + row * 512 + ((ch ^ (row & 7)) << 4));
        }
    } else {
        const unsigned call = __builtin_amdgcn_readfirstlane(*(LAS unsigned*)(L + LDS_MISC));
        const unsigned par = call & 1u;
        if (wid < 4) {
            const float s = scale;
            float y[32];
            unsigned long long nz[32];
#pragma unroll
            for (int i = 0; i < 32; ++i) {
                y[i] = acc[i >> 4][i & 15] * s;
                nz[i] = __builtin_amdgcn_fcmpf(y[i], 0.f, 10  );
            }
            unsigned mask = 0;
#pragma unroll
            for (int q = 3; q >= 0; --q)
                asm volatile("s_nop 1\n\t"
                             "v_addc_co_u32_e64 %0, vcc, %0, %0, %8\n\tv_addc_co_u32_e64 %0, vcc, %0, %0, %7\n\t"
                             "v_addc_co_u32_e64 %0, vcc, %0, %0, %6\n\tv_addc_co_u32_e64 %0, vcc, %0, %0, %5\n\t"
                             "v_addc_co_u32_e64 %0, vcc, %0, %0, %4\n\tv_addc_co_u32_e64 %0, vcc, %0, %0, %3\n\t"
                             "v_addc_co_u32_e64 %0, vcc, %0, %0, %2\n\tv_addc_co_u32_e64 %0, vcc, %0, %0, %1"
                             : "+v"(mask)
                             : "s"(nz[8 * q]), "s"(nz[8 * q + 1]), "s"(nz[8 * q + 2]), "s"(nz[8 * q + 3]),
                               "s"(nz[8 * q + 4]), "s"(nz[8 * q + 5]), "s"(nz[8 * q + 6]), "s"(nz[8 * q + 7])
                             : "vcc");
            const unsigned cnt = __builtin_popcount(mask);
            const unsigned incl = wave_scan(cnt);
            const unsigned total = __builtin_amdgcn_readlane(incl, 63);
            const unsigned reg = lds0 + wid * PK_SLOT;
            *(LAS unsigned*)(L + wid * PK_SLOT + lane * 4) = mask;
            unsigned ptr = reg + 256 + (incl - cnt) * 4;
#define PK1(m, v) "s_mov_b64 exec, %" #m "\n\tds_write_b32 %0, %" #v "\n\tv_add_u32 %0, 4, %0\n\t"
#pragma unroll
            for (int q = 0; q < 4; ++q)
                asm volatile(PK1(1, 9) PK1(2, 10) PK1(3, 11) PK1(4, 12) PK1(5, 13) PK1(6, 14) PK1(7, 15) PK1(8, 16)
                             "s_mov_b64 exec, -1"
                             : "+v"(ptr)
                             : "s"(nz[8 * q]), "s"(nz[8 * q + 1]), "s"(nz[8 * q + 2]), "s"(nz[8 * q + 3]),
                               "s"(nz[8 * q + 4]), "s"(nz[8 * q + 5]), "s"(nz[8 * q + 6]), "s"(nz[8 * q + 7]),
                               "v"(y[8 * q]), "v"(y[8 * q + 1]), "v"(y[8 * q + 2]), "v"(y[8 * q + 3]),
                               "v"(y[8 * q + 4]), "v"(y[8 * q + 5]), "v"(y[8 * q + 6]), "v"(y[8 * q + 7])
                             : "memory");
            WAIT_LGKM0();
            const unsigned bytes = 256 + 4 * total;
            const unsigned sidx = ((par * 8 + p.rank) * 32 + j) * 4 + wid;
            unsigned char* dslot = dws + WS_STAGE + (size_t)sidx * PK_SLOT;
            const unsigned lo = lane * 16;
            u32x4 c[5];
#pragma unroll
            for (int u = 0; u < 5; ++u) c[u] = *(LAS u32x4*)(L + wid * PK_SLOT + u * 1024 + lo);
#pragma unroll
            for (int u = 0; u < 5; ++u)
                if (u * 1024 + lo < bytes)
                    asm volatile("global_store_dwordx4 %0, %1, %2 sc0 sc1\n\ts_nop 1" :: "v"(lo + u * 1024), "v"(c[u]), "s"(dslot) : "memory");
#pragma unroll 1
            for (unsigned o = 5 * 1024; o < bytes; o += 1024) {
                const u32x4 v = *(LAS u32x4*)(L + wid * PK_SLOT + o + lo);
                if (o + lo < bytes)
                    asm volatile("global_store_dwordx4 %0, %1, %2 sc0 sc1\n\ts_nop 1" :: "v"(lo + o), "v"(v), "s"(dslot) : "memory");
            }
            WAIT_VM(0);
            if (lane == 0)
                __hip_atomic_store((unsigned long long*)(dws + WS_FLAG + (size_t)sidx * 128),
                                   ((unsigned long long)call << 32) | total, __ATOMIC_RELAXED, __HIP_MEMORY_SCOPE_SYSTEM);
            const unsigned ridx = ((par * 8 + dst) * 32 + j) * 4 + wid;
            unsigned rtotal;
            unsigned* tmo = (unsigned*)(myws + WS_TMO);
            {
                const unsigned char* flag = myws + WS_FLAG + (size_t)ridx * 128;
                const unsigned long long t0 = __builtin_amdgcn_s_memrealtime();
                for (;;) {
                    v2i f; unsigned t;
                    asm volatile("global_load_dwordx2 %0, %2, %3 sc0 sc1\n\tglobal_load_dword %1, %2, %4 sc1\n\ts_waitcnt vmcnt(0)"
                                 : "=&v"(f), "=&v"(t) : "v"(0u), "s"(flag), "s"(tmo) : "memory");
                    rtotal = __builtin_amdgcn_readfirstlane(f[0]);
                    if ((unsigned)__builtin_amdgcn_readfirstlane(f[1]) >= call) break;
                    if (__builtin_amdgcn_readfirstlane(t) == call || __builtin_amdgcn_s_memrealtime() - t0 > DEADLINE_TICKS) {
                        if (lane == 0) __hip_atomic_store(tmo, call, __ATOMIC_RELAXED, __HIP_MEMORY_SCOPE_AGENT);
                        rtotal = 0xffffffffu;
                        break;
                    }
                    __builtin_amdgcn_s_sleep(POLL_SLEEP);
                }
            }
            if (rtotal <= 2048u) {
                const unsigned rbytes = 256 + 4 * rtotal;
                const unsigned char* sslot = myws + WS_STAGE + (size_t)ridx * PK_SLOT;
#pragma unroll 1
                for (unsigned o = 0; o < rbytes; o += 1024)
                    asm volatile("s_mov_b32 m0, %2\n\ts_nop 0\n\tglobal_load_lds_dwordx4 %0, %1 sc0 sc1"
                                 :: "v"(lo + o), "s"(sslot), "s"(reg + o) : "memory");
                unsigned t;
                asm volatile("global_load_dword %0, %1, %2 sc1\n\ts_waitcnt vmcnt(0)" : "=&v"(t) : "v"(0u), "s"(tmo) : "memory");
                if (t == call) *(LAS unsigned*)(L + LDS_MISC + 4) = 1u;
                const unsigned rm = *(LAS unsigned*)(L + wid * PK_SLOT + lane * 4);
                const unsigned rc = __builtin_popcount(rm);
                const unsigned rb = wave_scan(rc) - rc;
                v2i tb; tb[0] = rm; tb[1] = wid * PK_SLOT + 256 + rb * 4;
                *(LAS v2i*)(L + LDS_TAB + wid * 512 + lane * 8) = tb;
            } else {
                *(LAS unsigned*)(L + LDS_MISC + 4) = 1u;
                *(LAS v2i*)(L + LDS_TAB + wid * 512 + lane * 8) = (v2i){0, 0};
            }
        }
        __syncthreads();
        const int w = wid & 3, mt = wid >> 2;
        float* ob = myout + (size_t)(dst * MS + tm * BM + mt * 32 + (lane >> 3)) * NS + tn * BN + w * 32 + (lane & 7) * 4;
        const bool bad = *(LAS unsigned*)(L + LDS_MISC + 4) != 0u;
        const unsigned bp = 16 * mt + 4 * ((lane & 7) >> 1);
        const unsigned low = (1u << bp) - 1u;
        u32x4 o[4];
#pragma unroll
        for (int u = 0; u < 4; ++u) {
            const v2i t = *(LAS v2i*)(L + LDS_TAB + w * 512 + (8 * u + (lane >> 3) + 32 * (lane & 1)) * 8);
            const unsigned m = (unsigned)t[0];
            int addr = t[1] + 4 * __builtin_popcount(m & low);
            const int ms = (int)(m >> bp);
#pragma unroll
            for (int e = 0; e < 4; ++e) {
                const int sel = __builtin_amdgcn_sbfe(ms, e, 1);
                o[u][e] = *(LAS unsigned*)(L + addr) & (unsigned)sel;
                addr -= 4 * sel;
            }
        }
        if (bad) {
#pragma unroll
            for (int u = 0; u < 4; ++u) o[u] = (u32x4){0x7fc00000u, 0x7fc00000u, 0x7fc00000u, 0x7fc00000u};
        }
#pragma unroll
        for (int u = 0; u < 4; ++u) {
            *(u32x4*)(ob + u * 8 * NS) = o[u];
        }
    }
}

extern "C" void kernel_launch(int rank, int nranks, ncclComm_t comm, hipStream_t stream,
                              void* const* d_in, const int* in_sizes, int n_in,
                              void* d_out, int out_size, const int* in_dtypes, int out_dtype,
                              void* const* d_out_all, void* const* d_ws_all, size_t ws_bytes) {
    if (nranks != NR || n_in != 4 || in_sizes[0] != MS * KK || in_sizes[1] != KK * NN || out_size != NR * MS * NS ||
        (unsigned)rank >= (unsigned)NR || ws_bytes < WS_END) {
        fprintf(stderr, "unexpected problem shape\n");
        return;
    }
    Params p;
    p.x = (const unsigned char*)d_in[0];
    p.w = (const unsigned char*)d_in[1];
    p.sx = (const float*)d_in[2];
    p.sw = (const float*)d_in[3];
    p.out = (float*)d_out;
    p.myws = (unsigned char*)d_ws_all[rank];
    for (int i = 0; i < NR; ++i) p.ws[i] = (unsigned char*)d_ws_all[i];
    p.rank = rank;
    p.pad = 0;
    gemm_a2a<<<GRID, 512, 0, stream>>>(p);
    HIP_CHECK(hipGetLastError());
}
```

```cpp
#include <hip/hip_runtime.h>
#include <rccl/rccl.h>
#include <cstdio>

#define HIP_CHECK(expr) do { hipError_t _e = (expr); if (_e != hipSuccess) { \
    fprintf(stderr, "HIP error %s at %s:%d\n", hipGetErrorString(_e), __FILE__, __LINE__); \
    return; } } while (0)

typedef int v8i __attribute__((ext_vector_type(8)));
typedef int v4i __attribute__((ext_vector_type(4)));
typedef int v2i __attribute__((ext_vector_type(2)));
typedef float v16f __attribute__((ext_vector_type(16)));
typedef float v4f __attribute__((ext_vector_type(4)));
typedef unsigned u32x4 __attribute__((ext_vector_type(4)));
#define LAS __attribute__((address_space(3)))

constexpr int NR = 8;
constexpr int MS = 512;
constexpr int KK = 4096;
constexpr int NN = 4096;
constexpr int NS = 512;
constexpr int BM = 64, BN = 128, BK = 128;
constexpr int NSTAGE = KK / BK;
constexpr int XROW0 = MS - BM;
constexpr int GRID = 256;
constexpr unsigned long long DEADLINE_TICKS = 50000000ull;

#define DEPTH_ 4
#define POLL_SLEEP 2
#define ROUND 16
#define OUTMODE 1
#define WIN_F 2
#define WIN_L 2
#define PRO 2
#define RA 1
#define RB 2
#define SS 8

#define LEAD0 3
#define PRETOUCH 1

constexpr int DEPTH = DEPTH_;
constexpr int ADEPTH = DEPTH + 1;
constexpr int A_BYTES = BM * BK;
constexpr int B_BYTES = BK * BN;
constexpr int A_RING = DEPTH * B_BYTES;
constexpr int RING_BYTES = DEPTH * B_BYTES + ADEPTH * A_BYTES;
constexpr int PK_SLOT = 5120;
constexpr int LDS_RECV = 8 * PK_SLOT;
constexpr int LDS_TAB = 16 * PK_SLOT;
constexpr int LDS_HAND = RING_BYTES;
constexpr int LDS_MISC = LDS_HAND + 4 * 4096;
constexpr int LDS_BYTES = LDS_MISC + 16;
static_assert(LDS_TAB + 8 * 512 <= RING_BYTES && LDS_BYTES > 80 * 1024 && LDS_BYTES <= 160 * 1024, "LDS layout");

constexpr size_t WS_CTR = 0;
constexpr size_t WS_TMO = 32768;
constexpr size_t WS_XFLAG = 36864;
constexpr size_t WS_STAMP = 65536;
constexpr size_t WS_FLAG = 131072;
constexpr size_t WS_XSTAGE = WS_FLAG + 2 * 2048 * 128;
constexpr size_t WS_STAGE = WS_XSTAGE + (size_t)16 * BM * KK;
constexpr size_t WS_END = WS_STAGE + (size_t)2 * 2048 * PK_SLOT;

struct Params {
    const unsigned char* x;
    const unsigned char* w;
    unsigned char* myws;
    int rank;
    int pad;
    const float* sx;
    const float* sw;
    float* out;
    unsigned char* ws[NR];
};

#define WAIT_VM(n) asm volatile("s_waitcnt vmcnt(%0)" :: "i"(n) : "memory")
#define WAIT_LGKM0() asm volatile("s_waitcnt lgkmcnt(0)" ::: "memory")
#define BAR() __builtin_amdgcn_s_barrier()
#define SCHED() __builtin_amdgcn_sched_barrier(0)
#define STAMP(k) do {} while (0)

#define KORD 1
template <int S>
__device__ __forceinline__ int korder(int lead, int tn) {
    const int rs = (S + tn) & 3;
    const int kl = S < 8 ? lead + S : (lead + (S & ~3) + rs) & (NSTAGE - 1);
    return lead < 0 ? (S >> 2) + 8 * rs : kl;
}

struct LoadCtx {
    unsigned va[2], vb[4];
    const unsigned char* xa;
    const unsigned char* wb;
    unsigned ldsA, ldsB;
    int lead, tn;
};
template <int S>
__device__ __forceinline__ void load_w(const LoadCtx& c) {
    const unsigned char* pb = c.wb + (size_t)korder<S>(c.lead, c.tn) * BK * NN;
    asm volatile("s_mov_b32 m0, %5\n\ts_nop 0\n\t"
                 "global_load_lds_dwordx4 %0, %4\n\tglobal_load_lds_dwordx4 %1, %4 offset:1024\n\t"
                 "global_load_lds_dwordx4 %2, %4 offset:2048\n\tglobal_load_lds_dwordx4 %3, %4 offset:3072"
                 :: "v"(c.vb[0]), "v"(c.vb[1]), "v"(c.vb[2]), "v"(c.vb[3]), "s"(pb), "s"(c.ldsB + (S % DEPTH) * B_BYTES) : "memory");
}
template <int S>
__device__ __forceinline__ void load_x(const LoadCtx& c) {
    const unsigned char* pa = c.xa + korder<S>(c.lead, c.tn) * BK;
    asm volatile("s_mov_b32 m0, %3\n\ts_nop 0\n\t"
                 "global_load_lds_dwordx4 %0, %2 sc0 sc1\n\tglobal_load_lds_dwordx4 %1, %2 offset:1024 sc0 sc1"
                 :: "v"(c.va[0]), "v"(c.va[1]), "s"(pa), "s"(c.ldsA + (S % ADEPTH) * A_BYTES) : "memory");
}
constexpr int mini(int a, int b) { return a < b ? a : b; }
constexpr int issued(int I) { return mini(NSTAGE, mini(PRO + 2 * I, I + DEPTH - 1)); }
constexpr int younger(int I) {
    return 4 * (issued(I) - 1 - I) + 2 * (mini(issued(I), NSTAGE - 1) - I);
}
template <int S, int E>
__device__ __forceinline__ void load_range(const LoadCtx& c) {
    if constexpr (S < E) {
        load_w<S>(c);
        if constexpr (S + 1 < NSTAGE) load_x<S + 1>(c);
        load_range<S + 1, E>(c);
    }
}
template <int I>
__device__ __forceinline__ void loader_loop(const LoadCtx& c) {
    WAIT_VM(younger(I));
    BAR();
    if constexpr (I + 1 < NSTAGE) {
        load_range<issued(I), issued(I + 1)>(c);
        loader_loop<I + 1>(c);
    }
}

__device__ __forceinline__ unsigned wave_scan(unsigned v) {
    v += __builtin_amdgcn_update_dpp(0u, v, 0x111, 0xf, 0xf, true);
    v += __builtin_amdgcn_update_dpp(0u, v, 0x112, 0xf, 0xf, true);
    v += __builtin_amdgcn_update_dpp(0u, v, 0x114, 0xf, 0xf, true);
    v += __builtin_amdgcn_update_dpp(0u, v, 0x118, 0xf, 0xf, true);
    v += __builtin_amdgcn_update_dpp(0u, v, 0x142, 0xa, 0xf, false);
    v += __builtin_amdgcn_update_dpp(0u, v, 0x143, 0xc, 0xf, false);
    return v;
}

__device__ __forceinline__ unsigned draw_ticket(unsigned char* ctr) {
    unsigned t;
    asm volatile("s_mov_b64 exec, 1\n\tglobal_atomic_add %0, %1, %2, %3 sc0\n\ts_mov_b64 exec, -1"
                 : "=&v"(t) : "v"(0u), "v"(1u), "s"(ctr) : "memory");
    return t;
}

__device__ __forceinline__ void put_flag(unsigned char* flag, unsigned call, unsigned low) {
    typedef __attribute__((address_space(1))) unsigned long long gu64;
    __hip_atomic_store((gu64*)flag, ((unsigned long long)call << 32) | low, __ATOMIC_RELAXED, __HIP_MEMORY_SCOPE_SYSTEM);
}

__device__ __forceinline__ unsigned wait_flag(const unsigned char* flag, unsigned voff, unsigned* tmo, unsigned call, int lane) {
    const unsigned long long t0 = __builtin_amdgcn_s_memrealtime();
    for (;;) {
        v2i f;
        unsigned t;
        asm volatile("global_load_dwordx2 %0, %2, %3 sc0 sc1\n\tglobal_load_dword %1, %4, %5 sc1\n\ts_waitcnt vmcnt(0)"
                     : "=&v"(f), "=&v"(t) : "v"(voff), "s"(flag), "v"(0u), "s"(tmo) : "memory");
        if (__builtin_amdgcn_ballot_w64((unsigned)f[1] < call) == 0ull) return __builtin_amdgcn_readfirstlane(f[0]);
        if (__builtin_amdgcn_readfirstlane(t) == call || __builtin_amdgcn_s_memrealtime() - t0 > DEADLINE_TICKS) {
            if (lane == 0) __hip_atomic_store(tmo, call, __ATOMIC_RELAXED, __HIP_MEMORY_SCOPE_AGENT);
            return 0xffffffffu;
        }
        __builtin_amdgcn_s_sleep(POLL_SLEEP);
    }
}

__device__ __forceinline__ void store_out(float* ptr, u32x4 v) {
    asm volatile("global_store_dwordx4 %0, %1, off sc0 sc1\n\ts_nop 1" :: "v"(ptr), "v"(v) : "memory");
}

struct Frag { v4i a[2][2][2]; v2i b[2][4]; };

__device__ __forceinline__ v8i cat8(v4i lo, v4i hi) { return __builtin_shufflevector(lo, hi, 0, 1, 2, 3, 4, 5, 6, 7); }
__device__ __forceinline__ v8i cat8b(v2i a, v2i b, v2i c, v2i d) {
    return cat8(__builtin_shufflevector(a, b, 0, 1, 2, 3), __builtin_shufflevector(c, d, 0, 1, 2, 3));
}

__global__ void __launch_bounds__(512, 2) gemm_a2a(Params p) {
    __shared__ __attribute__((aligned(1024))) unsigned char lds[LDS_BYTES];
    LAS unsigned char* L = (LAS unsigned char*)lds;
    const unsigned lds0 = (unsigned)(size_t)L;
    const int tid = threadIdx.x, lane = tid & 63;
    const int wid = __builtin_amdgcn_readfirstlane(tid >> 6);
    const int b = blockIdx.x;
    const int g = b & 7, j = b >> 3, tm = j >> 2, tn = j & 3;
    unsigned char* myws = p.myws;
    unsigned* tmo = (unsigned*)(myws + WS_TMO);
    const int rank = p.rank;
    if (wid == 0) STAMP(0);

    const unsigned char* xa = p.x + (size_t)tm * BM * KK;
    int wcol = g * NS + tn * BN;
    int orow = rank * MS + tm * BM, ocol = tn * BN;
    const bool gathered = g != rank && tm == 7;
    const bool local = g == rank || gathered;
    bool lost = false;

    if (gathered) {
        unsigned ticket = 0;
        if (wid == 0) ticket = draw_ticket(myws + WS_CTR + b * 128);
        const unsigned char* src = p.x + (size_t)(XROW0 + tn * 16 + wid * 2) * KK;
        u32x4 r[8];
#pragma unroll
        for (int u = 0; u < 8; ++u) r[u] = *(const u32x4*)(src + u * 1024 + lane * 16);
        if (wid == 0) {
            asm volatile("s_waitcnt vmcnt(0)" : "+v"(ticket) :: "memory");
            if (lane == 0) *(LAS v4i*)(L + LDS_MISC) = (v4i){(int)(ticket + 1u), 0, 0, 0};
        }
        __syncthreads();
        const unsigned call = __builtin_amdgcn_readfirstlane(*(LAS unsigned*)(L + LDS_MISC));
        const unsigned par = call & 1u;
        unsigned char* dws = p.ws[g];
        unsigned char* dx = dws + WS_XSTAGE + (size_t)((par * 8 + rank) * BM + tn * 16 + wid * 2) * KK;
        asm volatile("" : "+v"(r[0]), "+v"(r[1]), "+v"(r[2]), "+v"(r[3]), "+v"(r[4]), "+v"(r[5]), "+v"(r[6]), "+v"(r[7]));
#pragma unroll
        for (int u = 0; u < 8; ++u)
            asm volatile("global_store_dwordx4 %0, %1, %2 sc0 sc1\n\ts_nop 1" :: "v"(lane * 16 + u * 1024), "v"(r[u]), "s"(dx) : "memory");
        WAIT_VM(0);
        __syncthreads();
        const int u = ((g - rank - 1) & 7) * 4 + tn;
        const int src_rank = (rank + 1 + u % 7) & 7;
        if (wid == 0) {
            if (lane == 0) put_flag(dws + WS_XFLAG + ((par * 8 + rank) * 4 + tn) * 128, call, 0u);
            STAMP(2);
            const unsigned f = wait_flag(myws + WS_XFLAG + (par * 8 + src_rank) * 512, (lane & 3) * 128, tmo, call, lane);
            const unsigned t = __hip_atomic_load(tmo, __ATOMIC_RELAXED, __HIP_MEMORY_SCOPE_AGENT);
            if (f != 0u || t == call) *(LAS unsigned*)(L + LDS_MISC + 4) = 1u;
            STAMP(3);
        }
        __syncthreads();
        lost = __builtin_amdgcn_readfirstlane(*(LAS unsigned*)(L + LDS_MISC + 4)) != 0;
        xa = lost ? p.x : myws + WS_XSTAGE + (size_t)(par * 8 + src_rank) * BM * KK;
        ocol = (u / 7) * BN;
        wcol = rank * NS + ocol;
        orow = src_rank * MS + XROW0;
    }

    if (PRETOUCH && g == rank) {
        unsigned t;
        asm volatile("global_load_dword %0, %1, %2\n\ts_waitcnt vmcnt(0)" : "=&v"(t) : "v"((j * 512 + tid) * 128), "s"(p.x) : "memory");
    }

    v16f acc[2] = {};
    float scale = 0.f;
    unsigned char* dws = nullptr;

    if (wid >= 4) {
        const int lw = wid - 4;
        LoadCtx c;
#pragma unroll
        for (int i = 0; i < 2; ++i) {
            const int row = (lw * 2 + i) * 8 + (lane >> 3);
            c.va[i] = row * KK + (((lane & 7) ^ ((row >> 1) & 7)) << 4) - i * 1024;
        }
#pragma unroll
        for (int i = 0; i < 4; ++i) {
            const int row = (lw * 4 + i) * 8 + (lane >> 3);
            c.vb[i] = row * NN + (((lane & 7) ^ (row & 6)) << 4) - i * 1024;
        }
        c.xa = xa;
        c.wb = p.w + wcol;
        c.ldsA = lds0 + A_RING + lw * 2048;
        c.ldsB = lds0 + lw * 4096;
        c.lead = tm >= LEAD0 && tm < LEAD0 + 4 ? 8 * (tm - LEAD0) : -1;
        c.tn = tn;
        load_x<0>(c);
        load_range<0, issued(0)>(c);
        dws = p.ws[g];
        asm volatile("" : "+s"(dws));
        loader_loop<0>(c);
    } else {
        unsigned ticket = 0;
        if (wid == 0 && !gathered) ticket = draw_ticket(myws + WS_CTR + b * 128);
        scale = __builtin_bit_cast(float, __builtin_amdgcn_readfirstlane(lost ? 0x7fc00000 : __builtin_bit_cast(int, p.sx[0] * p.sw[0])));
        dws = p.ws[g];
        asm volatile("" : "+s"(scale), "+s"(dws));
        const int r = lane & 31, h = lane >> 5;
        unsigned ao[2][2];
#pragma unroll
        for (int ks = 0; ks < 2; ++ks)
#pragma unroll
            for (int e = 0; e < 2; ++e) ao[ks][e] = A_RING + r * 128 + (((4 * ks + 2 * h + e) ^ ((r >> 1) & 7)) << 4);
        const int q = (lane & 15) >> 1, pp = lane & 1, g1 = (lane >> 4) & 1;
        const unsigned bo = (32 * h + q) * 128 + (((2 * wid + g1) ^ (q & 6)) << 4) + pp * 8;

        auto rd = [&](Frag& f, int slot, int aslot, int ks) {
            const unsigned char LAS* base = L + slot * B_BYTES;
            const unsigned char LAS* abase = L + aslot * A_BYTES;
#pragma unroll
            for (int t = 0; t < 4; ++t)
                f.b[ks][t] = __builtin_amdgcn_ds_read_tr8_b64_v2i32((LAS v2i*)(base + bo + (64 * ks + 8 * t) * 128));
#pragma unroll
            for (int mt = 0; mt < 2; ++mt)
#pragma unroll
                for (int e = 0; e < 2; ++e) f.a[ks][mt][e] = *(const LAS v4i*)(abase + ao[ks][e] + mt * 4096);
        };
        auto mm = [&](const Frag& f, int ks, int mt) {
            acc[mt] = __builtin_amdgcn_mfma_scale_f32_32x32x64_f8f6f4(cat8b(f.b[ks][0], f.b[ks][1], f.b[ks][2], f.b[ks][3]),
                                                                      cat8(f.a[ks][mt][0], f.a[ks][mt][1]), acc[mt], 0, 0, 0, 0, 0, 0);
        };
        auto step = [&](Frag& fn, const Frag& fo, int slot, int aslot) {
            BAR(); SCHED();
            mm(fo, 0, 0); SCHED();
            rd(fn, slot, aslot, 0); SCHED();
            mm(fo, 0, 1); SCHED();
            rd(fn, slot, aslot, 1); SCHED();
            mm(fo, 1, 0); mm(fo, 1, 1); SCHED();
            WAIT_LGKM0(); SCHED();
        };
        Frag f0, f1;
        BAR();
        rd(f0, 0, 0, 0);
        rd(f0, 0, 0, 1);
        WAIT_LGKM0();
        SCHED();
        int slot = 1, aslot = 1;
#pragma unroll 1
        for (int i = 1; i < NSTAGE - 1; i += 2) {
            step(f1, f0, slot, aslot);
            slot = slot + 1 == DEPTH ? 0 : slot + 1;
            aslot = aslot + 1 == ADEPTH ? 0 : aslot + 1;
            step(f0, f1, slot, aslot);
            slot = slot + 1 == DEPTH ? 0 : slot + 1;
            aslot = aslot + 1 == ADEPTH ? 0 : aslot + 1;
        }
        step(f1, f0, slot, aslot);
        mm(f1, 0, 0); mm(f1, 0, 1); mm(f1, 1, 0); mm(f1, 1, 1);
        if (wid == 0 && !gathered) {
            asm volatile("s_waitcnt vmcnt(0)" : "+v"(ticket) :: "memory");
            if (lane == 0) *(LAS v4i*)(L + LDS_MISC) = (v4i){(int)(ticket + 1u), 0, 0, __builtin_bit_cast(int, scale)};
        }
        if (!local) {
#pragma unroll
            for (int gq = 0; gq < 4; ++gq)
                *(LAS v4f*)(L + LDS_HAND + wid * 4096 + gq * 1024 + lane * 16) =
                    (v4f){acc[1][4 * gq], acc[1][4 * gq + 1], acc[1][4 * gq + 2], acc[1][4 * gq + 3]};
        }
        WAIT_LGKM0();
    }
    BAR();
    if (wid == 0) STAMP(1);

    if (local) {
        if (wid < 4) {
            const int r = lane & 31, h = lane >> 5;
#pragma unroll
            for (int mt = 0; mt < 2; ++mt)
#pragma unroll
                for (int gq = 0; gq < 4; ++gq) {
                    v4f v;
#pragma unroll
                    for (int e = 0; e < 4; ++e) { const float y = acc[mt][4 * gq + e] * scale; v[e] = y <= 0.f ? 0.f : y; }
                    const int row = mt * 32 + r, ch = wid * 8 + 2 * gq + h;
                    *(LAS v4f*)(L + row * 512 + ((ch ^ (row & 7)) << 4)) = v;
                }
        }
        __syncthreads();
        float* ob = p.out + (size_t)orow * NS + ocol;
#pragma unroll
        for (int u = 0; u < 4; ++u) {
            const int row = (wid * 4 + u) * 2 + (lane >> 5), ch = lane & 31;
            store_out(ob + row * NS + ch * 4, *(LAS u32x4*)(L + row * 512 + ((ch ^ (row & 7)) << 4)));
        }
    } else {
        const unsigned lo = lane * 16;
        const unsigned callv = *(LAS unsigned*)(L + LDS_MISC);
        if (wid >= 4) {
            scale = *(LAS float*)(L + LDS_MISC + 12);
#pragma unroll
            for (int gq = 0; gq < 4; ++gq) {
                const v4f t = *(LAS v4f*)(L + LDS_HAND + (wid - 4) * 4096 + gq * 1024 + lo);
#pragma unroll
                for (int e = 0; e < 4; ++e) acc[0][4 * gq + e] = t[e];
            }
        }
        float y[16];
        unsigned long long nz[16];
#pragma unroll
        for (int i = 0; i < 16; ++i) {
            y[i] = acc[0][i] * scale;
            nz[i] = __builtin_amdgcn_fcmpf(y[i], 0.f, 10  );
        }
        unsigned mask = 0;
#pragma unroll
        for (int q = 1; q >= 0; --q)
            asm volatile("s_nop 1\n\t"
                         "v_addc_co_u32_e64 %0, vcc, %0, %0, %8\n\tv_addc_co_u32_e64 %0, vcc, %0, %0, %7\n\t"
                         "v_addc_co_u32_e64 %0, vcc, %0, %0, %6\n\tv_addc_co_u32_e64 %0, vcc, %0, %0, %5\n\t"
                         "v_addc_co_u32_e64 %0, vcc, %0, %0, %4\n\tv_addc_co_u32_e64 %0, vcc, %0, %0, %3\n\t"
                         "v_addc_co_u32_e64 %0, vcc, %0, %0, %2\n\tv_addc_co_u32_e64 %0, vcc, %0, %0, %1"
                         : "+v"(mask)
                         : "s"(nz[8 * q]), "s"(nz[8 * q + 1]), "s"(nz[8 * q + 2]), "s"(nz[8 * q + 3]),
                           "s"(nz[8 * q + 4]), "s"(nz[8 * q + 5]), "s"(nz[8 * q + 6]), "s"(nz[8 * q + 7])
                         : "vcc");
        const unsigned cnt = __builtin_popcount(mask);
        const unsigned incl = wave_scan(cnt);
        const unsigned total = __builtin_amdgcn_readlane(incl, 63);
        *(LAS unsigned short*)(L + wid * PK_SLOT + lane * 2) = (unsigned short)mask;
        unsigned ptr = lds0 + wid * PK_SLOT + 128 + (incl - cnt) * 4;
#define PK1(m, v) "s_mov_b64 exec, %" #m "\n\tds_write_b32 %0, %" #v "\n\tv_add_u32 %0, 4, %0\n\t"
#pragma unroll
        for (int q = 0; q < 2; ++q)
            asm volatile(PK1(1, 9) PK1(2, 10) PK1(3, 11) PK1(4, 12) PK1(5, 13) PK1(6, 14) PK1(7, 15) PK1(8, 16)
                         "s_mov_b64 exec, -1"
                         : "+v"(ptr)
                         : "s"(nz[8 * q]), "s"(nz[8 * q + 1]), "s"(nz[8 * q + 2]), "s"(nz[8 * q + 3]),
                           "s"(nz[8 * q + 4]), "s"(nz[8 * q + 5]), "s"(nz[8 * q + 6]), "s"(nz[8 * q + 7]),
                           "v"(y[8 * q]), "v"(y[8 * q + 1]), "v"(y[8 * q + 2]), "v"(y[8 * q + 3]),
                           "v"(y[8 * q + 4]), "v"(y[8 * q + 5]), "v"(y[8 * q + 6]), "v"(y[8 * q + 7])
                         : "memory");
        u32x4 c[3];
#pragma unroll
        for (int u = 0; u < 3; ++u) c[u] = *(LAS u32x4*)(L + wid * PK_SLOT + u * 1024 + lo);
        const unsigned call = __builtin_amdgcn_readfirstlane(callv);
        const unsigned par = call & 1u;
        const unsigned bytes = (128 + 4 * total + ROUND - 1) & ~(ROUND - 1u);
        const unsigned sidx = ((par * 8 + rank) * 32 + j) * 8 + wid;
        unsigned char* dslot = dws + WS_STAGE + (size_t)sidx * PK_SLOT;
        const int turn = wid - (tm >= LEAD0 ? WIN_L : WIN_F);
        if (turn > 0) {
            while ((int)__builtin_amdgcn_readfirstlane(__hip_atomic_load((LAS unsigned*)(L + LDS_MISC + 8), __ATOMIC_RELAXED,
                                                                         __HIP_MEMORY_SCOPE_WORKGROUP)) < turn)
                __builtin_amdgcn_s_sleep(1);
        }
        if (wid == 0) STAMP(2);
#pragma unroll
        for (int u = 0; u < 3; ++u)
            if (u * 1024 + lo < bytes)
                asm volatile("global_store_dwordx4 %0, %1, %2 sc0 sc1\n\ts_nop 1" :: "v"(lo + u * 1024), "v"(c[u]), "s"(dslot) : "memory");
#pragma unroll 1
        for (unsigned o = 3 * 1024; o < bytes; o += 1024) {
            const u32x4 v = *(LAS u32x4*)(L + wid * PK_SLOT + o + lo);
            if (o + lo < bytes)
                asm volatile("global_store_dwordx4 %0, %1, %2 sc0 sc1\n\ts_nop 1" :: "v"(lo + o), "v"(v), "s"(dslot) : "memory");
        }
        WAIT_VM(0);
        if (lane == 0) {
            put_flag(dws + WS_FLAG + (size_t)sidx * 128, call, total);
            __hip_atomic_fetch_add((LAS unsigned*)(L + LDS_MISC + 8), 1u, __ATOMIC_RELAXED, __HIP_MEMORY_SCOPE_WORKGROUP);
        }
        if (wid == 7) STAMP(3);
        const unsigned ridx = ((par * 8 + g) * 32 + j) * 8 + wid;
        const unsigned rtotal = wait_flag(myws + WS_FLAG + (size_t)ridx * 128, 0u, tmo, call, lane);
        if (wid == 7) STAMP(4);
        u32x4 o[4];
        bool bad = true;
        if (rtotal <= 1024u) {
            const unsigned rbytes = 128 + 4 * rtotal;
            const unsigned char* sslot = myws + WS_STAGE + (size_t)ridx * PK_SLOT;
#pragma unroll 1
            for (unsigned of = 0; of < rbytes; of += 1024)
                asm volatile("s_mov_b32 m0, %2\n\ts_nop 0\n\tglobal_load_lds_dwordx4 %0, %1 sc0 sc1"
                             :: "v"(lo + of), "s"(sslot), "s"(lds0 + LDS_RECV + wid * PK_SLOT + of) : "memory");
            unsigned t;
            asm volatile("global_load_dword %0, %1, %2 sc1\n\ts_waitcnt vmcnt(0)" : "=&v"(t) : "v"(0u), "s"(tmo) : "memory");
            if (wid == 7) STAMP(5);
            bad = __builtin_amdgcn_readfirstlane(t) == call;
            const unsigned rm = *(LAS unsigned short*)(L + LDS_RECV + wid * PK_SLOT + lane * 2);
            const unsigned rc = __builtin_popcount(rm);
            *(LAS v2i*)(L + LDS_TAB + wid * 512 + lane * 8) =
                (v2i){(int)rm, (int)(LDS_RECV + wid * PK_SLOT + 128 + (wave_scan(rc) - rc) * 4)};
            const unsigned bp = 4 * ((lane & 7) >> 1);
#pragma unroll
            for (int u = 0; u < 4; ++u) {
                const v2i tb = *(LAS v2i*)(L + LDS_TAB + wid * 512 + (8 * u + (lane >> 3) + 32 * (lane & 1)) * 8);
                const unsigned m = (unsigned)tb[0];
                int addr = tb[1] + 4 * __builtin_popcount(m & ((1u << bp) - 1u));
                const int ms = (int)(m >> bp);
#pragma unroll
                for (int e = 0; e < 4; ++e) {
                    const int sel = __builtin_amdgcn_sbfe(ms, e, 1);
                    o[u][e] = *(LAS unsigned*)(L + addr) & (unsigned)sel;
                    addr -= 4 * sel;
                }
            }
        }
        float* ob = p.out + (size_t)(g * MS + tm * BM + (wid >> 2) * 32 + (lane >> 3)) * NS + tn * BN + (wid & 3) * 32 + (lane & 7) * 4;
#pragma unroll
        for (int u = 0; u < 4; ++u) {
            if (bad) o[u] = (u32x4){0x7fc00000u, 0x7fc00000u, 0x7fc00000u, 0x7fc00000u};
            store_out(ob + u * 8 * NS, o[u]);
        }
    }
}

extern "C" void kernel_launch(int rank, int nranks, ncclComm_t comm, hipStream_t stream,
                              void* const* d_in, const int* in_sizes, int n_in,
                              void* d_out, int out_size, const int* in_dtypes, int out_dtype,
                              void* const* d_out_all, void* const* d_ws_all, size_t ws_bytes) {
    if (nranks != NR || n_in != 4 || in_sizes[0] != MS * KK || in_sizes[1] != KK * NN || out_size != NR * MS * NS ||
        (unsigned)rank >= (unsigned)NR || ws_bytes < WS_END) {
        fprintf(stderr, "unexpected problem shape\n");
        return;
    }
    Params p;
    p.x = (const unsigned char*)d_in[0];
    p.w = (const unsigned char*)d_in[1];
    p.sx = (const float*)d_in[2];
    p.sw = (const float*)d_in[3];
    p.out = (float*)d_out;
    p.myws = (unsigned char*)d_ws_all[rank];
    for (int i = 0; i < NR; ++i) p.ws[i] = (unsigned char*)d_ws_all[i];
    p.rank = rank;
    p.pad = 0;
    gemm_a2a<<<GRID, 512, 0, stream>>>(p);
    HIP_CHECK(hipGetLastError());
}
```

```cpp
#include <hip/hip_runtime.h>
#include <rccl/rccl.h>
#include <cstdio>

#define HIP_CHECK(expr) do { hipError_t _e = (expr); if (_e != hipSuccess) { \
    fprintf(stderr, "HIP error %s at %s:%d\n", hipGetErrorString(_e), __FILE__, __LINE__); \
    return; } } while (0)

typedef int v8i __attribute__((ext_vector_type(8)));
typedef int v4i __attribute__((ext_vector_type(4)));
typedef int v2i __attribute__((ext_vector_type(2)));
typedef float v16f __attribute__((ext_vector_type(16)));
typedef float v4f __attribute__((ext_vector_type(4)));
typedef unsigned u32x4 __attribute__((ext_vector_type(4)));
#define LAS __attribute__((address_space(3)))

constexpr int NR = 8;
constexpr int MS = 512;
constexpr int KK = 4096;
constexpr int NN = 4096;
constexpr int NS = 512;
constexpr int BM = 64, BN = 128, BK = 128;
constexpr int NSTAGE = KK / BK;
constexpr int XROW0 = MS - BM;
constexpr int GRID = 256;
constexpr unsigned DEADLINE_TICKS = 50000000u;

#define DEPTH_ 4
#define POLL_SLEEP 2
#define ROUND 16
#define SCHED0 100000
#define SCHEDD 0
#define OUTMODE 1
#define WIN_F 2
#define WIN_L 2
#define PRO 2
#define RA 1
#define RB 2
#define SS 8

#define TWARM 500
#define DENSE_F 0xfc
#define DENSE_L 0x7f
#define PKPRIO 3
#define CODEPF 1
#define LEAD0 3

constexpr int DEPTH = DEPTH_;
constexpr int ADEPTH = DEPTH + 1;
constexpr int A_BYTES = BM * BK;
constexpr int B_BYTES = BK * BN;
constexpr int A_RING = DEPTH * B_BYTES;
constexpr int RING_BYTES = DEPTH * B_BYTES + ADEPTH * A_BYTES;
constexpr int PK_SLOT = 5120;
constexpr int LDS_RECV = 8 * PK_SLOT;
constexpr int LDS_TAB = 16 * PK_SLOT;
constexpr int LDS_HAND = RING_BYTES;
constexpr int LDS_MISC = LDS_HAND + 4 * 4096;
constexpr int LDS_BYTES = LDS_MISC + 16;
static_assert(LDS_TAB + 8 * 512 <= RING_BYTES && LDS_BYTES > 80 * 1024 && LDS_BYTES <= 160 * 1024, "LDS layout");

constexpr size_t WS_CTR = 0;
constexpr size_t WS_TMO = 32768;
constexpr size_t WS_XFLAG = 36864;
constexpr size_t WS_STAMP = 65536;
constexpr size_t WS_FLAG = 131072;
constexpr size_t WS_XSTAGE = WS_FLAG + 2 * 2048 * 128;
constexpr size_t WS_STAGE = WS_XSTAGE + (size_t)16 * BM * KK;
constexpr size_t WS_END = WS_STAGE + (size_t)2 * 2048 * PK_SLOT;

struct Params {
    const unsigned char* x;
    const unsigned char* w;
    unsigned char* myws;
    int rank;
    int pad;
    const float* sx;
    const float* sw;
    float* out;
    unsigned char* ws[NR];
};

#define WAIT_VM(n) asm volatile("s_waitcnt vmcnt(%0)" :: "i"(n) : "memory")
#define WAIT_LGKM0() asm volatile("s_waitcnt lgkmcnt(0)" ::: "memory")
#define BAR() __builtin_amdgcn_s_barrier()
#define SLOTSTAMP(i) do {} while (0)
#define SCHED() __builtin_amdgcn_sched_barrier(0)
#define STAMP(k) do {} while (0)

#define KORD 1
template <int S>
__device__ __forceinline__ int korder(int lead, int tn) {
    const int rs = (S + tn) & 3;
    const int kl = S < 8 ? lead + S : (lead + (S & ~3) + rs) & (NSTAGE - 1);
    return lead < 0 ? (S >> 2) + 8 * rs : kl;
}

struct LoadCtx {
    unsigned va[2], vb[4];
    const unsigned char* xa;
    const unsigned char* wb;
    unsigned ldsA, ldsB;
    int lead, tn;
};
template <int S>
__device__ __forceinline__ void load_w(const LoadCtx& c) {
    const unsigned char* pb = c.wb + (size_t)korder<S>(c.lead, c.tn) * BK * NN;
    asm volatile("s_mov_b32 m0, %5\n\ts_nop 0\n\t"
                 "global_load_lds_dwordx4 %0, %4\n\tglobal_load_lds_dwordx4 %1, %4 offset:1024\n\t"
                 "global_load_lds_dwordx4 %2, %4 offset:2048\n\tglobal_load_lds_dwordx4 %3, %4 offset:3072"
                 :: "v"(c.vb[0]), "v"(c.vb[1]), "v"(c.vb[2]), "v"(c.vb[3]), "s"(pb), "s"(c.ldsB + (S % DEPTH) * B_BYTES) : "memory");
}
template <int S>
__device__ __forceinline__ void load_x(const LoadCtx& c) {
    const unsigned char* pa = c.xa + korder<S>(c.lead, c.tn) * BK;
    asm volatile("s_mov_b32 m0, %3\n\ts_nop 0\n\t"
                 "global_load_lds_dwordx4 %0, %2 sc0 sc1\n\tglobal_load_lds_dwordx4 %1, %2 offset:1024 sc0 sc1"
                 :: "v"(c.va[0]), "v"(c.va[1]), "s"(pa), "s"(c.ldsA + (S % ADEPTH) * A_BYTES) : "memory");
}
constexpr int mini(int a, int b) { return a < b ? a : b; }
constexpr int issued(int I) { return mini(NSTAGE, mini(PRO + 2 * I, I + DEPTH - 1)); }
constexpr int younger(int I) {
    return 4 * (issued(I) - 1 - I) + 2 * (mini(issued(I), NSTAGE - 1) - I);
}
template <int S, int E>
__device__ __forceinline__ void load_range(const LoadCtx& c) {
    if constexpr (S < E) {
        load_w<S>(c);
        if constexpr (S + 1 < NSTAGE) load_x<S + 1>(c);
        load_range<S + 1, E>(c);
    }
}
template <int I>
__device__ __forceinline__ void loader_loop(const LoadCtx& c) {
    WAIT_VM(younger(I));
    BAR();
    if constexpr (I + 1 < NSTAGE) {
        load_range<issued(I), issued(I + 1)>(c);
        loader_loop<I + 1>(c);
    }
}

__device__ __forceinline__ unsigned wave_scan(unsigned v) {
    v += __builtin_amdgcn_update_dpp(0u, v, 0x111, 0xf, 0xf, true);
    v += __builtin_amdgcn_update_dpp(0u, v, 0x112, 0xf, 0xf, true);
    v += __builtin_amdgcn_update_dpp(0u, v, 0x114, 0xf, 0xf, true);
    v += __builtin_amdgcn_update_dpp(0u, v, 0x118, 0xf, 0xf, true);
    v += __builtin_amdgcn_update_dpp(0u, v, 0x142, 0xa, 0xf, false);
    v += __builtin_amdgcn_update_dpp(0u, v, 0x143, 0xc, 0xf, false);
    return v;
}

__device__ __forceinline__ unsigned draw_ticket(unsigned char* ctr) {
    unsigned t;
    asm volatile("s_mov_b64 exec, 1\n\tglobal_atomic_add %0, %1, %2, %3 sc0\n\ts_mov_b64 exec, -1"
                 : "=&v"(t) : "v"(0u), "v"(1u), "s"(ctr) : "memory");
    return t;
}

__device__ __forceinline__ void touch(const unsigned char* base, unsigned voff, unsigned lds_dst) {
    asm volatile("s_mov_b32 m0, %2\n\ts_nop 0\n\tglobal_load_lds_dword %0, %1" :: "v"(voff), "s"(base), "s"(lds_dst) : "memory");
}

__device__ __forceinline__ void put_flag(unsigned char* flag, unsigned call, unsigned low) {
    typedef __attribute__((address_space(1))) unsigned long long gu64;
    __hip_atomic_store((gu64*)flag, ((unsigned long long)call << 32) | low, __ATOMIC_RELAXED, __HIP_MEMORY_SCOPE_SYSTEM);
}

__device__ __forceinline__ unsigned wait_flag(const unsigned char* flag, unsigned voff, unsigned* tmo, unsigned call, int lane) {
    const unsigned t0 = (unsigned)__builtin_amdgcn_s_memrealtime();
    unsigned low;
    bool seen;
    for (;;) {
        v2i f;
        unsigned t;
        asm volatile("global_load_dwordx2 %0, %2, %3 sc0 sc1\n\tglobal_load_dword %1, %4, %5 sc1\n\ts_waitcnt vmcnt(0)"
                     : "=&v"(f), "=&v"(t) : "v"(voff), "s"(flag), "v"(0u), "s"(tmo) : "memory");
        low = __builtin_amdgcn_readfirstlane(f[0]);
        seen = __builtin_amdgcn_ballot_w64((unsigned)f[1] < call) == 0ull;
        if (seen || __builtin_amdgcn_readfirstlane(t) == call || (unsigned)__builtin_amdgcn_s_memrealtime() - t0 > DEADLINE_TICKS) break;
        __builtin_amdgcn_s_sleep(POLL_SLEEP);
    }
    if (!seen && lane == 0) __hip_atomic_store(tmo, call, __ATOMIC_RELAXED, __HIP_MEMORY_SCOPE_AGENT);
    return seen ? low : 0xffffffffu;
}

__device__ __forceinline__ void store_out(float* ptr, u32x4 v) {
    asm volatile("global_store_dwordx4 %0, %1, off sc0 sc1\n\ts_nop 1" :: "v"(ptr), "v"(v) : "memory");
}

__device__ __forceinline__ unsigned compact(unsigned base, unsigned mask, const float (&y)[16], const unsigned long long (&nz)[16]) {
    const unsigned cnt = __builtin_popcount(mask);
    const unsigned incl = wave_scan(cnt);
    unsigned ptr = base + (incl - cnt) * 4;
#define PK1(m, v) "s_mov_b64 exec, %" #m "\n\tds_write_b32 %0, %" #v "\n\tv_add_u32 %0, 4, %0\n\t"
#pragma unroll
    for (int q = 0; q < 2; ++q)
        asm volatile(PK1(1, 9) PK1(2, 10) PK1(3, 11) PK1(4, 12) PK1(5, 13) PK1(6, 14) PK1(7, 15) PK1(8, 16)
                     "s_mov_b64 exec, -1"
                     : "+v"(ptr)
                     : "s"(nz[8 * q]), "s"(nz[8 * q + 1]), "s"(nz[8 * q + 2]), "s"(nz[8 * q + 3]),
                       "s"(nz[8 * q + 4]), "s"(nz[8 * q + 5]), "s"(nz[8 * q + 6]), "s"(nz[8 * q + 7]),
                       "v"(y[8 * q]), "v"(y[8 * q + 1]), "v"(y[8 * q + 2]), "v"(y[8 * q + 3]),
                       "v"(y[8 * q + 4]), "v"(y[8 * q + 5]), "v"(y[8 * q + 6]), "v"(y[8 * q + 7])
                     : "memory");
    return __builtin_amdgcn_readlane(incl, 63);
}

__device__ __forceinline__ int step4(int addr, int sel) {
    int r;
    asm("v_mad_i32_i24 %0, %1, -4, %2" : "=v"(r) : "v"(sel), "v"(addr));
    return r;
}

struct Frag { v4i a[2][2][2]; v2i b[2][4]; };

__device__ __forceinline__ v8i cat8(v4i lo, v4i hi) { return __builtin_shufflevector(lo, hi, 0, 1, 2, 3, 4, 5, 6, 7); }
__device__ __forceinline__ v8i cat8b(v2i a, v2i b, v2i c, v2i d) {
    return cat8(__builtin_shufflevector(a, b, 0, 1, 2, 3), __builtin_shufflevector(c, d, 0, 1, 2, 3));
}

__global__ void __launch_bounds__(512, 2) gemm_a2a(Params p) {
    __shared__ __attribute__((aligned(1024))) unsigned char lds[LDS_BYTES];
    LAS unsigned char* L = (LAS unsigned char*)lds;
    const unsigned lds0 = (unsigned)(size_t)L;
    const int tid = threadIdx.x, lane = tid & 63;
    const int wid = __builtin_amdgcn_readfirstlane(tid >> 6);
    const int b = blockIdx.x;
    const int g = b & 7, j = b >> 3, tm = j >> 2, tn = j & 3;
    unsigned char* myws = p.myws;
    unsigned* tmo = (unsigned*)(myws + WS_TMO);
    const int rank = p.rank;
    const unsigned t_begin = (unsigned)__builtin_amdgcn_s_memrealtime();
    if (wid == 0) STAMP(0);

    const unsigned char* xa = p.x + (size_t)tm * BM * KK;
    int wcol = g * NS + tn * BN;
    int orow = rank * MS + tm * BM, ocol = tn * BN;
    const bool early = g == rank || tm == 7;
    const bool local = early;
    bool lost = false;

    if (early) {
        unsigned ticket = 0;
        if (wid == 0) ticket = draw_ticket(myws + WS_CTR + b * 128);
        u32x4 r[8];
        if (g != rank) {
#pragma unroll
            for (int q = 0; q < 8; ++q)
                asm volatile("global_load_dwordx4 %0, %1, %2" : "=v"(r[q]) : "v"(lane * 16 + q * 1024), "s"(xa + (tn * 16 + wid * 2) * KK) : "memory");
        } else {
            touch(p.x, (j * 512 + tid) * 128, lds0);
            while ((unsigned)__builtin_amdgcn_s_memrealtime() - t_begin < TWARM) __builtin_amdgcn_s_sleep(4);
            touch(p.w + wcol, (tm * 512 + tid) * NN, lds0);
        }
        if (CODEPF && wid >= 5) {
            const unsigned char* pc;
            unsigned len;
            asm volatile("s_getpc_b64 %0\n.Lgemm_a2a_pc:\n\ts_mov_b32 %1, .Lgemm_a2a_end-.Lgemm_a2a_pc" : "=s"(pc), "=s"(len));
            touch(pc, min((unsigned)((wid - 5) * 8192 + lane * 128), len), lds0);
        }
        asm volatile("s_waitcnt vmcnt(0)"
                     : "+v"(ticket), "+v"(r[0]), "+v"(r[1]), "+v"(r[2]), "+v"(r[3]), "+v"(r[4]), "+v"(r[5]), "+v"(r[6]), "+v"(r[7])
                     :: "memory");
        if (wid == 0 && lane == 0) *(LAS v4i*)(L + LDS_MISC) = (v4i){(int)(ticket + 1u), 0, 0, 0};
        __syncthreads();
        const unsigned call = __builtin_amdgcn_readfirstlane(*(LAS unsigned*)(L + LDS_MISC));
        const unsigned par = call & 1u;
        if (g != rank) {
            unsigned char* pws = p.ws[g];
            unsigned char* dx = pws + WS_XSTAGE + (size_t)((par * 8 + rank) * BM + tn * 16 + wid * 2) * KK;
#pragma unroll
            for (int q = 0; q < 8; ++q)
                asm volatile("global_store_dwordx4 %0, %1, %2 sc0 sc1\n\ts_nop 1" :: "v"(lane * 16 + q * 1024), "v"(r[q]), "s"(dx) : "memory");
            WAIT_VM(0);
            __syncthreads();
            if (tid == 0) put_flag(pws + WS_XFLAG + ((par * 8 + rank) * 4 + tn) * 128, call, 0u);
            if (wid == 0) STAMP(2);
            const int u = ((g - rank - 1) & 7) * 4 + tn;
            xa = p.x + (size_t)(u % 7) * BM * KK;
            ocol = (u / 7) * BN;
            wcol = rank * NS + ocol;
            orow = rank * MS + (u % 7) * BM;
        } else if (tm < 7) {
            const int src_rank = (rank + 1 + tm) & 7;
            if (wid == 0) {
                STAMP(2);
                const unsigned f = wait_flag(myws + WS_XFLAG + (par * 8 + src_rank) * 512, (lane & 3) * 128, tmo, call, lane);
                const unsigned t = __hip_atomic_load(tmo, __ATOMIC_RELAXED, __HIP_MEMORY_SCOPE_AGENT);
                if (f != 0u || t == call) *(LAS unsigned*)(L + LDS_MISC + 4) = 1u;
                STAMP(3);
            }
            __syncthreads();
            lost = __builtin_amdgcn_readfirstlane(*(LAS unsigned*)(L + LDS_MISC + 4)) != 0;
            xa = lost ? p.x : myws + WS_XSTAGE + (size_t)(par * 8 + src_rank) * BM * KK;
            orow = src_rank * MS + XROW0;
        }
    }

    v16f acc[2];
    float scale;
    unsigned char* dws = nullptr;

    if (wid >= 4) {
        const int lw = wid - 4;
        LoadCtx c;
#pragma unroll
        for (int i = 0; i < 2; ++i) {
            const int row = (lw * 2 + i) * 8 + (lane >> 3);
            c.va[i] = row * KK + (((lane & 7) ^ ((row >> 1) & 7)) << 4) - i * 1024;
        }
#pragma unroll
        for (int i = 0; i < 4; ++i) {
            const int row = (lw * 4 + i) * 8 + (lane >> 3);
            c.vb[i] = row * NN + (((lane & 7) ^ (row & 6)) << 4) - i * 1024;
        }
        c.xa = xa;
        c.wb = p.w + wcol;
        c.ldsA = lds0 + A_RING + lw * 2048;
        c.ldsB = lds0 + lw * 4096;
        c.lead = !early && tm >= LEAD0 ? 8 * (tm - LEAD0) : -1;
        c.tn = tn;
        load_x<0>(c);
        load_range<0, issued(0)>(c);
        dws = p.ws[g];
        asm volatile("" : "+s"(dws));
        loader_loop<0>(c);
        scale = __builtin_bit_cast(float, __builtin_amdgcn_readfirstlane(*(LAS int*)(L + LDS_MISC + 12)));
    } else {
        unsigned ticket = 0;
        if (wid == 0 && !early) ticket = draw_ticket(myws + WS_CTR + b * 128);
        scale = __builtin_bit_cast(float, __builtin_amdgcn_readfirstlane(lost ? 0x7fc00000 : __builtin_bit_cast(int, p.sx[0] * p.sw[0])));
        dws = p.ws[g];
        asm volatile("" : "+s"(scale), "+s"(dws));
        if (tid == 0) *(LAS float*)(L + LDS_MISC + 12) = scale;
        acc[0] = acc[1] = (v16f)0.f;
        const int r = lane & 31, h = lane >> 5;
        unsigned ao[2][2];
#pragma unroll
        for (int ks = 0; ks < 2; ++ks)
#pragma unroll
            for (int e = 0; e < 2; ++e) ao[ks][e] = A_RING + r * 128 + (((4 * ks + 2 * h + e) ^ ((r >> 1) & 7)) << 4);
        const int q = (lane & 15) >> 1, pp = lane & 1, g1 = (lane >> 4) & 1;
        const unsigned bo = (32 * h + q) * 128 + (((2 * wid + g1) ^ (q & 6)) << 4) + pp * 8;

        auto rd = [&](Frag& f, int slot, int aslot, int ks) {
            const unsigned char LAS* base = L + slot * B_BYTES;
            const unsigned char LAS* abase = L + aslot * A_BYTES;
#pragma unroll
            for (int t = 0; t < 4; ++t)
                f.b[ks][t] = __builtin_amdgcn_ds_read_tr8_b64_v2i32((LAS v2i*)(base + bo + (64 * ks + 8 * t) * 128));
#pragma unroll
            for (int mt = 0; mt < 2; ++mt)
#pragma unroll
                for (int e = 0; e < 2; ++e) f.a[ks][mt][e] = *(const LAS v4i*)(abase + ao[ks][e] + mt * 4096);
        };
        auto mm = [&](const Frag& f, int ks, int mt) {
            acc[mt] = __builtin_amdgcn_mfma_scale_f32_32x32x64_f8f6f4(cat8b(f.b[ks][0], f.b[ks][1], f.b[ks][2], f.b[ks][3]),
                                                                      cat8(f.a[ks][mt][0], f.a[ks][mt][1]), acc[mt], 0, 0, 0, 0, 0, 0);
        };
        auto step = [&](Frag& fn, const Frag& fo, int slot, int aslot) {
            BAR(); SCHED();
            mm(fo, 0, 0); SCHED();
            rd(fn, slot, aslot, 0); SCHED();
            mm(fo, 0, 1); SCHED();
            rd(fn, slot, aslot, 1); SCHED();
            mm(fo, 1, 0); mm(fo, 1, 1); SCHED();
            WAIT_LGKM0(); SCHED();
        };
        Frag f0, f1;
        BAR();
        SLOTSTAMP(0);
        rd(f0, 0, 0, 0);
        rd(f0, 0, 0, 1);
        WAIT_LGKM0();
        SCHED();
        int slot = 1, aslot = 1;
#pragma unroll 1
        for (int i = 1; i < NSTAGE - 1; i += 2) {
            step(f1, f0, slot, aslot);
            slot = slot + 1 == DEPTH ? 0 : slot + 1;
            aslot = aslot + 1 == ADEPTH ? 0 : aslot + 1;
            step(f0, f1, slot, aslot);
            slot = slot + 1 == DEPTH ? 0 : slot + 1;
            aslot = aslot + 1 == ADEPTH ? 0 : aslot + 1;
        }
        step(f1, f0, slot, aslot);
        mm(f1, 0, 0); mm(f1, 0, 1); mm(f1, 1, 0); mm(f1, 1, 1);
        if (wid == 0 && !early) {
            asm volatile("s_waitcnt vmcnt(0)" : "+v"(ticket) :: "memory");
            if (lane == 0) *(LAS v4i*)(L + LDS_MISC) = (v4i){(int)(ticket + 1u), 0, 0, __builtin_bit_cast(int, scale)};
        }
        if (!local) {
#pragma unroll
            for (int gq = 0; gq < 4; ++gq)
                *(LAS v4f*)(L + LDS_HAND + wid * 4096 + gq * 1024 + lane * 16) =
                    (v4f){acc[1][4 * gq], acc[1][4 * gq + 1], acc[1][4 * gq + 2], acc[1][4 * gq + 3]};
        }
        WAIT_LGKM0();
    }
    BAR();
    if (wid == 0) STAMP(1);

    if (local) {
        if (wid < 4) {
            const int r = lane & 31, h = lane >> 5;
#pragma unroll
            for (int mt = 0; mt < 2; ++mt)
#pragma unroll
                for (int gq = 0; gq < 4; ++gq) {
                    v4f v;
#pragma unroll
                    for (int e = 0; e < 4; ++e) { const float y = acc[mt][4 * gq + e] * scale; v[e] = y <= 0.f ? 0.f : y; }
                    const int row = mt * 32 + r, ch = wid * 8 + 2 * gq + h;
                    *(LAS v4f*)(L + row * 512 + ((ch ^ (row & 7)) << 4)) = v;
                }
        }
        __syncthreads();
        float* ob = p.out + (size_t)orow * NS + ocol;
#pragma unroll
        for (int u = 0; u < 4; ++u) {
            const int row = (wid * 4 + u) * 2 + (lane >> 5), ch = lane & 31;
            store_out(ob + row * NS + ch * 4, *(LAS u32x4*)(L + row * 512 + ((ch ^ (row & 7)) << 4)));
        }
    } else {
        const unsigned lo = lane * 16;
        const unsigned callv = *(LAS unsigned*)(L + LDS_MISC);
        if (wid >= 4) {
            const LAS v4f* h = (const LAS v4f*)(L + LDS_HAND + (wid - 4) * 4096 + lo);
            acc[0] = __builtin_shufflevector(__builtin_shufflevector(h[0], h[64], 0, 1, 2, 3, 4, 5, 6, 7),
                                             __builtin_shufflevector(h[128], h[192], 0, 1, 2, 3, 4, 5, 6, 7),
                                             0, 1, 2, 3, 4, 5, 6, 7, 8, 9, 10, 11, 12, 13, 14, 15);
        }
        unsigned total = 0;
        bool dense = (((tm >= LEAD0 ? DENSE_L : DENSE_F) >> wid) & 1) &&
                     __builtin_bit_cast(unsigned, scale) - 0x0d800000u < 0x71800000u - 0x0d800000u;
        if (!dense) __builtin_amdgcn_s_setprio(PKPRIO);
        if (dense) {
            float y[16];
            unsigned long long nz[16];
            unsigned mask = 0, m1 = 0, m2 = 0, inexact = 0;
            float top = 0.f;
#pragma unroll
            for (int q = 3; q >= 0; --q) {
                unsigned long long s1[4], s2[4];
#pragma unroll
                for (int e = 0; e < 4; ++e) {
                    const float a = acc[0][4 * q + e], am = __builtin_elementwise_minimum(a, 32.f), u = am + 32.f;
                    nz[4 * q + e] = __builtin_amdgcn_fcmpf(a, 0.f, 2  );
                    s1[e] = __builtin_amdgcn_fcmpf(a, 32.f, 3  );
                    s2[e] = __builtin_amdgcn_fcmpf(a, 128.f, 3);
                    inexact = __builtin_amdgcn_bitop3_b32(inexact, __builtin_bit_cast(unsigned, u - 32.f), __builtin_bit_cast(unsigned, am), 0xf6);
                    y[4 * q + e] = __builtin_amdgcn_inverse_ballot_w64(s1[e]) ? a : u;
                    top = __builtin_elementwise_maximum(top, a);
                }
#define AC(d, m) "v_addc_co_u32_e64 %" #d ", vcc, %" #d ", %" #d ", %" #m "\n\t"
                asm volatile("s_nop 1\n\t" AC(0, 6) AC(1, 10) AC(2, 14) AC(0, 5) AC(1, 9) AC(2, 13) AC(0, 4) AC(1, 8) AC(2, 12) AC(0, 3) AC(1, 7) AC(2, 11)
                             : "+v"(mask), "+v"(m1), "+v"(m2)
                             : "s"(nz[4 * q]), "s"(nz[4 * q + 1]), "s"(nz[4 * q + 2]), "s"(nz[4 * q + 3]),
                               "s"(s1[0]), "s"(s1[1]), "s"(s1[2]), "s"(s1[3]), "s"(s2[0]), "s"(s2[1]), "s"(s2[2]), "s"(s2[3])
                             : "vcc");
            }
            dense = __builtin_amdgcn_ballot_w64((inexact << 1) != 0u || !(top < 512.f)) == 0ull;
            if (dense) {
                *(LAS unsigned*)(L + wid * PK_SLOT + lane * 4) = (mask ^ m1 ^ m2) | (m1 << 16);
                total = compact(lds0 + LDS_RECV + wid * PK_SLOT, mask, y, nz);
#pragma unroll 1
                for (unsigned o = 0; o < total; o += 256) {
                    const u32x4 v = *(LAS u32x4*)(L + LDS_RECV + wid * PK_SLOT + o * 4 + lo);
                    LAS unsigned* d = (LAS unsigned*)(L + wid * PK_SLOT + 256 + o * 3 + lane * 12);
                    d[0] = __builtin_amdgcn_perm(v[1], v[0], 0x04020100u);
                    d[1] = __builtin_amdgcn_perm(v[2], v[1], 0x05040201u);
                    d[2] = __builtin_amdgcn_perm(v[3], v[2], 0x06050402u);
                }
            }
        }
        if (!dense) {
            float y[16];
            unsigned long long nz[16];
#pragma unroll
            for (int i = 0; i < 16; ++i) {
                y[i] = acc[0][i] * scale;
                nz[i] = __builtin_amdgcn_fcmpf(y[i], 0.f, 10  );
            }
            unsigned mask = 0;
#pragma unroll
            for (int q = 1; q >= 0; --q)
                asm volatile("s_nop 1\n\t" AC(0, 8) AC(0, 7) AC(0, 6) AC(0, 5) AC(0, 4) AC(0, 3) AC(0, 2) AC(0, 1)
                             : "+v"(mask)
                             : "s"(nz[8 * q]), "s"(nz[8 * q + 1]), "s"(nz[8 * q + 2]), "s"(nz[8 * q + 3]),
                               "s"(nz[8 * q + 4]), "s"(nz[8 * q + 5]), "s"(nz[8 * q + 6]), "s"(nz[8 * q + 7])
                             : "vcc");
            *(LAS unsigned short*)(L + wid * PK_SLOT + lane * 2) = (unsigned short)mask;
            total = compact(lds0 + wid * PK_SLOT + 128, mask, y, nz);
        }
        u32x4 c[3];
#pragma unroll
        for (int u = 0; u < 3; ++u) c[u] = *(LAS u32x4*)(L + wid * PK_SLOT + u * 1024 + lo);
        const unsigned call = __builtin_amdgcn_readfirstlane(callv);
        const unsigned par = call & 1u;
        const unsigned bytes = dense ? 256 + 3 * total : 128 + 4 * total;
        const unsigned sidx = ((par * 8 + rank) * 32 + j) * 8 + wid;
        unsigned char* dslot = dws + WS_STAGE + (size_t)sidx * PK_SLOT;
        const int turn = wid - (tm >= LEAD0 ? WIN_L : WIN_F);
        if (turn > 0) {
            for (;;) {
                const int done = __builtin_amdgcn_readfirstlane(__hip_atomic_load((LAS unsigned*)(L + LDS_MISC + 8), __ATOMIC_RELAXED,
                                                                                  __HIP_MEMORY_SCOPE_WORKGROUP));
                const int late = (int)((unsigned)__builtin_amdgcn_s_memrealtime() - t_begin) - (SCHED0 + wid * SCHEDD);
                if (done >= turn || late > 0) break;
                __builtin_amdgcn_s_sleep(1);
            }
        }
        if (wid == 0) STAMP(2);
#pragma unroll
        for (int u = 0; u < 3; ++u)
            if (u * 1024 + lo < bytes)
                asm volatile("global_store_dwordx4 %0, %1, %2 sc0 sc1\n\ts_nop 1" :: "v"(lo + u * 1024), "v"(c[u]), "s"(dslot) : "memory");
#pragma unroll 1
        for (unsigned o = 3 * 1024; o < bytes; o += 1024) {
            const u32x4 v = *(LAS u32x4*)(L + wid * PK_SLOT + o + lo);
            if (o + lo < bytes)
                asm volatile("global_store_dwordx4 %0, %1, %2 sc0 sc1\n\ts_nop 1" :: "v"(lo + o), "v"(v), "s"(dslot) : "memory");
        }
        WAIT_VM(0);
        if (lane == 0) {
            put_flag(dws + WS_FLAG + (size_t)sidx * 128, call, total | (dense ? 0x10000u : 0u));
            __hip_atomic_fetch_add((LAS unsigned*)(L + LDS_MISC + 8), 1u, __ATOMIC_RELAXED, __HIP_MEMORY_SCOPE_WORKGROUP);
        }
        if (wid == 7) STAMP(3);
        const unsigned ridx = ((par * 8 + g) * 32 + j) * 8 + wid;
        const unsigned rword = wait_flag(myws + WS_FLAG + (size_t)ridx * 128, 0u, tmo, call, lane);
        if (wid == 7) STAMP(4);
        const unsigned rtotal = __builtin_amdgcn_readfirstlane(rword) & ~0x10000u;
        const bool rdense = __builtin_amdgcn_readfirstlane(rword) & 0x10000u;
        u32x4 o[4];
        bool bad = true;
        if (rtotal <= 1024u) {
            const unsigned rbytes = rdense ? 256 + 3 * rtotal : 128 + 4 * rtotal;
            const unsigned char* sslot = myws + WS_STAGE + (size_t)ridx * PK_SLOT;
            const unsigned rdst = __builtin_amdgcn_readfirstlane(lds0 + LDS_RECV + wid * PK_SLOT);
            asm volatile("s_mov_b32 m0, %2\n\ts_nop 0\n\tglobal_load_lds_dwordx4 %0, %1 sc0 sc1\n\t"
                         "s_cmp_le_u32 %3, 1024\n\ts_cbranch_scc1 1f\n\tglobal_load_lds_dwordx4 %0, %1 offset:1024 sc0 sc1\n\t"
                         "s_cmp_le_u32 %3, 2048\n\ts_cbranch_scc1 1f\n\tglobal_load_lds_dwordx4 %0, %1 offset:2048 sc0 sc1\n\t"
                         "s_cmp_le_u32 %3, 3072\n\ts_cbranch_scc1 1f\n\tglobal_load_lds_dwordx4 %0, %1 offset:3072 sc0 sc1\n\t"
                         "s_cmp_le_u32 %3, 4096\n\ts_cbranch_scc1 1f\n\ts_mov_b32 m0, %5\n\ts_nop 0\n\tglobal_load_lds_dwordx4 %4, %1 sc0 sc1\n"
                         "1:"
                         :: "v"(lo), "s"(sslot), "s"(rdst), "s"(rbytes), "v"(lo + 4096), "s"(rdst + 4096) : "memory", "scc");
            unsigned t;
            asm volatile("global_load_dword %0, %1, %2 sc1\n\ts_waitcnt vmcnt(0)" : "=&v"(t) : "v"(0u), "s"(tmo) : "memory");
            if (wid == 7) STAMP(5);
            bad = __builtin_amdgcn_readfirstlane(t) == call;
            const unsigned bp = 4 * ((lane & 7) >> 1);
            LAS v2i* tab = (LAS v2i*)(L + LDS_TAB + wid * 512);
            if (!rdense) {
                const unsigned rm = *(LAS unsigned short*)(L + LDS_RECV + wid * PK_SLOT + lane * 2);
                const unsigned rc = __builtin_popcount(rm);
                tab[lane] = (v2i){(int)rm, (int)(LDS_RECV + wid * PK_SLOT + 128 + (wave_scan(rc) - rc) * 4)};
#pragma unroll
                for (int u = 0; u < 4; ++u) {
                    const v2i tb = tab[8 * u + (lane >> 3) + 32 * (lane & 1)];
                    const unsigned m = (unsigned)tb[0];
                    int addr = tb[1] + 4 * __builtin_popcount(m & ((1u << bp) - 1u));
                    const int ms = (int)(m >> bp);
#pragma unroll
                    for (int e = 0; e < 4; ++e) {
                        const int sel = __builtin_amdgcn_sbfe(ms, e, 1);
                        o[u][e] = *(LAS unsigned*)(L + addr) & (unsigned)sel;
                        if (e < 3) addr = step4(addr, sel);
                    }
                }
            } else {
                const unsigned rm = *(LAS unsigned*)(L + LDS_RECV + wid * PK_SLOT + lane * 4);
#pragma unroll 1
                for (unsigned q = 0; q < rtotal; q += 256) {
                    const LAS unsigned* d = (const LAS unsigned*)(L + LDS_RECV + wid * PK_SLOT + 256 + q * 3 + lane * 12);
                    const unsigned d0 = d[0], d1 = d[1], d2 = d[2];
                    *(LAS u32x4*)(L + wid * PK_SLOT + q * 4 + lo) =
                        (u32x4){(d0 & 0xffffffu) | 0x42000000u, __builtin_amdgcn_perm(d1, d0, 0x0c050403u) | 0x42000000u,
                                __builtin_amdgcn_perm(d2, d1, 0x0c040302u) | 0x42000000u, __builtin_amdgcn_alignbyte(0x42u, d2, 1)};
                }
                const unsigned rc = __builtin_popcount((rm | (rm >> 16)) & 0xffffu);
                tab[lane] = (v2i){(int)rm, (int)(wid * PK_SLOT + (wave_scan(rc) - rc) * 4)};
                const unsigned sb = __builtin_bit_cast(unsigned, scale), nb = __builtin_bit_cast(unsigned, -32.f * scale);
#pragma unroll
                for (int u = 0; u < 4; ++u) {
                    const v2i tb = tab[8 * u + (lane >> 3) + 32 * (lane & 1)];
                    const unsigned m = (unsigned)tb[0];
                    int addr = tb[1] + 4 * __builtin_popcount((m | (m >> 16)) & ((1u << bp) - 1u));
                    const int ms = (int)(m >> bp);
#pragma unroll
                    for (int e = 0; e < 4; ++e) {
                        const unsigned b0 = __builtin_amdgcn_sbfe(ms, e, 1), b1 = __builtin_amdgcn_sbfe(ms, 16 + e, 1);
                        const unsigned code = *(LAS unsigned*)(L + addr) | __builtin_amdgcn_bitop3_b32(b0, b1, 0x01000000u, 0x80);
                        o[u][e] = __builtin_bit_cast(unsigned, __builtin_fmaf(__builtin_bit_cast(float, code),
                                                                              __builtin_bit_cast(float, __builtin_amdgcn_bitop3_b32(b0, b1, sb, 0xa8)),
                                                                              __builtin_bit_cast(float, __builtin_amdgcn_bitop3_b32(b0, b1, nb, 0x20))));
                        if (e < 3) addr = step4(addr, (int)(b0 | b1));
                    }
                }
            }
        }
        float* ob = p.out + (size_t)(g * MS + tm * BM + (wid >> 2) * 32 + (lane >> 3)) * NS + tn * BN + (wid & 3) * 32 + (lane & 7) * 4;
#pragma unroll
        for (int u = 0; u < 4; ++u) {
            if (bad) o[u] = (u32x4){0x7fc00000u, 0x7fc00000u, 0x7fc00000u, 0x7fc00000u};
            store_out(ob + u * 8 * NS, o[u]);
        }
    }
    asm volatile(".Lgemm_a2a_end:");
}

extern "C" void kernel_launch(int rank, int nranks, ncclComm_t comm, hipStream_t stream,
                              void* const* d_in, const int* in_sizes, int n_in,
                              void* d_out, int out_size, const int* in_dtypes, int out_dtype,
                              void* const* d_out_all, void* const* d_ws_all, size_t ws_bytes) {
    if (nranks != NR || n_in != 4 || in_sizes[0] != MS * KK || in_sizes[1] != KK * NN || out_size != NR * MS * NS ||
        (unsigned)rank >= (unsigned)NR || ws_bytes < WS_END) {
        fprintf(stderr, "unexpected problem shape\n");
        return;
    }
    Params p;
    p.x = (const unsigned char*)d_in[0];
    p.w = (const unsigned char*)d_in[1];
    p.sx = (const float*)d_in[2];
    p.sw = (const float*)d_in[3];
    p.out = (float*)d_out;
    p.myws = (unsigned char*)d_ws_all[rank];
    for (int i = 0; i < NR; ++i) p.ws[i] = (unsigned char*)d_ws_all[i];
    p.rank = rank;
    p.pad = 0;
    gemm_a2a<<<GRID, 512, 0, stream>>>(p);
    HIP_CHECK(hipGetLastError());
}
```

```cpp
#include <hip/hip_runtime.h>
#include <rccl/rccl.h>
#include <cstdio>

#define HIP_CHECK(expr) do { hipError_t _e = (expr); if (_e != hipSuccess) { \
    fprintf(stderr, "HIP error %s at %s:%d\n", hipGetErrorString(_e), __FILE__, __LINE__); \
    return; } } while (0)

typedef int v8i __attribute__((ext_vector_type(8)));
typedef int v4i __attribute__((ext_vector_type(4)));
typedef int v2i __attribute__((ext_vector_type(2)));
typedef float v16f __attribute__((ext_vector_type(16)));
typedef float v4f __attribute__((ext_vector_type(4)));
typedef unsigned u32x4 __attribute__((ext_vector_type(4)));
#define LAS __attribute__((address_space(3)))

constexpr int NR = 8;
constexpr int MS = 512;
constexpr int KK = 4096;
constexpr int NN = 4096;
constexpr int NS = 512;
constexpr int BM = 64, BN = 128, BK = 128;
constexpr int NSTAGE = KK / BK;
constexpr int XROW0 = MS - BM;
constexpr int GRID = 256;
constexpr unsigned DEADLINE_TICKS = 50000000u;

#define DEPTH_ 4
#define POLL_SLEEP 2
#define WIN 2
#define PRO 2

#define TWARM 500
#define DENSE_F 0xfc
#define DENSE_L 0x7f
#define PKPRIO 3
#define LEAD0 3

constexpr int DEPTH = DEPTH_;
constexpr int ADEPTH = DEPTH + 1;
constexpr int A_BYTES = BM * BK;
constexpr int B_BYTES = BK * BN;
constexpr int A_RING = DEPTH * B_BYTES;
constexpr int RING_BYTES = DEPTH * B_BYTES + ADEPTH * A_BYTES;
constexpr int PK_SLOT = 5120;
constexpr int LDS_RECV = 8 * PK_SLOT;
constexpr int LDS_TAB = 16 * PK_SLOT;
constexpr int LDS_HAND = RING_BYTES;
constexpr int LDS_MISC = LDS_HAND + 4 * 4096;
constexpr int LDS_BYTES = LDS_MISC + 16;
static_assert(LDS_TAB + 8 * 512 <= RING_BYTES && LDS_BYTES > 80 * 1024 && LDS_BYTES <= 160 * 1024, "LDS layout");

constexpr size_t WS_CTR = 0;
constexpr size_t WS_TMO = 32768;
constexpr size_t WS_XFLAG = 36864;
constexpr size_t WS_STAMP = 65536;
constexpr size_t WS_FLAG = 131072;
constexpr size_t WS_XSTAGE = WS_FLAG + 2 * 2048 * 128;
constexpr size_t WS_STAGE = WS_XSTAGE + (size_t)16 * BM * KK;
constexpr size_t WS_END = WS_STAGE + (size_t)2 * 2048 * PK_SLOT;

struct Params {
    const unsigned char* x;
    const unsigned char* w;
    unsigned char* myws;
    int rank;
    int pad;
    const float* sx;
    const float* sw;
    float* out;
    unsigned char* ws[NR];
};

#define WAIT_VM(n) asm volatile("s_waitcnt vmcnt(%0)" :: "i"(n) : "memory")
#define WAIT_LGKM0() asm volatile("s_waitcnt lgkmcnt(0)" ::: "memory")
#define BAR() __builtin_amdgcn_s_barrier()
#define SCHED() __builtin_amdgcn_sched_barrier(0)
#define STAMP(k) do {} while (0)

template <int S>
__device__ __forceinline__ int korder(int lead, int tn) {
    const int rs = (S + tn) & 3;
    const int kl = S < 8 ? lead + S : (lead + (S & ~3) + rs) & (NSTAGE - 1);
    return lead < 0 ? (S >> 2) + 8 * rs : kl;
}

struct LoadCtx {
    unsigned va[2], vb[4];
    const unsigned char* xa;
    const unsigned char* wb;
    unsigned ldsA, ldsB;
    int lead, tn;
};
template <int S>
__device__ __forceinline__ void load_w(const LoadCtx& c) {
    const unsigned char* pb = c.wb + (size_t)korder<S>(c.lead, c.tn) * BK * NN;
    asm volatile("s_mov_b32 m0, %5\n\ts_nop 3\n\t"
                 "global_load_lds_dwordx4 %0, %4\n\tglobal_load_lds_dwordx4 %1, %4 offset:1024\n\t"
                 "global_load_lds_dwordx4 %2, %4 offset:2048\n\tglobal_load_lds_dwordx4 %3, %4 offset:3072"
                 :: "v"(c.vb[0]), "v"(c.vb[1]), "v"(c.vb[2]), "v"(c.vb[3]), "s"(pb), "s"(c.ldsB + (S % DEPTH) * B_BYTES) : "memory");
}
template <int S>
__device__ __forceinline__ void load_x(const LoadCtx& c) {
    const unsigned char* pa = c.xa + korder<S>(c.lead, c.tn) * BK;
    asm volatile("s_mov_b32 m0, %3\n\ts_nop 3\n\t"
                 "global_load_lds_dwordx4 %0, %2 sc0 sc1\n\tglobal_load_lds_dwordx4 %1, %2 offset:1024 sc0 sc1"
                 :: "v"(c.va[0]), "v"(c.va[1]), "s"(pa), "s"(c.ldsA + (S % ADEPTH) * A_BYTES) : "memory");
}
constexpr int mini(int a, int b) { return a < b ? a : b; }
constexpr int issued(int I) { return mini(NSTAGE, mini(PRO + 2 * I, I + DEPTH - 1)); }
constexpr int younger(int I) {
    return 4 * (issued(I) - 1 - I) + 2 * (mini(issued(I), NSTAGE - 1) - I);
}
template <int S, int E>
__device__ __forceinline__ void load_range(const LoadCtx& c) {
    if constexpr (S < E) {
        load_w<S>(c);
        if constexpr (S + 1 < NSTAGE) load_x<S + 1>(c);
        load_range<S + 1, E>(c);
    }
}
template <int I>
__device__ __forceinline__ void loader_loop(const LoadCtx& c) {
    WAIT_VM(younger(I));
    BAR();
    if constexpr (I + 1 < NSTAGE) {
        load_range<issued(I), issued(I + 1)>(c);
        loader_loop<I + 1>(c);
    }
}

__device__ __forceinline__ unsigned wave_scan(unsigned v) {
    v += __builtin_amdgcn_update_dpp(0u, v, 0x111, 0xf, 0xf, true);
    v += __builtin_amdgcn_update_dpp(0u, v, 0x112, 0xf, 0xf, true);
    v += __builtin_amdgcn_update_dpp(0u, v, 0x114, 0xf, 0xf, true);
    v += __builtin_amdgcn_update_dpp(0u, v, 0x118, 0xf, 0xf, true);
    v += __builtin_amdgcn_update_dpp(0u, v, 0x142, 0xa, 0xf, false);
    v += __builtin_amdgcn_update_dpp(0u, v, 0x143, 0xc, 0xf, false);
    return v;
}

__device__ __forceinline__ unsigned draw_ticket(unsigned char* ctr) {
    unsigned t;
    asm volatile("s_mov_b64 exec, 1\n\ts_nop 3\n\tglobal_atomic_add %0, %1, %2, %3 sc0\n\ts_mov_b64 exec, -1"
                 : "=&v"(t) : "v"(0u), "v"(1u), "s"(ctr) : "memory");
    return t;
}

__device__ __forceinline__ void touch(const unsigned char* base, unsigned voff, unsigned lds_dst) {
    asm volatile("s_mov_b32 m0, %2\n\ts_nop 3\n\tglobal_load_lds_dword %0, %1" :: "v"(voff), "s"(base), "s"(lds_dst) : "memory");
}

__device__ __forceinline__ void put_flag(unsigned char* flag, unsigned call, unsigned low) {
    typedef __attribute__((address_space(1))) unsigned long long gu64;
    __hip_atomic_store((gu64*)flag, ((unsigned long long)call << 32) | low, __ATOMIC_RELAXED, __HIP_MEMORY_SCOPE_SYSTEM);
}

__device__ __forceinline__ unsigned wait_flag(const unsigned char* flag, unsigned voff, unsigned* tmo, unsigned call, int lane) {
    const unsigned t0 = (unsigned)__builtin_amdgcn_s_memrealtime();
    unsigned low;
    bool seen;
    for (;;) {
        v2i f;
        unsigned t;
        asm volatile("s_nop 4\n\tglobal_load_dwordx2 %0, %2, %3 sc0 sc1\n\tglobal_load_dword %1, %4, %5 sc1\n\ts_waitcnt vmcnt(0)"
                     : "=&v"(f), "=&v"(t) : "v"(voff), "s"(flag), "v"(0u), "s"(tmo) : "memory");
        low = __builtin_amdgcn_readfirstlane(f[0]);
        seen = __builtin_amdgcn_ballot_w64((unsigned)f[1] < call) == 0ull;
        if (seen || __builtin_amdgcn_readfirstlane(t) == call || (unsigned)__builtin_amdgcn_s_memrealtime() - t0 > DEADLINE_TICKS) break;
        __builtin_amdgcn_s_sleep(POLL_SLEEP);
    }
    if (!seen && lane == 0) __hip_atomic_store(tmo, call, __ATOMIC_RELAXED, __HIP_MEMORY_SCOPE_AGENT);
    return seen ? low : 0xffffffffu;
}

__device__ __forceinline__ void store_out(float* ptr, u32x4 v) {
    asm volatile("global_store_dwordx4 %0, %1, off sc0 sc1\n\ts_nop 1" :: "v"(ptr), "v"(v) : "memory");
}

__device__ __forceinline__ unsigned compact(unsigned base, unsigned mask, const float (&y)[16], const unsigned long long (&nz)[16]) {
    const unsigned cnt = __builtin_popcount(mask);
    const unsigned incl = wave_scan(cnt);
    unsigned ptr = base + (incl - cnt) * 4;
#define PK1(m, v) "s_mov_b64 exec, %" #m "\n\tds_write_b32 %0, %" #v "\n\tv_add_u32 %0, 4, %0\n\t"
#pragma unroll
    for (int q = 0; q < 2; ++q)
        asm volatile(PK1(1, 9) PK1(2, 10) PK1(3, 11) PK1(4, 12) PK1(5, 13) PK1(6, 14) PK1(7, 15) PK1(8, 16)
                     "s_mov_b64 exec, -1"
                     : "+v"(ptr)
                     : "s"(nz[8 * q]), "s"(nz[8 * q + 1]), "s"(nz[8 * q + 2]), "s"(nz[8 * q + 3]),
                       "s"(nz[8 * q + 4]), "s"(nz[8 * q + 5]), "s"(nz[8 * q + 6]), "s"(nz[8 * q + 7]),
                       "v"(y[8 * q]), "v"(y[8 * q + 1]), "v"(y[8 * q + 2]), "v"(y[8 * q + 3]),
                       "v"(y[8 * q + 4]), "v"(y[8 * q + 5]), "v"(y[8 * q + 6]), "v"(y[8 * q + 7])
                     : "memory");
    return __builtin_amdgcn_readlane(incl, 63);
}

__device__ __forceinline__ int step4(int addr, int sel) {
    int r;
    asm("v_mad_i32_i24 %0, %1, -4, %2" : "=v"(r) : "v"(sel), "v"(addr));
    return r;
}

struct Frag { v4i a[2][2][2]; v2i b[2][4]; };

__device__ __forceinline__ v8i cat8(v4i lo, v4i hi) { return __builtin_shufflevector(lo, hi, 0, 1, 2, 3, 4, 5, 6, 7); }
__device__ __forceinline__ v8i cat8b(v2i a, v2i b, v2i c, v2i d) {
    return cat8(__builtin_shufflevector(a, b, 0, 1, 2, 3), __builtin_shufflevector(c, d, 0, 1, 2, 3));
}

__global__ void __launch_bounds__(512, 2) gemm_a2a(Params p) {
    __shared__ __attribute__((aligned(1024))) unsigned char lds[LDS_BYTES];
    LAS unsigned char* L = (LAS unsigned char*)lds;
    const unsigned lds0 = (unsigned)(size_t)L;
    const int tid = threadIdx.x, lane = tid & 63;
    const int wid = __builtin_amdgcn_readfirstlane(tid >> 6);
    const int b = blockIdx.x;
    const int g = b & 7, j = b >> 3, tm = j >> 2, tn = j & 3;
    LoadCtx c;
    {
        const int lw = wid & 3;
#pragma unroll
        for (int i = 0; i < 2; ++i) {
            const int row = (lw * 2 + i) * 8 + (lane >> 3);
            c.va[i] = row * KK + (((lane & 7) ^ ((row >> 1) & 7)) << 4) - i * 1024;
        }
#pragma unroll
        for (int i = 0; i < 4; ++i) {
            const int row = (lw * 4 + i) * 8 + (lane >> 3);
            c.vb[i] = row * NN + (((lane & 7) ^ (row & 6)) << 4) - i * 1024;
        }
        c.ldsA = lds0 + A_RING + lw * 2048;
        c.ldsB = lds0 + lw * 4096;
        c.tn = tn;
    }
    const unsigned char* px = p.x;
    const unsigned char* pw = p.w;
    unsigned char* myws = p.myws;
    int rank = p.rank;
    asm volatile("" : "+v"(c.va[0]), "+v"(c.va[1]), "+v"(c.vb[0]), "+v"(c.vb[1]), "+v"(c.vb[2]), "+v"(c.vb[3]));
    asm volatile("" : "+s"(px), "+s"(pw), "+s"(rank));

    const unsigned char* xa = px + (size_t)tm * BM * KK;
    int wcol = g * NS + tn * BN;
    const bool early = g == rank || tm == 7;
    c.lead = !early && tm >= LEAD0 ? 8 * (tm - LEAD0) : -1;
    if (__builtin_expect(!early && wid >= 4, 1)) {
        c.xa = xa;
        c.wb = pw + wcol;
        load_x<0>(c);
        load_range<0, issued(0)>(c);
    }
    const unsigned t_begin = (unsigned)__builtin_amdgcn_s_memrealtime();
    if (wid == 0) STAMP(0);
    unsigned* tmo = (unsigned*)(myws + WS_TMO);
    int orow = rank * MS + tm * BM, ocol = tn * BN;
    const bool local = early;
    bool lost = false;

    if (early) {
        unsigned ticket = 0;
        if (wid == 0) ticket = draw_ticket(myws + WS_CTR + b * 128);
        u32x4 r[8];
        if (g != rank) {
#pragma unroll
            for (int q = 0; q < 8; ++q)
                asm volatile("s_nop 4\n\tglobal_load_dwordx4 %0, %1, %2" : "=v"(r[q]) : "v"(lane * 16 + q * 1024), "s"(xa + (tn * 16 + wid * 2) * KK) : "memory");
        } else {
            touch(px, (j * 512 + tid) * 128, lds0);
            while ((unsigned)__builtin_amdgcn_s_memrealtime() - t_begin < TWARM) __builtin_amdgcn_s_sleep(4);
            touch(pw + wcol, (tm * 512 + tid) * NN, lds0);
        }
        asm volatile("s_waitcnt vmcnt(0)"
                     : "+v"(ticket), "+v"(r[0]), "+v"(r[1]), "+v"(r[2]), "+v"(r[3]), "+v"(r[4]), "+v"(r[5]), "+v"(r[6]), "+v"(r[7])
                     :: "memory");
        if (wid == 0 && lane == 0) *(LAS v4i*)(L + LDS_MISC) = (v4i){(int)(ticket + 1u), 0, 0, 0};
        __syncthreads();
        const unsigned call = __builtin_amdgcn_readfirstlane(*(LAS unsigned*)(L + LDS_MISC));
        const unsigned par = call & 1u;
        if (g != rank) {
            unsigned char* pws = p.ws[g];
            unsigned char* dx = pws + WS_XSTAGE + (size_t)((par * 8 + rank) * BM + tn * 16 + wid * 2) * KK;
#pragma unroll
            for (int q = 0; q < 8; ++q)
                asm volatile("s_nop 4\n\tglobal_store_dwordx4 %0, %1, %2 sc0 sc1\n\ts_nop 1" :: "v"(lane * 16 + q * 1024), "v"(r[q]), "s"(dx) : "memory");
            WAIT_VM(0);
            __syncthreads();
            if (tid == 0) put_flag(pws + WS_XFLAG + ((par * 8 + rank) * 4 + tn) * 128, call, 0u);
            if (wid == 0) STAMP(2);
            const int u = ((g - rank - 1) & 7) * 4 + tn;
            xa = px + (size_t)(u % 7) * BM * KK;
            ocol = (u / 7) * BN;
            wcol = rank * NS + ocol;
            orow = rank * MS + (u % 7) * BM;
        } else if (tm < 7) {
            const int src_rank = (rank + 1 + tm) & 7;
            if (wid == 0) {
                STAMP(2);
                const unsigned f = wait_flag(myws + WS_XFLAG + (par * 8 + src_rank) * 512, (lane & 3) * 128, tmo, call, lane);
                const unsigned t = __hip_atomic_load(tmo, __ATOMIC_RELAXED, __HIP_MEMORY_SCOPE_AGENT);
                if (f != 0u || t == call) *(LAS unsigned*)(L + LDS_MISC + 4) = 1u;
                STAMP(3);
            }
            __syncthreads();
            lost = __builtin_amdgcn_readfirstlane(*(LAS unsigned*)(L + LDS_MISC + 4)) != 0;
            xa = lost ? px : myws + WS_XSTAGE + (size_t)(par * 8 + src_rank) * BM * KK;
            orow = src_rank * MS + XROW0;
        }
    }

    v16f acc[2];
    float scale;
    unsigned char* dws = nullptr;

    if (wid >= 4) {
        if (early) {
            c.xa = xa;
            c.wb = pw + wcol;
            load_x<0>(c);
            load_range<0, issued(0)>(c);
        }
        dws = p.ws[g];
        asm volatile("" : "+s"(dws));
        loader_loop<0>(c);
        scale = __builtin_bit_cast(float, __builtin_amdgcn_readfirstlane(*(LAS int*)(L + LDS_MISC + 12)));
    } else {
        unsigned ticket = 0;
        if (wid == 0 && !early) ticket = draw_ticket(myws + WS_CTR + b * 128);
        scale = __builtin_bit_cast(float, __builtin_amdgcn_readfirstlane(lost ? 0x7fc00000 : __builtin_bit_cast(int, p.sx[0] * p.sw[0])));
        dws = p.ws[g];
        asm volatile("" : "+s"(scale), "+s"(dws));
        if (tid == 0) *(LAS float*)(L + LDS_MISC + 12) = scale;
        acc[0] = acc[1] = (v16f)0.f;
        const int r = lane & 31, h = lane >> 5;
        unsigned ao[2][2];
#pragma unroll
        for (int ks = 0; ks < 2; ++ks)
#pragma unroll
            for (int e = 0; e < 2; ++e) ao[ks][e] = A_RING + r * 128 + (((4 * ks + 2 * h + e) ^ ((r >> 1) & 7)) << 4);
        const int q = (lane & 15) >> 1, pp = lane & 1, g1 = (lane >> 4) & 1;
        const unsigned bo = (32 * h + q) * 128 + (((2 * wid + g1) ^ (q & 6)) << 4) + pp * 8;

        auto rd = [&](Frag& f, int slot, int aslot, int ks) {
            const unsigned char LAS* base = L + slot * B_BYTES;
            const unsigned char LAS* abase = L + aslot * A_BYTES;
#pragma unroll
            for (int t = 0; t < 4; ++t)
                f.b[ks][t] = __builtin_amdgcn_ds_read_tr8_b64_v2i32((LAS v2i*)(base + bo + (64 * ks + 8 * t) * 128));
#pragma unroll
            for (int mt = 0; mt < 2; ++mt)
#pragma unroll
                for (int e = 0; e < 2; ++e) f.a[ks][mt][e] = *(const LAS v4i*)(abase + ao[ks][e] + mt * 4096);
        };
        auto mm = [&](const Frag& f, int ks, int mt) {
            acc[mt] = __builtin_amdgcn_mfma_scale_f32_32x32x64_f8f6f4(cat8b(f.b[ks][0], f.b[ks][1], f.b[ks][2], f.b[ks][3]),
                                                                      cat8(f.a[ks][mt][0], f.a[ks][mt][1]), acc[mt], 0, 0, 0, 0, 0, 0);
        };
        auto step = [&](Frag& fn, const Frag& fo, int slot, int aslot) {
            BAR(); SCHED();
            mm(fo, 0, 0); SCHED();
            rd(fn, slot, aslot, 0); SCHED();
            mm(fo, 0, 1); SCHED();
            rd(fn, slot, aslot, 1); SCHED();
            mm(fo, 1, 0); mm(fo, 1, 1); SCHED();
            WAIT_LGKM0(); SCHED();
        };
        Frag f0, f1;
        BAR();
        rd(f0, 0, 0, 0);
        rd(f0, 0, 0, 1);
        WAIT_LGKM0();
        SCHED();
        int slot = 1, aslot = 1;
#pragma unroll 1
        for (int i = 1; i < NSTAGE - 1; i += 2) {
            step(f1, f0, slot, aslot);
            slot = slot + 1 == DEPTH ? 0 : slot + 1;
            aslot = aslot + 1 == ADEPTH ? 0 : aslot + 1;
            step(f0, f1, slot, aslot);
            slot = slot + 1 == DEPTH ? 0 : slot + 1;
            aslot = aslot + 1 == ADEPTH ? 0 : aslot + 1;
        }
        step(f1, f0, slot, aslot);
        mm(f1, 0, 0); mm(f1, 0, 1); mm(f1, 1, 0); mm(f1, 1, 1);
        if (wid == 0 && !early) {
            asm volatile("s_waitcnt vmcnt(0)" : "+v"(ticket) :: "memory");
            if (lane == 0) *(LAS v4i*)(L + LDS_MISC) = (v4i){(int)(ticket + 1u), 0, 0, __builtin_bit_cast(int, scale)};
        }
        if (!local) {
#pragma unroll
            for (int gq = 0; gq < 4; ++gq)
                *(LAS v4f*)(L + LDS_HAND + wid * 4096 + gq * 1024 + lane * 16) =
                    (v4f){acc[1][4 * gq], acc[1][4 * gq + 1], acc[1][4 * gq + 2], acc[1][4 * gq + 3]};
        }
        WAIT_LGKM0();
    }
    BAR();
    if (wid == 0) STAMP(1);

    if (local) {
        if (wid < 4) {
            const int r = lane & 31, h = lane >> 5;
#pragma unroll
            for (int mt = 0; mt < 2; ++mt)
#pragma unroll
                for (int gq = 0; gq < 4; ++gq) {
                    v4f v;
#pragma unroll
                    for (int e = 0; e < 4; ++e) { const float y = acc[mt][4 * gq + e] * scale; v[e] = y <= 0.f ? 0.f : y; }
                    const int row = mt * 32 + r, ch = wid * 8 + 2 * gq + h;
                    *(LAS v4f*)(L + row * 512 + ((ch ^ (row & 7)) << 4)) = v;
                }
        }
        __syncthreads();
        float* ob = p.out + (size_t)orow * NS + ocol;
#pragma unroll
        for (int u = 0; u < 4; ++u) {
            const int row = (wid * 4 + u) * 2 + (lane >> 5), ch = lane & 31;
            store_out(ob + row * NS + ch * 4, *(LAS u32x4*)(L + row * 512 + ((ch ^ (row & 7)) << 4)));
        }
    } else {
        const unsigned lo = lane * 16;
        const unsigned callv = *(LAS unsigned*)(L + LDS_MISC);
        if (wid >= 4) {
            const LAS v4f* h = (const LAS v4f*)(L + LDS_HAND + (wid - 4) * 4096 + lo);
            acc[0] = __builtin_shufflevector(__builtin_shufflevector(h[0], h[64], 0, 1, 2, 3, 4, 5, 6, 7),
                                             __builtin_shufflevector(h[128], h[192], 0, 1, 2, 3, 4, 5, 6, 7),
                                             0, 1, 2, 3, 4, 5, 6, 7, 8, 9, 10, 11, 12, 13, 14, 15);
        }
        unsigned total = 0;
        bool dense = (((tm >= LEAD0 ? DENSE_L : DENSE_F) >> wid) & 1) &&
                     __builtin_bit_cast(unsigned, scale) - 0x0d800000u < 0x71800000u - 0x0d800000u;
        if (!dense) __builtin_amdgcn_s_setprio(PKPRIO);
        if (dense) {
            float y[16];
            unsigned long long nz[16];
            unsigned mask = 0, m1 = 0, m2 = 0, inexact = 0;
            float top = 0.f;
#pragma unroll
            for (int q = 3; q >= 0; --q) {
                unsigned long long s1[4], s2[4];
#pragma unroll
                for (int e = 0; e < 4; ++e) {
                    const float a = acc[0][4 * q + e], am = __builtin_elementwise_minimum(a, 32.f), u = am + 32.f;
                    nz[4 * q + e] = __builtin_amdgcn_fcmpf(a, 0.f, 2  );
                    s1[e] = __builtin_amdgcn_fcmpf(a, 32.f, 3  );
                    s2[e] = __builtin_amdgcn_fcmpf(a, 128.f, 3);
                    inexact = __builtin_amdgcn_bitop3_b32(inexact, __builtin_bit_cast(unsigned, u - 32.f), __builtin_bit_cast(unsigned, am), 0xf6);
                    y[4 * q + e] = __builtin_amdgcn_inverse_ballot_w64(s1[e]) ? a : u;
                    top = __builtin_elementwise_maximum(top, a);
                }
#define AC(d, m) "v_addc_co_u32_e64 %" #d ", vcc, %" #d ", %" #d ", %" #m "\n\t"
                asm volatile("s_nop 1\n\t" AC(0, 6) AC(1, 10) AC(2, 14) AC(0, 5) AC(1, 9) AC(2, 13) AC(0, 4) AC(1, 8) AC(2, 12) AC(0, 3) AC(1, 7) AC(2, 11)
                             : "+v"(mask), "+v"(m1), "+v"(m2)
                             : "s"(nz[4 * q]), "s"(nz[4 * q + 1]), "s"(nz[4 * q + 2]), "s"(nz[4 * q + 3]),
                               "s"(s1[0]), "s"(s1[1]), "s"(s1[2]), "s"(s1[3]), "s"(s2[0]), "s"(s2[1]), "s"(s2[2]), "s"(s2[3])
                             : "vcc");
            }
            dense = __builtin_amdgcn_ballot_w64((inexact << 1) != 0u || !(top < 512.f)) == 0ull;
            if (dense) {
                *(LAS unsigned*)(L + wid * PK_SLOT + lane * 4) = (mask ^ m1 ^ m2) | (m1 << 16);
                total = compact(lds0 + LDS_RECV + wid * PK_SLOT, mask, y, nz);
#pragma unroll 1
                for (unsigned o = 0; o < total; o += 256) {
                    const u32x4 v = *(LAS u32x4*)(L + LDS_RECV + wid * PK_SLOT + o * 4 + lo);
                    LAS unsigned* d = (LAS unsigned*)(L + wid * PK_SLOT + 256 + o * 3 + lane * 12);
                    d[0] = __builtin_amdgcn_perm(v[1], v[0], 0x04020100u);
                    d[1] = __builtin_amdgcn_perm(v[2], v[1], 0x05040201u);
                    d[2] = __builtin_amdgcn_perm(v[3], v[2], 0x06050402u);
                }
            }
        }
        if (!dense) {
            float y[16];
            unsigned long long nz[16];
#pragma unroll
            for (int i = 0; i < 16; ++i) {
                y[i] = acc[0][i] * scale;
                nz[i] = __builtin_amdgcn_fcmpf(y[i], 0.f, 10  );
            }
            unsigned mask = 0;
#pragma unroll
            for (int q = 1; q >= 0; --q)
                asm volatile("s_nop 1\n\t" AC(0, 8) AC(0, 7) AC(0, 6) AC(0, 5) AC(0, 4) AC(0, 3) AC(0, 2) AC(0, 1)
                             : "+v"(mask)
                             : "s"(nz[8 * q]), "s"(nz[8 * q + 1]), "s"(nz[8 * q + 2]), "s"(nz[8 * q + 3]),
                               "s"(nz[8 * q + 4]), "s"(nz[8 * q + 5]), "s"(nz[8 * q + 6]), "s"(nz[8 * q + 7])
                             : "vcc");
            *(LAS unsigned short*)(L + wid * PK_SLOT + lane * 2) = (unsigned short)mask;
            total = compact(lds0 + wid * PK_SLOT + 128, mask, y, nz);
        }
        u32x4 c[3];
#pragma unroll
        for (int u = 0; u < 3; ++u) c[u] = *(LAS u32x4*)(L + wid * PK_SLOT + u * 1024 + lo);
        const unsigned call = __builtin_amdgcn_readfirstlane(callv);
        const unsigned par = call & 1u;
        const unsigned bytes = dense ? 256 + 3 * total : 128 + 4 * total;
        const unsigned sidx = ((par * 8 + rank) * 32 + j) * 8 + wid;
        unsigned char* dslot = dws + WS_STAGE + (size_t)sidx * PK_SLOT;
        const int turn = wid - WIN;
        if (turn > 0)
            while ((int)__builtin_amdgcn_readfirstlane(__hip_atomic_load((LAS unsigned*)(L + LDS_MISC + 8), __ATOMIC_RELAXED,
                                                                         __HIP_MEMORY_SCOPE_WORKGROUP)) < turn)
                __builtin_amdgcn_s_sleep(1);
        if (wid == 0) STAMP(2);
#pragma unroll
        for (int u = 0; u < 3; ++u)
            if (u * 1024 + lo < bytes)
                asm volatile("s_nop 4\n\tglobal_store_dwordx4 %0, %1, %2 sc0 sc1\n\ts_nop 1" :: "v"(lo + u * 1024), "v"(c[u]), "s"(dslot) : "memory");
#pragma unroll 1
        for (unsigned o = 3 * 1024; o < bytes; o += 1024) {
            const u32x4 v = *(LAS u32x4*)(L + wid * PK_SLOT + o + lo);
            if (o + lo < bytes)
                asm volatile("s_nop 4\n\tglobal_store_dwordx4 %0, %1, %2 sc0 sc1\n\ts_nop 1" :: "v"(lo + o), "v"(v), "s"(dslot) : "memory");
        }
        WAIT_VM(0);
        if (lane == 0) {
            put_flag(dws + WS_FLAG + (size_t)sidx * 128, call, total | (dense ? 0x10000u : 0u));
            __hip_atomic_fetch_add((LAS unsigned*)(L + LDS_MISC + 8), 1u, __ATOMIC_RELAXED, __HIP_MEMORY_SCOPE_WORKGROUP);
        }
        if (wid == 7) STAMP(3);
        const unsigned ridx = ((par * 8 + g) * 32 + j) * 8 + wid;
        const unsigned rword = wait_flag(myws + WS_FLAG + (size_t)ridx * 128, 0u, tmo, call, lane);
        if (wid == 7) STAMP(4);
        const unsigned rtotal = __builtin_amdgcn_readfirstlane(rword) & ~0x10000u;
        const bool rdense = __builtin_amdgcn_readfirstlane(rword) & 0x10000u;
        u32x4 o[4];
        bool bad = true;
        if (rtotal <= 1024u) {
            const unsigned rbytes = rdense ? 256 + 3 * rtotal : 128 + 4 * rtotal;
            const unsigned char* sslot = myws + WS_STAGE + (size_t)ridx * PK_SLOT;
            const unsigned rdst = __builtin_amdgcn_readfirstlane(lds0 + LDS_RECV + wid * PK_SLOT);
            asm volatile("s_mov_b32 m0, %2\n\ts_nop 3\n\tglobal_load_lds_dwordx4 %0, %1 sc0 sc1\n\t"
                         "s_cmp_le_u32 %3, 1024\n\ts_cbranch_scc1 1f\n\tglobal_load_lds_dwordx4 %0, %1 offset:1024 sc0 sc1\n\t"
                         "s_cmp_le_u32 %3, 2048\n\ts_cbranch_scc1 1f\n\tglobal_load_lds_dwordx4 %0, %1 offset:2048 sc0 sc1\n\t"
                         "s_cmp_le_u32 %3, 3072\n\ts_cbranch_scc1 1f\n\tglobal_load_lds_dwordx4 %0, %1 offset:3072 sc0 sc1\n\t"
                         "s_cmp_le_u32 %3, 4096\n\ts_cbranch_scc1 1f\n\ts_mov_b32 m0, %5\n\ts_nop 0\n\tglobal_load_lds_dwordx4 %4, %1 sc0 sc1\n"
                         "1:"
                         :: "v"(lo), "s"(sslot), "s"(rdst), "s"(rbytes), "v"(lo + 4096), "s"(rdst + 4096) : "memory", "scc");
            unsigned t;
            asm volatile("s_nop 4\n\tglobal_load_dword %0, %1, %2 sc1\n\ts_waitcnt vmcnt(0)" : "=&v"(t) : "v"(0u), "s"(tmo) : "memory");
            if (wid == 7) STAMP(5);
            bad = __builtin_amdgcn_readfirstlane(t) == call;
            const unsigned bp = 4 * ((lane & 7) >> 1);
            LAS v2i* tab = (LAS v2i*)(L + LDS_TAB + wid * 512);
            if (!rdense) {
                const unsigned rm = *(LAS unsigned short*)(L + LDS_RECV + wid * PK_SLOT + lane * 2);
                const unsigned rc = __builtin_popcount(rm);
                tab[lane] = (v2i){(int)rm, (int)(LDS_RECV + wid * PK_SLOT + 128 + (wave_scan(rc) - rc) * 4)};
#pragma unroll
                for (int u = 0; u < 4; ++u) {
                    const v2i tb = tab[8 * u + (lane >> 3) + 32 * (lane & 1)];
                    const unsigned m = (unsigned)tb[0];
                    int addr = tb[1] + 4 * __builtin_popcount(m & ((1u << bp) - 1u));
                    const int ms = (int)(m >> bp);
#pragma unroll
                    for (int e = 0; e < 4; ++e) {
                        const int sel = __builtin_amdgcn_sbfe(ms, e, 1);
                        o[u][e] = *(LAS unsigned*)(L + addr) & (unsigned)sel;
                        if (e < 3) addr = step4(addr, sel);
                    }
                }
            } else {
                const unsigned rm = *(LAS unsigned*)(L + LDS_RECV + wid * PK_SLOT + lane * 4);
#pragma unroll 1
                for (unsigned q = 0; q < rtotal; q += 256) {
                    const LAS unsigned* d = (const LAS unsigned*)(L + LDS_RECV + wid * PK_SLOT + 256 + q * 3 + lane * 12);
                    const unsigned d0 = d[0], d1 = d[1], d2 = d[2];
                    *(LAS u32x4*)(L + wid * PK_SLOT + q * 4 + lo) =
                        (u32x4){(d0 & 0xffffffu) | 0x42000000u, __builtin_amdgcn_perm(d1, d0, 0x0c050403u) | 0x42000000u,
                                __builtin_amdgcn_perm(d2, d1, 0x0c040302u) | 0x42000000u, __builtin_amdgcn_alignbyte(0x42u, d2, 1)};
                }
                const unsigned rc = __builtin_popcount((rm | (rm >> 16)) & 0xffffu);
                tab[lane] = (v2i){(int)rm, (int)(wid * PK_SLOT + (wave_scan(rc) - rc) * 4)};
                const unsigned sb = __builtin_bit_cast(unsigned, scale), nb = __builtin_bit_cast(unsigned, -32.f * scale);
#pragma unroll
                for (int u = 0; u < 4; ++u) {
                    const v2i tb = tab[8 * u + (lane >> 3) + 32 * (lane & 1)];
                    const unsigned m = (unsigned)tb[0];
                    int addr = tb[1] + 4 * __builtin_popcount((m | (m >> 16)) & ((1u << bp) - 1u));
                    const int ms = (int)(m >> bp);
#pragma unroll
                    for (int e = 0; e < 4; ++e) {
                        const unsigned b0 = __builtin_amdgcn_sbfe(ms, e, 1), b1 = __builtin_amdgcn_sbfe(ms, 16 + e, 1);
                        const unsigned code = *(LAS unsigned*)(L + addr) | __builtin_amdgcn_bitop3_b32(b0, b1, 0x01000000u, 0x80);
                        o[u][e] = __builtin_bit_cast(unsigned, __builtin_fmaf(__builtin_bit_cast(float, code),
                                                                              __builtin_bit_cast(float, __builtin_amdgcn_bitop3_b32(b0, b1, sb, 0xa8)),
                                                                              __builtin_bit_cast(float, __builtin_amdgcn_bitop3_b32(b0, b1, nb, 0x20))));
                        if (e < 3) addr = step4(addr, (int)(b0 | b1));
                    }
                }
            }
        }
        float* ob = p.out + (size_t)(g * MS + tm * BM + (wid >> 2) * 32 + (lane >> 3)) * NS + tn * BN + (wid & 3) * 32 + (lane & 7) * 4;
#pragma unroll
        for (int u = 0; u < 4; ++u) {
            if (bad) o[u] = (u32x4){0x7fc00000u, 0x7fc00000u, 0x7fc00000u, 0x7fc00000u};
            store_out(ob + u * 8 * NS, o[u]);
        }
    }
}

extern "C" void kernel_launch(int rank, int nranks, ncclComm_t comm, hipStream_t stream,
                              void* const* d_in, const int* in_sizes, int n_in,
                              void* d_out, int out_size, const int* in_dtypes, int out_dtype,
                              void* const* d_out_all, void* const* d_ws_all, size_t ws_bytes) {
    if (nranks != NR || n_in != 4 || in_sizes[0] != MS * KK || in_sizes[1] != KK * NN || out_size != NR * MS * NS ||
        (unsigned)rank >= (unsigned)NR || ws_bytes < WS_END) {
        fprintf(stderr, "unexpected problem shape\n");
        return;
    }
    Params p;
    p.x = (const unsigned char*)d_in[0];
    p.w = (const unsigned char*)d_in[1];
    p.sx = (const float*)d_in[2];
    p.sw = (const float*)d_in[3];
    p.out = (float*)d_out;
    p.myws = (unsigned char*)d_ws_all[rank];
    for (int i = 0; i < NR; ++i) p.ws[i] = (unsigned char*)d_ws_all[i];
    p.rank = rank;
    p.pad = 0;
    gemm_a2a<<<GRID, 512, 0, stream>>>(p);
    HIP_CHECK(hipGetLastError());
}
```

```cpp
#include <hip/hip_runtime.h>
#include <rccl/rccl.h>
#include <cstdio>
#include <cstddef>

#define HIP_CHECK(expr) do { hipError_t _e = (expr); if (_e != hipSuccess) { \
    fprintf(stderr, "HIP error %s at %s:%d\n", hipGetErrorString(_e), __FILE__, __LINE__); \
    return; } } while (0)

typedef int v8i __attribute__((ext_vector_type(8)));
typedef int v4i __attribute__((ext_vector_type(4)));
typedef int v2i __attribute__((ext_vector_type(2)));
typedef float v16f __attribute__((ext_vector_type(16)));
typedef float v4f __attribute__((ext_vector_type(4)));
typedef unsigned u32x4 __attribute__((ext_vector_type(4)));
#define LAS __attribute__((address_space(3)))

constexpr int NR = 8;
constexpr int MS = 512;
constexpr int KK = 4096;
constexpr int NN = 4096;
constexpr int NS = 512;
constexpr int BM = 64, BN = 128, BK = 128;
constexpr int NSTAGE = KK / BK;
constexpr int XROW0 = MS - BM;
constexpr int GRID = 256;
constexpr unsigned DEADLINE_TICKS = 50000000u;

#define DEPTH_ 4
#define POLL_SLEEP 2
#define WIN 2
#define PRO 2

#define TWARM 500
#define DENSE_F 0xfc
#define DENSE_L 0x7f
#define PKPRIO 3
#define TPUSH 0
#define PAUSE_LEAD 16
#define WIN_B 2
#define DENSE_B 0x7f
#define T8 400
#define PI0 5
#define TAU_A 26
#define TAU_B 32
#define TEND 850
#define TAU_2 0
#define TAU_1 0
#define T2 270
#define TAU_E 0
#define T5_E 850
#define LEAD0 3

constexpr int DEPTH = DEPTH_;
constexpr int ADEPTH = DEPTH + 1;
constexpr int A_BYTES = BM * BK;
constexpr int B_BYTES = BK * BN;
constexpr int A_RING = DEPTH * B_BYTES;
constexpr int RING_BYTES = DEPTH * B_BYTES + ADEPTH * A_BYTES;
constexpr int PK_SLOT = 5120;
constexpr int LDS_RECV = 8 * PK_SLOT;
constexpr int LDS_TAB = 16 * PK_SLOT;
constexpr int LDS_HAND = RING_BYTES;
constexpr int LDS_MISC = LDS_HAND + 4 * 4096;
constexpr int LDS_BYTES = LDS_MISC + 16;
static_assert(LDS_TAB + 8 * 512 <= RING_BYTES && LDS_BYTES > 80 * 1024 && LDS_BYTES <= 160 * 1024, "LDS layout");

constexpr size_t WS_CTR = 0;
constexpr size_t WS_TMO = 32768;
constexpr size_t WS_XFLAG = 36864;
constexpr size_t WS_STAMP = 65536;
constexpr size_t WS_FLAG = 131072;
constexpr size_t WS_XSTAGE = WS_FLAG + 2 * 2048 * 128;
constexpr size_t WS_STAGE = WS_XSTAGE + (size_t)16 * BM * KK;
constexpr size_t WS_END = WS_STAGE + (size_t)2 * 2048 * PK_SLOT;

struct Params {
    const unsigned char* x;
    const unsigned char* w;
    unsigned char* myws;
    int rank;
    int pad;
    const float* sx;
    const float* sw;
    float* out;
    unsigned char* ws[NR];
};
static_assert(offsetof(Params, x) == 0 && offsetof(Params, w) == 8 && offsetof(Params, rank) == 24, "read as words in the kernel");

#define WAIT_VM(n) asm volatile("s_waitcnt vmcnt(%0)" :: "i"(n) : "memory")
#define WAIT_LGKM0() asm volatile("s_waitcnt lgkmcnt(0)" ::: "memory")
#define BAR() __builtin_amdgcn_s_barrier()
#define SCHED() __builtin_amdgcn_sched_barrier(0)
#define STAMP(k) do {} while (0)

template <int S>
__device__ __forceinline__ int korder(int lead, int tn) {
    const int rs = (S + tn) & 3;
    const int kl = S < 8 ? lead + S : (lead + (S & ~3) + rs) & (NSTAGE - 1);
    return lead < 0 ? (S >> 2) + 8 * rs : kl;
}

struct LoadCtx {
    unsigned va[2], vb[4];
    const unsigned char* xa;
    const unsigned char* wb;
    unsigned ldsA, ldsB;
    int lead, tn;
    unsigned t0, t5, tau;
};
__device__ __forceinline__ void load_w_at(const LoadCtx& c, const unsigned char* pb, int slot) {
    asm volatile("s_mov_b32 m0, %5\n\ts_nop 3\n\t"
                 "global_load_lds_dwordx4 %0, %4\n\tglobal_load_lds_dwordx4 %1, %4 offset:1024\n\t"
                 "global_load_lds_dwordx4 %2, %4 offset:2048\n\tglobal_load_lds_dwordx4 %3, %4 offset:3072"
                 :: "v"(c.vb[0]), "v"(c.vb[1]), "v"(c.vb[2]), "v"(c.vb[3]), "s"(pb), "s"(c.ldsB + slot * B_BYTES) : "memory");
}
__device__ __forceinline__ void load_x_at(const LoadCtx& c, const unsigned char* pa, int slot) {
    asm volatile("s_mov_b32 m0, %3\n\ts_nop 3\n\t"
                 "global_load_lds_dwordx4 %0, %2 sc0 sc1\n\tglobal_load_lds_dwordx4 %1, %2 offset:1024 sc0 sc1"
                 :: "v"(c.va[0]), "v"(c.va[1]), "s"(pa), "s"(c.ldsA + slot * A_BYTES) : "memory");
}
template <int S>
__device__ __forceinline__ void load_w(const LoadCtx& c) { load_w_at(c, c.wb + (unsigned)korder<S>(c.lead, c.tn) * (BK * NN), S % DEPTH); }
template <int S>
__device__ __forceinline__ void load_x(const LoadCtx& c) { load_x_at(c, c.xa + (unsigned)korder<S>(c.lead, c.tn) * BK, S % ADEPTH); }
constexpr int mini(int a, int b) { return a < b ? a : b; }
constexpr int issued(int I) { return mini(NSTAGE, mini(PRO + 2 * I, I + DEPTH - 1)); }
constexpr int younger(int I) {
    return 4 * (issued(I) - 1 - I) + 2 * (mini(issued(I), NSTAGE - 1) - I);
}
template <int S, int E>
__device__ __forceinline__ void load_range(const LoadCtx& c) {
    if constexpr (S < E) {
        load_w<S>(c);
        if constexpr (S + 1 < NSTAGE) load_x<S + 1>(c);
        load_range<S + 1, E>(c);
    }
}
template <int I>
__device__ __forceinline__ void loader_loop(const LoadCtx& c) {
    WAIT_VM(younger(I));
    BAR();
    if constexpr (I >= PI0 && I < NSTAGE - DEPTH + 1) {
        const unsigned due = c.t5 + (I - PI0) * c.tau;
        if (c.tau && due <= TEND)
            while ((unsigned)__builtin_amdgcn_s_memrealtime() - c.t0 < due) __builtin_amdgcn_s_sleep(1);
    }
    if constexpr (I + 1 < NSTAGE) {
        load_range<issued(I), issued(I + 1)>(c);
        loader_loop<I + 1>(c);
    }
}

__device__ __forceinline__ unsigned wave_scan(unsigned v) {
    v += __builtin_amdgcn_update_dpp(0u, v, 0x111, 0xf, 0xf, true);
    v += __builtin_amdgcn_update_dpp(0u, v, 0x112, 0xf, 0xf, true);
    v += __builtin_amdgcn_update_dpp(0u, v, 0x114, 0xf, 0xf, true);
    v += __builtin_amdgcn_update_dpp(0u, v, 0x118, 0xf, 0xf, true);
    v += __builtin_amdgcn_update_dpp(0u, v, 0x142, 0xa, 0xf, false);
    v += __builtin_amdgcn_update_dpp(0u, v, 0x143, 0xc, 0xf, false);
    return v;
}

__device__ __forceinline__ unsigned draw_ticket(unsigned char* ctr) {
    unsigned t;
    asm volatile("s_mov_b64 exec, 1\n\ts_nop 3\n\tglobal_atomic_add %0, %1, %2, %3 sc0\n\ts_mov_b64 exec, -1"
                 : "=&v"(t) : "v"(0u), "v"(1u), "s"(ctr) : "memory");
    return t;
}

__device__ __forceinline__ void touch(const unsigned char* base, unsigned voff, unsigned lds_dst) {
    asm volatile("s_mov_b32 m0, %2\n\ts_nop 3\n\tglobal_load_lds_dword %0, %1" :: "v"(voff), "s"(base), "s"(lds_dst) : "memory");
}

__device__ __forceinline__ void put_flag(unsigned char* flag, unsigned call, unsigned low) {
    typedef __attribute__((address_space(1))) unsigned long long gu64;
    __hip_atomic_store((gu64*)flag, ((unsigned long long)call << 32) | low, __ATOMIC_RELAXED, __HIP_MEMORY_SCOPE_SYSTEM);
}

__device__ __forceinline__ unsigned wait_flag(const unsigned char* flag, unsigned voff, unsigned* tmo, unsigned call, int lane) {
    const unsigned t0 = (unsigned)__builtin_amdgcn_s_memrealtime();
    unsigned low;
    bool seen;
    for (;;) {
        v2i f;
        unsigned t;
        asm volatile("s_nop 4\n\tglobal_load_dwordx2 %0, %2, %3 sc0 sc1\n\tglobal_load_dword %1, %4, %5 sc1\n\ts_waitcnt vmcnt(0)"
                     : "=&v"(f), "=&v"(t) : "v"(voff), "s"(flag), "v"(0u), "s"(tmo) : "memory");
        low = __builtin_amdgcn_readfirstlane(f[0]);
        seen = __builtin_amdgcn_ballot_w64((unsigned)f[1] < call) == 0ull;
        if (seen || __builtin_amdgcn_readfirstlane(t) == call || (unsigned)__builtin_amdgcn_s_memrealtime() - t0 > DEADLINE_TICKS) break;
        __builtin_amdgcn_s_sleep(POLL_SLEEP);
    }
    if (!seen && lane == 0) __hip_atomic_store(tmo, call, __ATOMIC_RELAXED, __HIP_MEMORY_SCOPE_AGENT);
    return seen ? low : 0xffffffffu;
}

__device__ __forceinline__ void store_out(float* ptr, u32x4 v) {
    asm volatile("global_store_dwordx4 %0, %1, off sc0 sc1\n\ts_nop 1" :: "v"(ptr), "v"(v) : "memory");
}

__device__ __forceinline__ unsigned compact(unsigned base, unsigned mask, const float (&y)[16], const unsigned long long (&nz)[16]) {
    const unsigned cnt = __builtin_popcount(mask);
    const unsigned incl = wave_scan(cnt);
    unsigned ptr = base + (incl - cnt) * 4;
#define PK1(m, v) "s_mov_b64 exec, %" #m "\n\tds_write_b32 %0, %" #v "\n\tv_add_u32 %0, 4, %0\n\t"
#pragma unroll
    for (int q = 0; q < 2; ++q)
        asm volatile(PK1(1, 9) PK1(2, 10) PK1(3, 11) PK1(4, 12) PK1(5, 13) PK1(6, 14) PK1(7, 15) PK1(8, 16)
                     "s_mov_b64 exec, -1"
                     : "+v"(ptr)
                     : "s"(nz[8 * q]), "s"(nz[8 * q + 1]), "s"(nz[8 * q + 2]), "s"(nz[8 * q + 3]),
                       "s"(nz[8 * q + 4]), "s"(nz[8 * q + 5]), "s"(nz[8 * q + 6]), "s"(nz[8 * q + 7]),
                       "v"(y[8 * q]), "v"(y[8 * q + 1]), "v"(y[8 * q + 2]), "v"(y[8 * q + 3]),
                       "v"(y[8 * q + 4]), "v"(y[8 * q + 5]), "v"(y[8 * q + 6]), "v"(y[8 * q + 7])
                     : "memory");
    return __builtin_amdgcn_readlane(incl, 63);
}

__device__ __forceinline__ int step4(int addr, int sel) {
    int r;
    asm("v_mad_i32_i24 %0, %1, -4, %2" : "=v"(r) : "v"(sel), "v"(addr));
    return r;
}

struct Frag { v4i a[2][2][2]; v2i b[2][4]; };

__device__ __forceinline__ v8i cat8(v4i lo, v4i hi) { return __builtin_shufflevector(lo, hi, 0, 1, 2, 3, 4, 5, 6, 7); }
__device__ __forceinline__ v8i cat8b(v2i a, v2i b, v2i c, v2i d) {
    return cat8(__builtin_shufflevector(a, b, 0, 1, 2, 3), __builtin_shufflevector(c, d, 0, 1, 2, 3));
}

__global__ void __launch_bounds__(512, 2) gemm_a2a(Params p) {
    __shared__ __attribute__((aligned(1024))) unsigned char lds[LDS_BYTES];
    v8i ka;
    asm volatile("s_load_dwordx8 %0, %1, 0x0" : "=s"(ka) : "s"(__builtin_amdgcn_kernarg_segment_ptr()));
    SCHED();
    LAS unsigned char* L = (LAS unsigned char*)lds;
    const unsigned lds0 = (unsigned)(size_t)L;
    const int tid = threadIdx.x, lane = tid & 63;
    const int wid = __builtin_amdgcn_readfirstlane(tid >> 6);
    const int b = blockIdx.x;
    const int g = b & 7, j = b >> 3, tm = j >> 2, tn = j & 3;
    LoadCtx c;
    {
        const int lw = wid & 3;
#pragma unroll
        for (int i = 0; i < 2; ++i) {
            const int row = (lw * 2 + i) * 8 + (lane >> 3);
            c.va[i] = row * KK + (((lane & 7) ^ ((row >> 1) & 7)) << 4) - i * 1024;
        }
#pragma unroll
        for (int i = 0; i < 4; ++i) {
            const int row = (lw * 4 + i) * 8 + (lane >> 3);
            c.vb[i] = row * NN + (((lane & 7) ^ (row & 6)) << 4) - i * 1024;
        }
        c.ldsA = lds0 + A_RING + lw * 2048;
        c.ldsB = lds0 + lw * 4096;
        c.tn = tn;
    }
    c.lead = tm >= LEAD0 ? 8 * (tm - LEAD0) : -1;
    int wcol = g * NS + tn * BN;
    unsigned xo = tm * BM * KK;
    unsigned x0 = xo + korder<0>(c.lead, tn) * BK, w0 = wcol + korder<0>(c.lead, tn) * BK * NN;
    int skip = tm == 7 || wid < 4 ? -1 : g;
    asm volatile("" : "+v"(c.va[0]), "+v"(c.va[1]), "+v"(c.vb[0]), "+v"(c.vb[1]), "+v"(c.vb[2]), "+v"(c.vb[3]));
    asm volatile("" : "+s"(x0), "+s"(w0), "+s"(skip), "+s"(c.ldsA), "+s"(c.ldsB));
    asm volatile("s_waitcnt lgkmcnt(0)" : "+s"(ka));
    const unsigned char* px = (const unsigned char*)(((unsigned long long)(unsigned)ka[1] << 32) | (unsigned)ka[0]);
    const unsigned char* pw = (const unsigned char*)(((unsigned long long)(unsigned)ka[3] << 32) | (unsigned)ka[2]);
    const int rank = ka[6];
    unsigned char* myws = p.myws;
    const unsigned char* xa = px + xo;
    if (__builtin_expect(skip >= 0 && skip != rank, 1)) {
        load_x_at(c, px + x0, 0);
        load_w_at(c, pw + w0, 0);
        c.xa = xa;
        c.wb = pw + wcol;
        load_x<1>(c);
        load_range<1, issued(0)>(c);
    }
    const bool early = g == rank || tm == 7;
    const unsigned t_begin = (unsigned)__builtin_amdgcn_s_memrealtime();
    c.t0 = t_begin;
    c.tau = early ? TAU_E : (tm >= LEAD0 + 2 ? TAU_B : (tm >= LEAD0 ? TAU_A : (tm == 2 ? TAU_2 : (tm == 1 ? TAU_1 : 0))));
    c.t5 = early ? T5_E : (tm >= LEAD0 ? T8 : T2);
    if (wid == 0) STAMP(0);
    unsigned* tmo = (unsigned*)(myws + WS_TMO);
    int orow = rank * MS + tm * BM, ocol = tn * BN;
    const bool local = early;
    bool lost = false;

    if (early) {
        unsigned ticket = 0;
        if (wid == 0) ticket = draw_ticket(myws + WS_CTR + b * 128);
        u32x4 r[8];
        if (g != rank) {
#pragma unroll
            for (int q = 0; q < 8; ++q)
                asm volatile("s_nop 4\n\tglobal_load_dwordx4 %0, %1, %2" : "=v"(r[q]) : "v"(lane * 16 + q * 1024), "s"(xa + (tn * 16 + wid * 2) * KK) : "memory");
        } else {
            touch(px, (j * 512 + tid) * 128, lds0);
            while ((unsigned)__builtin_amdgcn_s_memrealtime() - t_begin < TWARM) __builtin_amdgcn_s_sleep(4);
            touch(pw + wcol, (tm * 512 + tid) * NN, lds0);
        }
        asm volatile("s_waitcnt vmcnt(0)"
                     : "+v"(ticket), "+v"(r[0]), "+v"(r[1]), "+v"(r[2]), "+v"(r[3]), "+v"(r[4]), "+v"(r[5]), "+v"(r[6]), "+v"(r[7])
                     :: "memory");
        if (wid == 0 && lane == 0) *(LAS v4i*)(L + LDS_MISC) = (v4i){(int)(ticket + 1u), 0, 0, 0};
        __syncthreads();
        const unsigned call = __builtin_amdgcn_readfirstlane(*(LAS unsigned*)(L + LDS_MISC));
        const unsigned par = call & 1u;
        if (g != rank) {
            unsigned char* pws = p.ws[g];
            unsigned char* dx = pws + WS_XSTAGE + (size_t)((par * 8 + rank) * BM + tn * 16 + wid * 2) * KK;
#pragma unroll
            for (int q = 0; q < 8; ++q)
                asm volatile("s_nop 4\n\tglobal_store_dwordx4 %0, %1, %2 sc0 sc1\n\ts_nop 1" :: "v"(lane * 16 + q * 1024), "v"(r[q]), "s"(dx) : "memory");
            WAIT_VM(0);
            __syncthreads();
            if (tid == 0) put_flag(pws + WS_XFLAG + ((par * 8 + rank) * 4 + tn) * 128, call, 0u);
            if (wid == 0) STAMP(2);
            while ((unsigned)__builtin_amdgcn_s_memrealtime() - t_begin < TPUSH) __builtin_amdgcn_s_sleep(4);
            const int u = ((g - rank - 1) & 7) * 4 + tn;
            xa = px + (size_t)(u % 7) * BM * KK;
            ocol = (u / 7) * BN;
            wcol = rank * NS + ocol;
            orow = rank * MS + (u % 7) * BM;
        } else if (tm < 7) {
            const int src_rank = (rank + 1 + tm) & 7;
            if (wid == 0) {
                STAMP(2);
                const unsigned f = wait_flag(myws + WS_XFLAG + (par * 8 + src_rank) * 512, (lane & 3) * 128, tmo, call, lane);
                const unsigned t = __hip_atomic_load(tmo, __ATOMIC_RELAXED, __HIP_MEMORY_SCOPE_AGENT);
                if (f != 0u || t == call) *(LAS unsigned*)(L + LDS_MISC + 4) = 1u;
                STAMP(3);
            }
            __syncthreads();
            lost = __builtin_amdgcn_readfirstlane(*(LAS unsigned*)(L + LDS_MISC + 4)) != 0;
            xa = lost ? px : myws + WS_XSTAGE + (size_t)(par * 8 + src_rank) * BM * KK;
            orow = src_rank * MS + XROW0;
        }
    }

    v16f acc[2];
    float scale;
    unsigned char* dws = nullptr;

    if (wid >= 4) {
        if (early) {
            c.lead = -1;
            c.xa = xa;
            c.wb = pw + wcol;
            load_x<0>(c);
            load_range<0, issued(0)>(c);
        }
        dws = p.ws[g];
        asm volatile("" : "+s"(dws));
        loader_loop<0>(c);
        scale = __builtin_bit_cast(float, __builtin_amdgcn_readfirstlane(*(LAS int*)(L + LDS_MISC + 12)));
    } else {
        unsigned ticket = 0;
        if (wid == 0 && !early) ticket = draw_ticket(myws + WS_CTR + b * 128);
        scale = __builtin_bit_cast(float, __builtin_amdgcn_readfirstlane(lost ? 0x7fc00000 : __builtin_bit_cast(int, p.sx[0] * p.sw[0])));
        dws = p.ws[g];
        asm volatile("" : "+s"(scale), "+s"(dws));
        if (tid == 0) *(LAS float*)(L + LDS_MISC + 12) = scale;
        acc[0] = acc[1] = (v16f)0.f;
        const int r = lane & 31, h = lane >> 5;
        unsigned ao[2][2];
#pragma unroll
        for (int ks = 0; ks < 2; ++ks)
#pragma unroll
            for (int e = 0; e < 2; ++e) ao[ks][e] = A_RING + r * 128 + (((4 * ks + 2 * h + e) ^ ((r >> 1) & 7)) << 4);
        const int q = (lane & 15) >> 1, pp = lane & 1, g1 = (lane >> 4) & 1;
        const unsigned bo = (32 * h + q) * 128 + (((2 * wid + g1) ^ (q & 6)) << 4) + pp * 8;

        auto rd = [&](Frag& f, int slot, int aslot, int ks) {
            const unsigned char LAS* base = L + slot * B_BYTES;
            const unsigned char LAS* abase = L + aslot * A_BYTES;
#pragma unroll
            for (int t = 0; t < 4; ++t)
                f.b[ks][t] = __builtin_amdgcn_ds_read_tr8_b64_v2i32((LAS v2i*)(base + bo + (64 * ks + 8 * t) * 128));
#pragma unroll
            for (int mt = 0; mt < 2; ++mt)
#pragma unroll
                for (int e = 0; e < 2; ++e) f.a[ks][mt][e] = *(const LAS v4i*)(abase + ao[ks][e] + mt * 4096);
        };
        auto mm = [&](const Frag& f, int ks, int mt) {
            acc[mt] = __builtin_amdgcn_mfma_scale_f32_32x32x64_f8f6f4(cat8b(f.b[ks][0], f.b[ks][1], f.b[ks][2], f.b[ks][3]),
                                                                      cat8(f.a[ks][mt][0], f.a[ks][mt][1]), acc[mt], 0, 0, 0, 0, 0, 0);
        };
        auto step = [&](Frag& fn, const Frag& fo, int slot, int aslot) {
            BAR(); SCHED();
            mm(fo, 0, 0); SCHED();
            rd(fn, slot, aslot, 0); SCHED();
            mm(fo, 0, 1); SCHED();
            rd(fn, slot, aslot, 1); SCHED();
            mm(fo, 1, 0); mm(fo, 1, 1); SCHED();
            WAIT_LGKM0(); SCHED();
        };
        Frag f0, f1;
        BAR();
        rd(f0, 0, 0, 0);
        rd(f0, 0, 0, 1);
        WAIT_LGKM0();
        SCHED();
        int slot = 1, aslot = 1;
#pragma unroll 1
        for (int i = 1; i < NSTAGE - 1; i += 2) {
            step(f1, f0, slot, aslot);
            slot = slot + 1 == DEPTH ? 0 : slot + 1;
            aslot = aslot + 1 == ADEPTH ? 0 : aslot + 1;
            step(f0, f1, slot, aslot);
            slot = slot + 1 == DEPTH ? 0 : slot + 1;
            aslot = aslot + 1 == ADEPTH ? 0 : aslot + 1;
        }
        step(f1, f0, slot, aslot);
        mm(f1, 0, 0); mm(f1, 0, 1); mm(f1, 1, 0); mm(f1, 1, 1);
        if (wid == 0 && !early) {
            asm volatile("s_waitcnt vmcnt(0)" : "+v"(ticket) :: "memory");
            if (lane == 0) *(LAS v4i*)(L + LDS_MISC) = (v4i){(int)(ticket + 1u), 0, 0, __builtin_bit_cast(int, scale)};
        }
        if (!local) {
#pragma unroll
            for (int gq = 0; gq < 4; ++gq)
                *(LAS v4f*)(L + LDS_HAND + wid * 4096 + gq * 1024 + lane * 16) =
                    (v4f){acc[1][4 * gq], acc[1][4 * gq + 1], acc[1][4 * gq + 2], acc[1][4 * gq + 3]};
        }
        WAIT_LGKM0();
    }
    BAR();
    if (wid == 0) STAMP(1);

    if (local) {
        if (wid < 4) {
            const int r = lane & 31, h = lane >> 5;
#pragma unroll
            for (int mt = 0; mt < 2; ++mt)
#pragma unroll
                for (int gq = 0; gq < 4; ++gq) {
                    v4f v;
#pragma unroll
                    for (int e = 0; e < 4; ++e) { const float y = acc[mt][4 * gq + e] * scale; v[e] = y <= 0.f ? 0.f : y; }
                    const int row = mt * 32 + r, ch = wid * 8 + 2 * gq + h;
                    *(LAS v4f*)(L + row * 512 + ((ch ^ (row & 7)) << 4)) = v;
                }
        }
        __syncthreads();
        float* ob = p.out + (size_t)orow * NS + ocol;
#pragma unroll
        for (int u = 0; u < 4; ++u) {
            const int row = (wid * 4 + u) * 2 + (lane >> 5), ch = lane & 31;
            store_out(ob + row * NS + ch * 4, *(LAS u32x4*)(L + row * 512 + ((ch ^ (row & 7)) << 4)));
        }
    } else {
        const unsigned lo = lane * 16;
        const unsigned callv = *(LAS unsigned*)(L + LDS_MISC);
        if (wid >= 4) {
            const LAS v4f* h = (const LAS v4f*)(L + LDS_HAND + (wid - 4) * 4096 + lo);
            acc[0] = __builtin_shufflevector(__builtin_shufflevector(h[0], h[64], 0, 1, 2, 3, 4, 5, 6, 7),
                                             __builtin_shufflevector(h[128], h[192], 0, 1, 2, 3, 4, 5, 6, 7),
                                             0, 1, 2, 3, 4, 5, 6, 7, 8, 9, 10, 11, 12, 13, 14, 15);
        }
        unsigned total = 0;
        bool dense = (((tm >= LEAD0 ? (tm >= LEAD0 + PAUSE_LEAD / 8 ? DENSE_B : DENSE_L) : DENSE_F) >> wid) & 1) &&
                     __builtin_bit_cast(unsigned, scale) - 0x0d800000u < 0x71800000u - 0x0d800000u;
        if (!dense) __builtin_amdgcn_s_setprio(PKPRIO);
        if (dense) {
            float y[16];
            unsigned long long nz[16];
            unsigned mask = 0, m1 = 0, m2 = 0, inexact = 0;
            float top = 0.f;
#pragma unroll
            for (int q = 3; q >= 0; --q) {
                unsigned long long s1[4], s2[4];
#pragma unroll
                for (int e = 0; e < 4; ++e) {
                    const float a = acc[0][4 * q + e], am = __builtin_elementwise_minimum(a, 32.f), u = am + 32.f;
                    nz[4 * q + e] = __builtin_amdgcn_fcmpf(a, 0.f, 2  );
                    s1[e] = __builtin_amdgcn_fcmpf(a, 32.f, 3  );
                    s2[e] = __builtin_amdgcn_fcmpf(a, 128.f, 3);
                    inexact = __builtin_amdgcn_bitop3_b32(inexact, __builtin_bit_cast(unsigned, u - 32.f), __builtin_bit_cast(unsigned, am), 0xf6);
                    y[4 * q + e] = __builtin_amdgcn_inverse_ballot_w64(s1[e]) ? a : u;
                    top = __builtin_elementwise_maximum(top, a);
                }
#define AC(d, m) "v_addc_co_u32_e64 %" #d ", vcc, %" #d ", %" #d ", %" #m "\n\t"
                asm volatile("s_nop 1\n\t" AC(0, 6) AC(1, 10) AC(2, 14) AC(0, 5) AC(1, 9) AC(2, 13) AC(0, 4) AC(1, 8) AC(2, 12) AC(0, 3) AC(1, 7) AC(2, 11)
                             : "+v"(mask), "+v"(m1), "+v"(m2)
                             : "s"(nz[4 * q]), "s"(nz[4 * q + 1]), "s"(nz[4 * q + 2]), "s"(nz[4 * q + 3]),
                               "s"(s1[0]), "s"(s1[1]), "s"(s1[2]), "s"(s1[3]), "s"(s2[0]), "s"(s2[1]), "s"(s2[2]), "s"(s2[3])
                             : "vcc");
            }
            dense = __builtin_amdgcn_ballot_w64((inexact << 1) != 0u || !(top < 512.f)) == 0ull;
            if (dense) {
                *(LAS unsigned*)(L + wid * PK_SLOT + lane * 4) = (mask ^ m1 ^ m2) | (m1 << 16);
                total = compact(lds0 + LDS_RECV + wid * PK_SLOT, mask, y, nz);
#pragma unroll 1
                for (unsigned o = 0; o < total; o += 256) {
                    const u32x4 v = *(LAS u32x4*)(L + LDS_RECV + wid * PK_SLOT + o * 4 + lo);
                    LAS unsigned* d = (LAS unsigned*)(L + wid * PK_SLOT + 256 + o * 3 + lane * 12);
                    d[0] = __builtin_amdgcn_perm(v[1], v[0], 0x04020100u);
                    d[1] = __builtin_amdgcn_perm(v[2], v[1], 0x05040201u);
                    d[2] = __builtin_amdgcn_perm(v[3], v[2], 0x06050402u);
                }
            }
        }
        if (!dense) {
            float y[16];
            unsigned long long nz[16];
#pragma unroll
            for (int i = 0; i < 16; ++i) {
                y[i] = acc[0][i] * scale;
                nz[i] = __builtin_amdgcn_fcmpf(y[i], 0.f, 10  );
            }
            unsigned mask = 0;
#pragma unroll
            for (int q = 1; q >= 0; --q)
                asm volatile("s_nop 1\n\t" AC(0, 8) AC(0, 7) AC(0, 6) AC(0, 5) AC(0, 4) AC(0, 3) AC(0, 2) AC(0, 1)
                             : "+v"(mask)
                             : "s"(nz[8 * q]), "s"(nz[8 * q + 1]), "s"(nz[8 * q + 2]), "s"(nz[8 * q + 3]),
                               "s"(nz[8 * q + 4]), "s"(nz[8 * q + 5]), "s"(nz[8 * q + 6]), "s"(nz[8 * q + 7])
                             : "vcc");
            *(LAS unsigned short*)(L + wid * PK_SLOT + lane * 2) = (unsigned short)mask;
            total = compact(lds0 + wid * PK_SLOT + 128, mask, y, nz);
        }
        u32x4 c[3];
#pragma unroll
        for (int u = 0; u < 3; ++u) c[u] = *(LAS u32x4*)(L + wid * PK_SLOT + u * 1024 + lo);
        const unsigned call = __builtin_amdgcn_readfirstlane(callv);
        const unsigned par = call & 1u;
        const unsigned bytes = dense ? 256 + 3 * total : 128 + 4 * total;
        const unsigned sidx = ((par * 8 + rank) * 32 + j) * 8 + wid;
        unsigned char* dslot = dws + WS_STAGE + (size_t)sidx * PK_SLOT;
        const int turn = wid - (tm >= LEAD0 + PAUSE_LEAD / 8 ? WIN_B : WIN);
        if (turn > 0)
            while ((int)__builtin_amdgcn_readfirstlane(__hip_atomic_load((LAS unsigned*)(L + LDS_MISC + 8), __ATOMIC_RELAXED,
                                                                         __HIP_MEMORY_SCOPE_WORKGROUP)) < turn)
                __builtin_amdgcn_s_sleep(1);
        if (wid == 0) STAMP(2);
#pragma unroll
        for (int u = 0; u < 3; ++u)
            if (u * 1024 + lo < bytes)
                asm volatile("s_nop 4\n\tglobal_store_dwordx4 %0, %1, %2 sc0 sc1\n\ts_nop 1" :: "v"(lo + u * 1024), "v"(c[u]), "s"(dslot) : "memory");
#pragma unroll 1
        for (unsigned o = 3 * 1024; o < bytes; o += 1024) {
            const u32x4 v = *(LAS u32x4*)(L + wid * PK_SLOT + o + lo);
            if (o + lo < bytes)
                asm volatile("s_nop 4\n\tglobal_store_dwordx4 %0, %1, %2 sc0 sc1\n\ts_nop 1" :: "v"(lo + o), "v"(v), "s"(dslot) : "memory");
        }
        WAIT_VM(0);
        if (lane == 0) {
            put_flag(dws + WS_FLAG + (size_t)sidx * 128, call, total | (dense ? 0x10000u : 0u));
            __hip_atomic_fetch_add((LAS unsigned*)(L + LDS_MISC + 8), 1u, __ATOMIC_RELAXED, __HIP_MEMORY_SCOPE_WORKGROUP);
        }
        if (wid == 7) STAMP(3);
        const unsigned ridx = ((par * 8 + g) * 32 + j) * 8 + wid;
        const unsigned rword = wait_flag(myws + WS_FLAG + (size_t)ridx * 128, 0u, tmo, call, lane);
        if (wid == 7) STAMP(4);
        const unsigned rtotal = __builtin_amdgcn_readfirstlane(rword) & ~0x10000u;
        const bool rdense = __builtin_amdgcn_readfirstlane(rword) & 0x10000u;
        u32x4 o[4];
        bool bad = true;
        if (rtotal <= 1024u) {
            const unsigned rbytes = rdense ? 256 + 3 * rtotal : 128 + 4 * rtotal;
            const unsigned char* sslot = myws + WS_STAGE + (size_t)ridx * PK_SLOT;
            const unsigned rdst = __builtin_amdgcn_readfirstlane(lds0 + LDS_RECV + wid * PK_SLOT);
            asm volatile("s_mov_b32 m0, %2\n\ts_nop 3\n\tglobal_load_lds_dwordx4 %0, %1 sc0 sc1\n\t"
                         "s_cmp_le_u32 %3, 1024\n\ts_cbranch_scc1 1f\n\tglobal_load_lds_dwordx4 %0, %1 offset:1024 sc0 sc1\n\t"
                         "s_cmp_le_u32 %3, 2048\n\ts_cbranch_scc1 1f\n\tglobal_load_lds_dwordx4 %0, %1 offset:2048 sc0 sc1\n\t"
                         "s_cmp_le_u32 %3, 3072\n\ts_cbranch_scc1 1f\n\tglobal_load_lds_dwordx4 %0, %1 offset:3072 sc0 sc1\n\t"
                         "s_cmp_le_u32 %3, 4096\n\ts_cbranch_scc1 1f\n\ts_mov_b32 m0, %5\n\ts_nop 0\n\tglobal_load_lds_dwordx4 %4, %1 sc0 sc1\n"
                         "1:"
                         :: "v"(lo), "s"(sslot), "s"(rdst), "s"(rbytes), "v"(lo + 4096), "s"(rdst + 4096) : "memory", "scc");
            unsigned t;
            asm volatile("s_nop 4\n\tglobal_load_dword %0, %1, %2 sc1\n\ts_waitcnt vmcnt(0)" : "=&v"(t) : "v"(0u), "s"(tmo) : "memory");
            if (wid == 7) STAMP(5);
            bad = __builtin_amdgcn_readfirstlane(t) == call;
            const unsigned bp = 4 * ((lane & 7) >> 1);
            LAS v2i* tab = (LAS v2i*)(L + LDS_TAB + wid * 512);
            if (!rdense) {
                const unsigned rm = *(LAS unsigned short*)(L + LDS_RECV + wid * PK_SLOT + lane * 2);
                const unsigned rc = __builtin_popcount(rm);
                tab[lane] = (v2i){(int)rm, (int)(LDS_RECV + wid * PK_SLOT + 128 + (wave_scan(rc) - rc) * 4)};
#pragma unroll
                for (int u = 0; u < 4; ++u) {
                    const v2i tb = tab[8 * u + (lane >> 3) + 32 * (lane & 1)];
                    const unsigned m = (unsigned)tb[0];
                    int addr = tb[1] + 4 * __builtin_popcount(m & ((1u << bp) - 1u));
                    const int ms = (int)(m >> bp);
#pragma unroll
                    for (int e = 0; e < 4; ++e) {
                        const int sel = __builtin_amdgcn_sbfe(ms, e, 1);
                        o[u][e] = *(LAS unsigned*)(L + addr) & (unsigned)sel;
                        if (e < 3) addr = step4(addr, sel);
                    }
                }
            } else {
                const unsigned rm = *(LAS unsigned*)(L + LDS_RECV + wid * PK_SLOT + lane * 4);
#pragma unroll 1
                for (unsigned q = 0; q < rtotal; q += 256) {
                    const LAS unsigned* d = (const LAS unsigned*)(L + LDS_RECV + wid * PK_SLOT + 256 + q * 3 + lane * 12);
                    const unsigned d0 = d[0], d1 = d[1], d2 = d[2];
                    *(LAS u32x4*)(L + wid * PK_SLOT + q * 4 + lo) =
                        (u32x4){(d0 & 0xffffffu) | 0x42000000u, __builtin_amdgcn_perm(d1, d0, 0x0c050403u) | 0x42000000u,
                                __builtin_amdgcn_perm(d2, d1, 0x0c040302u) | 0x42000000u, __builtin_amdgcn_alignbyte(0x42u, d2, 1)};
                }
                const unsigned rc = __builtin_popcount((rm | (rm >> 16)) & 0xffffu);
                tab[lane] = (v2i){(int)rm, (int)(wid * PK_SLOT + (wave_scan(rc) - rc) * 4)};
                const unsigned sb = __builtin_bit_cast(unsigned, scale), nb = __builtin_bit_cast(unsigned, -32.f * scale);
#pragma unroll
                for (int u = 0; u < 4; ++u) {
                    const v2i tb = tab[8 * u + (lane >> 3) + 32 * (lane & 1)];
                    const unsigned m = (unsigned)tb[0];
                    int addr = tb[1] + 4 * __builtin_popcount((m | (m >> 16)) & ((1u << bp) - 1u));
                    const int ms = (int)(m >> bp);
#pragma unroll
                    for (int e = 0; e < 4; ++e) {
                        const unsigned b0 = __builtin_amdgcn_sbfe(ms, e, 1), b1 = __builtin_amdgcn_sbfe(ms, 16 + e, 1);
                        const unsigned code = *(LAS unsigned*)(L + addr) | __builtin_amdgcn_bitop3_b32(b0, b1, 0x01000000u, 0x80);
                        o[u][e] = __builtin_bit_cast(unsigned, __builtin_fmaf(__builtin_bit_cast(float, code),
                                                                              __builtin_bit_cast(float, __builtin_amdgcn_bitop3_b32(b0, b1, sb, 0xa8)),
                                                                              __builtin_bit_cast(float, __builtin_amdgcn_bitop3_b32(b0, b1, nb, 0x20))));
                        if (e < 3) addr = step4(addr, (int)(b0 | b1));
                    }
                }
            }
        }
        float* ob = p.out + (size_t)(g * MS + tm * BM + (wid >> 2) * 32 + (lane >> 3)) * NS + tn * BN + (wid & 3) * 32 + (lane & 7) * 4;
#pragma unroll
        for (int u = 0; u < 4; ++u) {
            if (bad) o[u] = (u32x4){0x7fc00000u, 0x7fc00000u, 0x7fc00000u, 0x7fc00000u};
            store_out(ob + u * 8 * NS, o[u]);
        }
    }
}

extern "C" void kernel_launch(int rank, int nranks, ncclComm_t comm, hipStream_t stream,
                              void* const* d_in, const int* in_sizes, int n_in,
                              void* d_out, int out_size, const int* in_dtypes, int out_dtype,
                              void* const* d_out_all, void* const* d_ws_all, size_t ws_bytes) {
    if (nranks != NR || n_in != 4 || in_sizes[0] != MS * KK || in_sizes[1] != KK * NN || out_size != NR * MS * NS ||
        (unsigned)rank >= (unsigned)NR || ws_bytes < WS_END) {
        fprintf(stderr, "unexpected problem shape\n");
        return;
    }
    Params p;
    p.x = (const unsigned char*)d_in[0];
    p.w = (const unsigned char*)d_in[1];
    p.sx = (const float*)d_in[2];
    p.sw = (const float*)d_in[3];
    p.out = (float*)d_out;
    p.myws = (unsigned char*)d_ws_all[rank];
    for (int i = 0; i < NR; ++i) p.ws[i] = (unsigned char*)d_ws_all[i];
    p.rank = rank;
    p.pad = 0;
    gemm_a2a<<<GRID, 512, 0, stream>>>(p);
    HIP_CHECK(hipGetLastError());
}
```

```cpp
#include <hip/hip_runtime.h>
#include <rccl/rccl.h>
#include <cstdio>
#include <cstddef>

#define HIP_CHECK(expr) do { hipError_t _e = (expr); if (_e != hipSuccess) { \
    fprintf(stderr, "HIP error %s at %s:%d\n", hipGetErrorString(_e), __FILE__, __LINE__); \
    return; } } while (0)

typedef int v8i __attribute__((ext_vector_type(8)));
typedef int v4i __attribute__((ext_vector_type(4)));
typedef int v2i __attribute__((ext_vector_type(2)));
typedef float v16f __attribute__((ext_vector_type(16)));
typedef float v4f __attribute__((ext_vector_type(4)));
typedef unsigned u32x4 __attribute__((ext_vector_type(4)));
#define LAS __attribute__((address_space(3)))

constexpr int NR = 8;
constexpr int MS = 512;
constexpr int KK = 4096;
constexpr int NN = 4096;
constexpr int NS = 512;
constexpr int BM = 64, BN = 128, BK = 128;
constexpr int NSTAGE = KK / BK;
constexpr int XROW0 = MS - BM;
constexpr int GRID = 256;
constexpr unsigned DEADLINE_TICKS = 50000000u;

constexpr int DEPTH_ = 4;
constexpr int LEAD0 = 3;
constexpr int TWARM = 500;
constexpr int PACE0 = 5;
constexpr int T_PACE0 = 400;
constexpr int T_PACE1 = 850;
constexpr int TAU_A = 26;
constexpr int TAU_B = 32;
constexpr int WIN = 2;
constexpr int DENSE_F = 0xfc;
constexpr int DENSE_L = 0x7f;

constexpr int DEPTH = DEPTH_;
constexpr int ADEPTH = DEPTH + 1;
constexpr int A_BYTES = BM * BK;
constexpr int B_BYTES = BK * BN;
constexpr int A_RING = DEPTH * B_BYTES;
constexpr int RING_BYTES = DEPTH * B_BYTES + ADEPTH * A_BYTES;
constexpr int PK_SLOT = 5120;
constexpr int LDS_RECV = 8 * PK_SLOT;
constexpr int LDS_TAB = 16 * PK_SLOT;
constexpr int LDS_HAND = RING_BYTES;
constexpr int LDS_MISC = LDS_HAND + 4 * 4096;
constexpr int LDS_BYTES = LDS_MISC + 16;
static_assert(LDS_TAB + 8 * 512 <= RING_BYTES && LDS_BYTES > 80 * 1024 && LDS_BYTES <= 160 * 1024, "LDS layout");

constexpr size_t WS_CTR = 0;
constexpr size_t WS_TMO = 32768;
constexpr size_t WS_XFLAG = 36864;
constexpr size_t WS_FLAG = 131072;
constexpr size_t WS_XSTAGE = WS_FLAG + 2 * 2048 * 128;
constexpr size_t WS_STAGE = WS_XSTAGE + (size_t)16 * BM * KK;
constexpr size_t WS_END = WS_STAGE + (size_t)2 * 2048 * PK_SLOT;

struct Params {
    const unsigned char* x;
    const unsigned char* w;
    unsigned char* myws;
    int rank;
    int pad;
    const float* sx;
    const float* sw;
    float* out;
    unsigned char* ws[NR];
};
static_assert(offsetof(Params, x) == 0 && offsetof(Params, w) == 8 && offsetof(Params, rank) == 24, "read as words in the kernel");

#define WAIT_VM(n) asm volatile("s_waitcnt vmcnt(%0)" :: "i"(n) : "memory")
#define WAIT_LGKM0() asm volatile("s_waitcnt lgkmcnt(0)" ::: "memory")
#define BAR() __builtin_amdgcn_s_barrier()
#define SCHED() __builtin_amdgcn_sched_barrier(0)

template <int S>
__device__ __forceinline__ int korder(int lead, int tn) {
    const int rs = (S + tn) & 3;
    const int kl = S < 8 ? lead + S : (lead + (S & ~3) + rs) & (NSTAGE - 1);
    return lead < 0 ? (S >> 2) + 8 * rs : kl;
}

struct LoadCtx {
    unsigned va[2], vb[4];
    const unsigned char* xa;
    const unsigned char* wb;
    unsigned ldsA, ldsB;
    int lead, tn;
    unsigned t0, tau;
};
__device__ __forceinline__ void load_w_at(const LoadCtx& c, const unsigned char* pb, int slot) {
    asm volatile("s_mov_b32 m0, %5\n\ts_nop 3\n\t"
                 "global_load_lds_dwordx4 %0, %4\n\tglobal_load_lds_dwordx4 %1, %4 offset:1024\n\t"
                 "global_load_lds_dwordx4 %2, %4 offset:2048\n\tglobal_load_lds_dwordx4 %3, %4 offset:3072"
                 :: "v"(c.vb[0]), "v"(c.vb[1]), "v"(c.vb[2]), "v"(c.vb[3]), "s"(pb), "s"(c.ldsB + slot * B_BYTES) : "memory");
}
__device__ __forceinline__ void load_x_at(const LoadCtx& c, const unsigned char* pa, int slot) {
    asm volatile("s_mov_b32 m0, %3\n\ts_nop 3\n\t"
                 "global_load_lds_dwordx4 %0, %2 sc0 sc1\n\tglobal_load_lds_dwordx4 %1, %2 offset:1024 sc0 sc1"
                 :: "v"(c.va[0]), "v"(c.va[1]), "s"(pa), "s"(c.ldsA + slot * A_BYTES) : "memory");
}
template <int S>
__device__ __forceinline__ void load_w(const LoadCtx& c) { load_w_at(c, c.wb + (unsigned)korder<S>(c.lead, c.tn) * (BK * NN), S % DEPTH); }
template <int S>
__device__ __forceinline__ void load_x(const LoadCtx& c) { load_x_at(c, c.xa + (unsigned)korder<S>(c.lead, c.tn) * BK, S % ADEPTH); }
constexpr int mini(int a, int b) { return a < b ? a : b; }
constexpr int issued(int I) { return mini(NSTAGE, mini(2 + 2 * I, I + DEPTH - 1)); }
constexpr int younger(int I) {
    return 4 * (issued(I) - 1 - I) + 2 * (mini(issued(I), NSTAGE - 1) - I);
}
template <int S, int E>
__device__ __forceinline__ void load_range(const LoadCtx& c) {
    if constexpr (S < E) {
        load_w<S>(c);
        if constexpr (S + 1 < NSTAGE) load_x<S + 1>(c);
        load_range<S + 1, E>(c);
    }
}
template <int I, int END>
__device__ __forceinline__ void loader_loop(const LoadCtx& c) {
    WAIT_VM(younger(I));
    BAR();
    if constexpr (I >= PACE0 && T_PACE0 + (I - PACE0) * TAU_A <= T_PACE1) {
        const unsigned due = T_PACE0 + (I - PACE0) * c.tau;
        if (c.lead >= 0 && due <= T_PACE1)
            while ((unsigned)__builtin_amdgcn_s_memrealtime() - c.t0 < due) __builtin_amdgcn_s_sleep(1);
    }
    if constexpr (I + 1 < NSTAGE) load_range<issued(I), issued(I + 1)>(c);
    if constexpr (I + 1 < END) loader_loop<I + 1, END>(c);
}

__device__ __forceinline__ unsigned wave_scan(unsigned v) {
    v += __builtin_amdgcn_update_dpp(0u, v, 0x111, 0xf, 0xf, true);
    v += __builtin_amdgcn_update_dpp(0u, v, 0x112, 0xf, 0xf, true);
    v += __builtin_amdgcn_update_dpp(0u, v, 0x114, 0xf, 0xf, true);
    v += __builtin_amdgcn_update_dpp(0u, v, 0x118, 0xf, 0xf, true);
    v += __builtin_amdgcn_update_dpp(0u, v, 0x142, 0xa, 0xf, false);
    v += __builtin_amdgcn_update_dpp(0u, v, 0x143, 0xc, 0xf, false);
    return v;
}

__device__ __forceinline__ unsigned draw_call(unsigned char* ctr) {
    unsigned t;
    asm volatile("s_mov_b64 exec, 1\n\ts_nop 3\n\tglobal_atomic_add %0, %1, %2, %3 sc0\n\ts_mov_b64 exec, -1\n\ts_waitcnt vmcnt(0)"
                 : "=&v"(t) : "v"(0u), "v"(1u), "s"(ctr) : "memory");
    return t + 1u;
}

__device__ __forceinline__ void touch(const unsigned char* base, unsigned voff, unsigned lds_dst) {
    asm volatile("s_mov_b32 m0, %2\n\ts_nop 3\n\tglobal_load_lds_dword %0, %1" :: "v"(voff), "s"(base), "s"(lds_dst) : "memory");
}

__device__ __forceinline__ void put_flag(unsigned char* flag, unsigned call, unsigned low) {
    typedef __attribute__((address_space(1))) unsigned long long gu64;
    __hip_atomic_store((gu64*)flag, ((unsigned long long)call << 32) | low, __ATOMIC_RELAXED, __HIP_MEMORY_SCOPE_SYSTEM);
}

__device__ __forceinline__ unsigned wait_flag(const unsigned char* flag, unsigned voff, unsigned* tmo, unsigned call, int lane) {
    const unsigned t0 = (unsigned)__builtin_amdgcn_s_memrealtime();
    unsigned low;
    bool seen;
    for (;;) {
        v2i f;
        unsigned t;
        asm volatile("s_nop 4\n\tglobal_load_dwordx2 %0, %2, %3 sc0 sc1\n\tglobal_load_dword %1, %4, %5 sc1\n\ts_waitcnt vmcnt(0)"
                     : "=&v"(f), "=&v"(t) : "v"(voff), "s"(flag), "v"(0u), "s"(tmo) : "memory");
        low = __builtin_amdgcn_readfirstlane(f[0]);
        seen = __builtin_amdgcn_ballot_w64((unsigned)f[1] < call) == 0ull;
        if (seen || __builtin_amdgcn_readfirstlane(t) == call || (unsigned)__builtin_amdgcn_s_memrealtime() - t0 > DEADLINE_TICKS) break;
        __builtin_amdgcn_s_sleep(2);
    }
    if (!seen && lane == 0) __hip_atomic_store(tmo, call, __ATOMIC_RELAXED, __HIP_MEMORY_SCOPE_AGENT);
    return seen ? low : 0xffffffffu;
}

__device__ __forceinline__ void store_out(float* ptr, u32x4 v) {
    asm volatile("global_store_dwordx4 %0, %1, off sc0 sc1\n\ts_nop 1" :: "v"(ptr), "v"(v) : "memory");
}

__device__ __forceinline__ unsigned compact(unsigned base, unsigned mask, const float (&y)[16], const unsigned long long (&nz)[16]) {
    const unsigned cnt = __builtin_popcount(mask);
    const unsigned incl = wave_scan(cnt);
    unsigned ptr = base + (incl - cnt) * 4;
#define PK1(m, v) "s_mov_b64 exec, %" #m "\n\tds_write_b32 %0, %" #v "\n\tv_add_u32 %0, 4, %0\n\t"
#pragma unroll
    for (int q = 0; q < 2; ++q)
        asm volatile(PK1(1, 9) PK1(2, 10) PK1(3, 11) PK1(4, 12) PK1(5, 13) PK1(6, 14) PK1(7, 15) PK1(8, 16)
                     "s_mov_b64 exec, -1"
                     : "+v"(ptr)
                     : "s"(nz[8 * q]), "s"(nz[8 * q + 1]), "s"(nz[8 * q + 2]), "s"(nz[8 * q + 3]),
                       "s"(nz[8 * q + 4]), "s"(nz[8 * q + 5]), "s"(nz[8 * q + 6]), "s"(nz[8 * q + 7]),
                       "v"(y[8 * q]), "v"(y[8 * q + 1]), "v"(y[8 * q + 2]), "v"(y[8 * q + 3]),
                       "v"(y[8 * q + 4]), "v"(y[8 * q + 5]), "v"(y[8 * q + 6]), "v"(y[8 * q + 7])
                     : "memory");
    return __builtin_amdgcn_readlane(incl, 63);
}

__device__ __forceinline__ int step4(int addr, int sel) {
    int r;
    asm("v_mad_i32_i24 %0, %1, -4, %2" : "=v"(r) : "v"(sel), "v"(addr));
    return r;
}

struct Frag { v4i a[2][2][2]; v2i b[2][4]; };

__device__ __forceinline__ v8i cat8(v4i lo, v4i hi) { return __builtin_shufflevector(lo, hi, 0, 1, 2, 3, 4, 5, 6, 7); }
__device__ __forceinline__ v8i cat8b(v2i a, v2i b, v2i c, v2i d) {
    return cat8(__builtin_shufflevector(a, b, 0, 1, 2, 3), __builtin_shufflevector(c, d, 0, 1, 2, 3));
}

__global__ void __launch_bounds__(512, 2) gemm_a2a(Params p) {
    __shared__ __attribute__((aligned(1024))) unsigned char lds[LDS_BYTES];
    v8i ka;
    asm volatile("s_load_dwordx8 %0, %1, 0x0" : "=s"(ka) : "s"(__builtin_amdgcn_kernarg_segment_ptr()));
    SCHED();
    LAS unsigned char* L = (LAS unsigned char*)lds;
    const unsigned lds0 = (unsigned)(size_t)L;
    const int tid = threadIdx.x, lane = tid & 63;
    const int wid = __builtin_amdgcn_readfirstlane(tid >> 6);
    const int b = blockIdx.x;
    const int g = b & 7, j = b >> 3, tm = j >> 2, tn = j & 3;
    LoadCtx c;
    {
        const int lw = wid & 3;
#pragma unroll
        for (int i = 0; i < 2; ++i) {
            const int row = (lw * 2 + i) * 8 + (lane >> 3);
            c.va[i] = row * KK + (((lane & 7) ^ ((row >> 1) & 7)) << 4) - i * 1024;
        }
#pragma unroll
        for (int i = 0; i < 4; ++i) {
            const int row = (lw * 4 + i) * 8 + (lane >> 3);
            c.vb[i] = row * NN + (((lane & 7) ^ (row & 6)) << 4) - i * 1024;
        }
        c.ldsA = lds0 + A_RING + lw * 2048;
        c.ldsB = lds0 + lw * 4096;
        c.tn = tn;
    }
    c.lead = tm >= LEAD0 ? 8 * (tm - LEAD0) : -1;
    int wcol = g * NS + tn * BN;
    unsigned xo = tm * BM * KK;
    unsigned x0 = xo + korder<0>(c.lead, tn) * BK, w0 = wcol + korder<0>(c.lead, tn) * BK * NN;
    int skip = tm == 7 || wid < 4 ? -1 : g;
    asm volatile("" : "+v"(c.va[0]), "+v"(c.va[1]), "+v"(c.vb[0]), "+v"(c.vb[1]), "+v"(c.vb[2]), "+v"(c.vb[3]));
    asm volatile("" : "+s"(x0), "+s"(w0), "+s"(skip), "+s"(c.ldsA), "+s"(c.ldsB));
    asm volatile("s_waitcnt lgkmcnt(0)" : "+s"(ka));
    const unsigned char* px = (const unsigned char*)(((unsigned long long)(unsigned)ka[1] << 32) | (unsigned)ka[0]);
    const unsigned char* pw = (const unsigned char*)(((unsigned long long)(unsigned)ka[3] << 32) | (unsigned)ka[2]);
    const int rank = ka[6];
    unsigned char* myws = p.myws;
    const unsigned char* xa = px + xo;
    if (__builtin_expect(skip >= 0 && skip != rank, 1)) {
        load_x_at(c, px + x0, 0);
        load_w_at(c, pw + w0, 0);
        c.xa = xa;
        c.wb = pw + wcol;
        load_x<1>(c);
        load_range<1, issued(0)>(c);
    }
    const bool early = g == rank || tm == 7;
    const unsigned t_begin = (unsigned)__builtin_amdgcn_s_memrealtime();
    c.t0 = t_begin;
    c.tau = tm >= LEAD0 + 2 ? TAU_B : TAU_A;
    unsigned* tmo = (unsigned*)(myws + WS_TMO);
    int orow = rank * MS + tm * BM, ocol = tn * BN;
    const bool local = early;
    bool lost = false;

    if (early) {
        u32x4 r[8];
        if (g != rank) {
            if (wid == 7) {
                touch((const unsigned char*)p.sx, 0u, lds0);
                touch((const unsigned char*)p.sw, 0u, lds0);
            }
            unsigned t;
            asm volatile("s_nop 4\n\t"
                         "global_load_dwordx4 %0, %9, %11\n\tglobal_load_dwordx4 %1, %9, %11 offset:1024\n\t"
                         "global_load_dwordx4 %2, %9, %11 offset:2048\n\tglobal_load_dwordx4 %3, %9, %11 offset:3072\n\t"
                         "global_load_dwordx4 %4, %10, %11\n\tglobal_load_dwordx4 %5, %10, %11 offset:1024\n\t"
                         "global_load_dwordx4 %6, %10, %11 offset:2048\n\tglobal_load_dwordx4 %7, %10, %11 offset:3072\n\t"
                         "s_cmp_lg_u32 %15, 0\n\ts_cbranch_scc1 1f\n\t"
                         "s_mov_b64 exec, 1\n\tglobal_atomic_add %8, %12, %13, %14 sc0\n\ts_mov_b64 exec, -1\n"
                         "1:\n\ts_waitcnt vmcnt(0)"
                         : "=&v"(r[0]), "=&v"(r[1]), "=&v"(r[2]), "=&v"(r[3]), "=&v"(r[4]), "=&v"(r[5]), "=&v"(r[6]), "=&v"(r[7]), "=&v"(t)
                         : "v"(lane * 16), "v"(lane * 16 + KK), "s"(xa + (tn * 16 + wid * 2) * KK),
                           "v"(0u), "v"(1u), "s"(myws + WS_CTR + b * 128), "s"(wid)
                         : "memory", "scc");
            if (tid == 0) *(LAS v4i*)(L + LDS_MISC) = (v4i){(int)(t + 1u), 0, 0, 0};
        } else {
            touch(px, (j * 512 + tid) * 128, lds0);
            while ((unsigned)__builtin_amdgcn_s_memrealtime() - t_begin < TWARM) __builtin_amdgcn_s_sleep(4);
            touch(pw + wcol, (tm * 512 + tid) * NN, lds0);
            if (wid == 0) {
                const unsigned t = draw_call(myws + WS_CTR + b * 128);
                if (lane == 0) *(LAS v4i*)(L + LDS_MISC) = (v4i){(int)t, 0, 0, 0};
            }
            WAIT_VM(0);
        }
        __syncthreads();
        const unsigned call = __builtin_amdgcn_readfirstlane(*(LAS unsigned*)(L + LDS_MISC));
        const unsigned par = call & 1u;
        if (g != rank) {
            unsigned char* pws = p.ws[g];
            unsigned char* dx = pws + WS_XSTAGE + (size_t)((par * 8 + rank) * BM + tn * 16 + wid * 2) * KK;
#pragma unroll
            for (int q = 0; q < 8; ++q)
                asm volatile("s_nop 4\n\tglobal_store_dwordx4 %0, %1, %2 sc0 sc1\n\ts_nop 1" :: "v"(lane * 16 + q * 1024), "v"(r[q]), "s"(dx) : "memory");
            WAIT_VM(0);
            __syncthreads();
            if (tid == 0) put_flag(pws + WS_XFLAG + ((par * 8 + rank) * 4 + tn) * 128, call, 0u);
            const int u = ((g - rank - 1) & 7) * 4 + tn;
            xa = px + (size_t)(u % 7) * BM * KK;
            ocol = (u / 7) * BN;
            wcol = rank * NS + ocol;
            orow = rank * MS + (u % 7) * BM;
        } else if (tm < 7) {
            const int src_rank = (rank + 1 + tm) & 7;
            if (wid == 0) {
                const unsigned f = wait_flag(myws + WS_XFLAG + (par * 8 + src_rank) * 512, (lane & 3) * 128, tmo, call, lane);
                const unsigned t = __hip_atomic_load(tmo, __ATOMIC_RELAXED, __HIP_MEMORY_SCOPE_AGENT);
                if (f != 0u || t == call) *(LAS unsigned*)(L + LDS_MISC + 4) = 1u;
            }
            __syncthreads();
            lost = __builtin_amdgcn_readfirstlane(*(LAS unsigned*)(L + LDS_MISC + 4)) != 0;
            xa = lost ? px : myws + WS_XSTAGE + (size_t)(par * 8 + src_rank) * BM * KK;
            orow = src_rank * MS + XROW0;
        }
    }

    v16f acc[2];
    unsigned char* dws = nullptr;

    if (wid >= 4) {
        if (early) {
            c.lead = -1;
            c.xa = xa;
            c.wb = pw + wcol;
            load_x<0>(c);
            load_range<0, issued(0)>(c);
        }
        dws = p.ws[g];
        asm volatile("" : "+s"(dws));
        loader_loop<0, NSTAGE - 2>(c);
        static_assert(issued(NSTAGE - 2) == NSTAGE && younger(NSTAGE - 2) == 6 && younger(NSTAGE - 1) == 0, "last two iterations");
        if (wid == 4) {
            unsigned t;
            float s0, s1;
            asm volatile("s_nop 4\n\tglobal_load_dword %1, %3, %6\n\tglobal_load_dword %2, %3, %7\n\t"
                         "s_mov_b64 exec, 1\n\tglobal_atomic_add %0, %3, %4, %5 sc0\n\ts_mov_b64 exec, -1\n\t"
                         "s_waitcnt vmcnt(9)\n\ts_barrier\n\ts_waitcnt vmcnt(3)\n\ts_barrier\n\ts_waitcnt vmcnt(0)"
                         : "=&v"(t), "=&v"(s0), "=&v"(s1)
                         : "v"(0u), "v"(early ? 0u : 1u), "s"(myws + WS_CTR + b * 128), "s"(p.sx), "s"(p.sw) : "memory");
            if (lane == 0)
                *(LAS v4i*)(L + LDS_MISC) = (v4i){(int)(t + 1u), 0, 0, lost ? 0x7fc00000 : __builtin_bit_cast(int, s0 * s1)};
            WAIT_LGKM0();
        } else {
            loader_loop<NSTAGE - 2, NSTAGE>(c);
        }
    } else {
        dws = p.ws[g];
        asm volatile("" : "+s"(dws));
        acc[0] = acc[1] = (v16f)0.f;
        const int r = lane & 31, h = lane >> 5;
        unsigned ao[2][2];
#pragma unroll
        for (int ks = 0; ks < 2; ++ks)
#pragma unroll
            for (int e = 0; e < 2; ++e) ao[ks][e] = A_RING + r * 128 + (((4 * ks + 2 * h + e) ^ ((r >> 1) & 7)) << 4);
        const int q = (lane & 15) >> 1, pp = lane & 1, g1 = (lane >> 4) & 1;
        const unsigned bo = (32 * h + q) * 128 + (((2 * wid + g1) ^ (q & 6)) << 4) + pp * 8;

        auto rd = [&](Frag& f, int slot, int aslot, int ks) {
            const unsigned char LAS* base = L + slot * B_BYTES;
            const unsigned char LAS* abase = L + aslot * A_BYTES;
#pragma unroll
            for (int t = 0; t < 4; ++t)
                f.b[ks][t] = __builtin_amdgcn_ds_read_tr8_b64_v2i32((LAS v2i*)(base + bo + (64 * ks + 8 * t) * 128));
#pragma unroll
            for (int mt = 0; mt < 2; ++mt)
#pragma unroll
                for (int e = 0; e < 2; ++e) f.a[ks][mt][e] = *(const LAS v4i*)(abase + ao[ks][e] + mt * 4096);
        };
        auto mm = [&](const Frag& f, int ks, int mt) {
            acc[mt] = __builtin_amdgcn_mfma_scale_f32_32x32x64_f8f6f4(cat8b(f.b[ks][0], f.b[ks][1], f.b[ks][2], f.b[ks][3]),
                                                                      cat8(f.a[ks][mt][0], f.a[ks][mt][1]), acc[mt], 0, 0, 0, 0, 0, 0);
        };
        auto step = [&](Frag& fn, const Frag& fo, int slot, int aslot) {
            BAR(); SCHED();
            mm(fo, 0, 0); SCHED();
            rd(fn, slot, aslot, 0); SCHED();
            mm(fo, 0, 1); SCHED();
            rd(fn, slot, aslot, 1); SCHED();
            mm(fo, 1, 0); mm(fo, 1, 1); SCHED();
            WAIT_LGKM0(); SCHED();
        };
        Frag f0, f1;
        BAR();
        rd(f0, 0, 0, 0);
        rd(f0, 0, 0, 1);
        WAIT_LGKM0();
        SCHED();
        int slot = 1, aslot = 1;
#pragma unroll 1
        for (int i = 1; i < NSTAGE - 1; i += 2) {
            step(f1, f0, slot, aslot);
            slot = slot + 1 == DEPTH ? 0 : slot + 1;
            aslot = aslot + 1 == ADEPTH ? 0 : aslot + 1;
            step(f0, f1, slot, aslot);
            slot = slot + 1 == DEPTH ? 0 : slot + 1;
            aslot = aslot + 1 == ADEPTH ? 0 : aslot + 1;
        }
        step(f1, f0, slot, aslot);
        mm(f1, 0, 0); mm(f1, 0, 1); mm(f1, 1, 0); mm(f1, 1, 1);
        if (!local) {
#pragma unroll
            for (int gq = 0; gq < 4; ++gq)
                *(LAS v4f*)(L + LDS_HAND + wid * 4096 + gq * 1024 + lane * 16) =
                    (v4f){acc[1][4 * gq], acc[1][4 * gq + 1], acc[1][4 * gq + 2], acc[1][4 * gq + 3]};
        }
        WAIT_LGKM0();
    }
    BAR();
    const v4i misc = *(LAS v4i*)(L + LDS_MISC);
    const float scale = __builtin_bit_cast(float, __builtin_amdgcn_readfirstlane(misc[3]));

    if (local) {
        if (wid < 4) {
            const int r = lane & 31, h = lane >> 5;
#pragma unroll
            for (int mt = 0; mt < 2; ++mt)
#pragma unroll
                for (int gq = 0; gq < 4; ++gq) {
                    v4f v;
#pragma unroll
                    for (int e = 0; e < 4; ++e) { const float y = acc[mt][4 * gq + e] * scale; v[e] = y <= 0.f ? 0.f : y; }
                    const int row = mt * 32 + r, ch = wid * 8 + 2 * gq + h;
                    *(LAS v4f*)(L + row * 512 + ((ch ^ (row & 7)) << 4)) = v;
                }
        }
        __syncthreads();
        float* ob = p.out + (size_t)orow * NS + ocol;
#pragma unroll
        for (int u = 0; u < 4; ++u) {
            const int row = (wid * 4 + u) * 2 + (lane >> 5), ch = lane & 31;
            store_out(ob + row * NS + ch * 4, *(LAS u32x4*)(L + row * 512 + ((ch ^ (row & 7)) << 4)));
        }
    } else {
        const unsigned lo = lane * 16;
        if (wid >= 4) {
            const LAS v4f* h = (const LAS v4f*)(L + LDS_HAND + (wid - 4) * 4096 + lo);
            acc[0] = __builtin_shufflevector(__builtin_shufflevector(h[0], h[64], 0, 1, 2, 3, 4, 5, 6, 7),
                                             __builtin_shufflevector(h[128], h[192], 0, 1, 2, 3, 4, 5, 6, 7),
                                             0, 1, 2, 3, 4, 5, 6, 7, 8, 9, 10, 11, 12, 13, 14, 15);
        }
        unsigned total = 0;
        bool dense = (((tm >= LEAD0 ? DENSE_L : DENSE_F) >> wid) & 1) &&
                     __builtin_bit_cast(unsigned, scale) - 0x0d800000u < 0x71800000u - 0x0d800000u;
        if (!dense) __builtin_amdgcn_s_setprio(3);
        if (dense) {
            float y[16];
            unsigned long long nz[16];
            unsigned mask = 0, m1 = 0, m2 = 0, inexact = 0;
            float top = 0.f;
#pragma unroll
            for (int q = 3; q >= 0; --q) {
                unsigned long long s1[4], s2[4];
#pragma unroll
                for (int e = 0; e < 4; ++e) {
                    const float a = acc[0][4 * q + e], am = __builtin_elementwise_minimum(a, 32.f), u = am + 32.f;
                    nz[4 * q + e] = __builtin_amdgcn_fcmpf(a, 0.f, 2  );
                    s1[e] = __builtin_amdgcn_fcmpf(a, 32.f, 3  );
                    s2[e] = __builtin_amdgcn_fcmpf(a, 128.f, 3);
                    inexact = __builtin_amdgcn_bitop3_b32(inexact, __builtin_bit_cast(unsigned, u - 32.f), __builtin_bit_cast(unsigned, am), 0xf6);
                    y[4 * q + e] = __builtin_amdgcn_inverse_ballot_w64(s1[e]) ? a : u;
                    top = __builtin_elementwise_maximum(top, a);
                }
#define AC(d, m) "v_addc_co_u32_e64 %" #d ", vcc, %" #d ", %" #d ", %" #m "\n\t"
                asm volatile("s_nop 1\n\t" AC(0, 6) AC(1, 10) AC(2, 14) AC(0, 5) AC(1, 9) AC(2, 13) AC(0, 4) AC(1, 8) AC(2, 12) AC(0, 3) AC(1, 7) AC(2, 11)
                             : "+v"(mask), "+v"(m1), "+v"(m2)
                             : "s"(nz[4 * q]), "s"(nz[4 * q + 1]), "s"(nz[4 * q + 2]), "s"(nz[4 * q + 3]),
                               "s"(s1[0]), "s"(s1[1]), "s"(s1[2]), "s"(s1[3]), "s"(s2[0]), "s"(s2[1]), "s"(s2[2]), "s"(s2[3])
                             : "vcc");
            }
            dense = __builtin_amdgcn_ballot_w64((inexact << 1) != 0u || !(top < 512.f)) == 0ull;
            if (dense) {
                *(LAS unsigned*)(L + wid * PK_SLOT + lane * 4) = (mask ^ m1 ^ m2) | (m1 << 16);
                total = compact(lds0 + LDS_RECV + wid * PK_SLOT, mask, y, nz);
#pragma unroll 1
                for (unsigned o = 0; o < total; o += 256) {
                    const u32x4 v = *(LAS u32x4*)(L + LDS_RECV + wid * PK_SLOT + o * 4 + lo);
                    LAS unsigned* d = (LAS unsigned*)(L + wid * PK_SLOT + 256 + o * 3 + lane * 12);
                    d[0] = __builtin_amdgcn_perm(v[1], v[0], 0x04020100u);
                    d[1] = __builtin_amdgcn_perm(v[2], v[1], 0x05040201u);
                    d[2] = __builtin_amdgcn_perm(v[3], v[2], 0x06050402u);
                }
            }
        }
        if (!dense) {
            float y[16];
            unsigned long long nz[16];
#pragma unroll
            for (int i = 0; i < 16; ++i) {
                y[i] = acc[0][i] * scale;
                nz[i] = __builtin_amdgcn_fcmpf(y[i], 0.f, 10  );
            }
            unsigned mask = 0;
#pragma unroll
            for (int q = 1; q >= 0; --q)
                asm volatile("s_nop 1\n\t" AC(0, 8) AC(0, 7) AC(0, 6) AC(0, 5) AC(0, 4) AC(0, 3) AC(0, 2) AC(0, 1)
                             : "+v"(mask)
                             : "s"(nz[8 * q]), "s"(nz[8 * q + 1]), "s"(nz[8 * q + 2]), "s"(nz[8 * q + 3]),
                               "s"(nz[8 * q + 4]), "s"(nz[8 * q + 5]), "s"(nz[8 * q + 6]), "s"(nz[8 * q + 7])
                             : "vcc");
            *(LAS unsigned short*)(L + wid * PK_SLOT + lane * 2) = (unsigned short)mask;
            total = compact(lds0 + wid * PK_SLOT + 128, mask, y, nz);
        }
        u32x4 c[3];
#pragma unroll
        for (int u = 0; u < 3; ++u) c[u] = *(LAS u32x4*)(L + wid * PK_SLOT + u * 1024 + lo);
        const unsigned call = __builtin_amdgcn_readfirstlane(misc[0]);
        const unsigned par = call & 1u;
        const unsigned bytes = dense ? 256 + 3 * total : 128 + 4 * total;
        const unsigned sidx = ((par * 8 + rank) * 32 + j) * 8 + wid;
        unsigned char* dslot = dws + WS_STAGE + (size_t)sidx * PK_SLOT;
        const int turn = wid - WIN;
        if (turn > 0)
            while ((int)__builtin_amdgcn_readfirstlane(__hip_atomic_load((LAS unsigned*)(L + LDS_MISC + 8), __ATOMIC_RELAXED,
                                                                         __HIP_MEMORY_SCOPE_WORKGROUP)) < turn)
                __builtin_amdgcn_s_sleep(1);
#pragma unroll
        for (int u = 0; u < 3; ++u)
            if (u * 1024 + lo < bytes)
                asm volatile("s_nop 4\n\tglobal_store_dwordx4 %0, %1, %2 sc0 sc1\n\ts_nop 1" :: "v"(lo + u * 1024), "v"(c[u]), "s"(dslot) : "memory");
#pragma unroll 1
        for (unsigned o = 3 * 1024; o < bytes; o += 1024) {
            const u32x4 v = *(LAS u32x4*)(L + wid * PK_SLOT + o + lo);
            if (o + lo < bytes)
                asm volatile("s_nop 4\n\tglobal_store_dwordx4 %0, %1, %2 sc0 sc1\n\ts_nop 1" :: "v"(lo + o), "v"(v), "s"(dslot) : "memory");
        }
        WAIT_VM(0);
        if (lane == 0) {
            put_flag(dws + WS_FLAG + (size_t)sidx * 128, call, total | (dense ? 0x10000u : 0u));
            __hip_atomic_fetch_add((LAS unsigned*)(L + LDS_MISC + 8), 1u, __ATOMIC_RELAXED, __HIP_MEMORY_SCOPE_WORKGROUP);
        }
        const unsigned ridx = ((par * 8 + g) * 32 + j) * 8 + wid;
        const unsigned rword = wait_flag(myws + WS_FLAG + (size_t)ridx * 128, 0u, tmo, call, lane);
        const unsigned rtotal = __builtin_amdgcn_readfirstlane(rword) & ~0x10000u;
        const bool rdense = __builtin_amdgcn_readfirstlane(rword) & 0x10000u;
        u32x4 o[4];
        bool bad = true;
        if (rtotal <= 1024u) {
            const unsigned rbytes = rdense ? 256 + 3 * rtotal : 128 + 4 * rtotal;
            const unsigned char* sslot = myws + WS_STAGE + (size_t)ridx * PK_SLOT;
            const unsigned rdst = __builtin_amdgcn_readfirstlane(lds0 + LDS_RECV + wid * PK_SLOT);
            asm volatile("s_mov_b32 m0, %2\n\ts_nop 3\n\tglobal_load_lds_dwordx4 %0, %1 sc0 sc1\n\t"
                         "s_cmp_le_u32 %3, 1024\n\ts_cbranch_scc1 1f\n\tglobal_load_lds_dwordx4 %0, %1 offset:1024 sc0 sc1\n\t"
                         "s_cmp_le_u32 %3, 2048\n\ts_cbranch_scc1 1f\n\tglobal_load_lds_dwordx4 %0, %1 offset:2048 sc0 sc1\n\t"
                         "s_cmp_le_u32 %3, 3072\n\ts_cbranch_scc1 1f\n\tglobal_load_lds_dwordx4 %0, %1 offset:3072 sc0 sc1\n\t"
                         "s_cmp_le_u32 %3, 4096\n\ts_cbranch_scc1 1f\n\ts_mov_b32 m0, %5\n\ts_nop 0\n\tglobal_load_lds_dwordx4 %4, %1 sc0 sc1\n"
                         "1:"
                         :: "v"(lo), "s"(sslot), "s"(rdst), "s"(rbytes), "v"(lo + 4096), "s"(rdst + 4096) : "memory", "scc");
            unsigned t;
            asm volatile("s_nop 4\n\tglobal_load_dword %0, %1, %2 sc1\n\ts_waitcnt vmcnt(0)" : "=&v"(t) : "v"(0u), "s"(tmo) : "memory");
            bad = __builtin_amdgcn_readfirstlane(t) == call;
            const unsigned bp = 4 * ((lane & 7) >> 1);
            LAS v2i* tab = (LAS v2i*)(L + LDS_TAB + wid * 512);
            if (!rdense) {
                const unsigned rm = *(LAS unsigned short*)(L + LDS_RECV + wid * PK_SLOT + lane * 2);
                const unsigned rc = __builtin_popcount(rm);
                tab[lane] = (v2i){(int)rm, (int)(LDS_RECV + wid * PK_SLOT + 128 + (wave_scan(rc) - rc) * 4)};
#pragma unroll
                for (int u = 0; u < 4; ++u) {
                    const v2i tb = tab[8 * u + (lane >> 3) + 32 * (lane & 1)];
                    const unsigned m = (unsigned)tb[0];
                    int addr = tb[1] + 4 * __builtin_popcount(m & ((1u << bp) - 1u));
                    const int ms = (int)(m >> bp);
#pragma unroll
                    for (int e = 0; e < 4; ++e) {
                        const int sel = __builtin_amdgcn_sbfe(ms, e, 1);
                        o[u][e] = *(LAS unsigned*)(L + addr) & (unsigned)sel;
                        if (e < 3) addr = step4(addr, sel);
                    }
                }
            } else {
                const unsigned rm = *(LAS unsigned*)(L + LDS_RECV + wid * PK_SLOT + lane * 4);
#pragma unroll 1
                for (unsigned q = 0; q <= rtotal; q += 256) {
                    const LAS unsigned* d = (const LAS unsigned*)(L + LDS_RECV + wid * PK_SLOT + 256 + q * 3 + lane * 12);
                    const unsigned d0 = d[0], d1 = d[1], d2 = d[2];
                    *(LAS u32x4*)(L + wid * PK_SLOT + q * 4 + lo) =
                        (u32x4){(d0 & 0xffffffu) | 0x42000000u, __builtin_amdgcn_perm(d1, d0, 0x0c050403u) | 0x42000000u,
                                __builtin_amdgcn_perm(d2, d1, 0x0c040302u) | 0x42000000u, __builtin_amdgcn_alignbyte(0x42u, d2, 1)};
                }
                const unsigned rc = __builtin_popcount((rm | (rm >> 16)) & 0xffffu);
                tab[lane] = (v2i){(int)rm, (int)(wid * PK_SLOT + (wave_scan(rc) - rc) * 4)};
                const unsigned sb = __builtin_bit_cast(unsigned, scale), nb = __builtin_bit_cast(unsigned, -32.f * scale);
#pragma unroll
                for (int u = 0; u < 4; ++u) {
                    const v2i tb = tab[8 * u + (lane >> 3) + 32 * (lane & 1)];
                    const unsigned m = (unsigned)tb[0];
                    int addr = tb[1] + 4 * __builtin_popcount((m | (m >> 16)) & ((1u << bp) - 1u));
                    const int ms = (int)(m >> bp);
#pragma unroll
                    for (int e = 0; e < 4; ++e) {
                        const unsigned b0 = __builtin_amdgcn_sbfe(ms, e, 1), b1 = __builtin_amdgcn_sbfe(ms, 16 + e, 1);
                        const unsigned code = *(LAS unsigned*)(L + addr) | __builtin_amdgcn_bitop3_b32(b0, b1, 0x01000000u, 0x80);
                        o[u][e] = __builtin_bit_cast(unsigned, __builtin_fmaf(__builtin_bit_cast(float, code),
                                                                              __builtin_bit_cast(float, __builtin_amdgcn_bitop3_b32(b0, b1, sb, 0xa8)),
                                                                              __builtin_bit_cast(float, __builtin_amdgcn_bitop3_b32(b0, b1, nb, 0x20))));
                        if (e < 3) addr = step4(addr, (int)(b0 | b1));
                    }
                }
            }
        }
        float* ob = p.out + (size_t)(g * MS + tm * BM + (wid >> 2) * 32 + (lane >> 3)) * NS + tn * BN + (wid & 3) * 32 + (lane & 7) * 4;
#pragma unroll
        for (int u = 0; u < 4; ++u) {
            if (bad) o[u] = (u32x4){0x7fc00000u, 0x7fc00000u, 0x7fc00000u, 0x7fc00000u};
            store_out(ob + u * 8 * NS, o[u]);
        }
    }
}

extern "C" void kernel_launch(int rank, int nranks, ncclComm_t comm, hipStream_t stream,
                              void* const* d_in, const int* in_sizes, int n_in,
                              void* d_out, int out_size, const int* in_dtypes, int out_dtype,
                              void* const* d_out_all, void* const* d_ws_all, size_t ws_bytes) {
    if (nranks != NR || n_in != 4 || in_sizes[0] != MS * KK || in_sizes[1] != KK * NN || out_size != NR * MS * NS ||
        (unsigned)rank >= (unsigned)NR || ws_bytes < WS_END) {
        fprintf(stderr, "unexpected problem shape\n");
        return;
    }
    Params p;
    p.x = (const unsigned char*)d_in[0];
    p.w = (const unsigned char*)d_in[1];
    p.sx = (const float*)d_in[2];
    p.sw = (const float*)d_in[3];
    p.out = (float*)d_out;
    p.myws = (unsigned char*)d_ws_all[rank];
    for (int i = 0; i < NR; ++i) p.ws[i] = (unsigned char*)d_ws_all[i];
    p.rank = rank;
    p.pad = 0;
    gemm_a2a<<<GRID, 512, 0, stream>>>(p);
    HIP_CHECK(hipGetLastError());
}
```

```cpp
#include <hip/hip_runtime.h>
#include <rccl/rccl.h>
#include <cstdio>
#include <cstddef>

#define HIP_CHECK(expr) do { hipError_t _e = (expr); if (_e != hipSuccess) { \
    fprintf(stderr, "HIP error %s at %s:%d\n", hipGetErrorString(_e), __FILE__, __LINE__); \
    return; } } while (0)

typedef int v8i __attribute__((ext_vector_type(8)));
typedef int v4i __attribute__((ext_vector_type(4)));
typedef int v2i __attribute__((ext_vector_type(2)));
typedef float v16f __attribute__((ext_vector_type(16)));
typedef float v4f __attribute__((ext_vector_type(4)));
typedef unsigned u32x4 __attribute__((ext_vector_type(4)));
#define LAS __attribute__((address_space(3)))

constexpr int NR = 8;
constexpr int MS = 512;
constexpr int KK = 4096;
constexpr int NN = 4096;
constexpr int NS = 512;
constexpr int BM = 64, BN = 128, BK = 128;
constexpr int NSTAGE = KK / BK;
constexpr int XROW0 = MS - BM;
constexpr int GRID = 256;
constexpr unsigned DEADLINE_TICKS = 50000000u;

constexpr int DEPTH_ = 4;
constexpr int LEAD0 = 3;
constexpr int TWARM = 500;
constexpr int PACE0 = 5;
constexpr int T_PACE0 = 400;
constexpr int T_PACE1 = 850;
constexpr int TAU_A = 26;
constexpr int TAU_B = 32;
constexpr int WIN = 2;
constexpr int DENSE_F = 0xfc;
constexpr int DENSE_L = 0x7f;

constexpr int DEPTH = DEPTH_;
constexpr int ADEPTH = DEPTH + 1;
constexpr int A_BYTES = BM * BK;
constexpr int B_BYTES = BK * BN;
constexpr int A_RING = DEPTH * B_BYTES;
constexpr int RING_BYTES = DEPTH * B_BYTES + ADEPTH * A_BYTES;
constexpr int PK_SLOT = 5120;
constexpr int LDS_RECV = 8 * PK_SLOT;
constexpr int LDS_TAB = 16 * PK_SLOT;
constexpr int LDS_HAND = RING_BYTES;
constexpr int LDS_MISC = LDS_HAND + 4 * 4096;
constexpr int LDS_BYTES = LDS_MISC + 16;
static_assert(LDS_TAB + 8 * 512 <= RING_BYTES && LDS_BYTES > 80 * 1024 && LDS_BYTES <= 160 * 1024, "LDS layout");

constexpr size_t WS_CTR = 0;
constexpr size_t WS_TMO = 32768;
constexpr size_t WS_XFLAG = 36864;
constexpr size_t WS_FLAG = 131072;
constexpr size_t WS_XSTAGE = WS_FLAG + 2 * 2048 * 128;
constexpr size_t WS_STAGE = WS_XSTAGE + (size_t)16 * BM * KK;
constexpr size_t WS_END = WS_STAGE + (size_t)2 * 2048 * PK_SLOT;

struct Params {
    const unsigned char* x;
    const unsigned char* w;
    unsigned char* myws;
    int rank;
    int pad;
    const float* sx;
    const float* sw;
    float* out;
    unsigned char* ws[NR];
};
static_assert(offsetof(Params, x) == 0 && offsetof(Params, w) == 8 && offsetof(Params, rank) == 24, "read as words in the kernel");

#define WAIT_VM(n) asm volatile("s_waitcnt vmcnt(%0)" :: "i"(n) : "memory")
#define WAIT_LGKM0() asm volatile("s_waitcnt lgkmcnt(0)" ::: "memory")
#define BAR() __builtin_amdgcn_s_barrier()
#define SCHED() __builtin_amdgcn_sched_barrier(0)

template <int S>
__device__ __forceinline__ int korder(int lead, int tn) {
    const int rs = (S + tn) & 3;
    const int kl = S < 8 ? lead + S : (lead + (S & ~3) + rs) & (NSTAGE - 1);
    return lead < 0 ? (S >> 2) + 8 * rs : kl;
}

struct LoadCtx {
    unsigned va[2], vb[4];
    const unsigned char* xa;
    const unsigned char* wb;
    unsigned ldsA, ldsB;
    int lead, tn;
    unsigned t0, tau, tend;
};
__device__ __forceinline__ void load_w_at(const LoadCtx& c, const unsigned char* pb, int slot) {
    asm volatile("s_mov_b32 m0, %5\n\ts_nop 3\n\t"
                 "global_load_lds_dwordx4 %0, %4\n\tglobal_load_lds_dwordx4 %1, %4 offset:1024\n\t"
                 "global_load_lds_dwordx4 %2, %4 offset:2048\n\tglobal_load_lds_dwordx4 %3, %4 offset:3072"
                 :: "v"(c.vb[0]), "v"(c.vb[1]), "v"(c.vb[2]), "v"(c.vb[3]), "s"(pb), "s"(c.ldsB + slot * B_BYTES) : "memory");
}
__device__ __forceinline__ void load_x_at(const LoadCtx& c, const unsigned char* pa, int slot) {
    asm volatile("s_mov_b32 m0, %3\n\ts_nop 3\n\t"
                 "global_load_lds_dwordx4 %0, %2 sc0 sc1\n\tglobal_load_lds_dwordx4 %1, %2 offset:1024 sc0 sc1"
                 :: "v"(c.va[0]), "v"(c.va[1]), "s"(pa), "s"(c.ldsA + slot * A_BYTES) : "memory");
}
template <int S>
__device__ __forceinline__ void load_w(const LoadCtx& c) { load_w_at(c, c.wb + (unsigned)korder<S>(c.lead, c.tn) * (BK * NN), S % DEPTH); }
template <int S>
__device__ __forceinline__ void load_x(const LoadCtx& c) { load_x_at(c, c.xa + (unsigned)korder<S>(c.lead, c.tn) * BK, S % ADEPTH); }
constexpr int mini(int a, int b) { return a < b ? a : b; }
constexpr int issued(int I) { return mini(NSTAGE, mini(2 + 2 * I, I + DEPTH - 1)); }
constexpr int younger(int I) {
    return 4 * (issued(I) - 1 - I) + 2 * (mini(issued(I), NSTAGE - 1) - I);
}
template <int S, int E>
__device__ __forceinline__ void load_range(const LoadCtx& c) {
    if constexpr (S < E) {
        load_w<S>(c);
        if constexpr (S + 1 < NSTAGE) load_x<S + 1>(c);
        load_range<S + 1, E>(c);
    }
}
template <int I, int END>
__device__ __forceinline__ void loader_loop(const LoadCtx& c) {
    WAIT_VM(younger(I));
    BAR();
    if constexpr (I >= PACE0 && T_PACE0 + (I - PACE0) * TAU_A <= T_PACE1) {
        const unsigned due = T_PACE0 + (I - PACE0) * c.tau;
        if (c.lead >= 0 && due <= c.tend)
            while ((unsigned)__builtin_amdgcn_s_memrealtime() - c.t0 < due) __builtin_amdgcn_s_sleep(1);
    }
    if constexpr (I + 1 < NSTAGE) load_range<issued(I), issued(I + 1)>(c);
    if constexpr (I + 1 < END) loader_loop<I + 1, END>(c);
}

__device__ __forceinline__ unsigned wave_scan(unsigned v) {
    v += __builtin_amdgcn_update_dpp(0u, v, 0x111, 0xf, 0xf, true);
    v += __builtin_amdgcn_update_dpp(0u, v, 0x112, 0xf, 0xf, true);
    v += __builtin_amdgcn_update_dpp(0u, v, 0x114, 0xf, 0xf, true);
    v += __builtin_amdgcn_update_dpp(0u, v, 0x118, 0xf, 0xf, true);
    v += __builtin_amdgcn_update_dpp(0u, v, 0x142, 0xa, 0xf, false);
    v += __builtin_amdgcn_update_dpp(0u, v, 0x143, 0xc, 0xf, false);
    return v;
}

__device__ __forceinline__ unsigned draw_call(unsigned char* ctr) {
    unsigned t;
    asm volatile("s_mov_b64 exec, 1\n\ts_nop 3\n\tglobal_atomic_add %0, %1, %2, %3 sc0\n\ts_mov_b64 exec, -1\n\ts_waitcnt vmcnt(0)"
                 : "=&v"(t) : "v"(0u), "v"(1u), "s"(ctr) : "memory");
    return t + 1u;
}

__device__ __forceinline__ void touch(const unsigned char* base, unsigned voff, unsigned lds_dst) {
    asm volatile("s_mov_b32 m0, %2\n\ts_nop 3\n\tglobal_load_lds_dword %0, %1" :: "v"(voff), "s"(base), "s"(lds_dst) : "memory");
}

__device__ __forceinline__ void put_flag(unsigned char* flag, unsigned call, unsigned low) {
    typedef __attribute__((address_space(1))) unsigned long long gu64;
    __hip_atomic_store((gu64*)flag, ((unsigned long long)call << 32) | low, __ATOMIC_RELAXED, __HIP_MEMORY_SCOPE_SYSTEM);
}

__device__ __forceinline__ unsigned wait_flag(const unsigned char* flag, unsigned voff, unsigned* tmo, unsigned call, int lane) {
    const unsigned t0 = (unsigned)__builtin_amdgcn_s_memrealtime();
    unsigned low;
    bool seen;
    for (;;) {
        v2i f;
        unsigned t;
        asm volatile("s_nop 4\n\tglobal_load_dwordx2 %0, %2, %3 sc0 sc1\n\tglobal_load_dword %1, %4, %5 sc1\n\ts_waitcnt vmcnt(0)"
                     : "=&v"(f), "=&v"(t) : "v"(voff), "s"(flag), "v"(0u), "s"(tmo) : "memory");
        low = __builtin_amdgcn_readfirstlane(f[0]);
        seen = __builtin_amdgcn_ballot_w64((unsigned)f[1] < call) == 0ull;
        if (seen || __builtin_amdgcn_readfirstlane(t) == call || (unsigned)__builtin_amdgcn_s_memrealtime() - t0 > DEADLINE_TICKS) break;
        __builtin_amdgcn_s_sleep(2);
    }
    if (!seen && lane == 0) __hip_atomic_store(tmo, call, __ATOMIC_RELAXED, __HIP_MEMORY_SCOPE_AGENT);
    return seen ? low : 0xffffffffu;
}

__device__ __forceinline__ void store_out(float* ptr, u32x4 v) {
    asm volatile("global_store_dwordx4 %0, %1, off sc0 sc1\n\ts_nop 1" :: "v"(ptr), "v"(v) : "memory");
}

__device__ __forceinline__ unsigned compact(unsigned base, unsigned mask, const float (&y)[16], const unsigned long long (&nz)[16]) {
    const unsigned cnt = __builtin_popcount(mask);
    const unsigned incl = wave_scan(cnt);
    unsigned ptr = base + (incl - cnt) * 4;
#define PK1(m, v) "s_mov_b64 exec, %" #m "\n\tds_write_b32 %0, %" #v "\n\tv_add_u32 %0, 4, %0\n\t"
#pragma unroll
    for (int q = 0; q < 2; ++q)
        asm volatile(PK1(1, 9) PK1(2, 10) PK1(3, 11) PK1(4, 12) PK1(5, 13) PK1(6, 14) PK1(7, 15) PK1(8, 16)
                     "s_mov_b64 exec, -1"
                     : "+v"(ptr)
                     : "s"(nz[8 * q]), "s"(nz[8 * q + 1]), "s"(nz[8 * q + 2]), "s"(nz[8 * q + 3]),
                       "s"(nz[8 * q + 4]), "s"(nz[8 * q + 5]), "s"(nz[8 * q + 6]), "s"(nz[8 * q + 7]),
                       "v"(y[8 * q]), "v"(y[8 * q + 1]), "v"(y[8 * q + 2]), "v"(y[8 * q + 3]),
                       "v"(y[8 * q + 4]), "v"(y[8 * q + 5]), "v"(y[8 * q + 6]), "v"(y[8 * q + 7])
                     : "memory");
    return __builtin_amdgcn_readlane(incl, 63);
}

__device__ __forceinline__ int step4(int addr, int sel) {
    int r;
    asm("v_mad_i32_i24 %0, %1, -4, %2" : "=v"(r) : "v"(sel), "v"(addr));
    return r;
}

struct Frag { v4i a[2][2][2]; v2i b[2][4]; };

__device__ __forceinline__ v8i cat8(v4i lo, v4i hi) { return __builtin_shufflevector(lo, hi, 0, 1, 2, 3, 4, 5, 6, 7); }
__device__ __forceinline__ v8i cat8b(v2i a, v2i b, v2i c, v2i d) {
    return cat8(__builtin_shufflevector(a, b, 0, 1, 2, 3), __builtin_shufflevector(c, d, 0, 1, 2, 3));
}

__global__ void __launch_bounds__(512, 2) gemm_a2a(Params p) {
    __shared__ __attribute__((aligned(1024))) unsigned char lds[LDS_BYTES];
    v8i ka;
    asm volatile("s_load_dwordx8 %0, %1, 0x0" : "=s"(ka) : "s"(__builtin_amdgcn_kernarg_segment_ptr()));
    SCHED();
    LAS unsigned char* L = (LAS unsigned char*)lds;
    const unsigned lds0 = (unsigned)(size_t)L;
    const int tid = threadIdx.x, lane = tid & 63;
    const int wid = __builtin_amdgcn_readfirstlane(tid >> 6);
    const int b = blockIdx.x;
    const int g = b & 7, j = b >> 3, tm = j >> 2, tn = j & 3;
    LoadCtx c;
    {
        const int lw = wid & 3;
#pragma unroll
        for (int i = 0; i < 2; ++i) {
            const int row = (lw * 2 + i) * 8 + (lane >> 3);
            c.va[i] = row * KK + (((lane & 7) ^ ((row >> 1) & 7)) << 4) - i * 1024;
        }
#pragma unroll
        for (int i = 0; i < 4; ++i) {
            const int row = (lw * 4 + i) * 8 + (lane >> 3);
            c.vb[i] = row * NN + (((lane & 7) ^ (row & 6)) << 4) - i * 1024;
        }
        c.ldsA = lds0 + A_RING + lw * 2048;
        c.ldsB = lds0 + lw * 4096;
        c.tn = tn;
    }
    c.lead = tm >= LEAD0 ? 8 * (tm - LEAD0) : -1;
    int wcol = g * NS + tn * BN;
    unsigned xo = tm * BM * KK;
    unsigned x0 = xo + korder<0>(c.lead, tn) * BK, w0 = wcol + korder<0>(c.lead, tn) * BK * NN;
    int skip = tm == 7 || wid < 4 ? -1 : g;
    asm volatile("" : "+v"(c.va[0]), "+v"(c.va[1]), "+v"(c.vb[0]), "+v"(c.vb[1]), "+v"(c.vb[2]), "+v"(c.vb[3]));
    asm volatile("" : "+s"(x0), "+s"(w0), "+s"(skip), "+s"(c.ldsA), "+s"(c.ldsB));
    asm volatile("s_waitcnt lgkmcnt(0)" : "+s"(ka));
    const unsigned char* px = (const unsigned char*)(((unsigned long long)(unsigned)ka[1] << 32) | (unsigned)ka[0]);
    const unsigned char* pw = (const unsigned char*)(((unsigned long long)(unsigned)ka[3] << 32) | (unsigned)ka[2]);
    const int rank = ka[6];
    unsigned char* myws = p.myws;
    const unsigned char* xa = px + xo;
    if (__builtin_expect(skip >= 0 && skip != rank, 1)) {
        load_x_at(c, px + x0, 0);
        load_w_at(c, pw + w0, 0);
        c.xa = xa;
        c.wb = pw + wcol;
        load_x<1>(c);
        load_range<1, issued(0)>(c);
    }
    const bool early = g == rank || tm == 7;
    const unsigned t_begin = (unsigned)__builtin_amdgcn_s_memrealtime();
    c.t0 = t_begin;
    c.tend = ((rank & 1) == 0) ? T_PACE1 : 0;
    c.tau = tm >= LEAD0 + 2 ? TAU_B : TAU_A;
    unsigned* tmo = (unsigned*)(myws + WS_TMO);
    int orow = rank * MS + tm * BM, ocol = tn * BN;
    const bool local = early;
    bool lost = false;

    if (early) {
        u32x4 r[8];
        if (g != rank) {
            if (wid == 7) {
                touch((const unsigned char*)p.sx, 0u, lds0);
                touch((const unsigned char*)p.sw, 0u, lds0);
            }
            unsigned t;
            asm volatile("s_nop 4\n\t"
                         "global_load_dwordx4 %0, %9, %11\n\tglobal_load_dwordx4 %1, %9, %11 offset:1024\n\t"
                         "global_load_dwordx4 %2, %9, %11 offset:2048\n\tglobal_load_dwordx4 %3, %9, %11 offset:3072\n\t"
                         "global_load_dwordx4 %4, %10, %11\n\tglobal_load_dwordx4 %5, %10, %11 offset:1024\n\t"
                         "global_load_dwordx4 %6, %10, %11 offset:2048\n\tglobal_load_dwordx4 %7, %10, %11 offset:3072\n\t"
                         "s_cmp_lg_u32 %15, 0\n\ts_cbranch_scc1 1f\n\t"
                         "s_mov_b64 exec, 1\n\tglobal_atomic_add %8, %12, %13, %14 sc0\n\ts_mov_b64 exec, -1\n"
                         "1:\n\ts_waitcnt vmcnt(0)"
                         : "=&v"(r[0]), "=&v"(r[1]), "=&v"(r[2]), "=&v"(r[3]), "=&v"(r[4]), "=&v"(r[5]), "=&v"(r[6]), "=&v"(r[7]), "=&v"(t)
                         : "v"(lane * 16), "v"(lane * 16 + KK), "s"(xa + (tn * 16 + wid * 2) * KK),
                           "v"(0u), "v"(1u), "s"(myws + WS_CTR + b * 128), "s"(wid)
                         : "memory", "scc");
            if (tid == 0) *(LAS v4i*)(L + LDS_MISC) = (v4i){(int)(t + 1u), 0, 0, 0};
        } else {
            touch(px, (j * 512 + tid) * 128, lds0);
            while ((unsigned)__builtin_amdgcn_s_memrealtime() - t_begin < TWARM) __builtin_amdgcn_s_sleep(4);
            touch(pw + wcol, (tm * 512 + tid) * NN, lds0);
            if (wid == 0) {
                const unsigned t = draw_call(myws + WS_CTR + b * 128);
                if (lane == 0) *(LAS v4i*)(L + LDS_MISC) = (v4i){(int)t, 0, 0, 0};
            }
            WAIT_VM(0);
        }
        __syncthreads();
        const unsigned call = __builtin_amdgcn_readfirstlane(*(LAS unsigned*)(L + LDS_MISC));
        const unsigned par = call & 1u;
        if (g != rank) {
            unsigned char* pws = p.ws[g];
            unsigned char* dx = pws + WS_XSTAGE + (size_t)((par * 8 + rank) * BM + tn * 16 + wid * 2) * KK;
#pragma unroll
            for (int q = 0; q < 8; ++q)
                asm volatile("s_nop 4\n\tglobal_store_dwordx4 %0, %1, %2 sc0 sc1\n\ts_nop 1" :: "v"(lane * 16 + q * 1024), "v"(r[q]), "s"(dx) : "memory");
            WAIT_VM(0);
            __syncthreads();
            if (tid == 0) put_flag(pws + WS_XFLAG + ((par * 8 + rank) * 4 + tn) * 128, call, 0u);
            const int u = ((g - rank - 1) & 7) * 4 + tn;
            xa = px + (size_t)(u % 7) * BM * KK;
            ocol = (u / 7) * BN;
            wcol = rank * NS + ocol;
            orow = rank * MS + (u % 7) * BM;
        } else if (tm < 7) {
            const int src_rank = (rank + 1 + tm) & 7;
            if (wid == 0) {
                const unsigned f = wait_flag(myws + WS_XFLAG + (par * 8 + src_rank) * 512, (lane & 3) * 128, tmo, call, lane);
                const unsigned t = __hip_atomic_load(tmo, __ATOMIC_RELAXED, __HIP_MEMORY_SCOPE_AGENT);
                if (f != 0u || t == call) *(LAS unsigned*)(L + LDS_MISC + 4) = 1u;
            }
            __syncthreads();
            lost = __builtin_amdgcn_readfirstlane(*(LAS unsigned*)(L + LDS_MISC + 4)) != 0;
            xa = lost ? px : myws + WS_XSTAGE + (size_t)(par * 8 + src_rank) * BM * KK;
            orow = src_rank * MS + XROW0;
        }
    }

    v16f acc[2];
    unsigned char* dws = nullptr;

    if (wid >= 4) {
        if (early) {
            c.lead = -1;
            c.xa = xa;
            c.wb = pw + wcol;
            load_x<0>(c);
            load_range<0, issued(0)>(c);
        }
        dws = p.ws[g];
        asm volatile("" : "+s"(dws));
        loader_loop<0, NSTAGE - 2>(c);
        static_assert(issued(NSTAGE - 2) == NSTAGE && younger(NSTAGE - 2) == 6 && younger(NSTAGE - 1) == 0, "last two iterations");
        if (wid == 4) {
            unsigned t;
            float s0, s1;
            asm volatile("s_nop 4\n\tglobal_load_dword %1, %3, %6\n\tglobal_load_dword %2, %3, %7\n\t"
                         "s_mov_b64 exec, 1\n\tglobal_atomic_add %0, %3, %4, %5 sc0\n\ts_mov_b64 exec, -1\n\t"
                         "s_waitcnt vmcnt(9)\n\ts_barrier\n\ts_waitcnt vmcnt(3)\n\ts_barrier\n\ts_waitcnt vmcnt(0)"
                         : "=&v"(t), "=&v"(s0), "=&v"(s1)
                         : "v"(0u), "v"(early ? 0u : 1u), "s"(myws + WS_CTR + b * 128), "s"(p.sx), "s"(p.sw) : "memory");
            if (lane == 0)
                *(LAS v4i*)(L + LDS_MISC) = (v4i){(int)(t + 1u), 0, 0, lost ? 0x7fc00000 : __builtin_bit_cast(int, s0 * s1)};
            WAIT_LGKM0();
        } else {
            loader_loop<NSTAGE - 2, NSTAGE>(c);
        }
    } else {
        dws = p.ws[g];
        asm volatile("" : "+s"(dws));
        acc[0] = acc[1] = (v16f)0.f;
        const int r = lane & 31, h = lane >> 5;
        unsigned ao[2][2];
#pragma unroll
        for (int ks = 0; ks < 2; ++ks)
#pragma unroll
            for (int e = 0; e < 2; ++e) ao[ks][e] = A_RING + r * 128 + (((4 * ks + 2 * h + e) ^ ((r >> 1) & 7)) << 4);
        const int q = (lane & 15) >> 1, pp = lane & 1, g1 = (lane >> 4) & 1;
        const unsigned bo = (32 * h + q) * 128 + (((2 * wid + g1) ^ (q & 6)) << 4) + pp * 8;

        auto rd = [&](Frag& f, int slot, int aslot, int ks) {
            const unsigned char LAS* base = L + slot * B_BYTES;
            const unsigned char LAS* abase = L + aslot * A_BYTES;
#pragma unroll
            for (int t = 0; t < 4; ++t)
                f.b[ks][t] = __builtin_amdgcn_ds_read_tr8_b64_v2i32((LAS v2i*)(base + bo + (64 * ks + 8 * t) * 128));
#pragma unroll
            for (int mt = 0; mt < 2; ++mt)
#pragma unroll
                for (int e = 0; e < 2; ++e) f.a[ks][mt][e] = *(const LAS v4i*)(abase + ao[ks][e] + mt * 4096);
        };
        auto mm = [&](const Frag& f, int ks, int mt) {
            acc[mt] = __builtin_amdgcn_mfma_scale_f32_32x32x64_f8f6f4(cat8b(f.b[ks][0], f.b[ks][1], f.b[ks][2], f.b[ks][3]),
                                                                      cat8(f.a[ks][mt][0], f.a[ks][mt][1]), acc[mt], 0, 0, 0, 0, 0, 0);
        };
        auto step = [&](Frag& fn, const Frag& fo, int slot, int aslot) {
            BAR(); SCHED();
            mm(fo, 0, 0); SCHED();
            rd(fn, slot, aslot, 0); SCHED();
            mm(fo, 0, 1); SCHED();
            rd(fn, slot, aslot, 1); SCHED();
            mm(fo, 1, 0); mm(fo, 1, 1); SCHED();
            WAIT_LGKM0(); SCHED();
        };
        Frag f0, f1;
        BAR();
        rd(f0, 0, 0, 0);
        rd(f0, 0, 0, 1);
        WAIT_LGKM0();
        SCHED();
        int slot = 1, aslot = 1;
#pragma unroll 1
        for (int i = 1; i < NSTAGE - 1; i += 2) {
            step(f1, f0, slot, aslot);
            slot = slot + 1 == DEPTH ? 0 : slot + 1;
            aslot = aslot + 1 == ADEPTH ? 0 : aslot + 1;
            step(f0, f1, slot, aslot);
            slot = slot + 1 == DEPTH ? 0 : slot + 1;
            aslot = aslot + 1 == ADEPTH ? 0 : aslot + 1;
        }
        step(f1, f0, slot, aslot);
        mm(f1, 0, 0); mm(f1, 0, 1); mm(f1, 1, 0); mm(f1, 1, 1);
        if (!local) {
#pragma unroll
            for (int gq = 0; gq < 4; ++gq)
                *(LAS v4f*)(L + LDS_HAND + wid * 4096 + gq * 1024 + lane * 16) =
                    (v4f){acc[1][4 * gq], acc[1][4 * gq + 1], acc[1][4 * gq + 2], acc[1][4 * gq + 3]};
        }
        WAIT_LGKM0();
    }
    BAR();
    const v4i misc = *(LAS v4i*)(L + LDS_MISC);
    const float scale = __builtin_bit_cast(float, __builtin_amdgcn_readfirstlane(misc[3]));

    if (local) {
        if (wid < 4) {
            const int r = lane & 31, h = lane >> 5;
#pragma unroll
            for (int mt = 0; mt < 2; ++mt)
#pragma unroll
                for (int gq = 0; gq < 4; ++gq) {
                    v4f v;
#pragma unroll
                    for (int e = 0; e < 4; ++e) { const float y = acc[mt][4 * gq + e] * scale; v[e] = y <= 0.f ? 0.f : y; }
                    const int row = mt * 32 + r, ch = wid * 8 + 2 * gq + h;
                    *(LAS v4f*)(L + row * 512 + ((ch ^ (row & 7)) << 4)) = v;
                }
        }
        __syncthreads();
        float* ob = p.out + (size_t)orow * NS + ocol;
#pragma unroll
        for (int u = 0; u < 4; ++u) {
            const int row = (wid * 4 + u) * 2 + (lane >> 5), ch = lane & 31;
            store_out(ob + row * NS + ch * 4, *(LAS u32x4*)(L + row * 512 + ((ch ^ (row & 7)) << 4)));
        }
    } else {
        const unsigned lo = lane * 16;
        if (wid >= 4) {
            const LAS v4f* h = (const LAS v4f*)(L + LDS_HAND + (wid - 4) * 4096 + lo);
            acc[0] = __builtin_shufflevector(__builtin_shufflevector(h[0], h[64], 0, 1, 2, 3, 4, 5, 6, 7),
                                             __builtin_shufflevector(h[128], h[192], 0, 1, 2, 3, 4, 5, 6, 7),
                                             0, 1, 2, 3, 4, 5, 6, 7, 8, 9, 10, 11, 12, 13, 14, 15);
        }
        unsigned total = 0;
        bool dense = (((tm >= LEAD0 ? DENSE_L : DENSE_F) >> wid) & 1) &&
                     __builtin_bit_cast(unsigned, scale) - 0x0d800000u < 0x71800000u - 0x0d800000u;
        if (!dense) __builtin_amdgcn_s_setprio(3);
        if (dense) {
            float y[16];
            unsigned long long nz[16];
            unsigned mask = 0, m1 = 0, m2 = 0, inexact = 0;
            float top = 0.f;
#pragma unroll
            for (int q = 3; q >= 0; --q) {
                unsigned long long s1[4], s2[4];
#pragma unroll
                for (int e = 0; e < 4; ++e) {
                    const float a = acc[0][4 * q + e], am = __builtin_elementwise_minimum(a, 32.f), u = am + 32.f;
                    nz[4 * q + e] = __builtin_amdgcn_fcmpf(a, 0.f, 2  );
                    s1[e] = __builtin_amdgcn_fcmpf(a, 32.f, 3  );
                    s2[e] = __builtin_amdgcn_fcmpf(a, 128.f, 3);
                    inexact = __builtin_amdgcn_bitop3_b32(inexact, __builtin_bit_cast(unsigned, u - 32.f), __builtin_bit_cast(unsigned, am), 0xf6);
                    y[4 * q + e] = __builtin_amdgcn_inverse_ballot_w64(s1[e]) ? a : u;
                    top = __builtin_elementwise_maximum(top, a);
                }
#define AC(d, m) "v_addc_co_u32_e64 %" #d ", vcc, %" #d ", %" #d ", %" #m "\n\t"
                asm volatile("s_nop 1\n\t" AC(0, 6) AC(1, 10) AC(2, 14) AC(0, 5) AC(1, 9) AC(2, 13) AC(0, 4) AC(1, 8) AC(2, 12) AC(0, 3) AC(1, 7) AC(2, 11)
                             : "+v"(mask), "+v"(m1), "+v"(m2)
                             : "s"(nz[4 * q]), "s"(nz[4 * q + 1]), "s"(nz[4 * q + 2]), "s"(nz[4 * q + 3]),
                               "s"(s1[0]), "s"(s1[1]), "s"(s1[2]), "s"(s1[3]), "s"(s2[0]), "s"(s2[1]), "s"(s2[2]), "s"(s2[3])
                             : "vcc");
            }
            dense = __builtin_amdgcn_ballot_w64((inexact << 1) != 0u || !(top < 512.f)) == 0ull;
            if (dense) {
                *(LAS unsigned*)(L + wid * PK_SLOT + lane * 4) = (mask ^ m1 ^ m2) | (m1 << 16);
                total = compact(lds0 + LDS_RECV + wid * PK_SLOT, mask, y, nz);
#pragma unroll 1
                for (unsigned o = 0; o < total; o += 256) {
                    const u32x4 v = *(LAS u32x4*)(L + LDS_RECV + wid * PK_SLOT + o * 4 + lo);
                    LAS unsigned* d = (LAS unsigned*)(L + wid * PK_SLOT + 256 + o * 3 + lane * 12);
                    d[0] = __builtin_amdgcn_perm(v[1], v[0], 0x04020100u);
                    d[1] = __builtin_amdgcn_perm(v[2], v[1], 0x05040201u);
                    d[2] = __builtin_amdgcn_perm(v[3], v[2], 0x06050402u);
                }
            }
        }
        if (!dense) {
            float y[16];
            unsigned long long nz[16];
#pragma unroll
            for (int i = 0; i < 16; ++i) {
                y[i] = acc[0][i] * scale;
                nz[i] = __builtin_amdgcn_fcmpf(y[i], 0.f, 10  );
            }
            unsigned mask = 0;
#pragma unroll
            for (int q = 1; q >= 0; --q)
                asm volatile("s_nop 1\n\t" AC(0, 8) AC(0, 7) AC(0, 6) AC(0, 5) AC(0, 4) AC(0, 3) AC(0, 2) AC(0, 1)
                             : "+v"(mask)
                             : "s"(nz[8 * q]), "s"(nz[8 * q + 1]), "s"(nz[8 * q + 2]), "s"(nz[8 * q + 3]),
                               "s"(nz[8 * q + 4]), "s"(nz[8 * q + 5]), "s"(nz[8 * q + 6]), "s"(nz[8 * q + 7])
                             : "vcc");
            *(LAS unsigned short*)(L + wid * PK_SLOT + lane * 2) = (unsigned short)mask;
            total = compact(lds0 + wid * PK_SLOT + 128, mask, y, nz);
        }
        u32x4 c[3];
#pragma unroll
        for (int u = 0; u < 3; ++u) c[u] = *(LAS u32x4*)(L + wid * PK_SLOT + u * 1024 + lo);
        const unsigned call = __builtin_amdgcn_readfirstlane(misc[0]);
        const unsigned par = call & 1u;
        const unsigned bytes = dense ? 256 + 3 * total : 128 + 4 * total;
        const unsigned sidx = ((par * 8 + rank) * 32 + j) * 8 + wid;
        unsigned char* dslot = dws + WS_STAGE + (size_t)sidx * PK_SLOT;
        const int turn = wid - WIN;
        if (turn > 0)
            while ((int)__builtin_amdgcn_readfirstlane(__hip_atomic_load((LAS unsigned*)(L + LDS_MISC + 8), __ATOMIC_RELAXED,
                                                                         __HIP_MEMORY_SCOPE_WORKGROUP)) < turn)
                __builtin_amdgcn_s_sleep(1);
#pragma unroll
        for (int u = 0; u < 3; ++u)
            if (u * 1024 + lo < bytes)
                asm volatile("s_nop 4\n\tglobal_store_dwordx4 %0, %1, %2 sc0 sc1\n\ts_nop 1" :: "v"(lo + u * 1024), "v"(c[u]), "s"(dslot) : "memory");
#pragma unroll 1
        for (unsigned o = 3 * 1024; o < bytes; o += 1024) {
            const u32x4 v = *(LAS u32x4*)(L + wid * PK_SLOT + o + lo);
            if (o + lo < bytes)
                asm volatile("s_nop 4\n\tglobal_store_dwordx4 %0, %1, %2 sc0 sc1\n\ts_nop 1" :: "v"(lo + o), "v"(v), "s"(dslot) : "memory");
        }
        WAIT_VM(0);
        if (lane == 0) {
            put_flag(dws + WS_FLAG + (size_t)sidx * 128, call, total | (dense ? 0x10000u : 0u));
            __hip_atomic_fetch_add((LAS unsigned*)(L + LDS_MISC + 8), 1u, __ATOMIC_RELAXED, __HIP_MEMORY_SCOPE_WORKGROUP);
        }
        {
            const unsigned d_sent = (unsigned)__builtin_amdgcn_s_memrealtime() - t_begin;
            if (lane == 0) __hip_atomic_fetch_max((unsigned*)(p.ws[0] + WS_TMO + 1024 + (rank & 1) * 128), (call << 16) | min(d_sent, 65535u), __ATOMIC_RELAXED, __HIP_MEMORY_SCOPE_SYSTEM);
        }
        const unsigned ridx = ((par * 8 + g) * 32 + j) * 8 + wid;
        const unsigned rword = wait_flag(myws + WS_FLAG + (size_t)ridx * 128, 0u, tmo, call, lane);
        const unsigned rtotal = __builtin_amdgcn_readfirstlane(rword) & ~0x10000u;
        const bool rdense = __builtin_amdgcn_readfirstlane(rword) & 0x10000u;
        u32x4 o[4];
        bool bad = true;
        if (rtotal <= 1024u) {
            const unsigned rbytes = rdense ? 256 + 3 * rtotal : 128 + 4 * rtotal;
            const unsigned char* sslot = myws + WS_STAGE + (size_t)ridx * PK_SLOT;
            const unsigned rdst = __builtin_amdgcn_readfirstlane(lds0 + LDS_RECV + wid * PK_SLOT);
            asm volatile("s_mov_b32 m0, %2\n\ts_nop 3\n\tglobal_load_lds_dwordx4 %0, %1 sc0 sc1\n\t"
                         "s_cmp_le_u32 %3, 1024\n\ts_cbranch_scc1 1f\n\tglobal_load_lds_dwordx4 %0, %1 offset:1024 sc0 sc1\n\t"
                         "s_cmp_le_u32 %3, 2048\n\ts_cbranch_scc1 1f\n\tglobal_load_lds_dwordx4 %0, %1 offset:2048 sc0 sc1\n\t"
                         "s_cmp_le_u32 %3, 3072\n\ts_cbranch_scc1 1f\n\tglobal_load_lds_dwordx4 %0, %1 offset:3072 sc0 sc1\n\t"
                         "s_cmp_le_u32 %3, 4096\n\ts_cbranch_scc1 1f\n\ts_mov_b32 m0, %5\n\ts_nop 0\n\tglobal_load_lds_dwordx4 %4, %1 sc0 sc1\n"
                         "1:"
                         :: "v"(lo), "s"(sslot), "s"(rdst), "s"(rbytes), "v"(lo + 4096), "s"(rdst + 4096) : "memory", "scc");
            unsigned t;
            asm volatile("s_nop 4\n\tglobal_load_dword %0, %1, %2 sc1\n\ts_waitcnt vmcnt(0)" : "=&v"(t) : "v"(0u), "s"(tmo) : "memory");
            bad = __builtin_amdgcn_readfirstlane(t) == call;
            const unsigned bp = 4 * ((lane & 7) >> 1);
            LAS v2i* tab = (LAS v2i*)(L + LDS_TAB + wid * 512);
            if (!rdense) {
                const unsigned rm = *(LAS unsigned short*)(L + LDS_RECV + wid * PK_SLOT + lane * 2);
                const unsigned rc = __builtin_popcount(rm);
                tab[lane] = (v2i){(int)rm, (int)(LDS_RECV + wid * PK_SLOT + 128 + (wave_scan(rc) - rc) * 4)};
#pragma unroll
                for (int u = 0; u < 4; ++u) {
                    const v2i tb = tab[8 * u + (lane >> 3) + 32 * (lane & 1)];
                    const unsigned m = (unsigned)tb[0];
                    int addr = tb[1] + 4 * __builtin_popcount(m & ((1u << bp) - 1u));
                    const int ms = (int)(m >> bp);
#pragma unroll
                    for (int e = 0; e < 4; ++e) {
                        const int sel = __builtin_amdgcn_sbfe(ms, e, 1);
                        o[u][e] = *(LAS unsigned*)(L + addr) & (unsigned)sel;
                        if (e < 3) addr = step4(addr, sel);
                    }
                }
            } else {
                const unsigned rm = *(LAS unsigned*)(L + LDS_RECV + wid * PK_SLOT + lane * 4);
#pragma unroll 1
                for (unsigned q = 0; q <= rtotal; q += 256) {
                    const LAS unsigned* d = (const LAS unsigned*)(L + LDS_RECV + wid * PK_SLOT + 256 + q * 3 + lane * 12);
                    const unsigned d0 = d[0], d1 = d[1], d2 = d[2];
                    *(LAS u32x4*)(L + wid * PK_SLOT + q * 4 + lo) =
                        (u32x4){(d0 & 0xffffffu) | 0x42000000u, __builtin_amdgcn_perm(d1, d0, 0x0c050403u) | 0x42000000u,
                                __builtin_amdgcn_perm(d2, d1, 0x0c040302u) | 0x42000000u, __builtin_amdgcn_alignbyte(0x42u, d2, 1)};
                }
                const unsigned rc = __builtin_popcount((rm | (rm >> 16)) & 0xffffu);
                tab[lane] = (v2i){(int)rm, (int)(wid * PK_SLOT + (wave_scan(rc) - rc) * 4)};
                const unsigned sb = __builtin_bit_cast(unsigned, scale), nb = __builtin_bit_cast(unsigned, -32.f * scale);
#pragma unroll
                for (int u = 0; u < 4; ++u) {
                    const v2i tb = tab[8 * u + (lane >> 3) + 32 * (lane & 1)];
                    const unsigned m = (unsigned)tb[0];
                    int addr = tb[1] + 4 * __builtin_popcount((m | (m >> 16)) & ((1u << bp) - 1u));
                    const int ms = (int)(m >> bp);
#pragma unroll
                    for (int e = 0; e < 4; ++e) {
                        const unsigned b0 = __builtin_amdgcn_sbfe(ms, e, 1), b1 = __builtin_amdgcn_sbfe(ms, 16 + e, 1);
                        const unsigned code = *(LAS unsigned*)(L + addr) | __builtin_amdgcn_bitop3_b32(b0, b1, 0x01000000u, 0x80);
                        o[u][e] = __builtin_bit_cast(unsigned, __builtin_fmaf(__builtin_bit_cast(float, code),
                                                                              __builtin_bit_cast(float, __builtin_amdgcn_bitop3_b32(b0, b1, sb, 0xa8)),
                                                                              __builtin_bit_cast(float, __builtin_amdgcn_bitop3_b32(b0, b1, nb, 0x20))));
                        if (e < 3) addr = step4(addr, (int)(b0 | b1));
                    }
                }
            }
        }
        float* ob = p.out + (size_t)(g * MS + tm * BM + (wid >> 2) * 32 + (lane >> 3)) * NS + tn * BN + (wid & 3) * 32 + (lane & 7) * 4;
#pragma unroll
        for (int u = 0; u < 4; ++u) {
            if (bad) o[u] = (u32x4){0x7fc00000u, 0x7fc00000u, 0x7fc00000u, 0x7fc00000u};
            store_out(ob + u * 8 * NS, o[u]);
        }
        if (rank == 0 && b == 1 && wid == 0) {
            while ((unsigned)__builtin_amdgcn_s_memrealtime() - t_begin < 20000u) __builtin_amdgcn_s_sleep(8);
            const unsigned e = __hip_atomic_load((unsigned*)(myws + WS_TMO + 1024), __ATOMIC_RELAXED, __HIP_MEMORY_SCOPE_SYSTEM) & 0xffffu;
            const unsigned od = __hip_atomic_load((unsigned*)(myws + WS_TMO + 1024 + 128), __ATOMIC_RELAXED, __HIP_MEMORY_SCOPE_SYSTEM) & 0xffffu;
            const unsigned qe = min(max((int)e - 1500, 0) / 5, 127), qo = min(max((int)od - 1500, 0) / 5, 127);
            const unsigned T = 30000u + (qe * 128u + qo) * 1000u;
            while ((unsigned)__builtin_amdgcn_s_memrealtime() - t_begin < __builtin_amdgcn_readfirstlane(T)) __builtin_amdgcn_s_sleep(8);
        }
    }
}

extern "C" void kernel_launch(int rank, int nranks, ncclComm_t comm, hipStream_t stream,
                              void* const* d_in, const int* in_sizes, int n_in,
                              void* d_out, int out_size, const int* in_dtypes, int out_dtype,
                              void* const* d_out_all, void* const* d_ws_all, size_t ws_bytes) {
    if (nranks != NR || n_in != 4 || in_sizes[0] != MS * KK || in_sizes[1] != KK * NN || out_size != NR * MS * NS ||
        (unsigned)rank >= (unsigned)NR || ws_bytes < WS_END) {
        fprintf(stderr, "unexpected problem shape\n");
        return;
    }
    Params p;
    p.x = (const unsigned char*)d_in[0];
    p.w = (const unsigned char*)d_in[1];
    p.sx = (const float*)d_in[2];
    p.sw = (const float*)d_in[3];
    p.out = (float*)d_out;
    p.myws = (unsigned char*)d_ws_all[rank];
    for (int i = 0; i < NR; ++i) p.ws[i] = (unsigned char*)d_ws_all[i];
    p.rank = rank;
    p.pad = 0;
    gemm_a2a<<<GRID, 512, 0, stream>>>(p);
    HIP_CHECK(hipGetLastError());
}
```

```cpp
#include <hip/hip_runtime.h>
#include <rccl/rccl.h>
#include <cstdio>
#include <cstddef>

#define HIP_CHECK(expr) do { hipError_t _e = (expr); if (_e != hipSuccess) { \
    fprintf(stderr, "HIP error %s at %s:%d\n", hipGetErrorString(_e), __FILE__, __LINE__); \
    return; } } while (0)

typedef int v8i __attribute__((ext_vector_type(8)));
typedef int v4i __attribute__((ext_vector_type(4)));
typedef int v2i __attribute__((ext_vector_type(2)));
typedef float v16f __attribute__((ext_vector_type(16)));
typedef float v4f __attribute__((ext_vector_type(4)));
typedef unsigned u32x4 __attribute__((ext_vector_type(4)));
#define LAS __attribute__((address_space(3)))

constexpr int NR = 8;
constexpr int MS = 512;
constexpr int KK = 4096;
constexpr int NN = 4096;
constexpr int NS = 512;
constexpr int BM = 64, BN = 128, BK = 128;
constexpr int NSTAGE = KK / BK;
constexpr int XROW0 = MS - BM;
constexpr int GRID = 256;
constexpr unsigned DEADLINE_TICKS = 50000000u;

constexpr int DEPTH_ = 4;
constexpr int LEAD0 = 3;
constexpr int TWARM = 500;
constexpr int PACE0 = 5;
constexpr int T_PACE0 = 400;
constexpr int T_PACE1 = 850;
constexpr int TAU_A = 26;
constexpr int TAU_B = 32;
constexpr int WIN = 2;
constexpr int DENSE_F = 0xfc;
constexpr int DENSE_L = 0x7f;

constexpr int DEPTH = DEPTH_;
constexpr int ADEPTH = DEPTH + 1;
constexpr int A_BYTES = BM * BK;
constexpr int B_BYTES = BK * BN;
constexpr int A_RING = DEPTH * B_BYTES;
constexpr int RING_BYTES = DEPTH * B_BYTES + ADEPTH * A_BYTES;
constexpr int PK_SLOT = 5120;
constexpr int LDS_RECV = 8 * PK_SLOT;
constexpr int LDS_TAB = 16 * PK_SLOT;
constexpr int LDS_HAND = RING_BYTES;
constexpr int LDS_MISC = LDS_HAND + 4 * 4096;
constexpr int LDS_BYTES = LDS_MISC + 16;
static_assert(LDS_TAB + 8 * 512 <= RING_BYTES && LDS_BYTES > 80 * 1024 && LDS_BYTES <= 160 * 1024, "LDS layout");

constexpr size_t WS_CTR = 0;
constexpr size_t WS_TMO = 32768;
constexpr size_t WS_XFLAG = 36864;
constexpr size_t WS_FLAG = 131072;
constexpr size_t WS_XSTAGE = WS_FLAG + 2 * 2048 * 128;
constexpr size_t WS_STAGE = WS_XSTAGE + (size_t)16 * BM * KK;
constexpr size_t WS_END = WS_STAGE + (size_t)2 * 2048 * PK_SLOT;

struct Params {
    const unsigned char* x;
    const unsigned char* w;
    unsigned char* myws;
    int rank;
    int pad;
    const float* sx;
    const float* sw;
    float* out;
    unsigned char* ws[NR];
};
static_assert(offsetof(Params, x) == 0 && offsetof(Params, w) == 8 && offsetof(Params, rank) == 24, "read as words in the kernel");

#define WAIT_VM(n) asm volatile("s_waitcnt vmcnt(%0)" :: "i"(n) : "memory")
#define WAIT_LGKM0() asm volatile("s_waitcnt lgkmcnt(0)" ::: "memory")
#define BAR() __builtin_amdgcn_s_barrier()
#define SCHED() __builtin_amdgcn_sched_barrier(0)

template <int S>
__device__ __forceinline__ int korder(int lead, int tn) {
    const int rs = (S + tn) & 3;
    const int kl = S < 8 ? lead + S : (lead + (S & ~3) + rs) & (NSTAGE - 1);
    return lead < 0 ? (S >> 2) + 8 * rs : kl;
}

struct LoadCtx {
    unsigned va[2], vb[4];
    const unsigned char* xa;
    const unsigned char* wb;
    unsigned ldsA, ldsB;
    int lead, tn;
    unsigned t0, tau, tend;
};
__device__ __forceinline__ void load_w_at(const LoadCtx& c, const unsigned char* pb, int slot) {
    asm volatile("s_mov_b32 m0, %5\n\ts_nop 3\n\t"
                 "global_load_lds_dwordx4 %0, %4\n\tglobal_load_lds_dwordx4 %1, %4 offset:1024\n\t"
                 "global_load_lds_dwordx4 %2, %4 offset:2048\n\tglobal_load_lds_dwordx4 %3, %4 offset:3072"
                 :: "v"(c.vb[0]), "v"(c.vb[1]), "v"(c.vb[2]), "v"(c.vb[3]), "s"(pb), "s"(c.ldsB + slot * B_BYTES) : "memory");
}
__device__ __forceinline__ void load_x_at(const LoadCtx& c, const unsigned char* pa, int slot) {
    asm volatile("s_mov_b32 m0, %3\n\ts_nop 3\n\t"
                 "global_load_lds_dwordx4 %0, %2 sc0 sc1\n\tglobal_load_lds_dwordx4 %1, %2 offset:1024 sc0 sc1"
                 :: "v"(c.va[0]), "v"(c.va[1]), "s"(pa), "s"(c.ldsA + slot * A_BYTES) : "memory");
}
template <int S>
__device__ __forceinline__ void load_w(const LoadCtx& c) { load_w_at(c, c.wb + (unsigned)korder<S>(c.lead, c.tn) * (BK * NN), S % DEPTH); }
template <int S>
__device__ __forceinline__ void load_x(const LoadCtx& c) { load_x_at(c, c.xa + (unsigned)korder<S>(c.lead, c.tn) * BK, S % ADEPTH); }
constexpr int mini(int a, int b) { return a < b ? a : b; }
constexpr int issued(int I) { return mini(NSTAGE, mini(2 + 2 * I, I + DEPTH - 1)); }
constexpr int younger(int I) {
    return 4 * (issued(I) - 1 - I) + 2 * (mini(issued(I), NSTAGE - 1) - I);
}
template <int S, int E>
__device__ __forceinline__ void load_range(const LoadCtx& c) {
    if constexpr (S < E) {
        load_w<S>(c);
        if constexpr (S + 1 < NSTAGE) load_x<S + 1>(c);
        load_range<S + 1, E>(c);
    }
}
template <int I, int END>
__device__ __forceinline__ void loader_loop(const LoadCtx& c) {
    WAIT_VM(younger(I));
    BAR();
    if constexpr (I >= PACE0 && T_PACE0 + (I - PACE0) * TAU_A <= T_PACE1) {
        const unsigned due = T_PACE0 + (I - PACE0) * c.tau;
        if (c.lead >= 0 && due <= c.tend)
            while ((unsigned)__builtin_amdgcn_s_memrealtime() - c.t0 < due) __builtin_amdgcn_s_sleep(1);
    }
    if constexpr (I + 1 < NSTAGE) load_range<issued(I), issued(I + 1)>(c);
    if constexpr (I + 1 < END) loader_loop<I + 1, END>(c);
}

__device__ __forceinline__ unsigned wave_scan(unsigned v) {
    v += __builtin_amdgcn_update_dpp(0u, v, 0x111, 0xf, 0xf, true);
    v += __builtin_amdgcn_update_dpp(0u, v, 0x112, 0xf, 0xf, true);
    v += __builtin_amdgcn_update_dpp(0u, v, 0x114, 0xf, 0xf, true);
    v += __builtin_amdgcn_update_dpp(0u, v, 0x118, 0xf, 0xf, true);
    v += __builtin_amdgcn_update_dpp(0u, v, 0x142, 0xa, 0xf, false);
    v += __builtin_amdgcn_update_dpp(0u, v, 0x143, 0xc, 0xf, false);
    return v;
}

__device__ __forceinline__ unsigned draw_call(unsigned char* ctr) {
    unsigned t;
    asm volatile("s_mov_b64 exec, 1\n\ts_nop 3\n\tglobal_atomic_add %0, %1, %2, %3 sc0\n\ts_mov_b64 exec, -1\n\ts_waitcnt vmcnt(0)"
                 : "=&v"(t) : "v"(0u), "v"(1u), "s"(ctr) : "memory");
    return t + 1u;
}

__device__ __forceinline__ void touch(const unsigned char* base, unsigned voff, unsigned lds_dst) {
    asm volatile("s_mov_b32 m0, %2\n\ts_nop 3\n\tglobal_load_lds_dword %0, %1" :: "v"(voff), "s"(base), "s"(lds_dst) : "memory");
}

__device__ __forceinline__ void put_flag(unsigned char* flag, unsigned call, unsigned low) {
    typedef __attribute__((address_space(1))) unsigned long long gu64;
    __hip_atomic_store((gu64*)flag, ((unsigned long long)call << 32) | low, __ATOMIC_RELAXED, __HIP_MEMORY_SCOPE_SYSTEM);
}

__device__ __forceinline__ unsigned wait_flag(const unsigned char* flag, unsigned voff, unsigned* tmo, unsigned call, int lane) {
    const unsigned t0 = (unsigned)__builtin_amdgcn_s_memrealtime();
    unsigned low;
    bool seen;
    for (;;) {
        v2i f;
        unsigned t;
        asm volatile("s_nop 4\n\tglobal_load_dwordx2 %0, %2, %3 sc0 sc1\n\tglobal_load_dword %1, %4, %5 sc1\n\ts_waitcnt vmcnt(0)"
                     : "=&v"(f), "=&v"(t) : "v"(voff), "s"(flag), "v"(0u), "s"(tmo) : "memory");
        low = __builtin_amdgcn_readfirstlane(f[0]);
        seen = __builtin_amdgcn_ballot_w64((unsigned)f[1] < call) == 0ull;
        if (seen || __builtin_amdgcn_readfirstlane(t) == call || (unsigned)__builtin_amdgcn_s_memrealtime() - t0 > DEADLINE_TICKS) break;
        __builtin_amdgcn_s_sleep(2);
    }
    if (!seen && lane == 0) __hip_atomic_store(tmo, call, __ATOMIC_RELAXED, __HIP_MEMORY_SCOPE_AGENT);
    return seen ? low : 0xffffffffu;
}

__device__ __forceinline__ void store_out(float* ptr, u32x4 v) {
    asm volatile("global_store_dwordx4 %0, %1, off sc0 sc1\n\ts_nop 1" :: "v"(ptr), "v"(v) : "memory");
}

__device__ __forceinline__ unsigned compact(unsigned base, unsigned mask, const float (&y)[16], const unsigned long long (&nz)[16]) {
    const unsigned cnt = __builtin_popcount(mask);
    const unsigned incl = wave_scan(cnt);
    unsigned ptr = base + (incl - cnt) * 4;
#define PK1(m, v) "s_mov_b64 exec, %" #m "\n\tds_write_b32 %0, %" #v "\n\tv_add_u32 %0, 4, %0\n\t"
#pragma unroll
    for (int q = 0; q < 2; ++q)
        asm volatile(PK1(1, 9) PK1(2, 10) PK1(3, 11) PK1(4, 12) PK1(5, 13) PK1(6, 14) PK1(7, 15) PK1(8, 16)
                     "s_mov_b64 exec, -1"
                     : "+v"(ptr)
                     : "s"(nz[8 * q]), "s"(nz[8 * q + 1]), "s"(nz[8 * q + 2]), "s"(nz[8 * q + 3]),
                       "s"(nz[8 * q + 4]), "s"(nz[8 * q + 5]), "s"(nz[8 * q + 6]), "s"(nz[8 * q + 7]),
                       "v"(y[8 * q]), "v"(y[8 * q + 1]), "v"(y[8 * q + 2]), "v"(y[8 * q + 3]),
                       "v"(y[8 * q + 4]), "v"(y[8 * q + 5]), "v"(y[8 * q + 6]), "v"(y[8 * q + 7])
                     : "memory");
    return __builtin_amdgcn_readlane(incl, 63);
}

__device__ __forceinline__ int step4(int addr, int sel) {
    int r;
    asm("v_mad_i32_i24 %0, %1, -4, %2" : "=v"(r) : "v"(sel), "v"(addr));
    return r;
}

struct Frag { v4i a[2][2][2]; v2i b[2][4]; };

__device__ __forceinline__ v8i cat8(v4i lo, v4i hi) { return __builtin_shufflevector(lo, hi, 0, 1, 2, 3, 4, 5, 6, 7); }
__device__ __forceinline__ v8i cat8b(v2i a, v2i b, v2i c, v2i d) {
    return cat8(__builtin_shufflevector(a, b, 0, 1, 2, 3), __builtin_shufflevector(c, d, 0, 1, 2, 3));
}

__global__ void __launch_bounds__(512, 2) gemm_a2a(Params p) {
    __shared__ __attribute__((aligned(1024))) unsigned char lds[LDS_BYTES];
    v8i ka;
    asm volatile("s_load_dwordx8 %0, %1, 0x0" : "=s"(ka) : "s"(__builtin_amdgcn_kernarg_segment_ptr()));
    SCHED();
    LAS unsigned char* L = (LAS unsigned char*)lds;
    const unsigned lds0 = (unsigned)(size_t)L;
    const int tid = threadIdx.x, lane = tid & 63;
    const int wid = __builtin_amdgcn_readfirstlane(tid >> 6);
    const int b = blockIdx.x;
    const int g = b & 7, j = b >> 3, tm = j >> 2, tn = j & 3;
    LoadCtx c;
    {
        const int lw = wid & 3;
#pragma unroll
        for (int i = 0; i < 2; ++i) {
            const int row = (lw * 2 + i) * 8 + (lane >> 3);
            c.va[i] = row * KK + (((lane & 7) ^ ((row >> 1) & 7)) << 4) - i * 1024;
        }
#pragma unroll
        for (int i = 0; i < 4; ++i) {
            const int row = (lw * 4 + i) * 8 + (lane >> 3);
            c.vb[i] = row * NN + (((lane & 7) ^ (row & 6)) << 4) - i * 1024;
        }
        c.ldsA = lds0 + A_RING + lw * 2048;
        c.ldsB = lds0 + lw * 4096;
        c.tn = tn;
    }
    c.lead = tm >= LEAD0 ? 8 * (tm - LEAD0) : -1;
    int wcol = g * NS + tn * BN;
    unsigned xo = tm * BM * KK;
    unsigned x0 = xo + korder<0>(c.lead, tn) * BK, w0 = wcol + korder<0>(c.lead, tn) * BK * NN;
    int skip = tm == 7 || wid < 4 ? -1 : g;
    asm volatile("" : "+v"(c.va[0]), "+v"(c.va[1]), "+v"(c.vb[0]), "+v"(c.vb[1]), "+v"(c.vb[2]), "+v"(c.vb[3]));
    asm volatile("" : "+s"(x0), "+s"(w0), "+s"(skip), "+s"(c.ldsA), "+s"(c.ldsB));
    asm volatile("s_waitcnt lgkmcnt(0)" : "+s"(ka));
    const unsigned char* px = (const unsigned char*)(((unsigned long long)(unsigned)ka[1] << 32) | (unsigned)ka[0]);
    const unsigned char* pw = (const unsigned char*)(((unsigned long long)(unsigned)ka[3] << 32) | (unsigned)ka[2]);
    const int rank = ka[6];
    unsigned char* myws = p.myws;
    const unsigned char* xa = px + xo;
    if (__builtin_expect(skip >= 0 && skip != rank, 1)) {
        load_x_at(c, px + x0, 0);
        load_w_at(c, pw + w0, 0);
        c.xa = xa;
        c.wb = pw + wcol;
        load_x<1>(c);
        load_range<1, issued(0)>(c);
    }
    const bool early = g == rank || tm == 7;
    const bool abB = (rank & 1) == 0;
    const unsigned t_begin = (unsigned)__builtin_amdgcn_s_memrealtime();
    c.t0 = t_begin;
    c.tend = abB ? 0 : T_PACE1;
    c.tau = tm >= LEAD0 + 2 ? TAU_B : TAU_A;
    unsigned* tmo = (unsigned*)(myws + WS_TMO);
    int orow = rank * MS + tm * BM, ocol = tn * BN;
    const bool local = early;
    bool lost = false;

    if (early) {
        u32x4 r[8];
        if (g != rank) {
            if (wid == 7) {
                touch((const unsigned char*)p.sx, 0u, lds0);
                touch((const unsigned char*)p.sw, 0u, lds0);
            }
            unsigned t;
            asm volatile("s_nop 4\n\t"
                         "global_load_dwordx4 %0, %9, %11\n\tglobal_load_dwordx4 %1, %9, %11 offset:1024\n\t"
                         "global_load_dwordx4 %2, %9, %11 offset:2048\n\tglobal_load_dwordx4 %3, %9, %11 offset:3072\n\t"
                         "global_load_dwordx4 %4, %10, %11\n\tglobal_load_dwordx4 %5, %10, %11 offset:1024\n\t"
                         "global_load_dwordx4 %6, %10, %11 offset:2048\n\tglobal_load_dwordx4 %7, %10, %11 offset:3072\n\t"
                         "s_cmp_lg_u32 %15, 0\n\ts_cbranch_scc1 1f\n\t"
                         "s_mov_b64 exec, 1\n\tglobal_atomic_add %8, %12, %13, %14 sc0\n\ts_mov_b64 exec, -1\n"
                         "1:\n\ts_waitcnt vmcnt(0)"
                         : "=&v"(r[0]), "=&v"(r[1]), "=&v"(r[2]), "=&v"(r[3]), "=&v"(r[4]), "=&v"(r[5]), "=&v"(r[6]), "=&v"(r[7]), "=&v"(t)
                         : "v"(lane * 16), "v"(lane * 16 + KK), "s"(xa + (tn * 16 + wid * 2) * KK),
                           "v"(0u), "v"(1u), "s"(myws + WS_CTR + b * 128), "s"(wid)
                         : "memory", "scc");
            if (tid == 0) *(LAS v4i*)(L + LDS_MISC) = (v4i){(int)(t + 1u), 0, 0, 0};
        } else {
            touch(px, (j * 512 + tid) * 128, lds0);
            while ((unsigned)__builtin_amdgcn_s_memrealtime() - t_begin < TWARM) __builtin_amdgcn_s_sleep(4);
            touch(pw + wcol, (tm * 512 + tid) * NN, lds0);
            if (wid == 0) {
                const unsigned t = draw_call(myws + WS_CTR + b * 128);
                if (lane == 0) *(LAS v4i*)(L + LDS_MISC) = (v4i){(int)t, 0, 0, 0};
            }
            WAIT_VM(0);
        }
        __syncthreads();
        const unsigned call = __builtin_amdgcn_readfirstlane(*(LAS unsigned*)(L + LDS_MISC));
        const unsigned par = call & 1u;
        if (g != rank) {
            unsigned char* pws = p.ws[g];
            unsigned char* dx = pws + WS_XSTAGE + (size_t)((par * 8 + rank) * BM + tn * 16 + wid * 2) * KK;
#pragma unroll
            for (int q = 0; q < 8; ++q)
                asm volatile("s_nop 4\n\tglobal_store_dwordx4 %0, %1, %2 sc0 sc1\n\ts_nop 1" :: "v"(lane * 16 + q * 1024), "v"(r[q]), "s"(dx) : "memory");
            WAIT_VM(0);
            __syncthreads();
            if (tid == 0) put_flag(pws + WS_XFLAG + ((par * 8 + rank) * 4 + tn) * 128, call, 0u);
            const int u = ((g - rank - 1) & 7) * 4 + tn;
            xa = px + (size_t)(u % 7) * BM * KK;
            ocol = (u / 7) * BN;
            wcol = rank * NS + ocol;
            orow = rank * MS + (u % 7) * BM;
        } else if (tm < 7) {
            const int src_rank = (rank + 1 + tm) & 7;
            if (wid == 0) {
                const unsigned f = wait_flag(myws + WS_XFLAG + (par * 8 + src_rank) * 512, (lane & 3) * 128, tmo, call, lane);
                const unsigned t = __hip_atomic_load(tmo, __ATOMIC_RELAXED, __HIP_MEMORY_SCOPE_AGENT);
                if (f != 0u || t == call) *(LAS unsigned*)(L + LDS_MISC + 4) = 1u;
            }
            __syncthreads();
            lost = __builtin_amdgcn_readfirstlane(*(LAS unsigned*)(L + LDS_MISC + 4)) != 0;
            xa = lost ? px : myws + WS_XSTAGE + (size_t)(par * 8 + src_rank) * BM * KK;
            orow = src_rank * MS + XROW0;
        }
    }

    v16f acc[2];
    unsigned char* dws = nullptr;

    if (wid >= 4) {
        if (early) {
            c.lead = -1;
            c.xa = xa;
            c.wb = pw + wcol;
            load_x<0>(c);
            load_range<0, issued(0)>(c);
        }
        dws = p.ws[g];
        asm volatile("" : "+s"(dws));
        loader_loop<0, NSTAGE - 2>(c);
        static_assert(issued(NSTAGE - 2) == NSTAGE && younger(NSTAGE - 2) == 6 && younger(NSTAGE - 1) == 0, "last two iterations");
        if (wid == 4) {
            unsigned t;
            float s0, s1;
            asm volatile("s_nop 4\n\tglobal_load_dword %1, %3, %6\n\tglobal_load_dword %2, %3, %7\n\t"
                         "s_mov_b64 exec, 1\n\tglobal_atomic_add %0, %3, %4, %5 sc0\n\ts_mov_b64 exec, -1\n\t"
                         "s_waitcnt vmcnt(9)\n\ts_barrier\n\ts_waitcnt vmcnt(3)\n\ts_barrier\n\ts_waitcnt vmcnt(0)"
                         : "=&v"(t), "=&v"(s0), "=&v"(s1)
                         : "v"(0u), "v"(early ? 0u : 1u), "s"(myws + WS_CTR + b * 128), "s"(p.sx), "s"(p.sw) : "memory");
            if (lane == 0)
                *(LAS v4i*)(L + LDS_MISC) = (v4i){(int)(t + 1u), 0, 0, lost ? 0x7fc00000 : __builtin_bit_cast(int, s0 * s1)};
            WAIT_LGKM0();
        } else {
            loader_loop<NSTAGE - 2, NSTAGE>(c);
        }
    } else {
        dws = p.ws[g];
        asm volatile("" : "+s"(dws));
        acc[0] = acc[1] = (v16f)0.f;
        const int r = lane & 31, h = lane >> 5;
        unsigned ao[2][2];
#pragma unroll
        for (int ks = 0; ks < 2; ++ks)
#pragma unroll
            for (int e = 0; e < 2; ++e) ao[ks][e] = A_RING + r * 128 + (((4 * ks + 2 * h + e) ^ ((r >> 1) & 7)) << 4);
        const int q = (lane & 15) >> 1, pp = lane & 1, g1 = (lane >> 4) & 1;
        const unsigned bo = (32 * h + q) * 128 + (((2 * wid + g1) ^ (q & 6)) << 4) + pp * 8;

        auto rd = [&](Frag& f, int slot, int aslot, int ks) {
            const unsigned char LAS* base = L + slot * B_BYTES;
            const unsigned char LAS* abase = L + aslot * A_BYTES;
#pragma unroll
            for (int t = 0; t < 4; ++t)
                f.b[ks][t] = __builtin_amdgcn_ds_read_tr8_b64_v2i32((LAS v2i*)(base + bo + (64 * ks + 8 * t) * 128));
#pragma unroll
            for (int mt = 0; mt < 2; ++mt)
#pragma unroll
                for (int e = 0; e < 2; ++e) f.a[ks][mt][e] = *(const LAS v4i*)(abase + ao[ks][e] + mt * 4096);
        };
        auto mm = [&](const Frag& f, int ks, int mt) {
            acc[mt] = __builtin_amdgcn_mfma_scale_f32_32x32x64_f8f6f4(cat8b(f.b[ks][0], f.b[ks][1], f.b[ks][2], f.b[ks][3]),
                                                                      cat8(f.a[ks][mt][0], f.a[ks][mt][1]), acc[mt], 0, 0, 0, 0, 0, 0);
        };
        auto step = [&](Frag& fn, const Frag& fo, int slot, int aslot) {
            BAR(); SCHED();
            mm(fo, 0, 0); SCHED();
            rd(fn, slot, aslot, 0); SCHED();
            mm(fo, 0, 1); SCHED();
            rd(fn, slot, aslot, 1); SCHED();
            mm(fo, 1, 0); mm(fo, 1, 1); SCHED();
            WAIT_LGKM0(); SCHED();
        };
        Frag f0, f1;
        BAR();
        rd(f0, 0, 0, 0);
        rd(f0, 0, 0, 1);
        WAIT_LGKM0();
        SCHED();
        int slot = 1, aslot = 1;
#pragma unroll 1
        for (int i = 1; i < NSTAGE - 1; i += 2) {
            step(f1, f0, slot, aslot);
            slot = slot + 1 == DEPTH ? 0 : slot + 1;
            aslot = aslot + 1 == ADEPTH ? 0 : aslot + 1;
            step(f0, f1, slot, aslot);
            slot = slot + 1 == DEPTH ? 0 : slot + 1;
            aslot = aslot + 1 == ADEPTH ? 0 : aslot + 1;
        }
        step(f1, f0, slot, aslot);
        mm(f1, 0, 0); mm(f1, 0, 1); mm(f1, 1, 0); mm(f1, 1, 1);
        if (!local) {
#pragma unroll
            for (int gq = 0; gq < 4; ++gq)
                *(LAS v4f*)(L + LDS_HAND + wid * 4096 + gq * 1024 + lane * 16) =
                    (v4f){acc[1][4 * gq], acc[1][4 * gq + 1], acc[1][4 * gq + 2], acc[1][4 * gq + 3]};
        }
        WAIT_LGKM0();
    }
    BAR();
    const unsigned d_gemm = (unsigned)__builtin_amdgcn_s_memrealtime() - t_begin;
    const v4i misc = *(LAS v4i*)(L + LDS_MISC);
    const float scale = __builtin_bit_cast(float, __builtin_amdgcn_readfirstlane(misc[3]));

    if (local) {
        if (wid < 4) {
            const int r = lane & 31, h = lane >> 5;
#pragma unroll
            for (int mt = 0; mt < 2; ++mt)
#pragma unroll
                for (int gq = 0; gq < 4; ++gq) {
                    v4f v;
#pragma unroll
                    for (int e = 0; e < 4; ++e) { const float y = acc[mt][4 * gq + e] * scale; v[e] = y <= 0.f ? 0.f : y; }
                    const int row = mt * 32 + r, ch = wid * 8 + 2 * gq + h;
                    *(LAS v4f*)(L + row * 512 + ((ch ^ (row & 7)) << 4)) = v;
                }
        }
        __syncthreads();
        float* ob = p.out + (size_t)orow * NS + ocol;
#pragma unroll
        for (int u = 0; u < 4; ++u) {
            const int row = (wid * 4 + u) * 2 + (lane >> 5), ch = lane & 31;
            store_out(ob + row * NS + ch * 4, *(LAS u32x4*)(L + row * 512 + ((ch ^ (row & 7)) << 4)));
        }
    } else {
        const unsigned lo = lane * 16;
        if (wid >= 4) {
            const LAS v4f* h = (const LAS v4f*)(L + LDS_HAND + (wid - 4) * 4096 + lo);
            acc[0] = __builtin_shufflevector(__builtin_shufflevector(h[0], h[64], 0, 1, 2, 3, 4, 5, 6, 7),
                                             __builtin_shufflevector(h[128], h[192], 0, 1, 2, 3, 4, 5, 6, 7),
                                             0, 1, 2, 3, 4, 5, 6, 7, 8, 9, 10, 11, 12, 13, 14, 15);
        }
        unsigned total = 0;
        bool dense = (((tm >= LEAD0 ? DENSE_L : DENSE_F) >> wid) & 1) &&
                     __builtin_bit_cast(unsigned, scale) - 0x0d800000u < 0x71800000u - 0x0d800000u;
        if (!dense) __builtin_amdgcn_s_setprio(3);
        if (dense) {
            float y[16];
            unsigned long long nz[16];
            unsigned mask = 0, m1 = 0, m2 = 0, inexact = 0;
            float top = 0.f;
#pragma unroll
            for (int q = 3; q >= 0; --q) {
                unsigned long long s1[4], s2[4];
#pragma unroll
                for (int e = 0; e < 4; ++e) {
                    const float a = acc[0][4 * q + e], am = __builtin_elementwise_minimum(a, 32.f), u = am + 32.f;
                    nz[4 * q + e] = __builtin_amdgcn_fcmpf(a, 0.f, 2  );
                    s1[e] = __builtin_amdgcn_fcmpf(a, 32.f, 3  );
                    s2[e] = __builtin_amdgcn_fcmpf(a, 128.f, 3);
                    inexact = __builtin_amdgcn_bitop3_b32(inexact, __builtin_bit_cast(unsigned, u - 32.f), __builtin_bit_cast(unsigned, am), 0xf6);
                    y[4 * q + e] = __builtin_amdgcn_inverse_ballot_w64(s1[e]) ? a : u;
                    top = __builtin_elementwise_maximum(top, a);
                }
#define AC(d, m) "v_addc_co_u32_e64 %" #d ", vcc, %" #d ", %" #d ", %" #m "\n\t"
                asm volatile("s_nop 1\n\t" AC(0, 6) AC(1, 10) AC(2, 14) AC(0, 5) AC(1, 9) AC(2, 13) AC(0, 4) AC(1, 8) AC(2, 12) AC(0, 3) AC(1, 7) AC(2, 11)
                             : "+v"(mask), "+v"(m1), "+v"(m2)
                             : "s"(nz[4 * q]), "s"(nz[4 * q + 1]), "s"(nz[4 * q + 2]), "s"(nz[4 * q + 3]),
                               "s"(s1[0]), "s"(s1[1]), "s"(s1[2]), "s"(s1[3]), "s"(s2[0]), "s"(s2[1]), "s"(s2[2]), "s"(s2[3])
                             : "vcc");
            }
            dense = __builtin_amdgcn_ballot_w64((inexact << 1) != 0u || !(top < 512.f)) == 0ull;
            if (dense) {
                *(LAS unsigned*)(L + wid * PK_SLOT + lane * 4) = (mask ^ m1 ^ m2) | (m1 << 16);
                total = compact(lds0 + LDS_RECV + wid * PK_SLOT, mask, y, nz);
#pragma unroll 1
                for (unsigned o = 0; o < total; o += 256) {
                    const u32x4 v = *(LAS u32x4*)(L + LDS_RECV + wid * PK_SLOT + o * 4 + lo);
                    LAS unsigned* d = (LAS unsigned*)(L + wid * PK_SLOT + 256 + o * 3 + lane * 12);
                    d[0] = __builtin_amdgcn_perm(v[1], v[0], 0x04020100u);
                    d[1] = __builtin_amdgcn_perm(v[2], v[1], 0x05040201u);
                    d[2] = __builtin_amdgcn_perm(v[3], v[2], 0x06050402u);
                }
            }
        }
        if (!dense) {
            float y[16];
            unsigned long long nz[16];
#pragma unroll
            for (int i = 0; i < 16; ++i) {
                y[i] = acc[0][i] * scale;
                nz[i] = __builtin_amdgcn_fcmpf(y[i], 0.f, 10  );
            }
            unsigned mask = 0;
#pragma unroll
            for (int q = 1; q >= 0; --q)
                asm volatile("s_nop 1\n\t" AC(0, 8) AC(0, 7) AC(0, 6) AC(0, 5) AC(0, 4) AC(0, 3) AC(0, 2) AC(0, 1)
                             : "+v"(mask)
                             : "s"(nz[8 * q]), "s"(nz[8 * q + 1]), "s"(nz[8 * q + 2]), "s"(nz[8 * q + 3]),
                               "s"(nz[8 * q + 4]), "s"(nz[8 * q + 5]), "s"(nz[8 * q + 6]), "s"(nz[8 * q + 7])
                             : "vcc");
            *(LAS unsigned short*)(L + wid * PK_SLOT + lane * 2) = (unsigned short)mask;
            total = compact(lds0 + wid * PK_SLOT + 128, mask, y, nz);
        }
        u32x4 c[3];
#pragma unroll
        for (int u = 0; u < 3; ++u) c[u] = *(LAS u32x4*)(L + wid * PK_SLOT + u * 1024 + lo);
        const unsigned call = __builtin_amdgcn_readfirstlane(misc[0]);
        const unsigned par = call & 1u;
        const unsigned bytes = dense ? 256 + 3 * total : 128 + 4 * total;
        const unsigned sidx = ((par * 8 + rank) * 32 + j) * 8 + wid;
        unsigned char* dslot = dws + WS_STAGE + (size_t)sidx * PK_SLOT;
        const int turn = wid - WIN;
        if (turn > 0)
            while ((int)__builtin_amdgcn_readfirstlane(__hip_atomic_load((LAS unsigned*)(L + LDS_MISC + 8), __ATOMIC_RELAXED,
                                                                         __HIP_MEMORY_SCOPE_WORKGROUP)) < turn)
                __builtin_amdgcn_s_sleep(1);
#pragma unroll
        for (int u = 0; u < 3; ++u)
            if (u * 1024 + lo < bytes)
                asm volatile("s_nop 4\n\tglobal_store_dwordx4 %0, %1, %2 sc0 sc1\n\ts_nop 1" :: "v"(lo + u * 1024), "v"(c[u]), "s"(dslot) : "memory");
#pragma unroll 1
        for (unsigned o = 3 * 1024; o < bytes; o += 1024) {
            const u32x4 v = *(LAS u32x4*)(L + wid * PK_SLOT + o + lo);
            if (o + lo < bytes)
                asm volatile("s_nop 4\n\tglobal_store_dwordx4 %0, %1, %2 sc0 sc1\n\ts_nop 1" :: "v"(lo + o), "v"(v), "s"(dslot) : "memory");
        }
        WAIT_VM(0);
        if (lane == 0) {
            put_flag(dws + WS_FLAG + (size_t)sidx * 128, call, total | (dense ? 0x10000u : 0u));
            __hip_atomic_fetch_add((LAS unsigned*)(L + LDS_MISC + 8), 1u, __ATOMIC_RELAXED, __HIP_MEMORY_SCOPE_WORKGROUP);
        }
        const unsigned d_sent = (unsigned)__builtin_amdgcn_s_memrealtime() - t_begin;
        const unsigned ridx = ((par * 8 + g) * 32 + j) * 8 + wid;
        const unsigned rword = wait_flag(myws + WS_FLAG + (size_t)ridx * 128, 0u, tmo, call, lane);
        const unsigned rtotal = __builtin_amdgcn_readfirstlane(rword) & ~0x10000u;
        const bool rdense = __builtin_amdgcn_readfirstlane(rword) & 0x10000u;
        u32x4 o[4];
        bool bad = true;
        if (rtotal <= 1024u) {
            const unsigned rbytes = rdense ? 256 + 3 * rtotal : 128 + 4 * rtotal;
            const unsigned char* sslot = myws + WS_STAGE + (size_t)ridx * PK_SLOT;
            const unsigned rdst = __builtin_amdgcn_readfirstlane(lds0 + LDS_RECV + wid * PK_SLOT);
            asm volatile("s_mov_b32 m0, %2\n\ts_nop 3\n\tglobal_load_lds_dwordx4 %0, %1 sc0 sc1\n\t"
                         "s_cmp_le_u32 %3, 1024\n\ts_cbranch_scc1 1f\n\tglobal_load_lds_dwordx4 %0, %1 offset:1024 sc0 sc1\n\t"
                         "s_cmp_le_u32 %3, 2048\n\ts_cbranch_scc1 1f\n\tglobal_load_lds_dwordx4 %0, %1 offset:2048 sc0 sc1\n\t"
                         "s_cmp_le_u32 %3, 3072\n\ts_cbranch_scc1 1f\n\tglobal_load_lds_dwordx4 %0, %1 offset:3072 sc0 sc1\n\t"
                         "s_cmp_le_u32 %3, 4096\n\ts_cbranch_scc1 1f\n\ts_mov_b32 m0, %5\n\ts_nop 0\n\tglobal_load_lds_dwordx4 %4, %1 sc0 sc1\n"
                         "1:"
                         :: "v"(lo), "s"(sslot), "s"(rdst), "s"(rbytes), "v"(lo + 4096), "s"(rdst + 4096) : "memory", "scc");
            unsigned t;
            asm volatile("s_nop 4\n\tglobal_load_dword %0, %1, %2 sc1\n\ts_waitcnt vmcnt(0)" : "=&v"(t) : "v"(0u), "s"(tmo) : "memory");
            bad = __builtin_amdgcn_readfirstlane(t) == call;
            const unsigned bp = 4 * ((lane & 7) >> 1);
            LAS v2i* tab = (LAS v2i*)(L + LDS_TAB + wid * 512);
            if (!rdense) {
                const unsigned rm = *(LAS unsigned short*)(L + LDS_RECV + wid * PK_SLOT + lane * 2);
                const unsigned rc = __builtin_popcount(rm);
                tab[lane] = (v2i){(int)rm, (int)(LDS_RECV + wid * PK_SLOT + 128 + (wave_scan(rc) - rc) * 4)};
#pragma unroll
                for (int u = 0; u < 4; ++u) {
                    const v2i tb = tab[8 * u + (lane >> 3) + 32 * (lane & 1)];
                    const unsigned m = (unsigned)tb[0];
                    int addr = tb[1] + 4 * __builtin_popcount(m & ((1u << bp) - 1u));
                    const int ms = (int)(m >> bp);
#pragma unroll
                    for (int e = 0; e < 4; ++e) {
                        const int sel = __builtin_amdgcn_sbfe(ms, e, 1);
                        o[u][e] = *(LAS unsigned*)(L + addr) & (unsigned)sel;
                        if (e < 3) addr = step4(addr, sel);
                    }
                }
            } else {
                const unsigned rm = *(LAS unsigned*)(L + LDS_RECV + wid * PK_SLOT + lane * 4);
#pragma unroll 1
                for (unsigned q = 0; q <= rtotal; q += 256) {
                    const LAS unsigned* d = (const LAS unsigned*)(L + LDS_RECV + wid * PK_SLOT + 256 + q * 3 + lane * 12);
                    const unsigned d0 = d[0], d1 = d[1], d2 = d[2];
                    *(LAS u32x4*)(L + wid * PK_SLOT + q * 4 + lo) =
                        (u32x4){(d0 & 0xffffffu) | 0x42000000u, __builtin_amdgcn_perm(d1, d0, 0x0c050403u) | 0x42000000u,
                                __builtin_amdgcn_perm(d2, d1, 0x0c040302u) | 0x42000000u, __builtin_amdgcn_alignbyte(0x42u, d2, 1)};
                }
                const unsigned rc = __builtin_popcount((rm | (rm >> 16)) & 0xffffu);
                tab[lane] = (v2i){(int)rm, (int)(wid * PK_SLOT + (wave_scan(rc) - rc) * 4)};
                const unsigned sb = __builtin_bit_cast(unsigned, scale), nb = __builtin_bit_cast(unsigned, -32.f * scale);
#pragma unroll
                for (int u = 0; u < 4; ++u) {
                    const v2i tb = tab[8 * u + (lane >> 3) + 32 * (lane & 1)];
                    const unsigned m = (unsigned)tb[0];
                    int addr = tb[1] + 4 * __builtin_popcount((m | (m >> 16)) & ((1u << bp) - 1u));
                    const int ms = (int)(m >> bp);
#pragma unroll
                    for (int e = 0; e < 4; ++e) {
                        const unsigned b0 = __builtin_amdgcn_sbfe(ms, e, 1), b1 = __builtin_amdgcn_sbfe(ms, 16 + e, 1);
                        const unsigned code = *(LAS unsigned*)(L + addr) | __builtin_amdgcn_bitop3_b32(b0, b1, 0x01000000u, 0x80);
                        o[u][e] = __builtin_bit_cast(unsigned, __builtin_fmaf(__builtin_bit_cast(float, code),
                                                                              __builtin_bit_cast(float, __builtin_amdgcn_bitop3_b32(b0, b1, sb, 0xa8)),
                                                                              __builtin_bit_cast(float, __builtin_amdgcn_bitop3_b32(b0, b1, nb, 0x20))));
                        if (e < 3) addr = step4(addr, (int)(b0 | b1));
                    }
                }
            }
        }
        float* ob = p.out + (size_t)(g * MS + tm * BM + (wid >> 2) * 32 + (lane >> 3)) * NS + tn * BN + (wid & 3) * 32 + (lane & 7) * 4;
#pragma unroll
        for (int u = 0; u < 4; ++u) {
            if (bad) o[u] = (u32x4){0x7fc00000u, 0x7fc00000u, 0x7fc00000u, 0x7fc00000u};
            store_out(ob + u * 8 * NS, o[u]);
        }
        while ((unsigned)__builtin_amdgcn_s_memrealtime() - t_begin < 8000u + b * 8u) __builtin_amdgcn_s_sleep(8);
        if (lane == 0) __hip_atomic_fetch_max((unsigned*)(p.ws[0] + WS_TMO + 1024 + (abB ? 128 : 0)), (call << 16) | min((unsigned)(d_sent), 65535u), __ATOMIC_RELAXED, __HIP_MEMORY_SCOPE_SYSTEM);
        if (rank == 0 && b == 1 && wid == 0) {
            while ((unsigned)__builtin_amdgcn_s_memrealtime() - t_begin < 25000u) __builtin_amdgcn_s_sleep(8);
            const unsigned e = __hip_atomic_load((unsigned*)(myws + WS_TMO + 1024), __ATOMIC_RELAXED, __HIP_MEMORY_SCOPE_SYSTEM) & 0xffffu;
            const unsigned od = __hip_atomic_load((unsigned*)(myws + WS_TMO + 1024 + 128), __ATOMIC_RELAXED, __HIP_MEMORY_SCOPE_SYSTEM) & 0xffffu;
            const unsigned qe = min(max((int)e - 1500, 0) / 5, 127), qo = min(max((int)od - 1500, 0) / 5, 127);
            const unsigned T = 30000u + (qe * 128u + qo) * 1000u;
            while ((unsigned)__builtin_amdgcn_s_memrealtime() - t_begin < __builtin_amdgcn_readfirstlane(T)) __builtin_amdgcn_s_sleep(8);
        }
    }
}

extern "C" void kernel_launch(int rank, int nranks, ncclComm_t comm, hipStream_t stream,
                              void* const* d_in, const int* in_sizes, int n_in,
                              void* d_out, int out_size, const int* in_dtypes, int out_dtype,
                              void* const* d_out_all, void* const* d_ws_all, size_t ws_bytes) {
    if (nranks != NR || n_in != 4 || in_sizes[0] != MS * KK || in_sizes[1] != KK * NN || out_size != NR * MS * NS ||
        (unsigned)rank >= (unsigned)NR || ws_bytes < WS_END) {
        fprintf(stderr, "unexpected problem shape\n");
        return;
    }
    Params p;
    p.x = (const unsigned char*)d_in[0];
    p.w = (const unsigned char*)d_in[1];
    p.sx = (const float*)d_in[2];
    p.sw = (const float*)d_in[3];
    p.out = (float*)d_out;
    p.myws = (unsigned char*)d_ws_all[rank];
    for (int i = 0; i < NR; ++i) p.ws[i] = (unsigned char*)d_ws_all[i];
    p.rank = rank;
    p.pad = 0;
    gemm_a2a<<<GRID, 512, 0, stream>>>(p);
    HIP_CHECK(hipGetLastError());
}
```

```cpp
#include <hip/hip_runtime.h>
#include <rccl/rccl.h>
#include <cstdio>
#include <cstddef>

#define HIP_CHECK(expr) do { hipError_t _e = (expr); if (_e != hipSuccess) { \
    fprintf(stderr, "HIP error %s at %s:%d\n", hipGetErrorString(_e), __FILE__, __LINE__); \
    return; } } while (0)

typedef int v8i __attribute__((ext_vector_type(8)));
typedef int v4i __attribute__((ext_vector_type(4)));
typedef int v2i __attribute__((ext_vector_type(2)));
typedef float v16f __attribute__((ext_vector_type(16)));
typedef float v4f __attribute__((ext_vector_type(4)));
typedef unsigned u32x4 __attribute__((ext_vector_type(4)));
#define LAS __attribute__((address_space(3)))

constexpr int NR = 8;
constexpr int MS = 512;
constexpr int KK = 4096;
constexpr int NN = 4096;
constexpr int NS = 512;
constexpr int BM = 64, BN = 128, BK = 128;
constexpr int NSTAGE = KK / BK;
constexpr int XROW0 = MS - BM;
constexpr int GRID = 256;
constexpr unsigned DEADLINE_TICKS = 50000000u;

constexpr int DEPTH_ = 4;
constexpr int LEAD0 = 3;
constexpr int TWARM = 500;
constexpr int PACE0 = 5;
constexpr int T_PACE0 = 400;
constexpr int T_PACE1 = 850;
constexpr int TAU_A = 26;
constexpr int TAU_B = 32;
constexpr int WIN = 2;
constexpr int DENSE_F = 0xfc;
constexpr int DENSE_L = 0x7f;

constexpr int DEPTH = DEPTH_;
constexpr int ADEPTH = DEPTH + 1;
constexpr int A_BYTES = BM * BK;
constexpr int B_BYTES = BK * BN;
constexpr int A_RING = DEPTH * B_BYTES;
constexpr int RING_BYTES = DEPTH * B_BYTES + ADEPTH * A_BYTES;
constexpr int PK_SLOT = 5120;
constexpr int LDS_RECV = 8 * PK_SLOT;
constexpr int LDS_TAB = 16 * PK_SLOT;
constexpr int LDS_HAND = RING_BYTES;
constexpr int LDS_MISC = LDS_HAND + 4 * 4096;
constexpr int LDS_BYTES = LDS_MISC + 16;
static_assert(LDS_TAB + 8 * 512 <= RING_BYTES && LDS_BYTES > 80 * 1024 && LDS_BYTES <= 160 * 1024, "LDS layout");

constexpr size_t WS_CTR = 0;
constexpr size_t WS_TMO = 32768;
constexpr size_t WS_XFLAG = 36864;
constexpr size_t WS_FLAG = 131072;
constexpr size_t WS_XSTAGE = WS_FLAG + 2 * 2048 * 128;
constexpr size_t WS_STAGE = WS_XSTAGE + (size_t)16 * BM * KK;
constexpr size_t WS_END = WS_STAGE + (size_t)2 * 2048 * PK_SLOT;

struct Params {
    const unsigned char* x;
    const unsigned char* w;
    unsigned char* myws;
    int rank;
    int pad;
    const float* sx;
    const float* sw;
    float* out;
    unsigned char* ws[NR];
};
static_assert(offsetof(Params, x) == 0 && offsetof(Params, w) == 8 && offsetof(Params, rank) == 24, "read as words in the kernel");

#define WAIT_VM(n) asm volatile("s_waitcnt vmcnt(%0)" :: "i"(n) : "memory")
#define WAIT_LGKM0() asm volatile("s_waitcnt lgkmcnt(0)" ::: "memory")
#define BAR() __builtin_amdgcn_s_barrier()
#define SCHED() __builtin_amdgcn_sched_barrier(0)

template <int S>
__device__ __forceinline__ int korder(int lead, int tn) {
    const int rs = (S + tn) & 3;
    const int kl = S < 8 ? lead + S : (lead + (S & ~3) + rs) & (NSTAGE - 1);
    return lead < 0 ? (S >> 2) + 8 * rs : kl;
}

struct LoadCtx {
    unsigned va[2], vb[4];
    const unsigned char* xa;
    const unsigned char* wb;
    unsigned ldsA, ldsB;
    int lead, tn;
    unsigned t0, tau;
};
__device__ __forceinline__ void load_w_at(const LoadCtx& c, const unsigned char* pb, int slot) {
    asm volatile("s_mov_b32 m0, %5\n\ts_nop 3\n\t"
                 "global_load_lds_dwordx4 %0, %4\n\tglobal_load_lds_dwordx4 %1, %4 offset:1024\n\t"
                 "global_load_lds_dwordx4 %2, %4 offset:2048\n\tglobal_load_lds_dwordx4 %3, %4 offset:3072"
                 :: "v"(c.vb[0]), "v"(c.vb[1]), "v"(c.vb[2]), "v"(c.vb[3]), "s"(pb), "s"(c.ldsB + slot * B_BYTES) : "memory");
}
__device__ __forceinline__ void load_x_at(const LoadCtx& c, const unsigned char* pa, int slot) {
    asm volatile("s_mov_b32 m0, %3\n\ts_nop 3\n\t"
                 "global_load_lds_dwordx4 %0, %2 sc0 sc1\n\tglobal_load_lds_dwordx4 %1, %2 offset:1024 sc0 sc1"
                 :: "v"(c.va[0]), "v"(c.va[1]), "s"(pa), "s"(c.ldsA + slot * A_BYTES) : "memory");
}
template <int S>
__device__ __forceinline__ void load_w(const LoadCtx& c) { load_w_at(c, c.wb + (unsigned)korder<S>(c.lead, c.tn) * (BK * NN), S % DEPTH); }
template <int S>
__device__ __forceinline__ void load_x(const LoadCtx& c) { load_x_at(c, c.xa + (unsigned)korder<S>(c.lead, c.tn) * BK, S % ADEPTH); }
constexpr int mini(int a, int b) { return a < b ? a : b; }
constexpr int issued(int I) { return mini(NSTAGE, mini(2 + 2 * I, I + DEPTH - 1)); }
constexpr int younger(int I) {
    return 4 * (issued(I) - 1 - I) + 2 * (mini(issued(I), NSTAGE - 1) - I);
}
template <int S, int E>
__device__ __forceinline__ void load_range(const LoadCtx& c) {
    if constexpr (S < E) {
        load_w<S>(c);
        if constexpr (S + 1 < NSTAGE) load_x<S + 1>(c);
        load_range<S + 1, E>(c);
    }
}
template <int I, int END>
__device__ __forceinline__ void loader_loop(const LoadCtx& c) {
    WAIT_VM(younger(I));
    BAR();
    if constexpr (I >= PACE0 && T_PACE0 + (I - PACE0) * TAU_A <= T_PACE1) {
        const unsigned due = T_PACE0 + (I - PACE0) * c.tau;
        if (c.lead >= 0 && due <= T_PACE1)
            while ((unsigned)__builtin_amdgcn_s_memrealtime() - c.t0 < due) __builtin_amdgcn_s_sleep(1);
    }
    if constexpr (I + 1 < NSTAGE) load_range<issued(I), issued(I + 1)>(c);
    if constexpr (I + 1 < END) loader_loop<I + 1, END>(c);
}

__device__ __forceinline__ unsigned wave_scan(unsigned v) {
    v += __builtin_amdgcn_update_dpp(0u, v, 0x111, 0xf, 0xf, true);
    v += __builtin_amdgcn_update_dpp(0u, v, 0x112, 0xf, 0xf, true);
    v += __builtin_amdgcn_update_dpp(0u, v, 0x114, 0xf, 0xf, true);
    v += __builtin_amdgcn_update_dpp(0u, v, 0x118, 0xf, 0xf, true);
    v += __builtin_amdgcn_update_dpp(0u, v, 0x142, 0xa, 0xf, false);
    v += __builtin_amdgcn_update_dpp(0u, v, 0x143, 0xc, 0xf, false);
    return v;
}

__device__ __forceinline__ unsigned draw_call(unsigned char* ctr) {
    unsigned t;
    asm volatile("s_mov_b64 exec, 1\n\ts_nop 3\n\tglobal_atomic_add %0, %1, %2, %3 sc0\n\ts_mov_b64 exec, -1\n\ts_waitcnt vmcnt(0)"
                 : "=&v"(t) : "v"(0u), "v"(1u), "s"(ctr) : "memory");
    return t + 1u;
}

__device__ __forceinline__ void touch(const unsigned char* base, unsigned voff, unsigned lds_dst) {
    asm volatile("s_mov_b32 m0, %2\n\ts_nop 3\n\tglobal_load_lds_dword %0, %1" :: "v"(voff), "s"(base), "s"(lds_dst) : "memory");
}

__device__ __forceinline__ void put_flag(unsigned char* flag, unsigned call, unsigned low) {
    typedef __attribute__((address_space(1))) unsigned long long gu64;
    __hip_atomic_store((gu64*)flag, ((unsigned long long)call << 32) | low, __ATOMIC_RELAXED, __HIP_MEMORY_SCOPE_SYSTEM);
}

__device__ __forceinline__ unsigned wait_flag(const unsigned char* flag, unsigned voff, unsigned* tmo, unsigned call, int lane) {
    const unsigned t0 = (unsigned)__builtin_amdgcn_s_memrealtime();
    unsigned low;
    bool seen;
    for (;;) {
        v2i f;
        unsigned t;
        asm volatile("s_nop 4\n\tglobal_load_dwordx2 %0, %2, %3 sc0 sc1\n\tglobal_load_dword %1, %4, %5 sc1\n\ts_waitcnt vmcnt(0)"
                     : "=&v"(f), "=&v"(t) : "v"(voff), "s"(flag), "v"(0u), "s"(tmo) : "memory");
        low = __builtin_amdgcn_readfirstlane(f[0]);
        seen = __builtin_amdgcn_ballot_w64((unsigned)f[1] < call) == 0ull;
        if (seen || __builtin_amdgcn_readfirstlane(t) == call || (unsigned)__builtin_amdgcn_s_memrealtime() - t0 > DEADLINE_TICKS) break;
        __builtin_amdgcn_s_sleep(2);
    }
    if (!seen && lane == 0) __hip_atomic_store(tmo, call, __ATOMIC_RELAXED, __HIP_MEMORY_SCOPE_AGENT);
    return seen ? low : 0xffffffffu;
}

__device__ __forceinline__ void store_out(float* ptr, u32x4 v) {
    asm volatile("global_store_dwordx4 %0, %1, off sc0 sc1\n\ts_nop 1" :: "v"(ptr), "v"(v) : "memory");
}

__device__ __forceinline__ unsigned compact(unsigned base, unsigned mask, const float (&y)[16], const unsigned long long (&nz)[16]) {
    const unsigned cnt = __builtin_popcount(mask);
    const unsigned incl = wave_scan(cnt);
    unsigned ptr = base + (incl - cnt) * 4;
#define PK1(m, v) "s_mov_b64 exec, %" #m "\n\tds_write_b32 %0, %" #v "\n\tv_add_u32 %0, 4, %0\n\t"
#pragma unroll
    for (int q = 0; q < 2; ++q)
        asm volatile(PK1(1, 9) PK1(2, 10) PK1(3, 11) PK1(4, 12) PK1(5, 13) PK1(6, 14) PK1(7, 15) PK1(8, 16)
                     "s_mov_b64 exec, -1"
                     : "+v"(ptr)
                     : "s"(nz[8 * q]), "s"(nz[8 * q + 1]), "s"(nz[8 * q + 2]), "s"(nz[8 * q + 3]),
                       "s"(nz[8 * q + 4]), "s"(nz[8 * q + 5]), "s"(nz[8 * q + 6]), "s"(nz[8 * q + 7]),
                       "v"(y[8 * q]), "v"(y[8 * q + 1]), "v"(y[8 * q + 2]), "v"(y[8 * q + 3]),
                       "v"(y[8 * q + 4]), "v"(y[8 * q + 5]), "v"(y[8 * q + 6]), "v"(y[8 * q + 7])
                     : "memory");
    return __builtin_amdgcn_readlane(incl, 63);
}

__device__ __forceinline__ int step4(int addr, int sel) {
    int r;
    asm("v_mad_i32_i24 %0, %1, -4, %2" : "=v"(r) : "v"(sel), "v"(addr));
    return r;
}

struct Frag { v4i a[2][2][2]; v2i b[2][4]; };

__device__ __forceinline__ v8i cat8(v4i lo, v4i hi) { return __builtin_shufflevector(lo, hi, 0, 1, 2, 3, 4, 5, 6, 7); }
__device__ __forceinline__ v8i cat8b(v2i a, v2i b, v2i c, v2i d) {
    return cat8(__builtin_shufflevector(a, b, 0, 1, 2, 3), __builtin_shufflevector(c, d, 0, 1, 2, 3));
}

__global__ void __launch_bounds__(512, 2) gemm_a2a(Params p) {
    __shared__ __attribute__((aligned(1024))) unsigned char lds[LDS_BYTES];
    v8i ka;
    asm volatile("s_load_dwordx8 %0, %1, 0x0" : "=s"(ka) : "s"(__builtin_amdgcn_kernarg_segment_ptr()));
    SCHED();
    LAS unsigned char* L = (LAS unsigned char*)lds;
    const unsigned lds0 = (unsigned)(size_t)L;
    const int tid = threadIdx.x, lane = tid & 63;
    const int wid = __builtin_amdgcn_readfirstlane(tid >> 6);
    const int b = blockIdx.x;
    const int g = b & 7, j = b >> 3, tm = j >> 2, tn = j & 3;
    LoadCtx c;
    {
        const int lw = wid & 3;
#pragma unroll
        for (int i = 0; i < 2; ++i) {
            const int row = (lw * 2 + i) * 8 + (lane >> 3);
            c.va[i] = row * KK + (((lane & 7) ^ ((row >> 1) & 7)) << 4) - i * 1024;
        }
#pragma unroll
        for (int i = 0; i < 4; ++i) {
            const int row = (lw * 4 + i) * 8 + (lane >> 3);
            c.vb[i] = row * NN + (((lane & 7) ^ (row & 6)) << 4) - i * 1024;
        }
        c.ldsA = lds0 + A_RING + lw * 2048;
        c.ldsB = lds0 + lw * 4096;
        c.tn = tn;
    }
    c.lead = tm >= LEAD0 ? 8 * (tm - LEAD0) : -1;
    int wcol = g * NS + tn * BN;
    unsigned xo = tm * BM * KK;
    unsigned x0 = xo + korder<0>(c.lead, tn) * BK, w0 = wcol + korder<0>(c.lead, tn) * BK * NN;
    int skip = tm == 7 || wid < 4 ? -1 : g;
    asm volatile("" : "+v"(c.va[0]), "+v"(c.va[1]), "+v"(c.vb[0]), "+v"(c.vb[1]), "+v"(c.vb[2]), "+v"(c.vb[3]));
    asm volatile("" : "+s"(x0), "+s"(w0), "+s"(skip), "+s"(c.ldsA), "+s"(c.ldsB));
    asm volatile("s_waitcnt lgkmcnt(0)" : "+s"(ka));
    const unsigned char* px = (const unsigned char*)(((unsigned long long)(unsigned)ka[1] << 32) | (unsigned)ka[0]);
    const unsigned char* pw = (const unsigned char*)(((unsigned long long)(unsigned)ka[3] << 32) | (unsigned)ka[2]);
    const int rank = ka[6];
    unsigned char* myws = p.myws;
    const unsigned char* xa = px + xo;
    if (__builtin_expect(skip >= 0 && skip != rank, 1)) {
        load_x_at(c, px + x0, 0);
        load_w_at(c, pw + w0, 0);
        c.xa = xa;
        c.wb = pw + wcol;
        load_x<1>(c);
        load_range<1, issued(0)>(c);
    }
    const bool early = g == rank || tm == 7;
    const unsigned t_begin = (unsigned)__builtin_amdgcn_s_memrealtime();
    c.t0 = t_begin;
    c.tau = tm >= LEAD0 + 2 ? TAU_B : TAU_A;
    unsigned* tmo = (unsigned*)(myws + WS_TMO);
    int orow = rank * MS + tm * BM, ocol = tn * BN;
    const bool local = early;
    bool lost = false;

    if (early) {
        u32x4 r[8];
        if (g != rank) {
            if (wid == 7) {
                touch((const unsigned char*)p.sx, 0u, lds0);
                touch((const unsigned char*)p.sw, 0u, lds0);
            }
            unsigned t;
            asm volatile("s_nop 4\n\t"
                         "global_load_dwordx4 %0, %9, %11\n\tglobal_load_dwordx4 %1, %9, %11 offset:1024\n\t"
                         "global_load_dwordx4 %2, %9, %11 offset:2048\n\tglobal_load_dwordx4 %3, %9, %11 offset:3072\n\t"
                         "global_load_dwordx4 %4, %10, %11\n\tglobal_load_dwordx4 %5, %10, %11 offset:1024\n\t"
                         "global_load_dwordx4 %6, %10, %11 offset:2048\n\tglobal_load_dwordx4 %7, %10, %11 offset:3072\n\t"
                         "s_cmp_lg_u32 %15, 0\n\ts_cbranch_scc1 1f\n\t"
                         "s_mov_b64 exec, 1\n\tglobal_atomic_add %8, %12, %13, %14 sc0\n\ts_mov_b64 exec, -1\n"
                         "1:\n\ts_waitcnt vmcnt(0)"
                         : "=&v"(r[0]), "=&v"(r[1]), "=&v"(r[2]), "=&v"(r[3]), "=&v"(r[4]), "=&v"(r[5]), "=&v"(r[6]), "=&v"(r[7]), "=&v"(t)
                         : "v"(lane * 16), "v"(lane * 16 + KK), "s"(xa + (tn * 16 + wid * 2) * KK),
                           "v"(0u), "v"(1u), "s"(myws + WS_CTR + b * 128), "s"(wid)
                         : "memory", "scc");
            if (tid == 0) *(LAS v4i*)(L + LDS_MISC) = (v4i){(int)(t + 1u), 0, 0, 0};
        } else {
            touch(px, (j * 512 + tid) * 128, lds0);
            while ((unsigned)__builtin_amdgcn_s_memrealtime() - t_begin < TWARM) __builtin_amdgcn_s_sleep(4);
            touch(pw + wcol, (tm * 512 + tid) * NN, lds0);
            if (wid == 0) {
                const unsigned t = draw_call(myws + WS_CTR + b * 128);
                if (lane == 0) *(LAS v4i*)(L + LDS_MISC) = (v4i){(int)t, 0, 0, 0};
            }
            WAIT_VM(0);
        }
        __syncthreads();
        const unsigned call = __builtin_amdgcn_readfirstlane(*(LAS unsigned*)(L + LDS_MISC));
        const unsigned par = call & 1u;
        if (g != rank) {
            unsigned char* pws = p.ws[g];
            unsigned char* dx = pws + WS_XSTAGE + (size_t)((par * 8 + rank) * BM + tn * 16 + wid * 2) * KK;
#pragma unroll
            for (int q = 0; q < 8; ++q)
                asm volatile("s_nop 4\n\tglobal_store_dwordx4 %0, %1, %2 sc0 sc1\n\ts_nop 1" :: "v"(lane * 16 + q * 1024), "v"(r[q]), "s"(dx) : "memory");
            WAIT_VM(0);
            __syncthreads();
            if (tid == 0) put_flag(pws + WS_XFLAG + ((par * 8 + rank) * 4 + tn) * 128, call, 0u);
            const int u = ((g - rank - 1) & 7) * 4 + tn;
            xa = px + (size_t)(u % 7) * BM * KK;
            ocol = (u / 7) * BN;
            wcol = rank * NS + ocol;
            orow = rank * MS + (u % 7) * BM;
        } else if (tm < 7) {
            const int src_rank = (rank + 1 + tm) & 7;
            if (wid == 0) {
                const unsigned f = wait_flag(myws + WS_XFLAG + (par * 8 + src_rank) * 512, (lane & 3) * 128, tmo, call, lane);
                const unsigned t = __hip_atomic_load(tmo, __ATOMIC_RELAXED, __HIP_MEMORY_SCOPE_AGENT);
                if (f != 0u || t == call) *(LAS unsigned*)(L + LDS_MISC + 4) = 1u;
            }
            __syncthreads();
            lost = __builtin_amdgcn_readfirstlane(*(LAS unsigned*)(L + LDS_MISC + 4)) != 0;
            xa = lost ? px : myws + WS_XSTAGE + (size_t)(par * 8 + src_rank) * BM * KK;
            orow = src_rank * MS + XROW0;
        }
    }

    v16f acc[2];
    unsigned char* dws = nullptr;

    if (wid >= 4) {
        if (early) {
            c.lead = -1;
            c.xa = xa;
            c.wb = pw + wcol;
            load_x<0>(c);
            load_range<0, issued(0)>(c);
        }
        dws = p.ws[g];
        asm volatile("" : "+s"(dws));
        loader_loop<0, NSTAGE - 2>(c);
        static_assert(issued(NSTAGE - 2) == NSTAGE && younger(NSTAGE - 2) == 6 && younger(NSTAGE - 1) == 0, "last two iterations");
        if (wid == 4) {
            unsigned t;
            float s0, s1;
            asm volatile("s_nop 4\n\tglobal_load_dword %1, %3, %6\n\tglobal_load_dword %2, %3, %7\n\t"
                         "s_mov_b64 exec, 1\n\tglobal_atomic_add %0, %3, %4, %5 sc0\n\ts_mov_b64 exec, -1\n\t"
                         "s_waitcnt vmcnt(9)\n\ts_barrier\n\ts_waitcnt vmcnt(3)\n\ts_barrier\n\ts_waitcnt vmcnt(0)"
                         : "=&v"(t), "=&v"(s0), "=&v"(s1)
                         : "v"(0u), "v"(early ? 0u : 1u), "s"(myws + WS_CTR + b * 128), "s"(p.sx), "s"(p.sw) : "memory");
            if (lane == 0)
                *(LAS v4i*)(L + LDS_MISC) = (v4i){(int)(t + 1u), 0, 0, lost ? 0x7fc00000 : __builtin_bit_cast(int, s0 * s1)};
            WAIT_LGKM0();
        } else {
            loader_loop<NSTAGE - 2, NSTAGE>(c);
        }
    } else {
        dws = p.ws[g];
        asm volatile("" : "+s"(dws));
        acc[0] = acc[1] = (v16f)0.f;
        const int r = lane & 31, h = lane >> 5;
        unsigned ao[2][2];
#pragma unroll
        for (int ks = 0; ks < 2; ++ks)
#pragma unroll
            for (int e = 0; e < 2; ++e) ao[ks][e] = A_RING + r * 128 + (((4 * ks + 2 * h + e) ^ ((r >> 1) & 7)) << 4);
        const int q = (lane & 15) >> 1, pp = lane & 1, g1 = (lane >> 4) & 1;
        const unsigned bo = (32 * h + q) * 128 + (((2 * wid + g1) ^ (q & 6)) << 4) + pp * 8;

        auto rd = [&](Frag& f, int slot, int aslot, int ks) {
            const unsigned char LAS* base = L + slot * B_BYTES;
            const unsigned char LAS* abase = L + aslot * A_BYTES;
#pragma unroll
            for (int t = 0; t < 4; ++t)
                f.b[ks][t] = __builtin_amdgcn_ds_read_tr8_b64_v2i32((LAS v2i*)(base + bo + (64 * ks + 8 * t) * 128));
#pragma unroll
            for (int mt = 0; mt < 2; ++mt)
#pragma unroll
                for (int e = 0; e < 2; ++e) f.a[ks][mt][e] = *(const LAS v4i*)(abase + ao[ks][e] + mt * 4096);
        };
        auto mm = [&](const Frag& f, int ks, int mt) {
            acc[mt] = __builtin_amdgcn_mfma_scale_f32_32x32x64_f8f6f4(cat8b(f.b[ks][0], f.b[ks][1], f.b[ks][2], f.b[ks][3]),
                                                                      cat8(f.a[ks][mt][0], f.a[ks][mt][1]), acc[mt], 0, 0, 0, 0, 0, 0);
        };
        auto step = [&](Frag& fn, const Frag& fo, int slot, int aslot) {
            BAR(); SCHED();
            mm(fo, 0, 0); SCHED();
            rd(fn, slot, aslot, 0); SCHED();
            mm(fo, 0, 1); SCHED();
            rd(fn, slot, aslot, 1); SCHED();
            mm(fo, 1, 0); mm(fo, 1, 1); SCHED();
            WAIT_LGKM0(); SCHED();
        };
        Frag f0, f1;
        BAR();
        rd(f0, 0, 0, 0);
        rd(f0, 0, 0, 1);
        WAIT_LGKM0();
        SCHED();
        int slot = 1, aslot = 1;
#pragma unroll 1
        for (int i = 1; i < NSTAGE - 1; i += 2) {
            step(f1, f0, slot, aslot);
            slot = slot + 1 == DEPTH ? 0 : slot + 1;
            aslot = aslot + 1 == ADEPTH ? 0 : aslot + 1;
            step(f0, f1, slot, aslot);
            slot = slot + 1 == DEPTH ? 0 : slot + 1;
            aslot = aslot + 1 == ADEPTH ? 0 : aslot + 1;
        }
        step(f1, f0, slot, aslot);
        mm(f1, 0, 0); mm(f1, 0, 1); mm(f1, 1, 0); mm(f1, 1, 1);
        if (!local) {
#pragma unroll
            for (int gq = 0; gq < 4; ++gq)
                *(LAS v4f*)(L + LDS_HAND + wid * 4096 + gq * 1024 + lane * 16) =
                    (v4f){acc[1][4 * gq], acc[1][4 * gq + 1], acc[1][4 * gq + 2], acc[1][4 * gq + 3]};
        }
        WAIT_LGKM0();
    }
    BAR();
    const unsigned d_gemm = (unsigned)__builtin_amdgcn_s_memrealtime() - t_begin;
    const v4i misc = *(LAS v4i*)(L + LDS_MISC);
    const float scale = __builtin_bit_cast(float, __builtin_amdgcn_readfirstlane(misc[3]));

    if (local) {
        if (wid < 4) {
            const int r = lane & 31, h = lane >> 5;
#pragma unroll
            for (int mt = 0; mt < 2; ++mt)
#pragma unroll
                for (int gq = 0; gq < 4; ++gq) {
                    v4f v;
#pragma unroll
                    for (int e = 0; e < 4; ++e) { const float y = acc[mt][4 * gq + e] * scale; v[e] = y <= 0.f ? 0.f : y; }
                    const int row = mt * 32 + r, ch = wid * 8 + 2 * gq + h;
                    *(LAS v4f*)(L + row * 512 + ((ch ^ (row & 7)) << 4)) = v;
                }
        }
        __syncthreads();
        float* ob = p.out + (size_t)orow * NS + ocol;
#pragma unroll
        for (int u = 0; u < 4; ++u) {
            const int row = (wid * 4 + u) * 2 + (lane >> 5), ch = lane & 31;
            store_out(ob + row * NS + ch * 4, *(LAS u32x4*)(L + row * 512 + ((ch ^ (row & 7)) << 4)));
        }
    } else {
        const unsigned lo = lane * 16;
        if (wid >= 4) {
            const LAS v4f* h = (const LAS v4f*)(L + LDS_HAND + (wid - 4) * 4096 + lo);
            acc[0] = __builtin_shufflevector(__builtin_shufflevector(h[0], h[64], 0, 1, 2, 3, 4, 5, 6, 7),
                                             __builtin_shufflevector(h[128], h[192], 0, 1, 2, 3, 4, 5, 6, 7),
                                             0, 1, 2, 3, 4, 5, 6, 7, 8, 9, 10, 11, 12, 13, 14, 15);
        }
        unsigned total = 0;
        bool dense = (((tm >= LEAD0 ? DENSE_L : DENSE_F) >> wid) & 1) &&
                     __builtin_bit_cast(unsigned, scale) - 0x0d800000u < 0x71800000u - 0x0d800000u;
        if (!dense) __builtin_amdgcn_s_setprio(3);
        if (dense) {
            float y[16];
            unsigned long long nz[16];
            unsigned mask = 0, m1 = 0, m2 = 0, inexact = 0;
            float top = 0.f;
#pragma unroll
            for (int q = 3; q >= 0; --q) {
                unsigned long long s1[4], s2[4];
#pragma unroll
                for (int e = 0; e < 4; ++e) {
                    const float a = acc[0][4 * q + e], am = __builtin_elementwise_minimum(a, 32.f), u = am + 32.f;
                    nz[4 * q + e] = __builtin_amdgcn_fcmpf(a, 0.f, 2  );
                    s1[e] = __builtin_amdgcn_fcmpf(a, 32.f, 3  );
                    s2[e] = __builtin_amdgcn_fcmpf(a, 128.f, 3);
                    inexact = __builtin_amdgcn_bitop3_b32(inexact, __builtin_bit_cast(unsigned, u - 32.f), __builtin_bit_cast(unsigned, am), 0xf6);
                    y[4 * q + e] = __builtin_amdgcn_inverse_ballot_w64(s1[e]) ? a : u;
                    top = __builtin_elementwise_maximum(top, a);
                }
#define AC(d, m) "v_addc_co_u32_e64 %" #d ", vcc, %" #d ", %" #d ", %" #m "\n\t"
                asm volatile("s_nop 1\n\t" AC(0, 6) AC(1, 10) AC(2, 14) AC(0, 5) AC(1, 9) AC(2, 13) AC(0, 4) AC(1, 8) AC(2, 12) AC(0, 3) AC(1, 7) AC(2, 11)
                             : "+v"(mask), "+v"(m1), "+v"(m2)
                             : "s"(nz[4 * q]), "s"(nz[4 * q + 1]), "s"(nz[4 * q + 2]), "s"(nz[4 * q + 3]),
                               "s"(s1[0]), "s"(s1[1]), "s"(s1[2]), "s"(s1[3]), "s"(s2[0]), "s"(s2[1]), "s"(s2[2]), "s"(s2[3])
                             : "vcc");
            }
            dense = __builtin_amdgcn_ballot_w64((inexact << 1) != 0u || !(top < 512.f)) == 0ull;
            if (dense) {
                *(LAS unsigned*)(L + wid * PK_SLOT + lane * 4) = (mask ^ m1 ^ m2) | (m1 << 16);
                total = compact(lds0 + LDS_RECV + wid * PK_SLOT, mask, y, nz);
#pragma unroll 1
                for (unsigned o = 0; o < total; o += 256) {
                    const u32x4 v = *(LAS u32x4*)(L + LDS_RECV + wid * PK_SLOT + o * 4 + lo);
                    LAS unsigned* d = (LAS unsigned*)(L + wid * PK_SLOT + 256 + o * 3 + lane * 12);
                    d[0] = __builtin_amdgcn_perm(v[1], v[0], 0x04020100u);
                    d[1] = __builtin_amdgcn_perm(v[2], v[1], 0x05040201u);
                    d[2] = __builtin_amdgcn_perm(v[3], v[2], 0x06050402u);
                }
            }
        }
        if (!dense) {
            float y[16];
            unsigned long long nz[16];
#pragma unroll
            for (int i = 0; i < 16; ++i) {
                y[i] = acc[0][i] * scale;
                nz[i] = __builtin_amdgcn_fcmpf(y[i], 0.f, 10  );
            }
            unsigned mask = 0;
#pragma unroll
            for (int q = 1; q >= 0; --q)
                asm volatile("s_nop 1\n\t" AC(0, 8) AC(0, 7) AC(0, 6) AC(0, 5) AC(0, 4) AC(0, 3) AC(0, 2) AC(0, 1)
                             : "+v"(mask)
                             : "s"(nz[8 * q]), "s"(nz[8 * q + 1]), "s"(nz[8 * q + 2]), "s"(nz[8 * q + 3]),
                               "s"(nz[8 * q + 4]), "s"(nz[8 * q + 5]), "s"(nz[8 * q + 6]), "s"(nz[8 * q + 7])
                             : "vcc");
            *(LAS unsigned short*)(L + wid * PK_SLOT + lane * 2) = (unsigned short)mask;
            total = compact(lds0 + wid * PK_SLOT + 128, mask, y, nz);
        }
        u32x4 c[3];
#pragma unroll
        for (int u = 0; u < 3; ++u) c[u] = *(LAS u32x4*)(L + wid * PK_SLOT + u * 1024 + lo);
        const unsigned call = __builtin_amdgcn_readfirstlane(misc[0]);
        const unsigned par = call & 1u;
        const unsigned bytes = dense ? 256 + 3 * total : 128 + 4 * total;
        const unsigned sidx = ((par * 8 + rank) * 32 + j) * 8 + wid;
        unsigned char* dslot = dws + WS_STAGE + (size_t)sidx * PK_SLOT;
        const int turn = wid - WIN;
        if (turn > 0)
            while ((int)__builtin_amdgcn_readfirstlane(__hip_atomic_load((LAS unsigned*)(L + LDS_MISC + 8), __ATOMIC_RELAXED,
                                                                         __HIP_MEMORY_SCOPE_WORKGROUP)) < turn)
                __builtin_amdgcn_s_sleep(1);
        const unsigned d_go = (unsigned)__builtin_amdgcn_s_memrealtime() - t_begin;
#pragma unroll
        for (int u = 0; u < 3; ++u)
            if (u * 1024 + lo < bytes)
                asm volatile("s_nop 4\n\tglobal_store_dwordx4 %0, %1, %2 sc0 sc1\n\ts_nop 1" :: "v"(lo + u * 1024), "v"(c[u]), "s"(dslot) : "memory");
#pragma unroll 1
        for (unsigned o = 3 * 1024; o < bytes; o += 1024) {
            const u32x4 v = *(LAS u32x4*)(L + wid * PK_SLOT + o + lo);
            if (o + lo < bytes)
                asm volatile("s_nop 4\n\tglobal_store_dwordx4 %0, %1, %2 sc0 sc1\n\ts_nop 1" :: "v"(lo + o), "v"(v), "s"(dslot) : "memory");
        }
        WAIT_VM(0);
        if (lane == 0) {
            put_flag(dws + WS_FLAG + (size_t)sidx * 128, call, total | (dense ? 0x10000u : 0u));
            __hip_atomic_fetch_add((LAS unsigned*)(L + LDS_MISC + 8), 1u, __ATOMIC_RELAXED, __HIP_MEMORY_SCOPE_WORKGROUP);
        }
        const unsigned d_sent = (unsigned)__builtin_amdgcn_s_memrealtime() - t_begin;
        const unsigned ridx = ((par * 8 + g) * 32 + j) * 8 + wid;
        const unsigned rword = wait_flag(myws + WS_FLAG + (size_t)ridx * 128, 0u, tmo, call, lane);
        const unsigned d_seen = (unsigned)__builtin_amdgcn_s_memrealtime() - t_begin;
        const unsigned rtotal = __builtin_amdgcn_readfirstlane(rword) & ~0x10000u;
        const bool rdense = __builtin_amdgcn_readfirstlane(rword) & 0x10000u;
        u32x4 o[4];
        bool bad = true;
        if (rtotal <= 1024u) {
            const unsigned rbytes = rdense ? 256 + 3 * rtotal : 128 + 4 * rtotal;
            const unsigned char* sslot = myws + WS_STAGE + (size_t)ridx * PK_SLOT;
            const unsigned rdst = __builtin_amdgcn_readfirstlane(lds0 + LDS_RECV + wid * PK_SLOT);
            asm volatile("s_mov_b32 m0, %2\n\ts_nop 3\n\tglobal_load_lds_dwordx4 %0, %1 sc0 sc1\n\t"
                         "s_cmp_le_u32 %3, 1024\n\ts_cbranch_scc1 1f\n\tglobal_load_lds_dwordx4 %0, %1 offset:1024 sc0 sc1\n\t"
                         "s_cmp_le_u32 %3, 2048\n\ts_cbranch_scc1 1f\n\tglobal_load_lds_dwordx4 %0, %1 offset:2048 sc0 sc1\n\t"
                         "s_cmp_le_u32 %3, 3072\n\ts_cbranch_scc1 1f\n\tglobal_load_lds_dwordx4 %0, %1 offset:3072 sc0 sc1\n\t"
                         "s_cmp_le_u32 %3, 4096\n\ts_cbranch_scc1 1f\n\ts_mov_b32 m0, %5\n\ts_nop 0\n\tglobal_load_lds_dwordx4 %4, %1 sc0 sc1\n"
                         "1:"
                         :: "v"(lo), "s"(sslot), "s"(rdst), "s"(rbytes), "v"(lo + 4096), "s"(rdst + 4096) : "memory", "scc");
            unsigned t;
            asm volatile("s_nop 4\n\tglobal_load_dword %0, %1, %2 sc1\n\ts_waitcnt vmcnt(0)" : "=&v"(t) : "v"(0u), "s"(tmo) : "memory");
            bad = __builtin_amdgcn_readfirstlane(t) == call;
            const unsigned bp = 4 * ((lane & 7) >> 1);
            LAS v2i* tab = (LAS v2i*)(L + LDS_TAB + wid * 512);
            if (!rdense) {
                const unsigned rm = *(LAS unsigned short*)(L + LDS_RECV + wid * PK_SLOT + lane * 2);
                const unsigned rc = __builtin_popcount(rm);
                tab[lane] = (v2i){(int)rm, (int)(LDS_RECV + wid * PK_SLOT + 128 + (wave_scan(rc) - rc) * 4)};
#pragma unroll
                for (int u = 0; u < 4; ++u) {
                    const v2i tb = tab[8 * u + (lane >> 3) + 32 * (lane & 1)];
                    const unsigned m = (unsigned)tb[0];
                    int addr = tb[1] + 4 * __builtin_popcount(m & ((1u << bp) - 1u));
                    const int ms = (int)(m >> bp);
#pragma unroll
                    for (int e = 0; e < 4; ++e) {
                        const int sel = __builtin_amdgcn_sbfe(ms, e, 1);
                        o[u][e] = *(LAS unsigned*)(L + addr) & (unsigned)sel;
                        if (e < 3) addr = step4(addr, sel);
                    }
                }
            } else {
                const unsigned rm = *(LAS unsigned*)(L + LDS_RECV + wid * PK_SLOT + lane * 4);
#pragma unroll 1
                for (unsigned q = 0; q <= rtotal; q += 256) {
                    const LAS unsigned* d = (const LAS unsigned*)(L + LDS_RECV + wid * PK_SLOT + 256 + q * 3 + lane * 12);
                    const unsigned d0 = d[0], d1 = d[1], d2 = d[2];
                    *(LAS u32x4*)(L + wid * PK_SLOT + q * 4 + lo) =
                        (u32x4){(d0 & 0xffffffu) | 0x42000000u, __builtin_amdgcn_perm(d1, d0, 0x0c050403u) | 0x42000000u,
                                __builtin_amdgcn_perm(d2, d1, 0x0c040302u) | 0x42000000u, __builtin_amdgcn_alignbyte(0x42u, d2, 1)};
                }
                const unsigned rc = __builtin_popcount((rm | (rm >> 16)) & 0xffffu);
                tab[lane] = (v2i){(int)rm, (int)(wid * PK_SLOT + (wave_scan(rc) - rc) * 4)};
                const unsigned sb = __builtin_bit_cast(unsigned, scale), nb = __builtin_bit_cast(unsigned, -32.f * scale);
#pragma unroll
                for (int u = 0; u < 4; ++u) {
                    const v2i tb = tab[8 * u + (lane >> 3) + 32 * (lane & 1)];
                    const unsigned m = (unsigned)tb[0];
                    int addr = tb[1] + 4 * __builtin_popcount((m | (m >> 16)) & ((1u << bp) - 1u));
                    const int ms = (int)(m >> bp);
#pragma unroll
                    for (int e = 0; e < 4; ++e) {
                        const unsigned b0 = __builtin_amdgcn_sbfe(ms, e, 1), b1 = __builtin_amdgcn_sbfe(ms, 16 + e, 1);
                        const unsigned code = *(LAS unsigned*)(L + addr) | __builtin_amdgcn_bitop3_b32(b0, b1, 0x01000000u, 0x80);
                        o[u][e] = __builtin_bit_cast(unsigned, __builtin_fmaf(__builtin_bit_cast(float, code),
                                                                              __builtin_bit_cast(float, __builtin_amdgcn_bitop3_b32(b0, b1, sb, 0xa8)),
                                                                              __builtin_bit_cast(float, __builtin_amdgcn_bitop3_b32(b0, b1, nb, 0x20))));
                        if (e < 3) addr = step4(addr, (int)(b0 | b1));
                    }
                }
            }
        }
        float* ob = p.out + (size_t)(g * MS + tm * BM + (wid >> 2) * 32 + (lane >> 3)) * NS + tn * BN + (wid & 3) * 32 + (lane & 7) * 4;
#pragma unroll
        for (int u = 0; u < 4; ++u) {
            if (bad) o[u] = (u32x4){0x7fc00000u, 0x7fc00000u, 0x7fc00000u, 0x7fc00000u};
            store_out(ob + u * 8 * NS, o[u]);
        }
        WAIT_VM(0);
        const unsigned d_end = (unsigned)__builtin_amdgcn_s_memrealtime() - t_begin;
        while ((unsigned)__builtin_amdgcn_s_memrealtime() - t_begin < 8000u + b * 8u) __builtin_amdgcn_s_sleep(8);
        if (lane == 0) {
            __hip_atomic_fetch_max((unsigned*)(p.ws[0] + WS_TMO + 1024), (call << 16) | min((unsigned)(d_sent), 65535u), __ATOMIC_RELAXED, __HIP_MEMORY_SCOPE_SYSTEM);
            __hip_atomic_fetch_max((unsigned*)(p.ws[0] + WS_TMO + 1024 + 128), (call << 16) | min((unsigned)(d_gemm + 500u), 65535u), __ATOMIC_RELAXED, __HIP_MEMORY_SCOPE_SYSTEM);
        }
        if (rank == 0 && b == 1 && wid == 0) {
            while ((unsigned)__builtin_amdgcn_s_memrealtime() - t_begin < 25000u) __builtin_amdgcn_s_sleep(8);
            const unsigned e = __hip_atomic_load((unsigned*)(myws + WS_TMO + 1024), __ATOMIC_RELAXED, __HIP_MEMORY_SCOPE_SYSTEM) & 0xffffu;
            const unsigned od = __hip_atomic_load((unsigned*)(myws + WS_TMO + 1024 + 128), __ATOMIC_RELAXED, __HIP_MEMORY_SCOPE_SYSTEM) & 0xffffu;
            const unsigned qe = min(max((int)e - 1500, 0) / 5, 127), qo = min(max((int)od - 1500, 0) / 5, 127);
            const unsigned T = 30000u + (qe * 128u + qo) * 1000u;
            while ((unsigned)__builtin_amdgcn_s_memrealtime() - t_begin < __builtin_amdgcn_readfirstlane(T)) __builtin_amdgcn_s_sleep(8);
        }
    }
}

extern "C" void kernel_launch(int rank, int nranks, ncclComm_t comm, hipStream_t stream,
                              void* const* d_in, const int* in_sizes, int n_in,
                              void* d_out, int out_size, const int* in_dtypes, int out_dtype,
                              void* const* d_out_all, void* const* d_ws_all, size_t ws_bytes) {
    if (nranks != NR || n_in != 4 || in_sizes[0] != MS * KK || in_sizes[1] != KK * NN || out_size != NR * MS * NS ||
        (unsigned)rank >= (unsigned)NR || ws_bytes < WS_END) {
        fprintf(stderr, "unexpected problem shape\n");
        return;
    }
    Params p;
    p.x = (const unsigned char*)d_in[0];
    p.w = (const unsigned char*)d_in[1];
    p.sx = (const float*)d_in[2];
    p.sw = (const float*)d_in[3];
    p.out = (float*)d_out;
    p.myws = (unsigned char*)d_ws_all[rank];
    for (int i = 0; i < NR; ++i) p.ws[i] = (unsigned char*)d_ws_all[i];
    p.rank = rank;
    p.pad = 0;
    gemm_a2a<<<GRID, 512, 0, stream>>>(p);
    HIP_CHECK(hipGetLastError());
}
```

```cpp
#include <hip/hip_runtime.h>
#include <rccl/rccl.h>
#include <cstdio>
#include <cstddef>

#define HIP_CHECK(expr) do { hipError_t _e = (expr); if (_e != hipSuccess) { \
    fprintf(stderr, "HIP error %s at %s:%d\n", hipGetErrorString(_e), __FILE__, __LINE__); \
    return; } } while (0)

typedef int v8i __attribute__((ext_vector_type(8)));
typedef int v4i __attribute__((ext_vector_type(4)));
typedef int v2i __attribute__((ext_vector_type(2)));
typedef float v16f __attribute__((ext_vector_type(16)));
typedef float v4f __attribute__((ext_vector_type(4)));
typedef unsigned u32x4 __attribute__((ext_vector_type(4)));
#define LAS __attribute__((address_space(3)))

constexpr int NR = 8;
constexpr int MS = 512;
constexpr int KK = 4096;
constexpr int NN = 4096;
constexpr int NS = 512;
constexpr int BM = 64, BN = 128, BK = 128;
constexpr int NSTAGE = KK / BK;
constexpr int XROW0 = MS - BM;
constexpr int GRID = 256;
constexpr unsigned DEADLINE_TICKS = 50000000u;

constexpr int LEAD0 = 3;
constexpr int TWARM = 500;
constexpr int PACE0 = 5;
constexpr int T_PACE0 = 400;
constexpr int T_PACE1 = 850;
constexpr int TAU_A = 26;
constexpr int TAU_B = 32;
constexpr int WIN = 2;
constexpr int DENSE_F = 0xfc;
constexpr int DENSE_M = 0xff;
constexpr int DENSE_L = 0x3f;

constexpr int DEPTH = 4;
constexpr int ADEPTH = DEPTH + 1;
constexpr int A_BYTES = BM * BK;
constexpr int B_BYTES = BK * BN;
constexpr int A_RING = DEPTH * B_BYTES;
constexpr int RING_BYTES = DEPTH * B_BYTES + ADEPTH * A_BYTES;
constexpr int PK_SLOT = 5120;
constexpr int LDS_RECV = 8 * PK_SLOT;
constexpr int LDS_TAB = 16 * PK_SLOT;
constexpr int LDS_HAND = RING_BYTES;
constexpr int LDS_MISC = LDS_HAND + 4 * 4096;
constexpr int LDS_BYTES = LDS_MISC + 16;
static_assert(LDS_TAB + 8 * 512 <= RING_BYTES && LDS_BYTES > 80 * 1024 && LDS_BYTES <= 160 * 1024, "LDS layout");

constexpr size_t WS_CTR = 0;
constexpr size_t WS_TMO = 32768;
constexpr size_t WS_XFLAG = 36864;
constexpr size_t WS_FLAG = 131072;
constexpr size_t WS_XSTAGE = WS_FLAG + 2 * 2048 * 128;
constexpr size_t WS_STAGE = WS_XSTAGE + (size_t)16 * BM * KK;
constexpr size_t WS_END = WS_STAGE + (size_t)2 * 2048 * PK_SLOT;

struct Params {
    const unsigned char* x;
    const unsigned char* w;
    unsigned char* myws;
    int rank;
    int pad;
    const float* sx;
    const float* sw;
    float* out;
    unsigned char* ws[NR];
};
static_assert(offsetof(Params, x) == 0 && offsetof(Params, w) == 8 && offsetof(Params, rank) == 24, "read as words in the kernel");

#define WAIT_VM(n) asm volatile("s_waitcnt vmcnt(%0)" :: "i"(n) : "memory")
#define WAIT_LGKM0() asm volatile("s_waitcnt lgkmcnt(0)" ::: "memory")
#define BAR() __builtin_amdgcn_s_barrier()
#define SCHED() __builtin_amdgcn_sched_barrier(0)

template <int S>
__device__ __forceinline__ int korder(int lead, int tn) {
    const int rs = (S + tn) & 3;
    const int kl = S < 8 ? lead + S : (lead + (S & ~3) + rs) & (NSTAGE - 1);
    return lead < 0 ? (S >> 2) + 8 * rs : kl;
}

struct LoadCtx {
    unsigned va[2], vb[4];
    const unsigned char* xa;
    const unsigned char* wb;
    unsigned ldsA, ldsB;
    int lead, tn;
    unsigned t0, tau;
};
__device__ __forceinline__ void load_w_at(const LoadCtx& c, const unsigned char* pb, int slot) {
    asm volatile("s_mov_b32 m0, %5\n\ts_nop 3\n\t"
                 "global_load_lds_dwordx4 %0, %4\n\tglobal_load_lds_dwordx4 %1, %4 offset:1024\n\t"
                 "global_load_lds_dwordx4 %2, %4 offset:2048\n\tglobal_load_lds_dwordx4 %3, %4 offset:3072"
                 :: "v"(c.vb[0]), "v"(c.vb[1]), "v"(c.vb[2]), "v"(c.vb[3]), "s"(pb), "s"(c.ldsB + slot * B_BYTES) : "memory");
}
__device__ __forceinline__ void load_x_at(const LoadCtx& c, const unsigned char* pa, int slot) {
    asm volatile("s_mov_b32 m0, %3\n\ts_nop 3\n\t"
                 "global_load_lds_dwordx4 %0, %2 sc0 sc1\n\tglobal_load_lds_dwordx4 %1, %2 offset:1024 sc0 sc1"
                 :: "v"(c.va[0]), "v"(c.va[1]), "s"(pa), "s"(c.ldsA + slot * A_BYTES) : "memory");
}
template <int S>
__device__ __forceinline__ void load_w(const LoadCtx& c) { load_w_at(c, c.wb + (unsigned)korder<S>(c.lead, c.tn) * (BK * NN), S % DEPTH); }
template <int S>
__device__ __forceinline__ void load_x(const LoadCtx& c) { load_x_at(c, c.xa + (unsigned)korder<S>(c.lead, c.tn) * BK, S % ADEPTH); }
constexpr int mini(int a, int b) { return a < b ? a : b; }
constexpr int issued(int I) { return mini(NSTAGE, mini(2 + 2 * I, I + DEPTH - 1)); }
constexpr int younger(int I) {
    return 4 * (issued(I) - 1 - I) + 2 * (mini(issued(I), NSTAGE - 1) - I);
}
template <int S, int E>
__device__ __forceinline__ void load_range(const LoadCtx& c) {
    if constexpr (S < E) {
        load_w<S>(c);
        if constexpr (S + 1 < NSTAGE) load_x<S + 1>(c);
        load_range<S + 1, E>(c);
    }
}
template <int I, int END>
__device__ __forceinline__ void loader_loop(const LoadCtx& c) {
    WAIT_VM(younger(I));
    BAR();
    if constexpr (I >= PACE0 && T_PACE0 + (I - PACE0) * TAU_A <= T_PACE1) {
        const unsigned due = T_PACE0 + (I - PACE0) * c.tau;
        if (c.lead >= 0 && due <= T_PACE1)
            while ((unsigned)__builtin_amdgcn_s_memrealtime() - c.t0 < due) __builtin_amdgcn_s_sleep(1);
    }
    if constexpr (I + 1 < NSTAGE) load_range<issued(I), issued(I + 1)>(c);
    if constexpr (I + 1 < END) loader_loop<I + 1, END>(c);
}

__device__ __forceinline__ unsigned wave_scan(unsigned v) {
    v += __builtin_amdgcn_update_dpp(0u, v, 0x111, 0xf, 0xf, true);
    v += __builtin_amdgcn_update_dpp(0u, v, 0x112, 0xf, 0xf, true);
    v += __builtin_amdgcn_update_dpp(0u, v, 0x114, 0xf, 0xf, true);
    v += __builtin_amdgcn_update_dpp(0u, v, 0x118, 0xf, 0xf, true);
    v += __builtin_amdgcn_update_dpp(0u, v, 0x142, 0xa, 0xf, false);
    v += __builtin_amdgcn_update_dpp(0u, v, 0x143, 0xc, 0xf, false);
    return v;
}

__device__ __forceinline__ unsigned draw_call(unsigned char* ctr) {
    unsigned t;
    asm volatile("s_mov_b64 exec, 1\n\ts_nop 3\n\tglobal_atomic_add %0, %1, %2, %3 sc0\n\ts_mov_b64 exec, -1\n\ts_waitcnt vmcnt(0)"
                 : "=&v"(t) : "v"(0u), "v"(1u), "s"(ctr) : "memory");
    return t + 1u;
}

__device__ __forceinline__ void touch(const unsigned char* base, unsigned voff, unsigned lds_dst) {
    asm volatile("s_mov_b32 m0, %2\n\ts_nop 3\n\tglobal_load_lds_dword %0, %1" :: "v"(voff), "s"(base), "s"(lds_dst) : "memory");
}

__device__ __forceinline__ void put_flag(unsigned char* flag, unsigned call, unsigned low) {
    typedef __attribute__((address_space(1))) unsigned long long gu64;
    __hip_atomic_store((gu64*)flag, ((unsigned long long)call << 32) | low, __ATOMIC_RELAXED, __HIP_MEMORY_SCOPE_SYSTEM);
}

__device__ __forceinline__ unsigned wait_flag(const unsigned char* flag, unsigned voff, unsigned* tmo, unsigned call, int lane) {
    const unsigned t0 = (unsigned)__builtin_amdgcn_s_memrealtime();
    unsigned low;
    bool seen;
    for (;;) {
        v2i f;
        unsigned t;
        asm volatile("s_nop 4\n\tglobal_load_dwordx2 %0, %2, %3 sc0 sc1\n\tglobal_load_dword %1, %4, %5 sc1\n\ts_waitcnt vmcnt(0)"
                     : "=&v"(f), "=&v"(t) : "v"(voff), "s"(flag), "v"(0u), "s"(tmo) : "memory");
        low = __builtin_amdgcn_readfirstlane(f[0]);
        seen = __builtin_amdgcn_ballot_w64((unsigned)f[1] < call) == 0ull;
        if (seen || __builtin_amdgcn_readfirstlane(t) == call || (unsigned)__builtin_amdgcn_s_memrealtime() - t0 > DEADLINE_TICKS) break;
        __builtin_amdgcn_s_sleep(2);
    }
    if (!seen && lane == 0) __hip_atomic_store(tmo, call, __ATOMIC_RELAXED, __HIP_MEMORY_SCOPE_AGENT);
    return seen ? low : 0xffffffffu;
}

__device__ __forceinline__ void store_out(float* ptr, u32x4 v) {
    asm volatile("global_store_dwordx4 %0, %1, off sc0 sc1\n\ts_nop 1" :: "v"(ptr), "v"(v) : "memory");
}

__device__ __forceinline__ unsigned compact(unsigned base, unsigned mask, const float (&y)[16], const unsigned long long (&nz)[16]) {
    const unsigned cnt = __builtin_popcount(mask);
    const unsigned incl = wave_scan(cnt);
    unsigned ptr = base + (incl - cnt) * 4;
#define PK1(m, v) "s_mov_b64 exec, %" #m "\n\tds_write_b32 %0, %" #v "\n\tv_add_u32 %0, 4, %0\n\t"
#pragma unroll
    for (int q = 0; q < 2; ++q)
        asm volatile(PK1(1, 9) PK1(2, 10) PK1(3, 11) PK1(4, 12) PK1(5, 13) PK1(6, 14) PK1(7, 15) PK1(8, 16)
                     "s_mov_b64 exec, -1"
                     : "+v"(ptr)
                     : "s"(nz[8 * q]), "s"(nz[8 * q + 1]), "s"(nz[8 * q + 2]), "s"(nz[8 * q + 3]),
                       "s"(nz[8 * q + 4]), "s"(nz[8 * q + 5]), "s"(nz[8 * q + 6]), "s"(nz[8 * q + 7]),
                       "v"(y[8 * q]), "v"(y[8 * q + 1]), "v"(y[8 * q + 2]), "v"(y[8 * q + 3]),
                       "v"(y[8 * q + 4]), "v"(y[8 * q + 5]), "v"(y[8 * q + 6]), "v"(y[8 * q + 7])
                     : "memory");
    return __builtin_amdgcn_readlane(incl, 63);
}

__device__ __forceinline__ int step4(int addr, int sel) {
    int r;
    asm("v_mad_i32_i24 %0, %1, -4, %2" : "=v"(r) : "v"(sel), "v"(addr));
    return r;
}

struct Frag { v4i a[2][2][2]; v2i b[2][4]; };

__device__ __forceinline__ v8i cat8(v4i lo, v4i hi) { return __builtin_shufflevector(lo, hi, 0, 1, 2, 3, 4, 5, 6, 7); }
__device__ __forceinline__ v8i cat8b(v2i a, v2i b, v2i c, v2i d) {
    return cat8(__builtin_shufflevector(a, b, 0, 1, 2, 3), __builtin_shufflevector(c, d, 0, 1, 2, 3));
}

__global__ void __launch_bounds__(512, 2) gemm_a2a(Params p) {
    __shared__ __attribute__((aligned(1024))) unsigned char lds[LDS_BYTES];
    v8i ka;
    asm volatile("s_load_dwordx8 %0, %1, 0x0" : "=s"(ka) : "s"(__builtin_amdgcn_kernarg_segment_ptr()));
    SCHED();
    LAS unsigned char* L = (LAS unsigned char*)lds;
    const unsigned lds0 = (unsigned)(size_t)L;
    const int tid = threadIdx.x, lane = tid & 63;
    const int wid = __builtin_amdgcn_readfirstlane(tid >> 6);
    const int b = blockIdx.x;
    const int g = b & 7, j = b >> 3, tm = j >> 2, tn = j & 3;
    LoadCtx c;
    {
        const int lw = wid & 3;
#pragma unroll
        for (int i = 0; i < 2; ++i) {
            const int row = (lw * 2 + i) * 8 + (lane >> 3);
            c.va[i] = row * KK + (((lane & 7) ^ ((row >> 1) & 7)) << 4) - i * 1024;
        }
#pragma unroll
        for (int i = 0; i < 4; ++i) {
            const int row = (lw * 4 + i) * 8 + (lane >> 3);
            c.vb[i] = row * NN + (((lane & 7) ^ (row & 6)) << 4) - i * 1024;
        }
        c.ldsA = lds0 + A_RING + lw * 2048;
        c.ldsB = lds0 + lw * 4096;
        c.tn = tn;
    }
    c.lead = tm >= LEAD0 ? 8 * (tm - LEAD0) : -1;
    int wcol = g * NS + tn * BN;
    unsigned xo = tm * BM * KK;
    unsigned x0 = xo + korder<0>(c.lead, tn) * BK, w0 = wcol + korder<0>(c.lead, tn) * BK * NN;
    int skip = tm == 7 || wid < 4 ? -1 : g;
    asm volatile("" : "+v"(c.va[0]), "+v"(c.va[1]), "+v"(c.vb[0]), "+v"(c.vb[1]), "+v"(c.vb[2]), "+v"(c.vb[3]));
    asm volatile("" : "+s"(x0), "+s"(w0), "+s"(skip), "+s"(c.ldsA), "+s"(c.ldsB));
    asm volatile("s_waitcnt lgkmcnt(0)" : "+s"(ka));
    const unsigned char* px = (const unsigned char*)(((unsigned long long)(unsigned)ka[1] << 32) | (unsigned)ka[0]);
    const unsigned char* pw = (const unsigned char*)(((unsigned long long)(unsigned)ka[3] << 32) | (unsigned)ka[2]);
    const int rank = ka[6];
    unsigned char* myws = p.myws;
    const unsigned char* xa = px + xo;
    if (__builtin_expect(skip >= 0 && skip != rank, 1)) {
        load_x_at(c, px + x0, 0);
        load_w_at(c, pw + w0, 0);
        c.xa = xa;
        c.wb = pw + wcol;
        load_x<1>(c);
        load_range<1, issued(0)>(c);
    }
    const bool early = g == rank || tm == 7;
    const unsigned t_begin = (unsigned)__builtin_amdgcn_s_memrealtime();
    c.t0 = t_begin;
    c.tau = tm >= LEAD0 + 2 ? TAU_B : TAU_A;
    unsigned* tmo = (unsigned*)(myws + WS_TMO);
    int orow = rank * MS + tm * BM, ocol = tn * BN;
    const bool local = early;
    bool lost = false;

    if (early) {
        u32x4 r[8];
        if (g != rank) {
            if (wid == 7) {
                touch((const unsigned char*)p.sx, 0u, lds0);
                touch((const unsigned char*)p.sw, 0u, lds0);
            }
            unsigned t;
            asm volatile("s_nop 4\n\t"
                         "global_load_dwordx4 %0, %9, %11\n\tglobal_load_dwordx4 %1, %9, %11 offset:1024\n\t"
                         "global_load_dwordx4 %2, %9, %11 offset:2048\n\tglobal_load_dwordx4 %3, %9, %11 offset:3072\n\t"
                         "global_load_dwordx4 %4, %10, %11\n\tglobal_load_dwordx4 %5, %10, %11 offset:1024\n\t"
                         "global_load_dwordx4 %6, %10, %11 offset:2048\n\tglobal_load_dwordx4 %7, %10, %11 offset:3072\n\t"
                         "s_cmp_lg_u32 %15, 0\n\ts_cbranch_scc1 1f\n\t"
                         "s_mov_b64 exec, 1\n\tglobal_atomic_add %8, %12, %13, %14 sc0\n\ts_mov_b64 exec, -1\n"
                         "1:\n\ts_waitcnt vmcnt(0)"
                         : "=&v"(r[0]), "=&v"(r[1]), "=&v"(r[2]), "=&v"(r[3]), "=&v"(r[4]), "=&v"(r[5]), "=&v"(r[6]), "=&v"(r[7]), "=&v"(t)
                         : "v"(lane * 16), "v"(lane * 16 + KK), "s"(xa + (tn * 16 + wid * 2) * KK),
                           "v"(0u), "v"(1u), "s"(myws + WS_CTR + b * 128), "s"(wid)
                         : "memory", "scc");
            if (tid == 0) *(LAS v4i*)(L + LDS_MISC) = (v4i){(int)(t + 1u), 0, 0, 0};
        } else {
            touch(px, (j * 512 + tid) * 128, lds0);
            while ((unsigned)__builtin_amdgcn_s_memrealtime() - t_begin < TWARM) __builtin_amdgcn_s_sleep(4);
            touch(pw + wcol, (tm * 512 + tid) * NN, lds0);
            if (wid == 0) {
                const unsigned t = draw_call(myws + WS_CTR + b * 128);
                if (lane == 0) *(LAS v4i*)(L + LDS_MISC) = (v4i){(int)t, 0, 0, 0};
            }
            WAIT_VM(0);
        }
        __syncthreads();
        const unsigned call = __builtin_amdgcn_readfirstlane(*(LAS unsigned*)(L + LDS_MISC));
        const unsigned par = call & 1u;
        if (g != rank) {
            unsigned char* pws = p.ws[g];
            unsigned char* dx = pws + WS_XSTAGE + (size_t)((par * 8 + rank) * BM + tn * 16 + wid * 2) * KK;
#pragma unroll
            for (int q = 0; q < 8; ++q)
                asm volatile("s_nop 4\n\tglobal_store_dwordx4 %0, %1, %2 sc0 sc1\n\ts_nop 1" :: "v"(lane * 16 + q * 1024), "v"(r[q]), "s"(dx) : "memory");
            WAIT_VM(0);
            __syncthreads();
            if (tid == 0) put_flag(pws + WS_XFLAG + ((par * 8 + rank) * 4 + tn) * 128, call, 0u);
            const int u = ((g - rank - 1) & 7) * 4 + tn;
            xa = px + (size_t)(u % 7) * BM * KK;
            ocol = (u / 7) * BN;
            wcol = rank * NS + ocol;
            orow = rank * MS + (u % 7) * BM;
        } else if (tm < 7) {
            const int src_rank = (rank + 1 + tm) & 7;
            if (wid == 0) {
                const unsigned f = wait_flag(myws + WS_XFLAG + (par * 8 + src_rank) * 512, (lane & 3) * 128, tmo, call, lane);
                const unsigned t = __hip_atomic_load(tmo, __ATOMIC_RELAXED, __HIP_MEMORY_SCOPE_AGENT);
                if (f != 0u || t == call) *(LAS unsigned*)(L + LDS_MISC + 4) = 1u;
            }
            __syncthreads();
            lost = __builtin_amdgcn_readfirstlane(*(LAS unsigned*)(L + LDS_MISC + 4)) != 0;
            xa = lost ? px : myws + WS_XSTAGE + (size_t)(par * 8 + src_rank) * BM * KK;
            orow = src_rank * MS + XROW0;
        }
    }

    v16f acc[2];
    unsigned char* dws = nullptr;

    if (wid >= 4) {
        if (early) {
            c.lead = -1;
            c.xa = xa;
            c.wb = pw + wcol;
            load_x<0>(c);
            load_range<0, issued(0)>(c);
        }
        dws = p.ws[g];
        asm volatile("" : "+s"(dws));
        loader_loop<0, NSTAGE - 2>(c);
        static_assert(issued(NSTAGE - 2) == NSTAGE && younger(NSTAGE - 2) == 6 && younger(NSTAGE - 1) == 0, "last two iterations");
        if (wid == 4) {
            unsigned t;
            float s0, s1;
            asm volatile("s_nop 4\n\tglobal_load_dword %1, %3, %6\n\tglobal_load_dword %2, %3, %7\n\t"
                         "s_mov_b64 exec, 1\n\tglobal_atomic_add %0, %3, %4, %5 sc0\n\ts_mov_b64 exec, -1\n\t"
                         "s_waitcnt vmcnt(9)\n\ts_barrier\n\ts_waitcnt vmcnt(3)\n\ts_barrier\n\ts_waitcnt vmcnt(0)"
                         : "=&v"(t), "=&v"(s0), "=&v"(s1)
                         : "v"(0u), "v"(early ? 0u : 1u), "s"(myws + WS_CTR + b * 128), "s"(p.sx), "s"(p.sw) : "memory");
            if (lane == 0)
                *(LAS v4i*)(L + LDS_MISC) = (v4i){(int)(t + 1u), 0, 0, lost ? 0x7fc00000 : __builtin_bit_cast(int, s0 * s1)};
            WAIT_LGKM0();
        } else {
            loader_loop<NSTAGE - 2, NSTAGE>(c);
        }
    } else {
        dws = p.ws[g];
        asm volatile("" : "+s"(dws));
        acc[0] = acc[1] = (v16f)0.f;
        const int r = lane & 31, h = lane >> 5;
        unsigned ao[2][2];
#pragma unroll
        for (int ks = 0; ks < 2; ++ks)
#pragma unroll
            for (int e = 0; e < 2; ++e) ao[ks][e] = A_RING + r * 128 + (((4 * ks + 2 * h + e) ^ ((r >> 1) & 7)) << 4);
        const int q = (lane & 15) >> 1, pp = lane & 1, g1 = (lane >> 4) & 1;
        const unsigned bo = (32 * h + q) * 128 + (((2 * wid + g1) ^ (q & 6)) << 4) + pp * 8;

        auto rd = [&](Frag& f, int slot, int aslot, int ks) {
            const unsigned char LAS* base = L + slot * B_BYTES;
            const unsigned char LAS* abase = L + aslot * A_BYTES;
#pragma unroll
            for (int t = 0; t < 4; ++t)
                f.b[ks][t] = __builtin_amdgcn_ds_read_tr8_b64_v2i32((LAS v2i*)(base + bo + (64 * ks + 8 * t) * 128));
#pragma unroll
            for (int mt = 0; mt < 2; ++mt)
#pragma unroll
                for (int e = 0; e < 2; ++e) f.a[ks][mt][e] = *(const LAS v4i*)(abase + ao[ks][e] + mt * 4096);
        };
        auto mm = [&](const Frag& f, int ks, int mt) {
            acc[mt] = __builtin_amdgcn_mfma_scale_f32_32x32x64_f8f6f4(cat8b(f.b[ks][0], f.b[ks][1], f.b[ks][2], f.b[ks][3]),
                                                                      cat8(f.a[ks][mt][0], f.a[ks][mt][1]), acc[mt], 0, 0, 0, 0, 0, 0);
        };
        auto step = [&](Frag& fn, const Frag& fo, int slot, int aslot) {
            BAR(); SCHED();
            mm(fo, 0, 0); SCHED();
            rd(fn, slot, aslot, 0); SCHED();
            mm(fo, 0, 1); SCHED();
            rd(fn, slot, aslot, 1); SCHED();
            mm(fo, 1, 0); mm(fo, 1, 1); SCHED();
            WAIT_LGKM0(); SCHED();
        };
        Frag f0, f1;
        BAR();
        rd(f0, 0, 0, 0);
        rd(f0, 0, 0, 1);
        WAIT_LGKM0();
        SCHED();
        int slot = 1, aslot = 1;
#pragma unroll 1
        for (int i = 1; i < NSTAGE - 1; i += 2) {
            step(f1, f0, slot, aslot);
            slot = slot + 1 == DEPTH ? 0 : slot + 1;
            aslot = aslot + 1 == ADEPTH ? 0 : aslot + 1;
            step(f0, f1, slot, aslot);
            slot = slot + 1 == DEPTH ? 0 : slot + 1;
            aslot = aslot + 1 == ADEPTH ? 0 : aslot + 1;
        }
        step(f1, f0, slot, aslot);
        mm(f1, 0, 0); mm(f1, 0, 1); mm(f1, 1, 0); mm(f1, 1, 1);
        if (!local) {
#pragma unroll
            for (int gq = 0; gq < 4; ++gq)
                *(LAS v4f*)(L + LDS_HAND + wid * 4096 + gq * 1024 + lane * 16) =
                    (v4f){acc[1][4 * gq], acc[1][4 * gq + 1], acc[1][4 * gq + 2], acc[1][4 * gq + 3]};
        }
        WAIT_LGKM0();
    }
    BAR();
    const v4i misc = *(LAS v4i*)(L + LDS_MISC);
    const float scale = __builtin_bit_cast(float, __builtin_amdgcn_readfirstlane(misc[3]));

    if (local) {
        if (wid < 4) {
            const int r = lane & 31, h = lane >> 5;
#pragma unroll
            for (int mt = 0; mt < 2; ++mt)
#pragma unroll
                for (int gq = 0; gq < 4; ++gq) {
                    v4f v;
#pragma unroll
                    for (int e = 0; e < 4; ++e) { const float y = acc[mt][4 * gq + e] * scale; v[e] = y <= 0.f ? 0.f : y; }
                    const int row = mt * 32 + r, ch = wid * 8 + 2 * gq + h;
                    *(LAS v4f*)(L + row * 512 + ((ch ^ (row & 7)) << 4)) = v;
                }
        }
        __syncthreads();
        float* ob = p.out + (size_t)orow * NS + ocol;
#pragma unroll
        for (int u = 0; u < 4; ++u) {
            const int row = (wid * 4 + u) * 2 + (lane >> 5), ch = lane & 31;
            store_out(ob + row * NS + ch * 4, *(LAS u32x4*)(L + row * 512 + ((ch ^ (row & 7)) << 4)));
        }
    } else {
        const unsigned lo = lane * 16;
        if (wid >= 4) {
            const LAS v4f* h = (const LAS v4f*)(L + LDS_HAND + (wid - 4) * 4096 + lo);
            acc[0] = __builtin_shufflevector(__builtin_shufflevector(h[0], h[64], 0, 1, 2, 3, 4, 5, 6, 7),
                                             __builtin_shufflevector(h[128], h[192], 0, 1, 2, 3, 4, 5, 6, 7),
                                             0, 1, 2, 3, 4, 5, 6, 7, 8, 9, 10, 11, 12, 13, 14, 15);
        }
        unsigned total = 0;
        bool dense = (((tm >= LEAD0 + 2 ? DENSE_L : (tm >= LEAD0 ? DENSE_M : DENSE_F)) >> wid) & 1) &&
                     __builtin_bit_cast(unsigned, scale) - 0x0d800000u < 0x71800000u - 0x0d800000u;
        if (!dense) __builtin_amdgcn_s_setprio(3);
        if (dense) {
            float y[16];
            unsigned long long nz[16];
            unsigned mask = 0, m1 = 0, m2 = 0, inexact = 0;
            float top = 0.f;
#pragma unroll
            for (int q = 3; q >= 0; --q) {
                unsigned long long s1[4], s2[4];
#pragma unroll
                for (int e = 0; e < 4; ++e) {
                    const float a = acc[0][4 * q + e], am = __builtin_elementwise_minimum(a, 32.f), u = am + 32.f;
                    nz[4 * q + e] = __builtin_amdgcn_fcmpf(a, 0.f, 2  );
                    s1[e] = __builtin_amdgcn_fcmpf(a, 32.f, 3  );
                    s2[e] = __builtin_amdgcn_fcmpf(a, 128.f, 3);
                    inexact = __builtin_amdgcn_bitop3_b32(inexact, __builtin_bit_cast(unsigned, u - 32.f), __builtin_bit_cast(unsigned, am), 0xf6);
                    y[4 * q + e] = __builtin_amdgcn_inverse_ballot_w64(s1[e]) ? a : u;
                    top = __builtin_elementwise_maximum(top, a);
                }
#define AC(d, m) "v_addc_co_u32_e64 %" #d ", vcc, %" #d ", %" #d ", %" #m "\n\t"
                asm volatile("s_nop 1\n\t" AC(0, 6) AC(1, 10) AC(2, 14) AC(0, 5) AC(1, 9) AC(2, 13) AC(0, 4) AC(1, 8) AC(2, 12) AC(0, 3) AC(1, 7) AC(2, 11)
                             : "+v"(mask), "+v"(m1), "+v"(m2)
                             : "s"(nz[4 * q]), "s"(nz[4 * q + 1]), "s"(nz[4 * q + 2]), "s"(nz[4 * q + 3]),
                               "s"(s1[0]), "s"(s1[1]), "s"(s1[2]), "s"(s1[3]), "s"(s2[0]), "s"(s2[1]), "s"(s2[2]), "s"(s2[3])
                             : "vcc");
            }
            dense = __builtin_amdgcn_ballot_w64((inexact << 1) != 0u || !(top < 512.f)) == 0ull;
            if (dense) {
                *(LAS unsigned*)(L + wid * PK_SLOT + lane * 4) = (mask ^ m1 ^ m2) | (m1 << 16);
                total = compact(lds0 + LDS_RECV + wid * PK_SLOT, mask, y, nz);
#pragma unroll 1
                for (unsigned o = 0; o < total; o += 256) {
                    const u32x4 v = *(LAS u32x4*)(L + LDS_RECV + wid * PK_SLOT + o * 4 + lo);
                    LAS unsigned* d = (LAS unsigned*)(L + wid * PK_SLOT + 256 + o * 3 + lane * 12);
                    d[0] = __builtin_amdgcn_perm(v[1], v[0], 0x04020100u);
                    d[1] = __builtin_amdgcn_perm(v[2], v[1], 0x05040201u);
                    d[2] = __builtin_amdgcn_perm(v[3], v[2], 0x06050402u);
                }
            }
        }
        if (!dense) {
            float y[16];
            unsigned long long nz[16];
#pragma unroll
            for (int i = 0; i < 16; ++i) {
                y[i] = acc[0][i] * scale;
                nz[i] = __builtin_amdgcn_fcmpf(y[i], 0.f, 10  );
            }
            unsigned mask = 0;
#pragma unroll
            for (int q = 1; q >= 0; --q)
                asm volatile("s_nop 1\n\t" AC(0, 8) AC(0, 7) AC(0, 6) AC(0, 5) AC(0, 4) AC(0, 3) AC(0, 2) AC(0, 1)
                             : "+v"(mask)
                             : "s"(nz[8 * q]), "s"(nz[8 * q + 1]), "s"(nz[8 * q + 2]), "s"(nz[8 * q + 3]),
                               "s"(nz[8 * q + 4]), "s"(nz[8 * q + 5]), "s"(nz[8 * q + 6]), "s"(nz[8 * q + 7])
                             : "vcc");
            *(LAS unsigned short*)(L + wid * PK_SLOT + lane * 2) = (unsigned short)mask;
            total = compact(lds0 + wid * PK_SLOT + 128, mask, y, nz);
        }
        u32x4 c[3];
#pragma unroll
        for (int u = 0; u < 3; ++u) c[u] = *(LAS u32x4*)(L + wid * PK_SLOT + u * 1024 + lo);
        const unsigned call = __builtin_amdgcn_readfirstlane(misc[0]);
        const unsigned par = call & 1u;
        const unsigned bytes = dense ? 256 + 3 * total : 128 + 4 * total;
        const unsigned sidx = ((par * 8 + rank) * 32 + j) * 8 + wid;
        unsigned char* dslot = dws + WS_STAGE + (size_t)sidx * PK_SLOT;
        const int turn = wid - WIN;
        if (turn > 0)
            while ((int)__builtin_amdgcn_readfirstlane(__hip_atomic_load((LAS unsigned*)(L + LDS_MISC + 8), __ATOMIC_RELAXED,
                                                                         __HIP_MEMORY_SCOPE_WORKGROUP)) < turn)
                __builtin_amdgcn_s_sleep(1);
#pragma unroll
        for (int u = 0; u < 3; ++u)
            if (u * 1024 + lo < bytes)
                asm volatile("s_nop 4\n\tglobal_store_dwordx4 %0, %1, %2 sc0 sc1\n\ts_nop 1" :: "v"(lo + u * 1024), "v"(c[u]), "s"(dslot) : "memory");
#pragma unroll 1
        for (unsigned o = 3 * 1024; o < bytes; o += 1024) {
            const u32x4 v = *(LAS u32x4*)(L + wid * PK_SLOT + o + lo);
            if (o + lo < bytes)
                asm volatile("s_nop 4\n\tglobal_store_dwordx4 %0, %1, %2 sc0 sc1\n\ts_nop 1" :: "v"(lo + o), "v"(v), "s"(dslot) : "memory");
        }
        WAIT_VM(0);
        if (lane == 0) {
            put_flag(dws + WS_FLAG + (size_t)sidx * 128, call, total | (dense ? 0x10000u : 0u));
            __hip_atomic_fetch_add((LAS unsigned*)(L + LDS_MISC + 8), 1u, __ATOMIC_RELAXED, __HIP_MEMORY_SCOPE_WORKGROUP);
        }
        const unsigned ridx = ((par * 8 + g) * 32 + j) * 8 + wid;
        const unsigned rword = wait_flag(myws + WS_FLAG + (size_t)ridx * 128, 0u, tmo, call, lane);
        const unsigned rtotal = __builtin_amdgcn_readfirstlane(rword) & ~0x10000u;
        const bool rdense = __builtin_amdgcn_readfirstlane(rword) & 0x10000u;
        u32x4 o[4];
        bool bad = true;
        if (rtotal <= 1024u) {
            const unsigned rbytes = rdense ? 256 + 3 * rtotal : 128 + 4 * rtotal;
            const unsigned char* sslot = myws + WS_STAGE + (size_t)ridx * PK_SLOT;
            const unsigned rdst = __builtin_amdgcn_readfirstlane(lds0 + LDS_RECV + wid * PK_SLOT);
            asm volatile("s_mov_b32 m0, %2\n\ts_nop 3\n\tglobal_load_lds_dwordx4 %0, %1 sc0 sc1\n\t"
                         "s_cmp_le_u32 %3, 1024\n\ts_cbranch_scc1 1f\n\tglobal_load_lds_dwordx4 %0, %1 offset:1024 sc0 sc1\n\t"
                         "s_cmp_le_u32 %3, 2048\n\ts_cbranch_scc1 1f\n\tglobal_load_lds_dwordx4 %0, %1 offset:2048 sc0 sc1\n\t"
                         "s_cmp_le_u32 %3, 3072\n\ts_cbranch_scc1 1f\n\tglobal_load_lds_dwordx4 %0, %1 offset:3072 sc0 sc1\n\t"
                         "s_cmp_le_u32 %3, 4096\n\ts_cbranch_scc1 1f\n\ts_mov_b32 m0, %5\n\ts_nop 0\n\tglobal_load_lds_dwordx4 %4, %1 sc0 sc1\n"
                         "1:"
                         :: "v"(lo), "s"(sslot), "s"(rdst), "s"(rbytes), "v"(lo + 4096), "s"(rdst + 4096) : "memory", "scc");
            unsigned t;
            asm volatile("s_nop 4\n\tglobal_load_dword %0, %1, %2 sc1\n\ts_waitcnt vmcnt(0)" : "=&v"(t) : "v"(0u), "s"(tmo) : "memory");
            bad = __builtin_amdgcn_readfirstlane(t) == call;
            const unsigned bp = 4 * ((lane & 7) >> 1);
            LAS v2i* tab = (LAS v2i*)(L + LDS_TAB + wid * 512);
            if (!rdense) {
                const unsigned rm = *(LAS unsigned short*)(L + LDS_RECV + wid * PK_SLOT + lane * 2);
                const unsigned rc = __builtin_popcount(rm);
                tab[lane] = (v2i){(int)rm, (int)(LDS_RECV + wid * PK_SLOT + 128 + (wave_scan(rc) - rc) * 4)};
#pragma unroll
                for (int u = 0; u < 4; ++u) {
                    const v2i tb = tab[8 * u + (lane >> 3) + 32 * (lane & 1)];
                    const unsigned m = (unsigned)tb[0];
                    int addr = tb[1] + 4 * __builtin_popcount(m & ((1u << bp) - 1u));
                    const int ms = (int)(m >> bp);
#pragma unroll
                    for (int e = 0; e < 4; ++e) {
                        const int sel = __builtin_amdgcn_sbfe(ms, e, 1);
                        o[u][e] = *(LAS unsigned*)(L + addr) & (unsigned)sel;
                        if (e < 3) addr = step4(addr, sel);
                    }
                }
            } else {
                const unsigned rm = *(LAS unsigned*)(L + LDS_RECV + wid * PK_SLOT + lane * 4);
#pragma unroll 1
                for (unsigned q = 0; q <= rtotal; q += 256) {
                    const LAS unsigned* d = (const LAS unsigned*)(L + LDS_RECV + wid * PK_SLOT + 256 + q * 3 + lane * 12);
                    const unsigned d0 = d[0], d1 = d[1], d2 = d[2];
                    *(LAS u32x4*)(L + wid * PK_SLOT + q * 4 + lo) =
                        (u32x4){(d0 & 0xffffffu) | 0x42000000u, __builtin_amdgcn_perm(d1, d0, 0x0c050403u) | 0x42000000u,
                                __builtin_amdgcn_perm(d2, d1, 0x0c040302u) | 0x42000000u, __builtin_amdgcn_alignbyte(0x42u, d2, 1)};
                }
                const unsigned rc = __builtin_popcount((rm | (rm >> 16)) & 0xffffu);
                tab[lane] = (v2i){(int)rm, (int)(wid * PK_SLOT + (wave_scan(rc) - rc) * 4)};
                const unsigned sb = __builtin_bit_cast(unsigned, scale), nb = __builtin_bit_cast(unsigned, -32.f * scale);
#pragma unroll
                for (int u = 0; u < 4; ++u) {
                    const v2i tb = tab[8 * u + (lane >> 3) + 32 * (lane & 1)];
                    const unsigned m = (unsigned)tb[0];
                    int addr = tb[1] + 4 * __builtin_popcount((m | (m >> 16)) & ((1u << bp) - 1u));
                    const int ms = (int)(m >> bp);
#pragma unroll
                    for (int e = 0; e < 4; ++e) {
                        const unsigned b0 = __builtin_amdgcn_sbfe(ms, e, 1), b1 = __builtin_amdgcn_sbfe(ms, 16 + e, 1);
                        const unsigned code = *(LAS unsigned*)(L + addr) | __builtin_amdgcn_bitop3_b32(b0, b1, 0x01000000u, 0x80);
                        o[u][e] = __builtin_bit_cast(unsigned, __builtin_fmaf(__builtin_bit_cast(float, code),
                                                                              __builtin_bit_cast(float, __builtin_amdgcn_bitop3_b32(b0, b1, sb, 0xa8)),
                                                                              __builtin_bit_cast(float, __builtin_amdgcn_bitop3_b32(b0, b1, nb, 0x20))));
                        if (e < 3) addr = step4(addr, (int)(b0 | b1));
                    }
                }
            }
        }
        float* ob = p.out + (size_t)(g * MS + tm * BM + (wid >> 2) * 32 + (lane >> 3)) * NS + tn * BN + (wid & 3) * 32 + (lane & 7) * 4;
#pragma unroll
        for (int u = 0; u < 4; ++u) {
            if (bad) o[u] = (u32x4){0x7fc00000u, 0x7fc00000u, 0x7fc00000u, 0x7fc00000u};
            store_out(ob + u * 8 * NS, o[u]);
        }
    }
}

extern "C" void kernel_launch(int rank, int nranks, ncclComm_t comm, hipStream_t stream,
                              void* const* d_in, const int* in_sizes, int n_in,
                              void* d_out, int out_size, const int* in_dtypes, int out_dtype,
                              void* const* d_out_all, void* const* d_ws_all, size_t ws_bytes) {
    if (nranks != NR || n_in != 4 || in_sizes[0] != MS * KK || in_sizes[1] != KK * NN || out_size != NR * MS * NS ||
        in_dtypes[0] != 3 || in_dtypes[1] != 3 || out_dtype != 0 || (unsigned)rank >= (unsigned)NR || ws_bytes < WS_END) {
        fprintf(stderr, "unexpected problem shape\n");
        return;
    }
    Params p;
    p.x = (const unsigned char*)d_in[0];
    p.w = (const unsigned char*)d_in[1];
    p.sx = (const float*)d_in[2];
    p.sw = (const float*)d_in[3];
    p.out = (float*)d_out;
    p.myws = (unsigned char*)d_ws_all[rank];
    for (int i = 0; i < NR; ++i) p.ws[i] = (unsigned char*)d_ws_all[i];
    p.rank = rank;
    p.pad = 0;
    gemm_a2a<<<GRID, 512, 0, stream>>>(p);
    HIP_CHECK(hipGetLastError());
}
```
